```python
import numpy as np

kernel_name = "gpu_gemm_rs_m4096_k4096_n32768_f32_none"


ABSMAX_RTOL = 0.001

COMPUTE_DTYPE = "float32"
EPILOGUE = "none"
IN_QUANT = None
OUT_QUANT = None
IN_DTYPES = {"x": "float32", "w_mat": "float32"}
WORLD_SIZE = 8

def setup_inputs():

    x = np.random.randn(4096, 4096).astype(np.float32)
    w_mat = (np.random.randn(4096, 32768) / np.sqrt(4096)).astype(np.float32)
    return {"x": x, "w_mat": w_mat}

def _epilogue(y):
    return y

def reference(x, w_mat):
    y = (x @ w_mat).astype(np.float32)
    return _epilogue(y)
```

```cpp
#include <hip/hip_runtime.h>
#include <rccl/rccl.h>
#include <cstdio>
#include <cstdint>

#define HIP_CHECK(expr) do { hipError_t _e = (expr); if (_e != hipSuccess) { \
    fprintf(stderr, "HIP error %s at %s:%d\n", hipGetErrorString(_e), __FILE__, __LINE__); \
    return; } } while (0)

#define LAS __attribute__((address_space(3)))
#define GAS __attribute__((address_space(1)))
typedef GAS char gchar;
typedef GAS unsigned gu32;
typedef _Float16 h16x8 __attribute__((ext_vector_type(8)));
typedef float f32x4 __attribute__((ext_vector_type(4)));
typedef float f32x2 __attribute__((ext_vector_type(2)));
typedef unsigned u32x4 __attribute__((ext_vector_type(4)));

constexpr int NR = 8;
constexpr int GM = 4096, GN = 32768, KS = 512, NL = 4096;
constexpr int GRID = 256, NTHR = 512;
constexpr unsigned UNIT = 32768;
constexpr unsigned HTB = 16384;
constexpr int NKT = 64;
constexpr size_t WS_TICKET = 0, WS_TMO = 128, WS_DONE = 1024  ;
constexpr size_t WS_FLAG = 4096;
constexpr size_t WS_DIAG = 65536;
constexpr size_t WS_STAGE = 1u << 20;
constexpr size_t OPND_BYTES = (size_t)16 * NKT * UNIT;
constexpr size_t WS_NEED = WS_STAGE + 4 * OPND_BYTES;
constexpr unsigned long long DEADLINE = 100000000ull;
constexpr unsigned L_CTL = 8 * HTB;
constexpr unsigned L_AMAX = L_CTL, L_CLS = L_CTL + 32, L_CALL = L_CTL + 48, L_HDR = L_CTL + 64, L_TAB = L_CTL + 128, LDS_BYTES = L_CTL + 512;

#define GLDS_AUX 17
struct Params {
    const float* x; const float* w;
    float* out[NR]; char* ws[NR];
    int rank; int pad;
};

#define STAMP(i) do {} while (0)

__device__ __forceinline__ void store16_sys(const gchar* sbase, unsigned voff, u32x4 v) {
    asm volatile("global_store_dwordx4 %0, %1, %2 sc0 sc1\n\ts_nop 1" :: "v"(voff), "v"(v), "s"(sbase) : "memory");
}
__device__ __forceinline__ unsigned ld_sys(const gu32* q) { return __hip_atomic_load(q, __ATOMIC_RELAXED, __HIP_MEMORY_SCOPE_SYSTEM); }
__device__ __forceinline__ unsigned ld_agt(const gu32* q) { return __hip_atomic_load(q, __ATOMIC_RELAXED, __HIP_MEMORY_SCOPE_AGENT); }
__device__ __forceinline__ unsigned rfl(unsigned v) { return (unsigned)__builtin_amdgcn_readfirstlane((int)v); }

template <class T> __device__ __forceinline__ gchar* uni(T q) {
    unsigned lo = rfl((unsigned)(uintptr_t)q), hi = rfl((unsigned)((uintptr_t)q >> 32));
    asm volatile("s_nop 4" : "+s"(lo), "+s"(hi));
    return (gchar*)(((uintptr_t)hi << 32) | lo);
}

__device__ __forceinline__ void item_decode(int i, int d, int& isW, int& j, int& u) {
    if (i < 224) { j = i / 28; const int r = i - j * 28; isW = r & 1; u = r >> 1; if (!isW && u >= 2 * d) u += 2; }
    else { const int r = i - 224; j = r >> 2; isW = (r >> 1) & 1; u = (isW ? 14 : 2 * d) + (r & 1); }
}

__device__ __forceinline__ void item_load(const Params& p, int i, int d, f32x4 (&L)[16]) {
    const bool valid = i < 256;
    int isW, j, u; item_decode(valid ? i : 0, d, isW, j, u);
    const unsigned t = threadIdx.x;
    const gchar* b = uni(isW ? (const char*)p.w + ((size_t)(j * 64) * GN + d * NL + u * 256) * 4 : (const char*)p.x + ((size_t)(u * 256) * KS + j * 64) * 4);
    unsigned vo = isW ? (t >> 6) * (4u * GN * 4u) + (t & 63u) * 16u : (t >> 3) * (KS * 4u) + ((t & 7u) ^ ((t >> 4) & 7u)) * 16u;
    if (!valid) vo = 0;
    const unsigned s0 = isW ? GN * 4u : 128u, s1 = isW ? 2u * GN * 4u : 64u * KS * 4u, s2 = isW ? 32u * GN * 4u : 128u * KS * 4u, s3 = isW ? 16u * GN * 4u : 32u * KS * 4u;
#pragma unroll
    for (int n = 0; n < 16; ++n)
        L[n] = *(const GAS f32x4*)(b + ((n & 1) * s0 + ((n >> 1) & 1) * s1 + ((n >> 2) & 1) * s2 + (n >> 3) * s3) + vo);
}

__device__ __forceinline__ unsigned pk2(float a, float b, int e) {
    typedef _Float16 h2 __attribute__((ext_vector_type(2)));
    h2 h; h.x = (_Float16)__builtin_amdgcn_ldexpf(a, e); h.y = (_Float16)__builtin_amdgcn_ldexpf(b, e);
    return __builtin_bit_cast(unsigned, h);
}

#define POLL_SLEEP 8
#define PARMASK 1u
#define THR_A 0
#define THR_C 6
#define PUSH_N 1
#define STR2(x) #x
#define STR(x) STR2(x)
#define LBAR do { asm volatile("s_waitcnt lgkmcnt(0)" ::: "memory"); __builtin_amdgcn_s_barrier(); asm volatile("" ::: "memory"); } while (0)

__device__ __forceinline__ void item_amax(LAS unsigned char* lds, const f32x4 (&L)[16], unsigned ph) {
    unsigned mu = 0;
#pragma unroll
    for (int n = 0; n < 16; ++n) { const u32x4 q = __builtin_bit_cast(u32x4, L[n]) & 0x7fffffffu; mu = max(max(mu, q.x), max(q.y, max(q.z, q.w))); }
    int mb = (int)(mu >> 23);
    mb = max(mb, __builtin_amdgcn_update_dpp(0, mb, 0x111, 0xf, 0xf, false));
    mb = max(mb, __builtin_amdgcn_update_dpp(0, mb, 0x112, 0xf, 0xf, false));
    mb = max(mb, __builtin_amdgcn_update_dpp(0, mb, 0x114, 0xf, 0xf, false));
    mb = max(mb, __builtin_amdgcn_update_dpp(0, mb, 0x118, 0xf, 0xf, false));
    mb = max(mb, __builtin_amdgcn_update_dpp(0, mb, 0x142, 0xa, 0xf, false));
    mb = max(mb, __builtin_amdgcn_update_dpp(0, mb, 0x143, 0xc, 0xf, false));
    if ((threadIdx.x & 63) == 63) *(LAS int*)(lds + L_AMAX + ph * 16 + (threadIdx.x >> 6) * 4) = mb;
}

__device__ __forceinline__ void conv_step(const Params& p, LAS unsigned char* lds, int d, int i, int stride, unsigned k, f32x4 (&L)[16], const f32x4 (&Ln)[16]) {
    const unsigned t = threadIdx.x, ph = k & 1u;
    const u32x4 am = *(const LAS u32x4*)(lds + L_AMAX + ph * 16);
    const int cls = (int)(max(max(am.x, am.y), max(am.z, am.w)) + 2u) >> 4;
    const int e = 128 - 16 * cls;
    if (t == 0) *(LAS int*)(lds + L_CLS + (k & 3u) * 4) = cls;
    int isW, j, u; item_decode(i, d, isW, j, u);
    LAS unsigned char* img = lds + ph * UNIT;
#pragma unroll
    for (int h = 0; h < 2; ++h) {
        const f32x4* Q = L + 8 * h;
        u32x4 H[4];
        if (isW) {
#pragma unroll
            for (int c = 0; c < 4; ++c) { H[c].x = pk2(Q[0][c], Q[1][c], e); H[c].y = pk2(Q[2][c], Q[3][c], e); H[c].z = pk2(Q[4][c], Q[5][c], e); H[c].w = pk2(Q[6][c], Q[7][c], e); }
        } else {
#pragma unroll
            for (int c = 0; c < 4; ++c) { H[c].x = pk2(Q[2 * c].x, Q[2 * c].y, e); H[c].y = pk2(Q[2 * c].z, Q[2 * c].w, e); H[c].z = pk2(Q[2 * c + 1].x, Q[2 * c + 1].y, e); H[c].w = pk2(Q[2 * c + 1].z, Q[2 * c + 1].w, e); }
        }
        const unsigned vo = isW ? (t >> 6) * 2048u + (t & 63u) * 16u + h * 8192u : t * 16u + h * 4096u;
        const unsigned c1 = isW ? 1024u : 8192u;
#pragma unroll
        for (int c = 0; c < 4; ++c) *(LAS u32x4*)(img + vo + (c >> 1) * HTB + (c & 1) * c1) = H[c];
    }
    item_load(p, i + 2 * stride, d, L);
    item_amax(lds, Ln, ph ^ 1u);
}

__device__ __forceinline__ gu32* item_flag(gchar* dws, int rank, int d, int i, unsigned par, const gchar*& ub) {
    int isW, j, u; item_decode(i, d, isW, j, u);
    const int srel = (rank - d - 1) & 7;
    const int idx = (srel == 7) ? j : 8 + j * 7 + srel;
    ub = dws + WS_STAGE + (size_t)(par * 2 + isW) * OPND_BYTES + (size_t)(u * NKT + idx) * UNIT;
    return (gu32*)(dws + WS_FLAG) + ((par * 2 + isW) * 16 + u) * NKT + idx;
}

__device__ __forceinline__ unsigned push_items(const Params& p, LAS unsigned char* lds, gchar* myws, int rank, int d, int first, int stride) {
    const int n = (256 - first + stride - 1) / stride;
    unsigned call;
    if (rfl(threadIdx.x >> 6) < 4) {
        f32x4 RA[16], RB[16];
        item_load(p, first, d, RA); item_load(p, first + stride, d, RB);
        item_amax(lds, RA, 0);
        LBAR;
        call = rfl(*(LAS unsigned*)(lds + L_CALL));
        for (int k = 0, i = first;;) {
            conv_step(p, lds, d, i, stride, k, RA, RB); LBAR; ++k; i += stride; if (k >= n) break;
            conv_step(p, lds, d, i, stride, k, RB, RA); LBAR; ++k; i += stride; if (k >= n) break;
        }
        LBAR;
    } else {
        if (threadIdx.x == 256) {
            const unsigned tk = __hip_atomic_fetch_add((gu32*)(myws + WS_TICKET), 1u, __ATOMIC_RELAXED, __HIP_MEMORY_SCOPE_AGENT);
            *(LAS unsigned*)(lds + L_CALL) = (tk / GRID + 1u) & 0xffffffu;
        }
        gchar* dws = uni(p.ws[d]);
        LBAR;
        call = rfl(*(LAS unsigned*)(lds + L_CALL));
        const unsigned par = call & PARMASK, vo = (threadIdx.x - 256u) * 16u;
        gu32* pf = nullptr;
        LBAR;
        for (int k = 0, i = first; k < n; ++k, i += stride) {
            const gchar* ub; gu32* f = item_flag(dws, rank, d, i, par, ub); ub = uni(ub);
            u32x4 H[8];
#pragma unroll
            for (int c = 0; c < 8; ++c) H[c] = *(const LAS u32x4*)(lds + (k & 1) * UNIT + c * 4096u + vo);
#pragma unroll
            for (int c = 0; c < 8; ++c) { store16_sys(ub + c * 4096u, vo, H[c]); asm volatile("s_waitcnt vmcnt(" STR(PUSH_N) ")" ::: "memory"); }
            if (k == n - 1) asm volatile("s_waitcnt vmcnt(0)" ::: "memory");
            LBAR;
            if (threadIdx.x == 256) {
                if (pf) __hip_atomic_store(pf, (call << 8) | *(const LAS unsigned*)(lds + L_CLS + ((k - 1) & 3) * 4), __ATOMIC_RELAXED, __HIP_MEMORY_SCOPE_SYSTEM);
                if (k == n - 1) __hip_atomic_store(f, (call << 8) | *(const LAS unsigned*)(lds + L_CLS + (k & 3) * 4), __ATOMIC_RELAXED, __HIP_MEMORY_SCOPE_SYSTEM);
            }
            pf = f;
        }
    }
    return call;
}

__global__ void __launch_bounds__(NTHR, 2) gemm_rs_kernel(const Params p) {
    extern __shared__ __attribute__((aligned(16))) unsigned char smem[];
    LAS unsigned char* lds = (LAS unsigned char*)smem;
    const int tid = threadIdx.x;
    const int rank = (int)rfl(p.rank), bid = (int)rfl(blockIdx.x);
    gchar* myws = uni(p.ws[rank]);
    STAMP(0);
    const int xcd = bid & 7, slot = bid >> 3;
    const int pd = slot < 7 ? (rank + 1 + slot) & 7 : rank, pfirst = slot < 7 ? xcd : (slot - 7) * 8 + xcd, pstride = slot < 7 ? 8 : 200;
    gu32* tmo = (gu32*)(myws + WS_TMO);

    int pm, pn;
    {
        int ar = -1;
        if (slot < 4) { pm = 2 * rank + (slot & 1); pn = 2 * xcd + (slot >> 1); }
        else if (slot < 7 || (slot == 31 && (xcd & 1))) {
            const int k = 7 * (xcd >> 1) + (slot == 31 ? 6 : (xcd & 1) * 3 + slot - 4);
            ar = k >> 1; pn = 14 + (k & 1);
        } else {
            const int q = xcd & 3, lin = (q * 49 + 1) / 2 + slot - 7;
            pn = lin / 7; ar = (xcd >> 2) * 7 + (lin - pn * 7);
        }
        if (ar >= 0) pm = ar + (ar >= 2 * rank ? 2 : 0);
    }
    const unsigned call = push_items(p, lds, myws, rank, pd, pfirst, pstride);
    STAMP(1);

    const int wid = rfl(tid >> 6), lane = tid & 63, wr = wid >> 2, wc = wid & 3, fr = lane & 15, fq = lane >> 4;
    const unsigned par = call & PARMASK;
    const gchar* cA = myws + WS_STAGE + (size_t)(par * 2) * OPND_BYTES + (size_t)pm * NKT * UNIT;
    const gchar* cB = myws + WS_STAGE + (size_t)(par * 2 + 1) * OPND_BYTES + (size_t)pn * NKT * UNIT;
    const gu32* fA = (const gu32*)(myws + WS_FLAG) + ((par * 2) * 16 + pm) * NKT;
    const gu32* fB = (const gu32*)(myws + WS_FLAG) + ((par * 2 + 1) * 16 + pn) * NKT;
    const unsigned voff[2] = {(unsigned)tid * 16u, (unsigned)tid * 16u + 8192u}; const unsigned ldsw = (unsigned)wid * 1024u;
    const unsigned aoff0 = (unsigned)((wr * 64 + fr) * 128 + ((fq ^ (fr >> 1)) * 16)), aoff1 = aoff0 ^ 64u;
    const unsigned boff = (unsigned)(fq * 2048 + (wc * 16 + fr) * 16);
    int ready = 0; unsigned long long chg = 0; int scur = 0, smax = 0;

#define SA(b, h) (((b) * 2 + (h)) * HTB)
#define SB(b, h) ((4 + (b) * 2 + (h)) * HTB)
#define STAGE(bufoff, gbase) do { _Pragma("unroll") for (int _i = 0; _i < 2; ++_i) \
        __builtin_amdgcn_global_load_lds((const gu32*)((gbase) + voff[_i]), (LAS unsigned*)(lds + (bufoff) + ldsw + _i * 8192), 16, 0, GLDS_AUX); } while (0)
#define LDA(dst, b, h) do { _Pragma("unroll") for (int m = 0; m < 4; ++m) { dst[m][0] = *(const LAS h16x8*)(lds + SA(b, h) + aoff0 + m * 2048); dst[m][1] = *(const LAS h16x8*)(lds + SA(b, h) + aoff1 + m * 2048); } } while (0)
#define LDB(dst, b, h) do { _Pragma("unroll") for (int n = 0; n < 2; ++n) _Pragma("unroll") for (int k = 0; k < 2; ++k) dst[n][k] = *(const LAS h16x8*)(lds + SB(b, h) + boff + n * 1024 + k * 8192); } while (0)
#define MMA(ai, bj, At, Bt) do { __builtin_amdgcn_s_setprio(1); _Pragma("unroll") for (int m = 0; m < 4; ++m) _Pragma("unroll") for (int n = 0; n < 2; ++n) _Pragma("unroll") for (int k = 0; k < 2; ++k) \
        acc[ai][bj][m][n] = __builtin_amdgcn_mfma_f32_16x16x32_f16(At[m][k], Bt[n][k], acc[ai][bj][m][n], 0, 0, 0); __builtin_amdgcn_s_setprio(0); } while (0)
#define WAIT_V(n) asm volatile("s_waitcnt vmcnt(" #n ")" ::: "memory")
#define WAIT_L(n) asm volatile("s_waitcnt lgkmcnt(" #n ")" ::: "memory")
#define BAR __builtin_amdgcn_s_barrier()
#define SCHED __builtin_amdgcn_sched_barrier(0)
#define HOOK(need) do { if (ready < (need)) { \
        if (wid == 0) { \
            const unsigned long long t0 = __builtin_amdgcn_s_memrealtime(); unsigned fa, fb; int rd; \
            unsigned ln = voff[0]; asm volatile("" : "+v"(ln)); ln >>= 4;     \
            for (;;) { \
                fa = ld_sys(fA + ln); fb = ld_sys(fB + ln); \
                const unsigned long long okm = __ballot((fa >> 8) >= call && (fb >> 8) >= call); \
                rd = (okm == ~0ull) ? 64 : (int)__builtin_ctzll(~okm); \
                if (rd >= (need)) break; \
                if (rfl(ld_agt(tmo)) == call || __builtin_amdgcn_s_memrealtime() - t0 > DEADLINE) { \
                    if (ln == 0) __hip_atomic_store(tmo, call, __ATOMIC_RELAXED, __HIP_MEMORY_SCOPE_AGENT); \
                    rd = 64; break; } \
                __builtin_amdgcn_s_sleep(POLL_SLEEP); \
            } \
            const int sv = (int)((fa & 31u) + (fb & 31u)); \
            const int sp = __builtin_amdgcn_update_dpp(-1, sv, 0x138, 0xf, 0xf, false);     \
            const unsigned long long cg = __ballot(sv != sp); \
            *(LAS int*)(lds + L_TAB + ln * 4) = sv; \
            if (ln == 0) { u32x4 h; h.x = (unsigned)rd; h.y = (unsigned)cg; h.z = (unsigned)(cg >> 32); h.w = 0; *(LAS u32x4*)(lds + L_HDR) = h; } \
            DIAG_WAIT(t0); \
            asm volatile("s_waitcnt vmcnt(0) lgkmcnt(0)" ::: "memory"); \
        } \
        asm volatile("" ::: "memory"); BAR; asm volatile("" ::: "memory"); \
        { const u32x4 h = *(const LAS u32x4*)(lds + L_HDR); ready = (int)rfl(h.x); chg = ((unsigned long long)rfl(h.z) << 32) | rfl(h.y); } \
    } } while (0)
#define DIAG_WAIT(t0) do {} while (0)
#define RS8(a, b) do { f32x2 q[16]; \
        _Pragma("unroll") for (int m = 0; m < 4; ++m) _Pragma("unroll") for (int n = 0; n < 2; ++n) { q[(m * 2 + n) * 2] = acc[a][b][m][n].xy; q[(m * 2 + n) * 2 + 1] = acc[a][b][m][n].zw; } \
        asm volatile("s_cmp_eq_u32 %17, 0\n\ts_cbranch_scc1 .Lrs%=\n\ts_nop 15\n\ts_nop 3\n\t" \
            "v_pk_mul_f32 %0, %0, %16\n\tv_pk_mul_f32 %1, %1, %16\n\tv_pk_mul_f32 %2, %2, %16\n\tv_pk_mul_f32 %3, %3, %16\n\t" \
            "v_pk_mul_f32 %4, %4, %16\n\tv_pk_mul_f32 %5, %5, %16\n\tv_pk_mul_f32 %6, %6, %16\n\tv_pk_mul_f32 %7, %7, %16\n\t" \
            "v_pk_mul_f32 %8, %8, %16\n\tv_pk_mul_f32 %9, %9, %16\n\tv_pk_mul_f32 %10, %10, %16\n\tv_pk_mul_f32 %11, %11, %16\n\t" \
            "v_pk_mul_f32 %12, %12, %16\n\tv_pk_mul_f32 %13, %13, %16\n\tv_pk_mul_f32 %14, %14, %16\n\tv_pk_mul_f32 %15, %15, %16\n\t" \
            "s_nop 4\n.Lrs%=:" \
            : "+v"(q[0]), "+v"(q[1]), "+v"(q[2]), "+v"(q[3]), "+v"(q[4]), "+v"(q[5]), "+v"(q[6]), "+v"(q[7]), \
              "+v"(q[8]), "+v"(q[9]), "+v"(q[10]), "+v"(q[11]), "+v"(q[12]), "+v"(q[13]), "+v"(q[14]), "+v"(q[15]) \
            : "s"(fac), "s"(de) : "scc"); \
        _Pragma("unroll") for (int m = 0; m < 4; ++m) _Pragma("unroll") for (int n = 0; n < 2; ++n) { acc[a][b][m][n].xy = q[(m * 2 + n) * 2]; acc[a][b][m][n].zw = q[(m * 2 + n) * 2 + 1]; } \
    } while (0)
#define RESCALE(kt) do { int de = 0; \
        if ((chg >> (kt)) & 1ull) { \
            const int s = (int)rfl(*(const LAS unsigned*)(lds + L_TAB + (kt) * 4)); \
            smax = max(smax, s); const int sn = max(s, smax - 4); de = max(16 * (scur - sn), -126); scur = sn; } \
        const unsigned long long fac = (unsigned long long)(unsigned)((de + 127) << 23) * 0x100000001ull; \
        RS8(0, 0); RS8(0, 1); RS8(1, 0); RS8(1, 1); \
    } while (0)

    f32x4 acc[2][2][4][2];
#pragma unroll
    for (int a = 0; a < 2; ++a)
#pragma unroll
        for (int b = 0; b < 2; ++b)
#pragma unroll
            for (int m = 0; m < 4; ++m)
#pragma unroll
                for (int n = 0; n < 2; ++n) acc[a][b][m][n] = (f32x4){0.f, 0.f, 0.f, 0.f};
    h16x8 At[4][2], B0[2][2], B1[2][2];

    HOOK(2);
    STAMP(2);
    STAGE(SB(0, 0), cB); STAGE(SA(0, 0), cA); STAGE(SB(0, 1), cB + HTB); STAGE(SA(0, 1), cA + HTB);
    if (wr == 1) BAR;
    WAIT_V(4); BAR;
    STAGE(SB(1, 0), cB + UNIT); STAGE(SA(1, 0), cA + UNIT); STAGE(SB(1, 1), cB + UNIT + HTB);
    WAIT_V(6); BAR;
    for (int t = 0; t < NKT; t += 2) {
        const int t2 = (t + 2) & (NKT - 1);
        const gchar* a1 = cA + (size_t)(t + 1) * UNIT;
        const gchar* a2 = cA + (size_t)t2 * UNIT; const gchar* b2 = cB + (size_t)t2 * UNIT;
        const gchar* a3 = a2 + UNIT; const gchar* b3 = b2 + UNIT;
        HOOK(min(t + 4, NKT));
        RESCALE(t);
        LDB(B0, 0, 0); SCHED; LDA(At, 0, 0); STAGE(SA(1, 1), a1 + HTB);
        WAIT_L(8); BAR; WAIT_L(0); MMA(0, 0, At, B0); BAR; SCHED;
        LDB(B1, 0, 1); STAGE(SB(0, 0), b2);
        BAR; WAIT_L(0); MMA(0, 1, At, B1); BAR;
        LDA(At, 0, 1); STAGE(SA(0, 0), a2);
        BAR; WAIT_L(0); MMA(1, 0, At, B0); BAR; SCHED;
        STAGE(SB(0, 1), b2 + HTB);
        WAIT_V(6); BAR; MMA(1, 1, At, B1); BAR;
        RESCALE(t + 1);
        LDB(B0, 1, 0); SCHED; LDA(At, 1, 0); STAGE(SA(0, 1), a2 + HTB);
        WAIT_L(8); BAR; WAIT_L(0); MMA(0, 0, At, B0); BAR; SCHED;
        LDB(B1, 1, 1); STAGE(SB(1, 0), b3);
        BAR; WAIT_L(0); MMA(0, 1, At, B1); BAR;
        LDA(At, 1, 1); STAGE(SA(1, 0), a3);
        BAR; WAIT_L(0); MMA(1, 0, At, B0); BAR; SCHED;
        STAGE(SB(1, 1), b3 + HTB);
        WAIT_V(6); BAR; MMA(1, 1, At, B1); BAR;
    }
    STAMP(3);

    const int dest = pm >> 1;
    {
        const bool bad = rfl(ld_agt(tmo)) == call;

#define ODEST dest
        const gchar* ob = uni((const char*)p.out[ODEST] + ((size_t)((pm & 1) * 256) * GN + rank * NL + pn * 256) * 4);
        const unsigned vo = (unsigned)(wr * 64 + 4 * fq) * (GN * 4u) + (unsigned)(wc * 16 + fr) * 16u;
        const int oe = 16 * scur - 256;
        const int thr = dest == rank ? 2 : pn >= 14 ? 1 : 0;
        const float qn = __builtin_nanf("");
#pragma unroll
        for (int a = 0; a < 2; ++a)
#pragma unroll
            for (int m = 0; m < 4; ++m)
#pragma unroll
                for (int r = 0; r < 4; ++r) {
                    f32x4 v;
                    v.x = __builtin_amdgcn_ldexpf(acc[a][0][m][0][r], oe); v.y = __builtin_amdgcn_ldexpf(acc[a][0][m][1][r], oe);
                    v.z = __builtin_amdgcn_ldexpf(acc[a][1][m][0][r], oe); v.w = __builtin_amdgcn_ldexpf(acc[a][1][m][1][r], oe);
                    if (bad) v = (f32x4){qn, qn, qn, qn};
                    store16_sys(ob + (size_t)(a * 128 + m * 16 + r) * (GN * 4), vo, __builtin_bit_cast(u32x4, v));
                    if (thr == 0) asm volatile("s_waitcnt vmcnt(" STR(THR_A) ")" ::: "memory");
                    else if (thr == 1) asm volatile("s_waitcnt vmcnt(" STR(THR_C) ")" ::: "memory");
                }
    }
    WAIT_V(0);
    if (wr == 0) BAR;
    BAR;
    if (tid == 0 && dest != rank)
        __hip_atomic_store((unsigned*)(p.ws[dest] + WS_DONE) + rank * 32 + (pm & 1) * 16 + pn, call, __ATOMIC_RELAXED, __HIP_MEMORY_SCOPE_SYSTEM);
    STAMP(4);

    if (bid == 0 && wid == 0) {
        const gu32* dn = (const gu32*)(myws + WS_DONE) + lane * 4;
        const unsigned long long t0 = __builtin_amdgcn_s_memrealtime();
        for (;;) {
            const unsigned a = ld_sys(dn), b = ld_sys(dn + 1), c = ld_sys(dn + 2), d = ld_sys(dn + 3);
            const bool ok = (lane >> 3) == rank || (a >= call && b >= call && c >= call && d >= call);
            if (__ballot(ok) == ~0ull) break;
            if (rfl(ld_agt(tmo)) == call || __builtin_amdgcn_s_memrealtime() - t0 > DEADLINE) {
                if (lane == 0) {
                    __hip_atomic_store(tmo, call, __ATOMIC_RELAXED, __HIP_MEMORY_SCOPE_AGENT);
                    p.out[rank][(size_t)rank * NL] = __builtin_nanf("");
                }
                break;
            }
            __builtin_amdgcn_s_sleep(4);
        }
        STAMP(5);
    }
}

extern "C" void kernel_launch(int rank, int nranks, ncclComm_t comm,
                              hipStream_t stream,
                              void* const* d_in, const int* in_sizes, int n_in,
                              void* d_out, int out_size,
                              const int* in_dtypes, int out_dtype,
                              void* const* d_out_all,
                              void* const* d_ws_all, size_t ws_bytes) {
    (void)comm; (void)in_dtypes; (void)out_dtype; (void)d_out;
    if (nranks != NR || (unsigned)rank >= (unsigned)NR || n_in < 2 || ws_bytes < WS_NEED ||
        in_sizes[0] != GM * KS || in_sizes[1] != KS * GN || out_size != (GM / NR) * GN) {
        fprintf(stderr, "gemm_rs: unexpected problem shape\n");
        return;
    }
    HIP_CHECK(hipFuncSetAttribute((const void*)gemm_rs_kernel, hipFuncAttributeMaxDynamicSharedMemorySize, (int)LDS_BYTES));
    Params p;
    p.x = (const float*)d_in[0]; p.w = (const float*)d_in[1];
    for (int i = 0; i < NR; ++i) { p.out[i] = (float*)d_out_all[i]; p.ws[i] = (char*)d_ws_all[i]; }
    p.rank = rank; p.pad = 0;
    hipLaunchKernelGGL(gemm_rs_kernel, dim3(GRID), dim3(NTHR), LDS_BYTES, stream, p);
    HIP_CHECK(hipGetLastError());
}
```

```cpp
#include <hip/hip_runtime.h>
#include <rccl/rccl.h>
#include <cstdio>
#include <cstdint>

#define HIP_CHECK(expr) do { hipError_t _e = (expr); if (_e != hipSuccess) { \
    fprintf(stderr, "HIP error %s at %s:%d\n", hipGetErrorString(_e), __FILE__, __LINE__); \
    return; } } while (0)

#define LAS __attribute__((address_space(3)))
#define GAS __attribute__((address_space(1)))
typedef GAS char gchar;
typedef GAS unsigned gu32;
typedef _Float16 h16x8 __attribute__((ext_vector_type(8)));
typedef float f32x4 __attribute__((ext_vector_type(4)));
typedef float f32x2 __attribute__((ext_vector_type(2)));
typedef unsigned u32x4 __attribute__((ext_vector_type(4)));

constexpr int NR = 8;
constexpr int GM = 4096, GN = 32768, KS = 512, NL = 4096;
constexpr int GRID = 256, NTHR = 512;
constexpr unsigned UNIT = 32768;
constexpr unsigned HTB = 16384;
constexpr int NKT = 64;
constexpr size_t WS_TICKET = 0, WS_TMO = 128, WS_LINKQ = 1024, WS_OTK = 2048, WS_OREL = 3072;
constexpr size_t WS_CFLAG = 24576;
constexpr size_t WS_FLAG = 4096;
constexpr size_t WS_DIAG = 65536;
constexpr size_t WS_STAGE = 1u << 20;
constexpr size_t OPND_BYTES = (size_t)16 * NKT * UNIT;
constexpr unsigned CHUNK = 36864;
constexpr size_t WS_PACK = WS_STAGE + 4 * OPND_BYTES;
constexpr size_t WS_NEED = WS_PACK + (size_t)NR * 256 * CHUNK;
constexpr unsigned long long DEADLINE = 100000000ull;
constexpr unsigned L_CTL = 8 * HTB;
constexpr unsigned L_AMAX = L_CTL, L_PROD = L_CTL + 32, L_CONS = L_CTL + 48, L_TURN = L_CTL + 64, L_CALL = L_CTL + 96, L_ITEM = L_CTL + 128, L_CLS = L_CTL + 192;
constexpr unsigned L_HDR = L_CTL + 256, L_TAB = L_CTL + 320, LDS_BYTES = L_CTL + 1024;

#define GLDS_AUX 17
struct Params {
    const float* x; const float* w;
    float* out[NR]; char* ws[NR];
    int rank; int pad;
};

#define STAMP(i) do {} while (0)

__device__ __forceinline__ void store16_sys(const gchar* sbase, unsigned voff, u32x4 v) {
    asm volatile("global_store_dwordx4 %0, %1, %2 sc0 sc1\n\ts_nop 1" :: "v"(voff), "v"(v), "s"(sbase) : "memory");
}
__device__ __forceinline__ unsigned ld_sys(const gu32* q) { return __hip_atomic_load(q, __ATOMIC_RELAXED, __HIP_MEMORY_SCOPE_SYSTEM); }
__device__ __forceinline__ unsigned ld_agt(const gu32* q) { return __hip_atomic_load(q, __ATOMIC_RELAXED, __HIP_MEMORY_SCOPE_AGENT); }
__device__ __forceinline__ unsigned rfl(unsigned v) { return (unsigned)__builtin_amdgcn_readfirstlane((int)v); }

template <class T> __device__ __forceinline__ gchar* uni(T q) {
    unsigned lo = rfl((unsigned)(uintptr_t)q), hi = rfl((unsigned)((uintptr_t)q >> 32));
    asm volatile("s_nop 4" : "+s"(lo), "+s"(hi));
    return (gchar*)(((uintptr_t)hi << 32) | lo);
}

__device__ __forceinline__ unsigned wave_max(unsigned x) {
    int mb = (int)x;
    mb = max(mb, __builtin_amdgcn_update_dpp(0, mb, 0x111, 0xf, 0xf, false));
    mb = max(mb, __builtin_amdgcn_update_dpp(0, mb, 0x112, 0xf, 0xf, false));
    mb = max(mb, __builtin_amdgcn_update_dpp(0, mb, 0x114, 0xf, 0xf, false));
    mb = max(mb, __builtin_amdgcn_update_dpp(0, mb, 0x118, 0xf, 0xf, false));
    mb = max(mb, __builtin_amdgcn_update_dpp(0, mb, 0x142, 0xa, 0xf, false));
    mb = max(mb, __builtin_amdgcn_update_dpp(0, mb, 0x143, 0xc, 0xf, false));
    return (unsigned)__builtin_amdgcn_readlane(mb, 63);
}
__device__ __forceinline__ u32x4 byte_swap(u32x4 r) {
    u32x4 o;
    o.x = __builtin_amdgcn_perm(r.w, r.x, 0x04020100u); o.y = __builtin_amdgcn_perm(r.w, r.y, 0x05020100u); o.z = __builtin_amdgcn_perm(r.w, r.z, 0x06020100u);
    o.w = __builtin_amdgcn_perm(r.y, r.x, 0x0c0c0703u) | __builtin_amdgcn_perm(r.w, r.z, 0x07030c0cu);
    return o;
}

__device__ __forceinline__ void item_decode(int i, int d, int& isW, int& j, int& u) {
    if (i < 224) { j = i / 28; const int r = i - j * 28; isW = r & 1; u = r >> 1; if (!isW) { u = u < 7 ? 2 * u : 2 * u - 13; if (u >= 2 * d) u += 2; } }
    else { const int r = i - 224; j = r >> 2; isW = (r >> 1) & 1; u = (isW ? 14 : 2 * d) + (r & 1); }
}

__device__ __forceinline__ void item_load(const Params& p, int i, int d, f32x4 (&L)[16]) {
    const bool valid = i < 256;
    int isW, j, u; item_decode(valid ? i : 0, d, isW, j, u);
    const unsigned t = threadIdx.x;
    const gchar* b = uni(isW ? (const char*)p.w + ((size_t)(j * 64) * GN + d * NL + u * 256) * 4 : (const char*)p.x + ((size_t)(u * 256) * KS + j * 64) * 4);
    unsigned vo = isW ? (t >> 6) * (4u * GN * 4u) + (t & 63u) * 16u : (t >> 3) * (KS * 4u) + ((t & 7u) ^ ((t >> 4) & 7u)) * 16u;
    if (!valid) vo = 0;
    const unsigned s0 = isW ? GN * 4u : 128u, s1 = isW ? 2u * GN * 4u : 64u * KS * 4u, s2 = isW ? 32u * GN * 4u : 128u * KS * 4u, s3 = isW ? 16u * GN * 4u : 32u * KS * 4u;
#pragma unroll
    for (int n = 0; n < 16; ++n)
        L[n] = *(const GAS f32x4*)(b + ((n & 1) * s0 + ((n >> 1) & 1) * s1 + ((n >> 2) & 1) * s2 + (n >> 3) * s3) + vo);
}

__device__ __forceinline__ unsigned pk2(float a, float b, int e) {
    typedef _Float16 h2 __attribute__((ext_vector_type(2)));
    h2 h; h.x = (_Float16)__builtin_amdgcn_ldexpf(a, e); h.y = (_Float16)__builtin_amdgcn_ldexpf(b, e);
    return __builtin_bit_cast(unsigned, h);
}

#define POLL_SLEEP 8
#define PARMASK 1u
#define SELF_DELAY 4
#define C_BYPASS 0
#define FLAG_SLEEP 16
#define CASCADE_FROM 225u
#define THR_L 8
#define OUT_W 3
#define REL_AT 24
#define LINK_EXTRA 8u
#define WG_SENDERS 1
#define WG_SENDERS_C 4
#define PUSH_N 6
#define STR2(x) #x
#define STR(x) STR2(x)
#define LBAR do { asm volatile("s_waitcnt lgkmcnt(0)" ::: "memory"); __builtin_amdgcn_s_barrier(); asm volatile("" ::: "memory"); } while (0)

typedef volatile LAS unsigned vl32;
typedef volatile LAS u32x4 vl128;
#define LGKM0 asm volatile("s_waitcnt lgkmcnt(0)" ::: "memory")
__device__ __forceinline__ u32x4 lds_wait4(LAS unsigned char* lds, unsigned off, unsigned v) {
    for (;;) {
        const u32x4 q = *(vl128*)(lds + off);
        if (rfl(min(min(q.x, q.y), min(q.z, q.w))) >= v) { asm volatile("" ::: "memory"); return q; }
        __builtin_amdgcn_s_sleep(1);
    }
}
__device__ __forceinline__ void item_amax(LAS unsigned char* lds, const f32x4 (&L)[16], unsigned k) {
    unsigned mu = 0;
#pragma unroll
    for (int n = 0; n < 16; ++n) { const u32x4 q = __builtin_bit_cast(u32x4, L[n]) & 0x7fffffffu; mu = max(max(mu, q.x), max(q.y, max(q.z, q.w))); }
    const unsigned mb = wave_max(mu >> 23);
    LGKM0;
    if ((threadIdx.x & 63) == 63) *(vl32*)(lds + L_AMAX + (k & 1u) * 16 + (threadIdx.x >> 6) * 4) = ((k + 1u) << 8) | mb;
}

__device__ __forceinline__ int conv_step(const Params& p, LAS unsigned char* lds, gu32* tick, int d, unsigned k, f32x4 (&L)[16], const f32x4 (&Ln)[16]) {
    const unsigned t = threadIdx.x;
    const u32x4 am = lds_wait4(lds, L_AMAX + (k & 1u) * 16, (k + 1u) << 8) & 255u;
    const int i = (int)rfl(*(vl32*)(lds + L_ITEM + (k & 15u) * 4)), i1 = (int)rfl(*(vl32*)(lds + L_ITEM + ((k + 1u) & 15u) * 4)), i2 = (int)rfl(*(vl32*)(lds + L_ITEM + ((k + 2u) & 15u) * 4));
    const bool valid = i < 256;
    if (valid) {
        const int cls = (int)(max(max(am.x, am.y), max(am.z, am.w)) + 2u) >> 4;
        const int e = 128 - 16 * cls;
        if (k >= 4) while (rfl(*(vl32*)(lds + L_CONS)) < k - 3u) __builtin_amdgcn_s_sleep(2);
        asm volatile("" ::: "memory");
        if (t == 0) *(vl32*)(lds + L_CLS + (k & 15u) * 4) = (unsigned)cls;
        int isW, j, u; item_decode(i, d, isW, j, u);
        LAS unsigned char* img = lds + (k & 3u) * UNIT;
#pragma unroll
        for (int h = 0; h < 2; ++h) {
            const f32x4* Q = L + 8 * h;
            u32x4 H[4];
            if (isW) {
#pragma unroll
                for (int c = 0; c < 4; ++c) { H[c].x = pk2(Q[0][c], Q[1][c], e); H[c].y = pk2(Q[2][c], Q[3][c], e); H[c].z = pk2(Q[4][c], Q[5][c], e); H[c].w = pk2(Q[6][c], Q[7][c], e); }
            } else {
#pragma unroll
                for (int c = 0; c < 4; ++c) { H[c].x = pk2(Q[2 * c].x, Q[2 * c].y, e); H[c].y = pk2(Q[2 * c].z, Q[2 * c].w, e); H[c].z = pk2(Q[2 * c + 1].x, Q[2 * c + 1].y, e); H[c].w = pk2(Q[2 * c + 1].z, Q[2 * c + 1].w, e); }
            }
            const unsigned vo = isW ? (t >> 6) * 2048u + (t & 63u) * 16u + h * 8192u : t * 16u + h * 4096u;
            const unsigned c1 = isW ? 1024u : 8192u;
#pragma unroll
            for (int c = 0; c < 4; ++c) *(LAS u32x4*)(img + vo + (c >> 1) * HTB + (c & 1) * c1) = H[c];
        }
        LGKM0;
        if ((t & 63u) == 0) *(vl32*)(lds + L_PROD + (t >> 6) * 4) = k + 1u;
    }
    unsigned tk = 255u;
    gu32* q = tick; asm volatile("" : "+v"(q));
    if (t == 0 && tick && valid) tk = __hip_atomic_fetch_add(q, 1u, __ATOMIC_RELAXED, __HIP_MEMORY_SCOPE_AGENT);
    item_load(p, i2, d, L);
    __builtin_amdgcn_s_waitcnt(0x4F70);
    asm volatile("" : "+v"(tk));
    if (t == 0) *(vl32*)(lds + L_ITEM + ((k + 3u) & 15u) * 4) = 24u + (tk & 255u);
    item_amax(lds, Ln, k + 1u);
    return i1;
}

__device__ __forceinline__ gu32* item_flag(gchar* dws, int rank, int d, int i, unsigned par, const gchar*& ub) {
    int isW, j, u; item_decode(i, d, isW, j, u);
    const int srel = (rank - d - 1) & 7;
    const int idx = (srel == 7) ? j : 8 + j * 7 + srel;
    ub = dws + WS_STAGE + (size_t)(par * 2 + isW) * OPND_BYTES + (size_t)(u * NKT + idx) * UNIT;
    return (gu32*)(dws + WS_FLAG) + ((par * 2 + isW) * 16 + u) * NKT + idx;
}
__device__ __forceinline__ void send_fetch(LAS unsigned char* lds, unsigned k, int q, u32x4 (&H)[8]) {
#pragma unroll
    for (int c = 0; c < 8; ++c) H[c] = *(const LAS u32x4*)(lds + (k & 3u) * UNIT + q * 8192u + c * 1024u + (threadIdx.x & 63u) * 16u);
}
template <int Q> __device__ __forceinline__ void send8(const gchar* ub, const u32x4 (&H)[8], gu32* pf, unsigned pv) {
#pragma unroll
    for (int c = 0; c < 8; ++c) {
        store16_sys(ub + (Q * 8192u + c * 1024u), (threadIdx.x & 63u) * 16u, H[c]);
        asm volatile("s_waitcnt vmcnt(" STR(PUSH_N) ")" ::: "memory");
        if (Q * 8 + c == PUSH_N - 1 && pf && (threadIdx.x & 63u) == 0) __hip_atomic_store(pf, pv, __ATOMIC_RELAXED, __HIP_MEMORY_SCOPE_SYSTEM);
    }
}

__device__ __forceinline__ unsigned push_items(const Params& p, LAS unsigned char* lds, gchar* myws, int rank, int d, int first, int stride, gu32* tick) {
    unsigned call;
#define PSTAMP(i) do {} while (0)
    if (rfl(threadIdx.x >> 6) < 4) {
        f32x4 RA[16], RB[16];
        if (!tick) for (int z = 0; z < SELF_DELAY; ++z) __builtin_amdgcn_s_sleep(32);
        item_load(p, first, d, RA);
        PSTAMP(12);
        LBAR;
        PSTAMP(13);
        item_amax(lds, RA, 0);
        PSTAMP(14);
        item_load(p, first + stride, d, RB);
        PSTAMP(15);
        int nx; unsigned k = 0;
        do { conv_step(p, lds, tick, d, k, RA, RB); nx = conv_step(p, lds, tick, d, k + 1u, RB, RA); k += 2; } while (nx < 256);
        LBAR;
        call = rfl(*(vl32*)(lds + L_CALL));
    } else {
        const unsigned ts = threadIdx.x - 256u;
        if (ts < 64) *(vl32*)(lds + L_CTL + ts * 4) = (ts >= (L_ITEM - L_CTL) / 4 && ts < (L_ITEM - L_CTL) / 4 + 3) ? (unsigned)(first + stride * (int)(ts - (L_ITEM - L_CTL) / 4)) : 0u;
        unsigned tk = 0;
        gu32* q = (gu32*)(myws + WS_TICKET); asm volatile("" : "+v"(q));
        if (ts == 0) tk = __hip_atomic_fetch_add(q, 1u, __ATOMIC_RELAXED, __HIP_MEMORY_SCOPE_AGENT);
        gchar* dws = uni(p.ws[d]);
        LBAR;
        if (ts < 64) {
            asm volatile("" : "+v"(tk));
            call = (rfl(tk) / GRID + 1u) & 0xffffffu;
            if (ts == 0) *(vl32*)(lds + L_CALL) = call;
            u32x4 HA[8], HB[8]; gu32* pf = nullptr; unsigned pv = 0;
            int i = first; unsigned k = 0;
            lds_wait4(lds, L_PROD, 1u); send_fetch(lds, 0, 0, HA);
            do {
                const gchar* ub; gu32* f = item_flag(dws, rank, d, i, call & PARMASK, ub); ub = uni(ub);
                const unsigned fv = (call << 8) | rfl(*(vl32*)(lds + L_CLS + (k & 15u) * 4));
                i = (int)rfl(*(vl32*)(lds + L_ITEM + ((k + 1u) & 15u) * 4));
                send_fetch(lds, k, 1, HB); send8<0>(ub, HA, pf, pv);
                send_fetch(lds, k, 2, HA); send8<1>(ub, HB, pf, pv);
                send_fetch(lds, k, 3, HB); send8<2>(ub, HA, pf, pv);
                LGKM0;
                if (ts == 0) *(vl32*)(lds + L_CONS) = k + 1u;
                if (i < 256) { lds_wait4(lds, L_PROD, k + 2u); send_fetch(lds, k + 1u, 0, HA); }
                else if (tick && ts == 0) __hip_atomic_fetch_add(tick + (WS_OREL - WS_LINKQ) / 4, 1u, __ATOMIC_RELAXED, __HIP_MEMORY_SCOPE_AGENT);
                send8<3>(ub, HB, pf, pv);
                pf = f; pv = fv; ++k;
            } while (i < 256);
            asm volatile("s_waitcnt vmcnt(0)" ::: "memory");
            if (ts == 0) __hip_atomic_store(pf, pv, __ATOMIC_RELAXED, __HIP_MEMORY_SCOPE_SYSTEM);
        }
        LBAR;
        call = rfl(*(vl32*)(lds + L_CALL));
    }
    return call;
}

__global__ void __launch_bounds__(NTHR, 2) gemm_rs_kernel(const Params p) {
    extern __shared__ __attribute__((aligned(16))) unsigned char smem[];
    LAS unsigned char* lds = (LAS unsigned char*)smem;
    const int tid = threadIdx.x;
    const int rank = (int)rfl(p.rank), bid = (int)rfl(blockIdx.x);
    gchar* myws = uni(p.ws[rank]);
    STAMP(0);
    const int xcd = bid & 7, slot = bid >> 3;
    const int pd = slot < 7 ? (rank + 1 + slot) & 7 : rank, pfirst = slot < 7 ? xcd : (slot - 7) * 8 + xcd, pstride = slot < 7 ? 8 : 200;
    gu32* tmo = (gu32*)(myws + WS_TMO);

    int pm, pn;
    {
        int ar = -1;
        if ((xcd & 1) && (slot < 7 || slot == 31)) { const int k = slot & 7; pm = 2 * rank + (k & 1); pn = 4 * (xcd >> 1) + (k >> 1); }
        else if (slot < 7) {
            const int k = 7 * (xcd >> 1) + slot;
            ar = k >> 1; pn = 14 + (k & 1);
        } else {
            const int q = xcd & 3, lin = (q * 49 + 1) / 2 + slot - 7;
            pn = lin / 7; ar = (xcd >> 2) * 7 + (lin - pn * 7);
        }
        if (ar >= 0) pm = ar + (ar >= 2 * rank ? 2 : 0);
    }
    const unsigned call = push_items(p, lds, myws, rank, pd, pfirst, pstride, slot < 7 ? (gu32*)(myws + WS_LINKQ + slot * 128) : nullptr);
    STAMP(1);

    const int wid = rfl(tid >> 6), lane = tid & 63, wr = wid >> 2, wc = wid & 3, fr = lane & 15, fq = lane >> 4;
    const unsigned par = call & PARMASK;
    const gchar* cA = myws + WS_STAGE + (size_t)(par * 2) * OPND_BYTES + (size_t)pm * NKT * UNIT;
    const gchar* cB = myws + WS_STAGE + (size_t)(par * 2 + 1) * OPND_BYTES + (size_t)pn * NKT * UNIT;
    const gu32* fA = (const gu32*)(myws + WS_FLAG) + ((par * 2) * 16 + pm) * NKT;
    const gu32* fB = (const gu32*)(myws + WS_FLAG) + ((par * 2 + 1) * 16 + pn) * NKT;
    const unsigned voff[2] = {(unsigned)tid * 16u, (unsigned)tid * 16u + 8192u}; const unsigned ldsw = (unsigned)wid * 1024u;
    const unsigned aoff0 = (unsigned)((wr * 64 + fr) * 128 + ((fq ^ (fr >> 1)) * 16)), aoff1 = aoff0 ^ 64u;
    const unsigned boff = (unsigned)(fq * 2048 + (wc * 16 + fr) * 16);
    int ready = 0; unsigned long long chg = 0; int scur = 0, smax = 0;

#define SA(b, h) (((b) * 2 + (h)) * HTB)
#define SB(b, h) ((4 + (b) * 2 + (h)) * HTB)
#define STAGE(bufoff, gbase) do { _Pragma("unroll") for (int _i = 0; _i < 2; ++_i) \
        __builtin_amdgcn_global_load_lds((const gu32*)((gbase) + voff[_i]), (LAS unsigned*)(lds + (bufoff) + ldsw + _i * 8192), 16, 0, GLDS_AUX); } while (0)
#define LDA(dst, b, h) do { _Pragma("unroll") for (int m = 0; m < 4; ++m) { dst[m][0] = *(const LAS h16x8*)(lds + SA(b, h) + aoff0 + m * 2048); dst[m][1] = *(const LAS h16x8*)(lds + SA(b, h) + aoff1 + m * 2048); } } while (0)
#define LDB(dst, b, h) do { _Pragma("unroll") for (int n = 0; n < 2; ++n) _Pragma("unroll") for (int k = 0; k < 2; ++k) dst[n][k] = *(const LAS h16x8*)(lds + SB(b, h) + boff + n * 1024 + k * 8192); } while (0)
#define MMA(ai, bj, At, Bt) do { __builtin_amdgcn_s_setprio(1); _Pragma("unroll") for (int m = 0; m < 4; ++m) _Pragma("unroll") for (int n = 0; n < 2; ++n) _Pragma("unroll") for (int k = 0; k < 2; ++k) \
        acc[ai][bj][m][n] = __builtin_amdgcn_mfma_f32_16x16x32_f16(At[m][k], Bt[n][k], acc[ai][bj][m][n], 0, 0, 0); __builtin_amdgcn_s_setprio(0); } while (0)
#define WAIT_V(n) asm volatile("s_waitcnt vmcnt(" #n ")" ::: "memory")
#define WAIT_L(n) asm volatile("s_waitcnt lgkmcnt(" #n ")" ::: "memory")
#define BAR __builtin_amdgcn_s_barrier()
#define SCHED __builtin_amdgcn_sched_barrier(0)
#define HOOK(need) do { if (ready < (need)) { \
        if (wid == 0) { \
            const unsigned long long t0 = __builtin_amdgcn_s_memrealtime(); unsigned fa, fb; int rd; \
            unsigned ln = voff[0]; asm volatile("" : "+v"(ln)); ln >>= 4;     \
            for (;;) { \
                fa = ld_sys(fA + ln); fb = ld_sys(fB + ln); \
                const unsigned long long okm = __ballot((fa >> 8) >= call && (fb >> 8) >= call); \
                rd = (okm == ~0ull) ? 64 : (int)__builtin_ctzll(~okm); \
                if (rd >= (need)) break; \
                if (rfl(ld_agt(tmo)) == call || __builtin_amdgcn_s_memrealtime() - t0 > DEADLINE) { \
                    if (ln == 0) __hip_atomic_store(tmo, call, __ATOMIC_RELAXED, __HIP_MEMORY_SCOPE_AGENT); \
                    rd = 64; break; } \
                __builtin_amdgcn_s_sleep(POLL_SLEEP); \
            } \
            const int sv = (int)((fa & 31u) + (fb & 31u)); \
            const int sp = __builtin_amdgcn_update_dpp(-1, sv, 0x138, 0xf, 0xf, false);     \
            const unsigned long long cg = __ballot(sv != sp); \
            *(LAS int*)(lds + L_TAB + ln * 4) = sv; \
            if (ln == 0) { u32x4 h; h.x = (unsigned)rd; h.y = (unsigned)cg; h.z = (unsigned)(cg >> 32); h.w = 0; *(LAS u32x4*)(lds + L_HDR) = h; } \
            DIAG_WAIT(t0); \
            asm volatile("s_waitcnt vmcnt(0) lgkmcnt(0)" ::: "memory"); \
        } \
        asm volatile("" ::: "memory"); BAR; asm volatile("" ::: "memory"); \
        { const u32x4 h = *(const LAS u32x4*)(lds + L_HDR); ready = (int)rfl(h.x); chg = ((unsigned long long)rfl(h.z) << 32) | rfl(h.y); } \
    } } while (0)
#define DIAG_WAIT(t0) do {} while (0)
#define PKMUL4(op) ".set rs_i, 0\n\t.set rs_d, 0\n\t.irpc c, \"" op "\"\n\t.ifc \\c,:\n\t.set rs_d, 1\n\t.endif\n\t" \
        ".if rs_d == 0\n\t.ifnc \\c,v\n\t.ifnc \\c,[\n\t.set rs_i, rs_i*10 + \\c\n\t.endif\n\t.endif\n\t.endif\n\t.endr\n\t" \
        "v_pk_mul_f32 v[rs_i:rs_i+1], v[rs_i:rs_i+1], %16\n\tv_pk_mul_f32 v[rs_i+2:rs_i+3], v[rs_i+2:rs_i+3], %16\n\t"
#define RS16(a) asm volatile("s_cmp_eq_u32 %17, 0\n\ts_cbranch_scc1 .Lrs%=\n\ts_nop 15\n\ts_nop 3\n\t" \
            PKMUL4("%0") PKMUL4("%1") PKMUL4("%2") PKMUL4("%3") PKMUL4("%4") PKMUL4("%5") PKMUL4("%6") PKMUL4("%7") \
            PKMUL4("%8") PKMUL4("%9") PKMUL4("%10") PKMUL4("%11") PKMUL4("%12") PKMUL4("%13") PKMUL4("%14") PKMUL4("%15") \
            "s_nop 4\n.Lrs%=:" \
            : "+v"(acc[a][0][0][0]), "+v"(acc[a][0][0][1]), "+v"(acc[a][0][1][0]), "+v"(acc[a][0][1][1]), "+v"(acc[a][0][2][0]), "+v"(acc[a][0][2][1]), "+v"(acc[a][0][3][0]), "+v"(acc[a][0][3][1]), \
              "+v"(acc[a][1][0][0]), "+v"(acc[a][1][0][1]), "+v"(acc[a][1][1][0]), "+v"(acc[a][1][1][1]), "+v"(acc[a][1][2][0]), "+v"(acc[a][1][2][1]), "+v"(acc[a][1][3][0]), "+v"(acc[a][1][3][1]) \
            : "s"(fac), "s"(de) : "scc")
#define RESCALE(kt) do { int de = 0; \
        if ((chg >> (kt)) & 1ull) { \
            const int s = (int)rfl(*(const LAS unsigned*)(lds + L_TAB + (kt) * 4)); \
            smax = max(smax, s); const int sn = max(s, smax - 4); de = max(16 * (scur - sn), -126); scur = sn; } \
        const unsigned long long fac = (unsigned long long)(unsigned)((de + 127) << 23) * 0x100000001ull; \
        RS16(0); RS16(1); \
    } while (0)
#define TRIP do { \
        const int t2 = (t + 2) & (NKT - 1);             \
        const gchar* a1 = cA + (size_t)(t + 1) * UNIT; \
        const gchar* a2 = cA + (size_t)t2 * UNIT; const gchar* b2 = cB + (size_t)t2 * UNIT; \
        const gchar* a3 = a2 + UNIT; const gchar* b3 = b2 + UNIT; \
        RESCALE(t); \
        LDB(B0, 0, 0); SCHED; LDA(At, 0, 0); STAGE(SA(1, 1), a1 + HTB); \
        WAIT_L(8); BAR; WAIT_L(0); MMA(0, 0, At, B0); BAR; SCHED; \
        LDB(B1, 0, 1); STAGE(SB(0, 0), b2); \
        BAR; WAIT_L(0); MMA(0, 1, At, B1); BAR; \
        LDA(At, 0, 1); STAGE(SA(0, 0), a2); \
        BAR; WAIT_L(0); MMA(1, 0, At, B0); BAR; SCHED; \
        STAGE(SB(0, 1), b2 + HTB); \
        WAIT_V(6); BAR; MMA(1, 1, At, B1); BAR; \
        RESCALE(t + 1); \
        LDB(B0, 1, 0); SCHED; LDA(At, 1, 0); STAGE(SA(0, 1), a2 + HTB); \
        WAIT_L(8); BAR; WAIT_L(0); MMA(0, 0, At, B0); BAR; SCHED; \
        LDB(B1, 1, 1); STAGE(SB(1, 0), b3); \
        BAR; WAIT_L(0); MMA(0, 1, At, B1); BAR; \
        LDA(At, 1, 1); STAGE(SA(1, 0), a3); \
        BAR; WAIT_L(0); MMA(1, 0, At, B0); BAR; SCHED; \
        STAGE(SB(1, 1), b3 + HTB); \
        WAIT_V(6); BAR; MMA(1, 1, At, B1); BAR; \
    } while (0)

    f32x4 acc[2][2][4][2];
#pragma unroll
    for (int a = 0; a < 2; ++a)
#pragma unroll
        for (int b = 0; b < 2; ++b)
#pragma unroll
            for (int m = 0; m < 4; ++m)
#pragma unroll
                for (int n = 0; n < 2; ++n) acc[a][b][m][n] = (f32x4){0.f, 0.f, 0.f, 0.f};
    h16x8 At[4][2], B0[2][2], B1[2][2];

    HOOK(2);
    STAMP(2);
    STAGE(SB(0, 0), cB); STAGE(SA(0, 0), cA); STAGE(SB(0, 1), cB + HTB); STAGE(SA(0, 1), cA + HTB);
    if (wr == 1) BAR;
    WAIT_V(4); BAR;
    STAGE(SB(1, 0), cB + UNIT); STAGE(SA(1, 0), cA + UNIT); STAGE(SB(1, 1), cB + UNIT + HTB);
    WAIT_V(6); BAR;
    for (int t = 0; t < NKT; t += 2) {
        HOOK(min(t + 4, NKT));
        TRIP;
    }
    if (wr == 0) BAR;
    STAMP(3);

    const int dest = pm >> 1;
    const unsigned ovo = (unsigned)(wr * 64 + 4 * fq) * (GN * 4u) + (unsigned)(wc * 16 + fr) * 16u;
    const float qn = __builtin_nanf("");
    u32x4 V[32];
    {
        const bool bad = rfl(ld_agt(tmo)) == call;
        const int oe = 16 * scur - 256;
#pragma unroll
        for (int a = 0; a < 2; ++a)
#pragma unroll
            for (int m = 0; m < 4; ++m)
#pragma unroll
                for (int r = 0; r < 4; ++r) {
                    f32x4 v;
                    v.x = __builtin_amdgcn_ldexpf(acc[a][0][m][0][r], oe); v.y = __builtin_amdgcn_ldexpf(acc[a][0][m][1][r], oe);
                    v.z = __builtin_amdgcn_ldexpf(acc[a][1][m][0][r], oe); v.w = __builtin_amdgcn_ldexpf(acc[a][1][m][1][r], oe);
                    if (bad) v = (f32x4){qn, qn, qn, qn};
                    V[(a * 4 + m) * 4 + r] = __builtin_bit_cast(u32x4, v);
                }
    }
#define OSTAMP(k) do {} while (0)
#define DIAG_END do {} while (0)
    if (dest == rank) {
        const gchar* ob = uni((const char*)p.out[rank] + ((size_t)((pm & 1) * 256) * GN + rank * NL + pn * 256) * 4);
#pragma unroll
        for (int q = 0; q < 32; ++q) { store16_sys(ob + (size_t)((q >> 4) * 128 + ((q >> 2) & 3) * 16 + (q & 3)) * (GN * 4), ovo, V[q]); asm volatile("s_waitcnt vmcnt(" STR(THR_L) ")" ::: "memory"); }
    }
    int remote = dest - rank;
    asm volatile("" : "+s"(remote));
    if (remote) {
        const unsigned cidx = (unsigned)(((pm & 1) * 16 + pn) * 8 + wid);
        gchar* dws = uni(p.ws[dest]);
        const gchar* pb = uni(dws + WS_PACK + (size_t)(rank * 256 + cidx) * CHUNK);
        unsigned mx = 0;
#pragma unroll
        for (int q = 0; q < 32; ++q) {
#pragma unroll
            for (int c = 0; c < 4; ++c) V[q][c] = __builtin_amdgcn_alignbit(V[q][c], V[q][c], 31);
            mx = max(max(mx, V[q].x), max(V[q].y, max(V[q].z, V[q].w)));
        }
        const unsigned base = wave_max(mx >> 24), base4 = base * 0x01010101u;
        u32x4 PK[28];
#pragma unroll
        for (int g = 0; g < 4; ++g) {
            unsigned E[8], D[8], any = 0;
#pragma unroll
            for (int v = 0; v < 8; ++v) {
                const u32x4 w = byte_swap(V[g * 8 + v]);
#pragma unroll
                for (int i = 0; i < 3; ++i) PK[g * 7 + ((3 * v + i) >> 2)][(3 * v + i) & 3] = w[i];
                E[v] = w.w; D[v] = base4 - w.w; any |= D[v];
            }
            u32x4 nb;
#pragma unroll
            for (int j = 0; j < 4; ++j) nb[j] = D[2 * j] | (D[2 * j + 1] << 4);
            if ((any & 0xf0f0f0f0u) != 0u || nb.x == 0xffffffffu) {
                nb.x = 0xffffffffu;
                store16_sys(pb + (28672u + g * 2048u), (unsigned)lane * 32u, (u32x4){E[0], E[1], E[2], E[3]});
                store16_sys(pb + (28672u + g * 2048u + 16u), (unsigned)lane * 32u, (u32x4){E[4], E[5], E[6], E[7]});
            }
            PK[g * 7 + 6] = nb;
        }
        const int link = (dest - rank - 1) & 7;
        while ((int)rfl(*(vl32*)(lds + L_TURN)) + (pn < 14 ? WG_SENDERS : WG_SENDERS_C) <= wid) __builtin_amdgcn_s_sleep(8);
        gu32* rel = (gu32*)(myws + WS_OREL + link * 128);
        bool late;
        {
            unsigned tk = 0;
            if (lane == 0) tk = __hip_atomic_fetch_add((gu32*)(myws + WS_OTK + link * 128), 1u, __ATOMIC_RELAXED, __HIP_MEMORY_SCOPE_AGENT);
            const unsigned need = (call - 1u) * 264u + (rfl(tk) & 255u) + 1u - LINK_EXTRA;
            late = (rfl(tk) & 255u) >= CASCADE_FROM;
            const unsigned long long t0 = __builtin_amdgcn_s_memrealtime();
            for (;;) {
                const int ahead = (int)(need - rfl(ld_agt(rel)));
                if (ahead <= 0 || (C_BYPASS && pn >= 14) || __builtin_amdgcn_s_memrealtime() - t0 > DEADLINE) break;
                if (ahead > 4) __builtin_amdgcn_s_sleep(32); else __builtin_amdgcn_s_sleep(4);
            }
        }
#pragma unroll
        for (int q = 0; q < 28; ++q) {
            store16_sys(pb + q * 1024u, (unsigned)lane * 16u, PK[q]);
            asm volatile("s_waitcnt vmcnt(" STR(OUT_W) ")" ::: "memory");
            if ((q == 0 || q == REL_AT) && late == (q == 0) && lane == 0) __hip_atomic_fetch_add((LAS unsigned*)(lds + L_TURN), 1u, __ATOMIC_RELAXED, __HIP_MEMORY_SCOPE_WORKGROUP);
            if (q == REL_AT && lane == 0) __hip_atomic_fetch_add(rel, 1u, __ATOMIC_RELAXED, __HIP_MEMORY_SCOPE_AGENT);
            OSTAMP(q);
        }
        WAIT_V(0);
        if (lane == 0) __hip_atomic_store((gu32*)(dws + WS_CFLAG) + rank * 256 + cidx, (call << 8) | base, __ATOMIC_RELAXED, __HIP_MEMORY_SCOPE_SYSTEM);
        OSTAMP(28);

        const gu32* cf = (const gu32*)(myws + WS_CFLAG) + dest * 256 + cidx;
        unsigned fv; bool bad = false;
        {
            const unsigned long long t0 = __builtin_amdgcn_s_memrealtime();
            for (;;) {
                fv = rfl(ld_sys(cf));
                if ((fv >> 8) >= call) break;
                if (rfl(ld_agt(tmo)) == call || __builtin_amdgcn_s_memrealtime() - t0 > DEADLINE) {
                    if (lane == 0) __hip_atomic_store(tmo, call, __ATOMIC_RELAXED, __HIP_MEMORY_SCOPE_AGENT);
                    break;
                }
                __builtin_amdgcn_s_sleep(FLAG_SLEEP);
            }
            bad = rfl(ld_agt(tmo)) == call;
            asm volatile("" ::: "memory");
        }
        OSTAMP(29);
        const __amdgpu_buffer_rsrc_t rs = __builtin_amdgcn_make_buffer_rsrc((void*)uni(myws + WS_PACK + (size_t)(dest * 256 + cidx) * CHUNK), 0, bad ? 0 : CHUNK, 0x00020000);
        const gchar* ob = uni((const char*)p.out[rank] + ((size_t)((pm & 1) * 256) * GN + dest * NL + pn * 256) * 4);
        const unsigned ib4 = (fv & 255u) * 0x01010101u;
#define PLOAD(vo, so) __builtin_bit_cast(u32x4, __builtin_amdgcn_raw_buffer_load_b128(rs, vo, so, 17))
#pragma unroll
        for (int g = 0; g < 4; ++g) PK[g * 7 + 6] = PLOAD(lane * 16, (g * 7 + 6) * 1024);
#pragma unroll
        for (int q = 0; q < 28; ++q) if (q % 7 != 6) PK[q] = PLOAD(lane * 16, q * 1024);
        u32x4 E[8];
#pragma unroll
        for (int g = 0; g < 4; ++g) {
            const u32x4 nb = PK[g * 7 + 6];
            E[2 * g].x = ib4 - (nb.x & 0x0f0f0f0fu); E[2 * g].y = ib4 - ((nb.x >> 4) & 0x0f0f0f0fu); E[2 * g].z = ib4 - (nb.y & 0x0f0f0f0fu); E[2 * g].w = ib4 - ((nb.y >> 4) & 0x0f0f0f0fu);
            E[2 * g + 1].x = ib4 - (nb.z & 0x0f0f0f0fu); E[2 * g + 1].y = ib4 - ((nb.z >> 4) & 0x0f0f0f0fu); E[2 * g + 1].z = ib4 - (nb.w & 0x0f0f0f0fu); E[2 * g + 1].w = ib4 - ((nb.w >> 4) & 0x0f0f0f0fu);
            if (nb.x == 0xffffffffu) { E[2 * g] = PLOAD(lane * 32, 28672 + g * 2048); E[2 * g + 1] = PLOAD(lane * 32 + 16, 28672 + g * 2048); }
        }
        __builtin_amdgcn_s_waitcnt(0x0F70);
#pragma unroll
        for (int q = 0; q < 32; ++q) {
            const int g = q >> 3, v = q & 7;
            u32x4 w;
#pragma unroll
            for (int i = 0; i < 3; ++i) w[i] = PK[g * 7 + ((3 * v + i) >> 2)][(3 * v + i) & 3];
            w.w = E[q >> 2][q & 3];
            w = byte_swap(w);
#pragma unroll
            for (int c = 0; c < 4; ++c) w[c] = bad ? 0x7fc00000u : __builtin_amdgcn_alignbit(w[c], w[c], 1);
            store16_sys(ob + (size_t)((q >> 4) * 128 + ((q >> 2) & 3) * 16 + (q & 3)) * (GN * 4), ovo, w);
        }
    }
    DIAG_END;
}

extern "C" void kernel_launch(int rank, int nranks, ncclComm_t comm,
                              hipStream_t stream,
                              void* const* d_in, const int* in_sizes, int n_in,
                              void* d_out, int out_size,
                              const int* in_dtypes, int out_dtype,
                              void* const* d_out_all,
                              void* const* d_ws_all, size_t ws_bytes) {
    (void)comm; (void)in_dtypes; (void)out_dtype; (void)d_out;
    if (nranks != NR || (unsigned)rank >= (unsigned)NR || n_in < 2 || ws_bytes < WS_NEED ||
        in_sizes[0] != GM * KS || in_sizes[1] != KS * GN || out_size != (GM / NR) * GN) {
        fprintf(stderr, "gemm_rs: unexpected problem shape\n");
        return;
    }
    HIP_CHECK(hipFuncSetAttribute((const void*)gemm_rs_kernel, hipFuncAttributeMaxDynamicSharedMemorySize, (int)LDS_BYTES));
    Params p;
    p.x = (const float*)d_in[0]; p.w = (const float*)d_in[1];
    for (int i = 0; i < NR; ++i) { p.out[i] = (float*)d_out_all[i]; p.ws[i] = (char*)d_ws_all[i]; }
    p.rank = rank; p.pad = 0;
    hipLaunchKernelGGL(gemm_rs_kernel, dim3(GRID), dim3(NTHR), LDS_BYTES, stream, p);
    HIP_CHECK(hipGetLastError());
}
```

```cpp
#include <hip/hip_runtime.h>
#include <rccl/rccl.h>
#include <cstdio>
#include <cstdint>

#define HIP_CHECK(expr) do { hipError_t _e = (expr); if (_e != hipSuccess) { \
    fprintf(stderr, "HIP error %s at %s:%d\n", hipGetErrorString(_e), __FILE__, __LINE__); \
    return; } } while (0)

#define LAS __attribute__((address_space(3)))
#define GAS __attribute__((address_space(1)))
typedef GAS char gchar;
typedef GAS unsigned gu32;
typedef _Float16 h16x8 __attribute__((ext_vector_type(8)));
typedef float f32x4 __attribute__((ext_vector_type(4)));
typedef float f32x2 __attribute__((ext_vector_type(2)));
typedef unsigned u32x4 __attribute__((ext_vector_type(4)));

constexpr int NR = 8;
constexpr int GM = 4096, GN = 32768, KS = 512, NL = 4096;
constexpr int GRID = 256, NTHR = 512;
constexpr unsigned UNIT = 32768;
constexpr unsigned HTB = 16384;
constexpr int NKT = 64;
constexpr size_t WS_TICKET = 0, WS_TMO = 128, WS_LINKQ = 1024, WS_OTK = 2048, WS_OREL = 3072;
constexpr size_t WS_FLAG = 4096;
constexpr size_t WS_CFLAG = 16384;
constexpr size_t WS_AFLAG = 32768;
constexpr size_t WS_DIAG = 65536;
constexpr size_t OPND_BYTES = (size_t)16 * NKT * UNIT;
constexpr size_t WS_STAGE = 1u << 20;
constexpr size_t WS_PIN = WS_STAGE + 2 * OPND_BYTES;
constexpr unsigned CHUNK = 36864;
constexpr size_t WS_PACK = WS_PIN + (size_t)2 * 8 * 256 * UNIT;
constexpr size_t WS_NEED = WS_PACK + (size_t)NR * 256 * CHUNK;
constexpr int NEC = 12;
constexpr unsigned long long DEADLINE = 100000000ull;
constexpr unsigned L_CTL = 8 * HTB;
constexpr unsigned L_AMAX = L_CTL, L_CONS = L_CTL + 48, L_TURN = L_CTL + 64, L_CALL = L_CTL + 96, L_ITEM = L_CTL + 128, L_CLS = L_CTL + 192;
constexpr unsigned L_QI = L_CTL + 256, L_HDR = L_CTL + 320, L_TAB = L_CTL + 384, LDS_BYTES = L_CTL + 1024;

#define GLDS_AUX 17
struct Params {
    const float* x; const float* w;
    float* out[NR]; char* ws[NR];
    int rank; int pad;
};

#define STAMP(i) do {} while (0)

__device__ __forceinline__ void store16_sys(const gchar* sbase, unsigned voff, u32x4 v) {
    asm volatile("global_store_dwordx4 %0, %1, %2 sc0 sc1\n\ts_nop 1" :: "v"(voff), "v"(v), "s"(sbase) : "memory");
}
__device__ __forceinline__ unsigned ld_sys(const gu32* q) { return __hip_atomic_load(q, __ATOMIC_RELAXED, __HIP_MEMORY_SCOPE_SYSTEM); }
__device__ __forceinline__ unsigned ld_agt(const gu32* q) { return __hip_atomic_load(q, __ATOMIC_RELAXED, __HIP_MEMORY_SCOPE_AGENT); }
__device__ __forceinline__ unsigned rfl(unsigned v) { return (unsigned)__builtin_amdgcn_readfirstlane((int)v); }

template <class T> __device__ __forceinline__ gchar* uni(T q) {
    unsigned lo = rfl((unsigned)(uintptr_t)q), hi = rfl((unsigned)((uintptr_t)q >> 32));
    asm volatile("s_nop 4" : "+s"(lo), "+s"(hi));
    return (gchar*)(((uintptr_t)hi << 32) | lo);
}

__device__ __forceinline__ unsigned wave_max(unsigned x) {
    int mb = (int)x;
    mb = max(mb, __builtin_amdgcn_update_dpp(0, mb, 0x111, 0xf, 0xf, false));
    mb = max(mb, __builtin_amdgcn_update_dpp(0, mb, 0x112, 0xf, 0xf, false));
    mb = max(mb, __builtin_amdgcn_update_dpp(0, mb, 0x114, 0xf, 0xf, false));
    mb = max(mb, __builtin_amdgcn_update_dpp(0, mb, 0x118, 0xf, 0xf, false));
    mb = max(mb, __builtin_amdgcn_update_dpp(0, mb, 0x142, 0xa, 0xf, false));
    mb = max(mb, __builtin_amdgcn_update_dpp(0, mb, 0x143, 0xc, 0xf, false));
    return (unsigned)__builtin_amdgcn_readlane(mb, 63);
}
__device__ __forceinline__ u32x4 byte_swap(u32x4 r) {
    u32x4 o;
    o.x = __builtin_amdgcn_perm(r.w, r.x, 0x04020100u); o.y = __builtin_amdgcn_perm(r.w, r.y, 0x05020100u); o.z = __builtin_amdgcn_perm(r.w, r.z, 0x06020100u);
    o.w = __builtin_amdgcn_perm(r.y, r.x, 0x0c0c0703u) | __builtin_amdgcn_perm(r.w, r.z, 0x07030c0cu);
    return o;
}

typedef unsigned short u16x2 __attribute__((ext_vector_type(2)));
typedef unsigned u32x2 __attribute__((ext_vector_type(2)));
constexpr unsigned Q_P6 = 6144, Q_LEN = 6656, Q_STR = 6720, Q_BYTES = 8192;
__device__ __forceinline__ u16x2 as16(unsigned a) { return __builtin_bit_cast(u16x2, a); }
__device__ __forceinline__ unsigned as32(u16x2 a) { return __builtin_bit_cast(unsigned, a); }
__device__ __forceinline__ unsigned pk_sub(unsigned a, unsigned b) { return as32(as16(a) - as16(b)); }
__device__ __forceinline__ unsigned pk_subs(unsigned a, unsigned b) { return as32(__builtin_elementwise_sub_sat(as16(a), as16(b))); }
__device__ __forceinline__ unsigned pk_min(unsigned a, unsigned b) { return as32(__builtin_elementwise_min(as16(a), as16(b))); }
__device__ __forceinline__ unsigned wave_scan(unsigned x) {
    int v = (int)x;
    v += __builtin_amdgcn_update_dpp(0, v, 0x111, 0xf, 0xf, false);
    v += __builtin_amdgcn_update_dpp(0, v, 0x112, 0xf, 0xf, false);
    v += __builtin_amdgcn_update_dpp(0, v, 0x114, 0xf, 0xf, false);
    v += __builtin_amdgcn_update_dpp(0, v, 0x118, 0xf, 0xf, false);
    v += __builtin_amdgcn_update_dpp(0, v, 0x142, 0xa, 0xf, false);
    v += __builtin_amdgcn_update_dpp(0, v, 0x143, 0xc, 0xf, false);
    return (unsigned)v;
}
__device__ __forceinline__ unsigned q_encode(const u32x4 (&W)[8], unsigned B, LAS unsigned char* dst, unsigned lane) {
    const unsigned B1 = (B + 1u) * 0x04000400u;
    unsigned v[32], seg[4], pos[4];
    u32x4 X[4];
#pragma unroll
    for (int s = 0; s < 4; ++s) {
        unsigned a = 0, p = 0;
        X[s].x = __builtin_amdgcn_perm(W[2 * s].y, W[2 * s].x, 0x07050301u); X[s].y = __builtin_amdgcn_perm(W[2 * s].w, W[2 * s].z, 0x07050301u);
        X[s].z = __builtin_amdgcn_perm(W[2 * s + 1].y, W[2 * s + 1].x, 0x07050301u); X[s].w = __builtin_amdgcn_perm(W[2 * s + 1].w, W[2 * s + 1].z, 0x07050301u);
#pragma unroll
        for (int k = 8 * s; k < 8 * s + 8; ++k) {
            const unsigned w = W[k >> 2][k & 3];
            const unsigned dq = as32(as16(pk_sub(B1, w & 0x7c007c00u)) >> 10);
            const unsigned ix = pk_min(dq, 0x00070007u) | 0x0c000c00u;
            const unsigned len = __builtin_amdgcn_perm(0x05040201u, 0x00000003u, ix) + pk_subs(dq, 0x00070007u);
            v[k] = (w & 0x83ff83ffu) | (__builtin_amdgcn_perm(0u, 0x03020100u, ix) << 10);
            p += len & 0xffffu; a |= 1u << (p & 31u);
            p += len >> 16;     a |= 1u << (p & 31u);
        }
        seg[s] = a >> 1; pos[s] = p;
        asm volatile("" : "+v"(v[8 * s]), "+v"(v[8 * s + 1]), "+v"(v[8 * s + 2]), "+v"(v[8 * s + 3]), "+v"(v[8 * s + 4]), "+v"(v[8 * s + 5]), "+v"(v[8 * s + 6]), "+v"(v[8 * s + 7]),
                     "+v"(X[s]), "+v"(seg[s]), "+v"(pos[s]));
    }
    const bool ovf = max(max(pos[0], pos[1]), max(pos[2], pos[3])) > 31u;
    const unsigned tot = ovf ? 0u : pos[0] + pos[1] + pos[2] + pos[3];
    const unsigned incl = wave_scan(tot), total = (unsigned)__builtin_amdgcn_readlane((int)incl, 63);
    const unsigned long long om = __ballot(ovf);
    const unsigned xb = Q_STR + (((total + 127u) >> 7) << 4), size = xb + 64u * (unsigned)__popcll(om);
    if (size >= Q_BYTES) return Q_BYTES;
    *(LAS u32x4*)(dst + Q_STR + lane * 16u) = (u32x4){0u, 0u, 0u, 0u};
#pragma unroll
    for (int j = 0; j < 6; ++j)
#pragma unroll
        for (int i = 0; i < 4; ++i) v[4 * j + i] |= (v[26 + j] << (12 - 3 * i)) & 0x70007000u;
#pragma unroll
    for (int j = 0; j < 6; ++j) v[24 + j / 3] |= (v[26 + j] >> (1 + j % 3)) & (0x80008000u >> (1 + j % 3));
#pragma unroll
    for (int j = 0; j < 6; ++j) *(LAS u32x4*)(dst + j * 1024u + lane * 16u) = (u32x4){v[4 * j], v[4 * j + 1], v[4 * j + 2], v[4 * j + 3]};
    *(LAS u32x2*)(dst + Q_P6 + lane * 8u) = (u32x2){v[24], v[25]};
    const unsigned slot = __builtin_amdgcn_mbcnt_hi((unsigned)(om >> 32), __builtin_amdgcn_mbcnt_lo((unsigned)om, 0u));
    *(LAS unsigned char*)(dst + Q_LEN + lane) = (unsigned char)(ovf ? 128u + slot : tot);
    if (!ovf) {
        unsigned off = incl - tot;
#pragma unroll
        for (int s = 0; s < 4; ++s) {
            const unsigned long long x = (unsigned long long)seg[s] << (off & 31u);
            LAS unsigned* q = (LAS unsigned*)(dst + Q_STR + (off >> 5) * 4u);
            __hip_atomic_fetch_or(q, (unsigned)x, __ATOMIC_RELAXED, __HIP_MEMORY_SCOPE_WORKGROUP);
            __hip_atomic_fetch_or(q + 1, (unsigned)(x >> 32), __ATOMIC_RELAXED, __HIP_MEMORY_SCOPE_WORKGROUP);
            off += pos[s];
        }
    } else {
#pragma unroll
        for (int j = 0; j < 4; ++j) *(LAS u32x4*)(dst + xb + slot * 64u + j * 16u) = X[j];
    }
    return size;
}
#define RLOAD4(rs, vo, so) __builtin_bit_cast(u32x4, __builtin_amdgcn_raw_buffer_load_b128(rs, vo, so, 17))
#define RLOAD2(rs, vo, so) __builtin_bit_cast(u32x2, __builtin_amdgcn_raw_buffer_load_b64(rs, vo, so, 17))
#define RLOAD1(rs, vo, so) __builtin_amdgcn_raw_buffer_load_b32(rs, vo, so, 17)
struct QIn { u32x4 P[6]; u32x2 P6; unsigned lb, off, xo, w4; u32x4 win; };
__device__ __forceinline__ void q_load1(__amdgpu_buffer_rsrc_t rs, unsigned so, unsigned lane, QIn& q) {
#pragma unroll
    for (int j = 0; j < 6; ++j) q.P[j] = RLOAD4(rs, lane * 16u, so + j * 1024u);
    q.P6 = RLOAD2(rs, lane * 8u, so + Q_P6);
}
__device__ __forceinline__ void q_load2(__amdgpu_buffer_rsrc_t rs, unsigned so, QIn& q) {
    const unsigned tot = q.lb >= 128u ? 0u : q.lb, incl = wave_scan(tot), total = (unsigned)__builtin_amdgcn_readlane((int)incl, 63);
    q.off = incl - tot;
    q.xo = Q_STR + (((total + 127u) >> 7) << 4);
    q.win = RLOAD4(rs, (q.off >> 5) * 4u, so + Q_STR);
    q.w4 = RLOAD1(rs, (q.off >> 5) * 4u + 16u, so + Q_STR);
}
__device__ __forceinline__ void q_finish(__amdgpu_buffer_rsrc_t rs, unsigned so, unsigned B, const QIn& q, u32x4 (&W)[8]) {
    const bool ovf = q.lb >= 128u;
    u32x4 X[4];
    if (ovf) {
#pragma unroll
        for (int j = 0; j < 4; ++j) X[j] = RLOAD4(rs, q.xo + (q.lb - 128u) * 64u + j * 16u, so);
    }
    unsigned v[32];
#pragma unroll
    for (int k = 0; k < 24; ++k) v[k] = q.P[k >> 2][k & 3];
    v[24] = q.P6.x; v[25] = q.P6.y;
#pragma unroll
    for (int j = 0; j < 6; ++j) {
        unsigned d = (v[24 + j / 3] << (1 + j % 3)) & 0x80008000u;
#pragma unroll
        for (int i = 0; i < 4; ++i) d |= (v[4 * j + i] & 0x70007000u) >> (12 - 3 * i);
        v[26 + j] = d;
    }
    u32x4 win;
    win.x = __builtin_amdgcn_alignbit(q.win.y, q.win.x, q.off); win.y = __builtin_amdgcn_alignbit(q.win.z, q.win.y, q.off);
    win.z = __builtin_amdgcn_alignbit(q.win.w, q.win.z, q.off); win.w = __builtin_amdgcn_alignbit(q.w4, q.win.w, q.off);
    const unsigned B1 = (B + 1u) * 0x04000400u;
#pragma unroll
    for (int s = 0; s < 4; ++s) {
        unsigned a = win.x << 1, used = 0;
#pragma unroll
        for (int k = 8 * s; k < 8 * s + 8; ++k) {
            const unsigned c = v[k] & 0x0c000c00u;
            const unsigned l0 = (c & 0xffffu) ? 0u : (unsigned)__builtin_ctz(a | 0x80000000u);
            a = (a >> l0) & ~1u;
            const unsigned l1 = (c >> 16) ? 0u : (unsigned)__builtin_ctz(a | 0x80000000u);
            a = (a >> l1) & ~1u;
            const unsigned len = l0 | (l1 << 16);
            used += len;
            const unsigned dq = __builtin_amdgcn_perm(0x00000006u, 0x00050400u, pk_min(len, 0x00040004u) | 0x0c000c00u) + pk_subs(len, 0x00040004u);
            W[k >> 2][k & 3] = (v[k] & 0x83ff83ffu) | (pk_sub(B1, c + (dq << 10)) & 0x7c007c00u);
        }
        if (s < 3) {
            const unsigned sh = (used + (used >> 16)) & 31u;
            win.x = __builtin_amdgcn_alignbit(win.y, win.x, sh); win.y = __builtin_amdgcn_alignbit(win.z, win.y, sh);
            win.z = __builtin_amdgcn_alignbit(win.w, win.z, sh); win.w >>= sh;
        }
    }
    if (ovf) {
#pragma unroll
        for (int k = 0; k < 32; ++k) W[k >> 2][k & 3] = __builtin_amdgcn_perm(X[k >> 3][(k >> 1) & 3], v[k], (k & 1) ? 0x07020600u : 0x05020400u);
    }
}

__device__ __forceinline__ void item_decode(int i, int d, int& isW, int& j, int& u) {
    if (i < 208) { j = i / 26; const int r = i - j * 26; isW = r < 24 ? r & 1 : 0; u = r < 24 ? r >> 1 : r - 12; if (!isW) { u = u < 7 ? 2 * u : 2 * u - 13; if (u >= 2 * d) u += 2; } }
    else { const int r = i - 208; j = r / 6; const int t = r - j * 6; isW = t < 4; u = isW ? NEC + t : 2 * d + t - 4; }
}

__device__ __forceinline__ void item_load(const Params& p, int i, int d, f32x4 (&L)[16]) {
    const bool valid = i < 256;
    int isW, j, u; item_decode(valid ? i : 0, d, isW, j, u);
    const unsigned t = threadIdx.x;
    const gchar* b = uni(isW ? (const char*)p.w + ((size_t)(j * 64) * GN + d * NL + u * 256) * 4 : (const char*)p.x + ((size_t)(u * 256) * KS + j * 64) * 4);
    unsigned vo = isW ? (t >> 6) * (4u * GN * 4u) + (t & 63u) * 16u : (t >> 3) * (KS * 4u) + ((t & 7u) ^ ((t >> 4) & 7u)) * 16u;
    if (!valid) vo = 0;
    const unsigned s0 = isW ? GN * 4u : 128u, s1 = isW ? 2u * GN * 4u : 64u * KS * 4u, s2 = isW ? 32u * GN * 4u : 128u * KS * 4u, s3 = isW ? 16u * GN * 4u : 32u * KS * 4u;
#pragma unroll
    for (int n = 0; n < 16; ++n)
        L[n] = *(const GAS f32x4*)(b + ((n & 1) * s0 + ((n >> 1) & 1) * s1 + ((n >> 2) & 1) * s2 + (n >> 3) * s3) + vo);
}

__device__ __forceinline__ unsigned pk2(float a, float b, int e) {
    typedef _Float16 h2 __attribute__((ext_vector_type(2)));
    h2 h; h.x = (_Float16)__builtin_amdgcn_ldexpf(a, e); h.y = (_Float16)__builtin_amdgcn_ldexpf(b, e);
    return __builtin_bit_cast(unsigned, h);
}

#define POLL_SLEEP 8
#define SELF_DELAY 4
#define C_BYPASS 0
#define FLAG_SLEEP 16
#define CASCADE_FROM 225u
#define PUSH_REL 1u
#define RAW_FIRST 1u
#define SEND_PRIO 3
#define THR_L 8
#define OUT_W 3
#define REL_AT 24
#define LINK_EXTRA 8u
#define WG_SENDERS 1
#define WG_SENDERS_C 4
#define PUSH_N 7
#define STR2(x) #x
#define STR(x) STR2(x)
#define LBAR do { asm volatile("s_waitcnt lgkmcnt(0)" ::: "memory"); __builtin_amdgcn_s_barrier(); asm volatile("" ::: "memory"); } while (0)

typedef volatile LAS unsigned vl32;
typedef volatile LAS u32x4 vl128;
#define LGKM0 asm volatile("s_waitcnt lgkmcnt(0)" ::: "memory")
__device__ __forceinline__ u32x4 lds_wait4(LAS unsigned char* lds, unsigned off, unsigned v) {
    for (;;) {
        const u32x4 q = *(vl128*)(lds + off);
        if (rfl(min(min(q.x, q.y), min(q.z, q.w))) >= v) { asm volatile("" ::: "memory"); return q; }
        __builtin_amdgcn_s_sleep(1);
    }
}
__device__ __forceinline__ unsigned item_amax(LAS unsigned char* lds, const f32x4 (&L)[16], unsigned k) {
    unsigned mu = 0; float sm = 0.f;
#pragma unroll
    for (int n = 0; n < 16; ++n) {
        const u32x4 q = __builtin_bit_cast(u32x4, L[n]) & 0x7fffffffu; mu = max(max(mu, q.x), max(q.y, max(q.z, q.w)));
        const f32x4 f = __builtin_bit_cast(f32x4, q); sm += (f.x + f.y) + (f.z + f.w);
    }
    const unsigned mb = wave_max(mu >> 23);
#define DPP_ADD(ctl, rm) sm += __builtin_bit_cast(float, __builtin_amdgcn_update_dpp(0, __builtin_bit_cast(int, sm), ctl, rm, 0xf, false))
    DPP_ADD(0x111, 0xf); DPP_ADD(0x112, 0xf); DPP_ADD(0x114, 0xf); DPP_ADD(0x118, 0xf); DPP_ADD(0x142, 0xa); DPP_ADD(0x143, 0xc);
    const unsigned em = ((unsigned)__builtin_amdgcn_readlane(__builtin_bit_cast(int, sm * (2.32f / 4096.f)), 63) >> 23) & 255u;
    LGKM0;
    if ((threadIdx.x & 63) == 63) *(vl32*)(lds + L_AMAX + (k & 1u) * 16 + (threadIdx.x >> 6) * 4) = ((k + 1u) << 8) | mb;
    return min(em, mb);
}

__device__ __forceinline__ unsigned img_off(int isW, unsigned q, int c) {
    return (isW ? q * 2048u + (c >> 2) * 8192u + (c & 1) * 1024u : q * 1024u + (c >> 2) * 4096u + (c & 1) * 8192u) + ((c >> 1) & 1) * HTB;
}
__device__ __forceinline__ int conv_step(const Params& p, LAS unsigned char* lds, gu32* tick, int d, unsigned k, f32x4 (&L)[16], const f32x4 (&Ln)[16], unsigned& top) {
    unsigned t = threadIdx.x; asm volatile("" : "+v"(t));
    const u32x4 am = lds_wait4(lds, L_AMAX + (k & 1u) * 16, (k + 1u) << 8) & 255u;
    const int i = (int)rfl(*(vl32*)(lds + L_ITEM + (k & 15u) * 4)), i1 = (int)rfl(*(vl32*)(lds + L_ITEM + ((k + 1u) & 15u) * 4)), i2 = (int)rfl(*(vl32*)(lds + L_ITEM + ((k + 2u) & 15u) * 4));
    const bool valid = i < 256;
    const int cls = (int)rfl(max(max(am.x, am.y), max(am.z, am.w)) + 2u) >> 4;
    const int e = 128 - 16 * cls;
    int isW, j, u; item_decode(valid ? i : 0, d, isW, j, u);
    u32x4 H[8];
#pragma unroll
    for (int h = 0; h < 2; ++h) {
        const f32x4* Q = L + 8 * h; u32x4* G = H + 4 * h;
        if (isW) {
#pragma unroll
            for (int c = 0; c < 4; ++c) { G[c].x = pk2(Q[0][c], Q[1][c], e); G[c].y = pk2(Q[2][c], Q[3][c], e); G[c].z = pk2(Q[4][c], Q[5][c], e); G[c].w = pk2(Q[6][c], Q[7][c], e); }
        } else {
#pragma unroll
            for (int c = 0; c < 4; ++c) { G[c].x = pk2(Q[2 * c].x, Q[2 * c].y, e); G[c].y = pk2(Q[2 * c].z, Q[2 * c].w, e); G[c].z = pk2(Q[2 * c + 1].x, Q[2 * c + 1].y, e); G[c].w = pk2(Q[2 * c + 1].z, Q[2 * c + 1].w, e); }
        }
    }
    unsigned tk = 255u;
    gu32* q = tick; asm volatile("" : "+v"(q));
    if (t == 0 && tick && valid) tk = __hip_atomic_fetch_add(q, 1u, __ATOMIC_RELAXED, __HIP_MEMORY_SCOPE_AGENT);
    if (valid) {
        if (k >= 4) while (rfl(*(vl32*)(lds + L_CONS)) < k - 3u) __builtin_amdgcn_s_sleep(2);
        asm volatile("" ::: "memory");
        if (t == 0) *(vl32*)(lds + L_CLS + (k & 15u) * 4) = (unsigned)cls;
        LAS unsigned char* img = lds + (k & 3u) * UNIT;
#pragma unroll
        for (int c = 0; c < 8; ++c) *(LAS u32x4*)(img + (tick ? rfl(t >> 6) * Q_BYTES + c * 1024u : img_off(isW, rfl(t >> 6), c)) + (t & 63u) * 16u) = H[c];
        LGKM0;
        if ((t & 63u) == 0) *(vl32*)(lds + L_QI + (k & 3u) * 16 + (t >> 6) * 4) =
            (tick && k >= RAW_FIRST) ? ((2u * k + 1u) << 20) | ((unsigned)min(max((int)top + e - 112, 2), 31) << 14) : ((2u * k + 2u) << 20) | Q_BYTES;
    }
    item_load(p, i2, d, L);
    __builtin_amdgcn_s_waitcnt(0x4F70);
    asm volatile("" : "+v"(tk));
    if (t == 0) *(vl32*)(lds + L_ITEM + ((k + 3u) & 15u) * 4) = 24u + (tk & 255u);
    top = item_amax(lds, Ln, k + 1u);
    return i1;
}

__device__ __forceinline__ gu32* stage_unit(gchar* myws, int rank, int srel, int i, int& isW, const gchar*& ub) {
    int j, u; item_decode(i, rank, isW, j, u);
    const int idx = (srel == 7) ? j : 8 + j * 7 + srel;
    ub = myws + WS_STAGE + (size_t)isW * OPND_BYTES + (size_t)(u * NKT + idx) * UNIT;
    return (gu32*)(myws + WS_FLAG) + (isW * 16 + u) * NKT + idx;
}
__device__ __forceinline__ void publish(gu32* f, unsigned call, unsigned info, bool remote) {
    if ((threadIdx.x & 63u) == 0) {
        if (remote) __hip_atomic_store((GAS unsigned long long*)f, ((unsigned long long)call << 32) | info, __ATOMIC_RELAXED, __HIP_MEMORY_SCOPE_SYSTEM);
        else __hip_atomic_store(f, (call << 8) | (info & 31u), __ATOMIC_RELAXED, __HIP_MEMORY_SCOPE_SYSTEM);
    }
}
__device__ __forceinline__ void send_fetch(LAS unsigned char* lds, unsigned k, int q, u32x4 (&H)[8]) {
#pragma unroll
    for (int c = 0; c < 8; ++c) H[c] = *(const LAS u32x4*)(lds + (k & 3u) * UNIT + q * 8192u + c * 1024u + (threadIdx.x & 63u) * 16u);
}
template <int Q> __device__ __forceinline__ void send8(const gchar* ub, const u32x4 (&H)[8], unsigned size, gu32* pf, unsigned call, unsigned pv, bool remote) {
#pragma unroll
    for (int c = 0; c < 8; ++c) {
        if (c < 6 || c * 1024u + (threadIdx.x & 63u) * 16u < size) store16_sys(ub + (Q * 8192u + c * 1024u), (threadIdx.x & 63u) * 16u, H[c]);
        asm volatile("s_waitcnt vmcnt(" STR(PUSH_N) ")" ::: "memory");
        if (c < 6 && Q * 6 + c == PUSH_N - 1 && pf) publish(pf, call, pv, remote);
    }
}

__device__ __forceinline__ unsigned push_items(const Params& p, LAS unsigned char* lds, gchar* myws, int rank, int d, int first, int stride, gu32* tick) {
    unsigned call;
#define PSTAMP(i) do {} while (0)
    if (rfl(threadIdx.x >> 6) < 4) {
        f32x4 RA[16], RB[16];
        if (!tick) for (int z = 0; z < SELF_DELAY; ++z) __builtin_amdgcn_s_sleep(32);
        item_load(p, first, d, RA);
        PSTAMP(12);
        LBAR;
        PSTAMP(13);
        unsigned top = item_amax(lds, RA, 0);
        PSTAMP(14);
        item_load(p, first + stride, d, RB);
        PSTAMP(15);
        int nx; unsigned k = 0;
        do { conv_step(p, lds, tick, d, k, RA, RB, top); nx = conv_step(p, lds, tick, d, k + 1u, RB, RA, top); k += 2; } while (nx < 256);
        LBAR;
        call = rfl(*(vl32*)(lds + L_CALL));
    } else {
        const unsigned ts = threadIdx.x - 256u;
        if (ts < 80) *(vl32*)(lds + L_CTL + ts * 4) = (ts >= (L_ITEM - L_CTL) / 4 && ts < (L_ITEM - L_CTL) / 4 + 3) ? (unsigned)(first + stride * (int)(ts - (L_ITEM - L_CTL) / 4)) : 0u;
        unsigned tk = 0;
        gu32* q = (gu32*)(myws + WS_TICKET); asm volatile("" : "+v"(q));
        if (ts == 0) tk = __hip_atomic_fetch_add(q, 1u, __ATOMIC_RELAXED, __HIP_MEMORY_SCOPE_AGENT);
        gchar* dws = uni(p.ws[d]);
        LBAR;
        if (ts < 64) {
            asm volatile("" : "+v"(tk));
            call = (rfl(tk) / GRID + 1u) & 0xffffffu;
            if (ts == 0) *(vl32*)(lds + L_CALL) = call;
            u32x4 HA[8], HB[8]; gu32* pf = nullptr; unsigned pv = 0;
            const bool remote = tick != nullptr;
            __builtin_amdgcn_s_setprio(SEND_PRIO);
            int i = first; unsigned k = 0;
            lds_wait4(lds, L_QI, 2u << 20); send_fetch(lds, 0, 0, HA);
            do {
                const gchar* ub; gu32* f;
                if (remote) { const unsigned ui = ((call & 1u) * 8 + ((rank - d - 1) & 7)) * 256 + i; ub = dws + WS_PIN + (size_t)ui * UNIT; f = (gu32*)(dws + WS_AFLAG + ui * 8); }
                else { int isW; f = stage_unit(myws, rank, 7, i, isW, ub); }
                ub = uni(ub);
                const u32x4 qv = *(vl128*)(lds + L_QI + (k & 3u) * 16) & 0xfffffu;
                const unsigned q0 = rfl(qv.x), q1 = rfl(qv.y), q2 = rfl(qv.z), q3 = rfl(qv.w);
                const unsigned fv = rfl(*(vl32*)(lds + L_CLS + (k & 15u) * 4)) | (q0 >> 14 << 5) | (q1 >> 14 << 10) | (q2 >> 14 << 15) | (q3 >> 14 << 20);
                i = (int)rfl(*(vl32*)(lds + L_ITEM + ((k + 1u) & 15u) * 4));
                send_fetch(lds, k, 1, HB); send8<0>(ub, HA, q0 & 0x3fffu, pf, call, pv, remote);
                send_fetch(lds, k, 2, HA); send8<1>(ub, HB, q1 & 0x3fffu, pf, call, pv, remote);
                send_fetch(lds, k, 3, HB); send8<2>(ub, HA, q2 & 0x3fffu, pf, call, pv, remote);
                LGKM0;
                if (ts == 0) *(vl32*)(lds + L_CONS) = k + 1u;
                if (i < 256) { lds_wait4(lds, L_QI + ((k + 1u) & 3u) * 16, (2u * k + 4u) << 20);
                    send_fetch(lds, k + 1u, 0, HA); }
                else if (tick && ts == 0) __hip_atomic_fetch_add(tick + (WS_OREL - WS_LINKQ) / 4, PUSH_REL, __ATOMIC_RELAXED, __HIP_MEMORY_SCOPE_AGENT);
                send8<3>(ub, HB, q3 & 0x3fffu, pf, call, pv, remote);
                pf = f; pv = fv; ++k;
            } while (i < 256);
            asm volatile("s_waitcnt vmcnt(0)" ::: "memory");
            publish(pf, call, pv, remote);
            __builtin_amdgcn_s_setprio(0);
        } else if (tick) {
            for (unsigned n = rfl(ts >> 6) - 1u;; n += 3u) {
                const unsigned k = n >> 2, qq = n & 3u;
                if (rfl(*(vl32*)(lds + L_ITEM + (k & 15u) * 4)) >= 256u) break;
                unsigned st;
                while ((st = rfl(*(vl32*)(lds + L_QI + (k & 3u) * 16 + qq * 4))) < ((2u * k + 1u) << 20)) __builtin_amdgcn_s_sleep(1);
                if (st >= ((2u * k + 2u) << 20)) continue;
                asm volatile("" ::: "memory");
                u32x4 H[8];
                send_fetch(lds, k, (int)qq, H);
                const unsigned B = (st >> 14) & 31u, size = q_encode(H, B, lds + (k & 3u) * UNIT + qq * Q_BYTES, ts & 63u);
                LGKM0;
                if ((ts & 63u) == 0) *(vl32*)(lds + L_QI + (k & 3u) * 16 + qq * 4) = ((2u * k + 2u) << 20) | (size < Q_BYTES ? B << 14 : 0u) | size;
            }
        }
        LBAR;
        call = rfl(*(vl32*)(lds + L_CALL));
    }
    return call;
}

__device__ __forceinline__ void unpack_units(gchar* myws, int rank, unsigned call, int g0) {
    const unsigned lane = threadIdx.x & 63u;
    gu32* tmo = (gu32*)(myws + WS_TMO);
    for (int g = g0; g < 7 * 256; g += 256) {
        const int i = g / 7, srel = g - i * 7;
        const unsigned ui = ((call & 1u) * 8 + srel) * 256 + i;
        const GAS unsigned long long* af = (const GAS unsigned long long*)(myws + WS_AFLAG) + ui;
        unsigned info;
        const unsigned long long t0 = __builtin_amdgcn_s_memrealtime();
        for (;;) {
            const unsigned long long fv = __hip_atomic_load(af, __ATOMIC_RELAXED, __HIP_MEMORY_SCOPE_SYSTEM);
            info = rfl((unsigned)fv);
            if (rfl((unsigned)(fv >> 32)) >= call) break;
            if (rfl(ld_agt(tmo)) == call || __builtin_amdgcn_s_memrealtime() - t0 > DEADLINE) {
                if (lane == 0) __hip_atomic_store(tmo, call, __ATOMIC_RELAXED, __HIP_MEMORY_SCOPE_AGENT);
                return;
            }
            __builtin_amdgcn_s_sleep(FLAG_SLEEP);
        }
        asm volatile("" ::: "memory");
        const __amdgpu_buffer_rsrc_t rs = __builtin_amdgcn_make_buffer_rsrc((void*)uni(myws + WS_PIN + (size_t)ui * UNIT), 0, UNIT, 0x00020000);
        int isW; const gchar* ub; gu32* f = stage_unit(myws, rank, srel, i, isW, ub); ub = uni(ub);
        QIn Q[4];
#pragma unroll
        for (int q = 0; q < 4; ++q) Q[q].lb = __builtin_amdgcn_raw_buffer_load_b8(rs, lane, q * Q_BYTES + Q_LEN, 17);
#pragma unroll
        for (int q = 0; q < 4; ++q) q_load1(rs, q * Q_BYTES, lane, Q[q]);
#pragma unroll
        for (int q = 0; q < 4; ++q) q_load2(rs, q * Q_BYTES, Q[q]);
        __builtin_amdgcn_s_waitcnt(0x0F70);
#pragma unroll
        for (int q = 0; q < 4; ++q) {
            const unsigned B = (info >> (5 + 5 * q)) & 31u;
            u32x4 W[8];
            if (B) q_finish(rs, q * Q_BYTES, B, Q[q], W);
            else {
#pragma unroll
                for (int j = 0; j < 6; ++j) W[j] = Q[q].P[j];
                W[6] = RLOAD4(rs, lane * 16u, q * Q_BYTES + 6144u); W[7] = RLOAD4(rs, lane * 16u, q * Q_BYTES + 7168u);
                __builtin_amdgcn_s_waitcnt(0x0F70);
            }
#pragma unroll
            for (int c = 0; c < 8; ++c) store16_sys(ub + img_off(isW, q, c), lane * 16u, W[c]);
            __builtin_amdgcn_sched_barrier(0);
        }
        asm volatile("s_waitcnt vmcnt(0)" ::: "memory");
        publish(f, call, info, false);
    }
}

__global__ void __launch_bounds__(NTHR, 2) gemm_rs_kernel(const Params p) {
    extern __shared__ __attribute__((aligned(16))) unsigned char smem[];
    LAS unsigned char* lds = (LAS unsigned char*)smem;
    const int tid = threadIdx.x;
    const int rank = (int)rfl(p.rank), bid = (int)rfl(blockIdx.x);
    gchar* myws = uni(p.ws[rank]);
    STAMP(0);
    const int xcd = bid & 7, slot = bid >> 3;
    const int pd = slot < 7 ? (rank + 1 + slot) & 7 : rank, pfirst = slot < 7 ? xcd : (31 - slot) * 8 + xcd, pstride = slot < 7 ? 8 : 200;
    gu32* tmo = (gu32*)(myws + WS_TMO);

    int pm, pn;
    {
        int ar = -1;
        if (slot < 11) {
            const int c = xcd * 11 + slot;
            if (c < 32) { pm = 2 * rank + (c & 1); pn = c >> 1; } else { ar = (c - 32) >> 2; pn = NEC + (c & 3); }
        } else {
            const int li = slot - 11, q = li / 7;
            pn = 3 * (xcd & 3) + q; ar = (xcd >> 2) * 7 + (li - q * 7);
        }
        if (ar >= 0) pm = ar + (ar >= 2 * rank ? 2 : 0);
    }
    const unsigned call = push_items(p, lds, myws, rank, pd, pfirst, pstride, slot < 7 ? (gu32*)(myws + WS_LINKQ + slot * 128) : nullptr);
    if (slot >= 7 && slot < 11) unpack_units(myws, rank, call, (int)rfl(tid >> 6) * 32 + (slot - 7) * 8 + xcd);
    STAMP(1);

    const int wid = rfl(tid >> 6), lane = tid & 63, wr = wid >> 2, wc = wid & 3, fr = lane & 15, fq = lane >> 4;
    const gchar* cA = myws + WS_STAGE + (size_t)pm * NKT * UNIT;
    const gchar* cB = myws + WS_STAGE + OPND_BYTES + (size_t)pn * NKT * UNIT;
    const gu32* fA = (const gu32*)(myws + WS_FLAG) + pm * NKT;
    const gu32* fB = (const gu32*)(myws + WS_FLAG) + (16 + pn) * NKT;
    const unsigned voff[2] = {(unsigned)tid * 16u, (unsigned)tid * 16u + 8192u}; const unsigned ldsw = (unsigned)wid * 1024u;
    const unsigned aoff0 = (unsigned)((wr * 64 + fr) * 128 + ((fq ^ (fr >> 1)) * 16)), aoff1 = aoff0 ^ 64u;
    const unsigned boff = (unsigned)(fq * 2048 + (wc * 16 + fr) * 16);
    int ready = 0; unsigned long long chg = 0; int scur = 0, smax = 0;

#define SA(b, h) (((b) * 2 + (h)) * HTB)
#define SB(b, h) ((4 + (b) * 2 + (h)) * HTB)
#define STAGE(bufoff, gbase) do { _Pragma("unroll") for (int _i = 0; _i < 2; ++_i) \
        __builtin_amdgcn_global_load_lds((const gu32*)((gbase) + voff[_i]), (LAS unsigned*)(lds + (bufoff) + ldsw + _i * 8192), 16, 0, GLDS_AUX); } while (0)
#define LDA(dst, b, h) do { _Pragma("unroll") for (int m = 0; m < 4; ++m) { dst[m][0] = *(const LAS h16x8*)(lds + SA(b, h) + aoff0 + m * 2048); dst[m][1] = *(const LAS h16x8*)(lds + SA(b, h) + aoff1 + m * 2048); } } while (0)
#define LDB(dst, b, h) do { _Pragma("unroll") for (int n = 0; n < 2; ++n) _Pragma("unroll") for (int k = 0; k < 2; ++k) dst[n][k] = *(const LAS h16x8*)(lds + SB(b, h) + boff + n * 1024 + k * 8192); } while (0)
#define MMA(ai, bj, At, Bt) do { __builtin_amdgcn_s_setprio(1); _Pragma("unroll") for (int m = 0; m < 4; ++m) _Pragma("unroll") for (int n = 0; n < 2; ++n) _Pragma("unroll") for (int k = 0; k < 2; ++k) \
        acc[ai][bj][m][n] = __builtin_amdgcn_mfma_f32_16x16x32_f16(At[m][k], Bt[n][k], acc[ai][bj][m][n], 0, 0, 0); __builtin_amdgcn_s_setprio(0); } while (0)
#define WAIT_V(n) asm volatile("s_waitcnt vmcnt(" #n ")" ::: "memory")
#define WAIT_L(n) asm volatile("s_waitcnt lgkmcnt(" #n ")" ::: "memory")
#define BAR __builtin_amdgcn_s_barrier()
#define SCHED __builtin_amdgcn_sched_barrier(0)
#define HOOK(need) do { if (ready < (need)) { \
        if (wid == 0) { \
            const unsigned long long t0 = __builtin_amdgcn_s_memrealtime(); unsigned fa, fb; int rd; \
            unsigned ln = voff[0]; asm volatile("" : "+v"(ln)); ln >>= 4;     \
            for (;;) { \
                fa = ld_sys(fA + ln); fb = ld_sys(fB + ln); \
                const unsigned long long okm = __ballot((fa >> 8) >= call && (fb >> 8) >= call); \
                rd = (okm == ~0ull) ? 64 : (int)__builtin_ctzll(~okm); \
                if (rd >= (need)) break; \
                if (rfl(ld_agt(tmo)) == call || __builtin_amdgcn_s_memrealtime() - t0 > DEADLINE) { \
                    if (ln == 0) __hip_atomic_store(tmo, call, __ATOMIC_RELAXED, __HIP_MEMORY_SCOPE_AGENT); \
                    rd = 64; break; } \
                __builtin_amdgcn_s_sleep(POLL_SLEEP); \
            } \
            const int sv = (int)((fa & 31u) + (fb & 31u)); \
            const int sp = __builtin_amdgcn_update_dpp(-1, sv, 0x138, 0xf, 0xf, false);     \
            const unsigned long long cg = __ballot(sv != sp); \
            *(LAS int*)(lds + L_TAB + ln * 4) = sv; \
            if (ln == 0) { u32x4 h; h.x = (unsigned)rd; h.y = (unsigned)cg; h.z = (unsigned)(cg >> 32); h.w = 0; *(LAS u32x4*)(lds + L_HDR) = h; } \
            DIAG_WAIT(t0); \
            asm volatile("s_waitcnt vmcnt(0) lgkmcnt(0)" ::: "memory"); \
        } \
        asm volatile("" ::: "memory"); BAR; asm volatile("" ::: "memory"); \
        { const u32x4 h = *(const LAS u32x4*)(lds + L_HDR); ready = (int)rfl(h.x); chg = ((unsigned long long)rfl(h.z) << 32) | rfl(h.y); } \
    } } while (0)
#define DIAG_WAIT(t0) do {} while (0)
#define PKMUL4(op) ".set rs_i, 0\n\t.set rs_d, 0\n\t.irpc c, \"" op "\"\n\t.ifc \\c,:\n\t.set rs_d, 1\n\t.endif\n\t" \
        ".if rs_d == 0\n\t.ifnc \\c,v\n\t.ifnc \\c,[\n\t.set rs_i, rs_i*10 + \\c\n\t.endif\n\t.endif\n\t.endif\n\t.endr\n\t" \
        "v_pk_mul_f32 v[rs_i:rs_i+1], v[rs_i:rs_i+1], %16\n\tv_pk_mul_f32 v[rs_i+2:rs_i+3], v[rs_i+2:rs_i+3], %16\n\t"
#define RS16(a) asm volatile("s_cmp_eq_u32 %17, 0\n\ts_cbranch_scc1 .Lrs%=\n\ts_nop 15\n\ts_nop 3\n\t" \
            PKMUL4("%0") PKMUL4("%1") PKMUL4("%2") PKMUL4("%3") PKMUL4("%4") PKMUL4("%5") PKMUL4("%6") PKMUL4("%7") \
            PKMUL4("%8") PKMUL4("%9") PKMUL4("%10") PKMUL4("%11") PKMUL4("%12") PKMUL4("%13") PKMUL4("%14") PKMUL4("%15") \
            "s_nop 4\n.Lrs%=:" \
            : "+v"(acc[a][0][0][0]), "+v"(acc[a][0][0][1]), "+v"(acc[a][0][1][0]), "+v"(acc[a][0][1][1]), "+v"(acc[a][0][2][0]), "+v"(acc[a][0][2][1]), "+v"(acc[a][0][3][0]), "+v"(acc[a][0][3][1]), \
              "+v"(acc[a][1][0][0]), "+v"(acc[a][1][0][1]), "+v"(acc[a][1][1][0]), "+v"(acc[a][1][1][1]), "+v"(acc[a][1][2][0]), "+v"(acc[a][1][2][1]), "+v"(acc[a][1][3][0]), "+v"(acc[a][1][3][1]) \
            : "s"(fac), "s"(de) : "scc")
#define RESCALE(kt) do { int de = 0; \
        if ((chg >> (kt)) & 1ull) { \
            const int s = (int)rfl(*(const LAS unsigned*)(lds + L_TAB + (kt) * 4)); \
            smax = max(smax, s); const int sn = max(s, smax - 4); de = max(16 * (scur - sn), -126); scur = sn; } \
        const unsigned long long fac = (unsigned long long)(unsigned)((de + 127) << 23) * 0x100000001ull; \
        RS16(0); RS16(1); \
    } while (0)
#define TRIP do { \
        const int t2 = (t + 2) & (NKT - 1);             \
        const gchar* a1 = cA + (size_t)(t + 1) * UNIT; \
        const gchar* a2 = cA + (size_t)t2 * UNIT; const gchar* b2 = cB + (size_t)t2 * UNIT; \
        const gchar* a3 = a2 + UNIT; const gchar* b3 = b2 + UNIT; \
        RESCALE(t); \
        LDB(B0, 0, 0); SCHED; LDA(At, 0, 0); STAGE(SA(1, 1), a1 + HTB); \
        WAIT_L(8); BAR; WAIT_L(0); MMA(0, 0, At, B0); BAR; SCHED; \
        LDB(B1, 0, 1); STAGE(SB(0, 0), b2); \
        BAR; WAIT_L(0); MMA(0, 1, At, B1); BAR; \
        LDA(At, 0, 1); STAGE(SA(0, 0), a2); \
        BAR; WAIT_L(0); MMA(1, 0, At, B0); BAR; SCHED; \
        STAGE(SB(0, 1), b2 + HTB); \
        WAIT_V(6); BAR; MMA(1, 1, At, B1); BAR; \
        RESCALE(t + 1); \
        LDB(B0, 1, 0); SCHED; LDA(At, 1, 0); STAGE(SA(0, 1), a2 + HTB); \
        WAIT_L(8); BAR; WAIT_L(0); MMA(0, 0, At, B0); BAR; SCHED; \
        LDB(B1, 1, 1); STAGE(SB(1, 0), b3); \
        BAR; WAIT_L(0); MMA(0, 1, At, B1); BAR; \
        LDA(At, 1, 1); STAGE(SA(1, 0), a3); \
        BAR; WAIT_L(0); MMA(1, 0, At, B0); BAR; SCHED; \
        STAGE(SB(1, 1), b3 + HTB); \
        WAIT_V(6); BAR; MMA(1, 1, At, B1); BAR; \
    } while (0)

    f32x4 acc[2][2][4][2];
#pragma unroll
    for (int a = 0; a < 2; ++a)
#pragma unroll
        for (int b = 0; b < 2; ++b)
#pragma unroll
            for (int m = 0; m < 4; ++m)
#pragma unroll
                for (int n = 0; n < 2; ++n) acc[a][b][m][n] = (f32x4){0.f, 0.f, 0.f, 0.f};
    h16x8 At[4][2], B0[2][2], B1[2][2];

    HOOK(2);
    STAMP(2);
    STAGE(SB(0, 0), cB); STAGE(SA(0, 0), cA); STAGE(SB(0, 1), cB + HTB); STAGE(SA(0, 1), cA + HTB);
    if (wr == 1) BAR;
    WAIT_V(4); BAR;
    STAGE(SB(1, 0), cB + UNIT); STAGE(SA(1, 0), cA + UNIT); STAGE(SB(1, 1), cB + UNIT + HTB);
    WAIT_V(6); BAR;
    for (int t = 0; t < NKT; t += 2) {
        HOOK(min(t + 4, NKT));
        TRIP;
    }
    if (wr == 0) BAR;
    STAMP(3);

    const int dest = pm >> 1;
    const unsigned ovo = (unsigned)(wr * 64 + 4 * fq) * (GN * 4u) + (unsigned)(wc * 16 + fr) * 16u;
    const float qn = __builtin_nanf("");
    u32x4 V[32];
    {
        const bool bad = rfl(ld_agt(tmo)) == call;
        const int oe = 16 * scur - 256;
#pragma unroll
        for (int a = 0; a < 2; ++a)
#pragma unroll
            for (int m = 0; m < 4; ++m)
#pragma unroll
                for (int r = 0; r < 4; ++r) {
                    f32x4 v;
                    v.x = __builtin_amdgcn_ldexpf(acc[a][0][m][0][r], oe); v.y = __builtin_amdgcn_ldexpf(acc[a][0][m][1][r], oe);
                    v.z = __builtin_amdgcn_ldexpf(acc[a][1][m][0][r], oe); v.w = __builtin_amdgcn_ldexpf(acc[a][1][m][1][r], oe);
                    if (bad) v = (f32x4){qn, qn, qn, qn};
                    V[(a * 4 + m) * 4 + r] = __builtin_bit_cast(u32x4, v);
                }
    }
#define OSTAMP(k) do {} while (0)
#define DIAG_END do {} while (0)
    if (dest == rank) {
        const gchar* ob = uni((const char*)p.out[rank] + ((size_t)((pm & 1) * 256) * GN + rank * NL + pn * 256) * 4);
#pragma unroll
        for (int q = 0; q < 32; ++q) { store16_sys(ob + (size_t)((q >> 4) * 128 + ((q >> 2) & 3) * 16 + (q & 3)) * (GN * 4), ovo, V[q]); asm volatile("s_waitcnt vmcnt(" STR(THR_L) ")" ::: "memory"); }
    }
    int remote = dest - rank;
    asm volatile("" : "+s"(remote));
    if (remote) {
        const unsigned cidx = (unsigned)(((pm & 1) * 16 + pn) * 8 + wid);
        gchar* dws = uni(p.ws[dest]);
        const gchar* pb = uni(dws + WS_PACK + (size_t)(rank * 256 + cidx) * CHUNK);
        unsigned mx = 0;
#pragma unroll
        for (int q = 0; q < 32; ++q) {
#pragma unroll
            for (int c = 0; c < 4; ++c) V[q][c] = __builtin_amdgcn_alignbit(V[q][c], V[q][c], 31);
            mx = max(max(mx, V[q].x), max(V[q].y, max(V[q].z, V[q].w)));
        }
        const unsigned base = wave_max(mx >> 24), base4 = base * 0x01010101u;
        u32x4 PK[28];
#pragma unroll
        for (int g = 0; g < 4; ++g) {
            unsigned E[8], D[8], any = 0;
#pragma unroll
            for (int v = 0; v < 8; ++v) {
                const u32x4 w = byte_swap(V[g * 8 + v]);
#pragma unroll
                for (int i = 0; i < 3; ++i) PK[g * 7 + ((3 * v + i) >> 2)][(3 * v + i) & 3] = w[i];
                E[v] = w.w; D[v] = base4 - w.w; any |= D[v];
            }
            u32x4 nb;
#pragma unroll
            for (int j = 0; j < 4; ++j) nb[j] = D[2 * j] | (D[2 * j + 1] << 4);
            if ((any & 0xf0f0f0f0u) != 0u || nb.x == 0xffffffffu) {
                nb.x = 0xffffffffu;
                store16_sys(pb + (28672u + g * 2048u), (unsigned)lane * 32u, (u32x4){E[0], E[1], E[2], E[3]});
                store16_sys(pb + (28672u + g * 2048u + 16u), (unsigned)lane * 32u, (u32x4){E[4], E[5], E[6], E[7]});
            }
            PK[g * 7 + 6] = nb;
        }
        const int link = (dest - rank - 1) & 7;
        while ((int)rfl(*(vl32*)(lds + L_TURN)) + (pn < NEC ? WG_SENDERS : WG_SENDERS_C) <= wid) __builtin_amdgcn_s_sleep(8);
        gu32* rel = (gu32*)(myws + WS_OREL + link * 128);
        bool late;
        {
            unsigned tk = 0;
            if (lane == 0) tk = __hip_atomic_fetch_add((gu32*)(myws + WS_OTK + link * 128), 1u, __ATOMIC_RELAXED, __HIP_MEMORY_SCOPE_AGENT);
            const unsigned need = (call - 1u) * (256u + 8u * PUSH_REL) + (rfl(tk) & 255u) + 1u - LINK_EXTRA;
            late = (rfl(tk) & 255u) >= CASCADE_FROM;
            const unsigned long long t0 = __builtin_amdgcn_s_memrealtime();
            for (;;) {
                const int ahead = (int)(need - rfl(ld_agt(rel)));
                if (ahead <= 0 || (C_BYPASS && pn >= NEC) || __builtin_amdgcn_s_memrealtime() - t0 > DEADLINE) break;
                if (ahead > 4) __builtin_amdgcn_s_sleep(32); else __builtin_amdgcn_s_sleep(4);
            }
        }
        __builtin_amdgcn_s_setprio(SEND_PRIO);
#pragma unroll
        for (int q = 0; q < 28; ++q) {
            store16_sys(pb + q * 1024u, (unsigned)lane * 16u, PK[q]);
            asm volatile("s_waitcnt vmcnt(" STR(OUT_W) ")" ::: "memory");
            if ((q == 0 || q == REL_AT) && late == (q == 0) && lane == 0) __hip_atomic_fetch_add((LAS unsigned*)(lds + L_TURN), 1u, __ATOMIC_RELAXED, __HIP_MEMORY_SCOPE_WORKGROUP);
            if (q == REL_AT && lane == 0) __hip_atomic_fetch_add(rel, 1u, __ATOMIC_RELAXED, __HIP_MEMORY_SCOPE_AGENT);
            OSTAMP(q);
        }
        WAIT_V(0);
        if (lane == 0) __hip_atomic_store((gu32*)(dws + WS_CFLAG) + rank * 256 + cidx, (call << 8) | base, __ATOMIC_RELAXED, __HIP_MEMORY_SCOPE_SYSTEM);
        __builtin_amdgcn_s_setprio(0);
        OSTAMP(28);

        const gu32* cf = (const gu32*)(myws + WS_CFLAG) + dest * 256 + cidx;
        unsigned fv; bool bad = false;
        {
            const unsigned long long t0 = __builtin_amdgcn_s_memrealtime();
            for (;;) {
                fv = rfl(ld_sys(cf));
                if ((fv >> 8) >= call) break;
                if (rfl(ld_agt(tmo)) == call || __builtin_amdgcn_s_memrealtime() - t0 > DEADLINE) {
                    if (lane == 0) __hip_atomic_store(tmo, call, __ATOMIC_RELAXED, __HIP_MEMORY_SCOPE_AGENT);
                    break;
                }
                __builtin_amdgcn_s_sleep(FLAG_SLEEP);
            }
            bad = rfl(ld_agt(tmo)) == call;
            asm volatile("" ::: "memory");
        }
        OSTAMP(29);
        const __amdgpu_buffer_rsrc_t rs = __builtin_amdgcn_make_buffer_rsrc((void*)uni(myws + WS_PACK + (size_t)(dest * 256 + cidx) * CHUNK), 0, bad ? 0 : CHUNK, 0x00020000);
        const gchar* ob = uni((const char*)p.out[rank] + ((size_t)((pm & 1) * 256) * GN + dest * NL + pn * 256) * 4);
        const unsigned ib4 = (fv & 255u) * 0x01010101u;
#define PLOAD(vo, so) RLOAD4(rs, vo, so)
#pragma unroll
        for (int g = 0; g < 4; ++g) PK[g * 7 + 6] = PLOAD(lane * 16, (g * 7 + 6) * 1024);
#pragma unroll
        for (int q = 0; q < 28; ++q) if (q % 7 != 6) PK[q] = PLOAD(lane * 16, q * 1024);
        u32x4 E[8];
#pragma unroll
        for (int g = 0; g < 4; ++g) {
            const u32x4 nb = PK[g * 7 + 6];
            E[2 * g].x = ib4 - (nb.x & 0x0f0f0f0fu); E[2 * g].y = ib4 - ((nb.x >> 4) & 0x0f0f0f0fu); E[2 * g].z = ib4 - (nb.y & 0x0f0f0f0fu); E[2 * g].w = ib4 - ((nb.y >> 4) & 0x0f0f0f0fu);
            E[2 * g + 1].x = ib4 - (nb.z & 0x0f0f0f0fu); E[2 * g + 1].y = ib4 - ((nb.z >> 4) & 0x0f0f0f0fu); E[2 * g + 1].z = ib4 - (nb.w & 0x0f0f0f0fu); E[2 * g + 1].w = ib4 - ((nb.w >> 4) & 0x0f0f0f0fu);
            if (nb.x == 0xffffffffu) { E[2 * g] = PLOAD(lane * 32, 28672 + g * 2048); E[2 * g + 1] = PLOAD(lane * 32 + 16, 28672 + g * 2048); }
        }
        __builtin_amdgcn_s_waitcnt(0x0F70);
#pragma unroll
        for (int q = 0; q < 32; ++q) {
            const int g = q >> 3, v = q & 7;
            u32x4 w;
#pragma unroll
            for (int i = 0; i < 3; ++i) w[i] = PK[g * 7 + ((3 * v + i) >> 2)][(3 * v + i) & 3];
            w.w = E[q >> 2][q & 3];
            w = byte_swap(w);
#pragma unroll
            for (int c = 0; c < 4; ++c) w[c] = bad ? 0x7fc00000u : __builtin_amdgcn_alignbit(w[c], w[c], 1);
            store16_sys(ob + (size_t)((q >> 4) * 128 + ((q >> 2) & 3) * 16 + (q & 3)) * (GN * 4), ovo, w);
        }
    }
    DIAG_END;
}

extern "C" void kernel_launch(int rank, int nranks, ncclComm_t comm,
                              hipStream_t stream,
                              void* const* d_in, const int* in_sizes, int n_in,
                              void* d_out, int out_size,
                              const int* in_dtypes, int out_dtype,
                              void* const* d_out_all,
                              void* const* d_ws_all, size_t ws_bytes) {
    (void)comm; (void)in_dtypes; (void)out_dtype; (void)d_out;
    if (nranks != NR || (unsigned)rank >= (unsigned)NR || n_in < 2 || ws_bytes < WS_NEED ||
        in_sizes[0] != GM * KS || in_sizes[1] != KS * GN || out_size != (GM / NR) * GN) {
        fprintf(stderr, "gemm_rs: unexpected problem shape\n");
        return;
    }
    HIP_CHECK(hipFuncSetAttribute((const void*)gemm_rs_kernel, hipFuncAttributeMaxDynamicSharedMemorySize, (int)LDS_BYTES));
    Params p;
    p.x = (const float*)d_in[0]; p.w = (const float*)d_in[1];
    for (int i = 0; i < NR; ++i) { p.out[i] = (float*)d_out_all[i]; p.ws[i] = (char*)d_ws_all[i]; }
    p.rank = rank; p.pad = 0;
    hipLaunchKernelGGL(gemm_rs_kernel, dim3(GRID), dim3(NTHR), LDS_BYTES, stream, p);
    HIP_CHECK(hipGetLastError());
}
```

```cpp
#include <hip/hip_runtime.h>
#include <rccl/rccl.h>
#include <cstdio>
#include <cstdint>

#define HIP_CHECK(expr) do { hipError_t _e = (expr); if (_e != hipSuccess) { \
    fprintf(stderr, "HIP error %s at %s:%d\n", hipGetErrorString(_e), __FILE__, __LINE__); \
    return; } } while (0)

#define LAS __attribute__((address_space(3)))
#define GAS __attribute__((address_space(1)))
typedef GAS char gchar;
typedef GAS unsigned gu32;
typedef _Float16 h16x8 __attribute__((ext_vector_type(8)));
typedef float f32x4 __attribute__((ext_vector_type(4)));
typedef float f32x2 __attribute__((ext_vector_type(2)));
typedef unsigned u32x4 __attribute__((ext_vector_type(4)));

constexpr int NR = 8;
constexpr int GM = 4096, GN = 32768, KS = 512, NL = 4096;
constexpr int GRID = 256, NTHR = 512;
constexpr unsigned UNIT = 32768;
constexpr unsigned HTB = 16384;
constexpr int NKT = 64;
constexpr size_t WS_TICKET = 0, WS_TMO = 128, WS_LINKQ = 1024, WS_OTK = 2048, WS_OREL = 3072;
constexpr size_t WS_FLAG = 4096;
constexpr size_t WS_CFLAG = 16384;
constexpr size_t WS_AFLAG = 32768;
constexpr size_t WS_DIAG = 65536;
constexpr size_t OPND_BYTES = (size_t)16 * NKT * UNIT;
constexpr size_t WS_STAGE = 1u << 20;
constexpr size_t WS_PIN = WS_STAGE + 2 * OPND_BYTES;
constexpr unsigned CHUNK = 37888;
constexpr size_t WS_PACK = WS_PIN + (size_t)2 * 8 * 256 * UNIT;
constexpr size_t WS_NEED = WS_PACK + (size_t)NR * 256 * CHUNK;
#define DEFER 1
constexpr int NEC = 12;
constexpr unsigned long long DEADLINE = 100000000ull;
constexpr unsigned L_CTL = 8 * HTB;
constexpr unsigned L_AMAX = L_CTL, L_CONS = L_CTL + 48, L_TURN = L_CTL + 64, L_CALL = L_CTL + 96, L_ITEM = L_CTL + 128, L_CLS = L_CTL + 192;
constexpr unsigned L_QI = L_CTL + 256, L_UCNT = L_CTL + 320, L_HDR = L_CTL + 576, L_TAB = L_CTL + 640, L_SCR = L_CTL + 1024, LDS_BYTES = L_SCR + 8 * 3072;

#define GLDS_AUX 17
struct Params {
    const float* x; const float* w;
    float* out[NR]; char* ws[NR];
    int rank; int pad;
};

#define STAMP(i) do {} while (0)

__device__ __forceinline__ void store16_sys(const gchar* sbase, unsigned voff, u32x4 v) {
    asm volatile("global_store_dwordx4 %0, %1, %2 sc0 sc1\n\ts_nop 1" :: "v"(voff), "v"(v), "s"(sbase) : "memory");
}
__device__ __forceinline__ unsigned ld_sys(const gu32* q) { return __hip_atomic_load(q, __ATOMIC_RELAXED, __HIP_MEMORY_SCOPE_SYSTEM); }
__device__ __forceinline__ unsigned ld_agt(const gu32* q) { return __hip_atomic_load(q, __ATOMIC_RELAXED, __HIP_MEMORY_SCOPE_AGENT); }
__device__ __forceinline__ unsigned rfl(unsigned v) { return (unsigned)__builtin_amdgcn_readfirstlane((int)v); }

template <class T> __device__ __forceinline__ gchar* uni(T q) {
    unsigned lo = rfl((unsigned)(uintptr_t)q), hi = rfl((unsigned)((uintptr_t)q >> 32));
    asm volatile("s_nop 4" : "+s"(lo), "+s"(hi));
    return (gchar*)(((uintptr_t)hi << 32) | lo);
}

__device__ __forceinline__ unsigned wave_max(unsigned x) {
    int mb = (int)x;
    mb = max(mb, __builtin_amdgcn_update_dpp(0, mb, 0x111, 0xf, 0xf, false));
    mb = max(mb, __builtin_amdgcn_update_dpp(0, mb, 0x112, 0xf, 0xf, false));
    mb = max(mb, __builtin_amdgcn_update_dpp(0, mb, 0x114, 0xf, 0xf, false));
    mb = max(mb, __builtin_amdgcn_update_dpp(0, mb, 0x118, 0xf, 0xf, false));
    mb = max(mb, __builtin_amdgcn_update_dpp(0, mb, 0x142, 0xa, 0xf, false));
    mb = max(mb, __builtin_amdgcn_update_dpp(0, mb, 0x143, 0xc, 0xf, false));
    return (unsigned)__builtin_amdgcn_readlane(mb, 63);
}
__device__ __forceinline__ unsigned wave_mean_exp(float sm) {
#define DPP_ADD(ctl, rm) sm += __builtin_bit_cast(float, __builtin_amdgcn_update_dpp(0, __builtin_bit_cast(int, sm), ctl, rm, 0xf, false))
    DPP_ADD(0x111, 0xf); DPP_ADD(0x112, 0xf); DPP_ADD(0x114, 0xf); DPP_ADD(0x118, 0xf); DPP_ADD(0x142, 0xa); DPP_ADD(0x143, 0xc);
    return ((unsigned)__builtin_amdgcn_readlane(__builtin_bit_cast(int, sm), 63) >> 23) & 255u;
}

typedef unsigned short u16x2 __attribute__((ext_vector_type(2)));
typedef unsigned u32x2 __attribute__((ext_vector_type(2)));
constexpr unsigned Q_P6 = 6144, Q_LEN = 6656, Q_STR = 6720, Q_BYTES = 8192;
__device__ __forceinline__ u16x2 as16(unsigned a) { return __builtin_bit_cast(u16x2, a); }
__device__ __forceinline__ unsigned as32(u16x2 a) { return __builtin_bit_cast(unsigned, a); }
__device__ __forceinline__ unsigned pk_sub(unsigned a, unsigned b) { return as32(as16(a) - as16(b)); }
__device__ __forceinline__ unsigned pk_subs(unsigned a, unsigned b) { return as32(__builtin_elementwise_sub_sat(as16(a), as16(b))); }
__device__ __forceinline__ unsigned pk_min(unsigned a, unsigned b) { return as32(__builtin_elementwise_min(as16(a), as16(b))); }
__device__ __forceinline__ unsigned ffbl(unsigned a) { unsigned r; asm("v_ffbl_b32 %0, %1" : "=v"(r) : "v"(a)); return r; }
__device__ __forceinline__ unsigned wave_scan(unsigned x) {
    int v = (int)x;
    v += __builtin_amdgcn_update_dpp(0, v, 0x111, 0xf, 0xf, false);
    v += __builtin_amdgcn_update_dpp(0, v, 0x112, 0xf, 0xf, false);
    v += __builtin_amdgcn_update_dpp(0, v, 0x114, 0xf, 0xf, false);
    v += __builtin_amdgcn_update_dpp(0, v, 0x118, 0xf, 0xf, false);
    v += __builtin_amdgcn_update_dpp(0, v, 0x142, 0xa, 0xf, false);
    v += __builtin_amdgcn_update_dpp(0, v, 0x143, 0xc, 0xf, false);
    return (unsigned)v;
}
__device__ __forceinline__ unsigned q_encode(const u32x4 (&W)[8], unsigned B, LAS unsigned char* dst, unsigned lane) {
    const unsigned B1 = (B + 1u) * 0x04000400u;
    unsigned v[32], seg[4], pos[4];
    u32x4 X[4];
#pragma unroll
    for (int s = 0; s < 4; ++s) {
        unsigned a = 0, p = 0;
        X[s].x = __builtin_amdgcn_perm(W[2 * s].y, W[2 * s].x, 0x07050301u); X[s].y = __builtin_amdgcn_perm(W[2 * s].w, W[2 * s].z, 0x07050301u);
        X[s].z = __builtin_amdgcn_perm(W[2 * s + 1].y, W[2 * s + 1].x, 0x07050301u); X[s].w = __builtin_amdgcn_perm(W[2 * s + 1].w, W[2 * s + 1].z, 0x07050301u);
#pragma unroll
        for (int k = 8 * s; k < 8 * s + 8; ++k) {
            const unsigned w = W[k >> 2][k & 3];
            const unsigned dq = as32(as16(pk_sub(B1, w & 0x7c007c00u)) >> 10);
            const unsigned ix = pk_min(dq, 0x00070007u) | 0x0c000c00u;
            const unsigned len = __builtin_amdgcn_perm(0x05040201u, 0x00000003u, ix) + pk_subs(dq, 0x00070007u);
            v[k] = (w & 0x83ff83ffu) | (__builtin_amdgcn_perm(0u, 0x03020100u, ix) << 10);
            p += len & 0xffffu; a |= 1u << (p & 31u);
            p += len >> 16;     a |= 1u << (p & 31u);
        }
        seg[s] = a >> 1; pos[s] = p;
        asm volatile("" : "+v"(v[8 * s]), "+v"(v[8 * s + 1]), "+v"(v[8 * s + 2]), "+v"(v[8 * s + 3]), "+v"(v[8 * s + 4]), "+v"(v[8 * s + 5]), "+v"(v[8 * s + 6]), "+v"(v[8 * s + 7]),
                     "+v"(X[s]), "+v"(seg[s]), "+v"(pos[s]));
    }
    const bool ovf = max(max(pos[0], pos[1]), max(pos[2], pos[3])) > 31u;
    const unsigned tot = ovf ? 0u : pos[0] + pos[1] + pos[2] + pos[3];
    const unsigned incl = wave_scan(tot), total = (unsigned)__builtin_amdgcn_readlane((int)incl, 63);
    const unsigned long long om = __ballot(ovf);
    const unsigned xb = Q_STR + (((total + 127u) >> 7) << 4), size = xb + 64u * (unsigned)__popcll(om);
    if (size >= Q_BYTES) return Q_BYTES;
    *(LAS u32x4*)(dst + Q_STR + lane * 16u) = (u32x4){0u, 0u, 0u, 0u};
#pragma unroll
    for (int j = 0; j < 6; ++j)
#pragma unroll
        for (int i = 0; i < 4; ++i) v[4 * j + i] |= (v[26 + j] << (12 - 3 * i)) & 0x70007000u;
#pragma unroll
    for (int j = 0; j < 6; ++j) v[24 + j / 3] |= (v[26 + j] >> (1 + j % 3)) & (0x80008000u >> (1 + j % 3));
#pragma unroll
    for (int j = 0; j < 6; ++j) *(LAS u32x4*)(dst + j * 1024u + lane * 16u) = (u32x4){v[4 * j], v[4 * j + 1], v[4 * j + 2], v[4 * j + 3]};
    *(LAS u32x2*)(dst + Q_P6 + lane * 8u) = (u32x2){v[24], v[25]};
    const unsigned slot = __builtin_amdgcn_mbcnt_hi((unsigned)(om >> 32), __builtin_amdgcn_mbcnt_lo((unsigned)om, 0u));
    *(LAS unsigned char*)(dst + Q_LEN + lane) = (unsigned char)(ovf ? 128u + slot : tot);
    if (!ovf) {
        unsigned off = incl - tot;
#pragma unroll
        for (int s = 0; s < 4; ++s) {
            const unsigned long long x = (unsigned long long)seg[s] << (off & 31u);
            LAS unsigned* q = (LAS unsigned*)(dst + Q_STR + (off >> 5) * 4u);
            __hip_atomic_fetch_or(q, (unsigned)x, __ATOMIC_RELAXED, __HIP_MEMORY_SCOPE_WORKGROUP);
            __hip_atomic_fetch_or(q + 1, (unsigned)(x >> 32), __ATOMIC_RELAXED, __HIP_MEMORY_SCOPE_WORKGROUP);
            off += pos[s];
        }
    } else {
#pragma unroll
        for (int j = 0; j < 4; ++j) *(LAS u32x4*)(dst + xb + slot * 64u + j * 16u) = X[j];
    }
    return size;
}
#define RLOAD4(rs, vo, so) __builtin_bit_cast(u32x4, __builtin_amdgcn_raw_buffer_load_b128(rs, vo, so, 17))
#define RLOAD2(rs, vo, so) __builtin_bit_cast(u32x2, __builtin_amdgcn_raw_buffer_load_b64(rs, vo, so, 17))
#define RLOAD1(rs, vo, so) __builtin_amdgcn_raw_buffer_load_b32(rs, vo, so, 17)
struct QIn { u32x4 P[6]; u32x2 P6; unsigned lb, off, w4; u32x4 win, X[4]; };
__device__ __forceinline__ void q_load1(__amdgpu_buffer_rsrc_t rs, unsigned so, unsigned lane, QIn& q) {
#pragma unroll
    for (int j = 0; j < 6; ++j) q.P[j] = RLOAD4(rs, lane * 16u, so + j * 1024u);
    q.P6 = RLOAD2(rs, lane * 8u, so + Q_P6);
}
__device__ __forceinline__ void q_load2(__amdgpu_buffer_rsrc_t rs, unsigned so, unsigned lane, bool raw, QIn& q) {
    const unsigned tot = q.lb >= 128u ? 0u : q.lb, incl = wave_scan(tot), total = (unsigned)__builtin_amdgcn_readlane((int)incl, 63);
    q.off = incl - tot;
    q.win = RLOAD4(rs, (q.off >> 5) * 4u, so + Q_STR);
    q.w4 = RLOAD1(rs, (q.off >> 5) * 4u + 16u, so + Q_STR);
    if (raw || q.lb >= 128u) {
        const unsigned xo = raw ? Q_P6 + lane * 16u : Q_STR + (((total + 127u) >> 7) << 4) + (q.lb - 128u) * 64u;
#pragma unroll
        for (int j = 0; j < 4; ++j) q.X[j] = RLOAD4(rs, xo, so + j * (raw ? 1024u : 16u));
    }
}
__device__ __forceinline__ void q_finish(unsigned B, const QIn& q, u32x4 (&W)[8]) {
    unsigned v[32];
#pragma unroll
    for (int k = 0; k < 24; ++k) v[k] = q.P[k >> 2][k & 3];
    v[24] = q.P6.x; v[25] = q.P6.y;
#pragma unroll
    for (int j = 0; j < 6; ++j) {
        unsigned d = (v[24 + j / 3] << (1 + j % 3)) & 0x80008000u;
#pragma unroll
        for (int i = 0; i < 4; ++i) d |= (v[4 * j + i] & 0x70007000u) >> (12 - 3 * i);
        v[26 + j] = d;
    }
    u32x4 win;
    win.x = __builtin_amdgcn_alignbit(q.win.y, q.win.x, q.off); win.y = __builtin_amdgcn_alignbit(q.win.z, q.win.y, q.off);
    win.z = __builtin_amdgcn_alignbit(q.win.w, q.win.z, q.off); win.w = __builtin_amdgcn_alignbit(q.w4, q.win.w, q.off);
    const unsigned B1 = (B + 1u) * 0x04000400u;
#pragma unroll
    for (int s = 0; s < 4; ++s) {
        unsigned a = win.x << 1, used = 0;
#pragma unroll
        for (int k = 8 * s; k < 8 * s + 8; ++k) {
            const unsigned nz = v[k] | (v[k] >> 1);
            const unsigned l0 = ffbl((a & ~1u) | ((nz >> 10) & 1u));
            a >>= l0;
            const unsigned l1 = ffbl((a & ~1u) | ((nz >> 26) & 1u));
            a >>= l1;
            const unsigned len = l0 | (l1 << 16);
            used += len;
            const unsigned dq = __builtin_amdgcn_perm(0x00000006u, 0x00050400u, pk_min(len, 0x00040004u) | 0x0c000c00u) + pk_subs(len, 0x00040004u);
            W[k >> 2][k & 3] = (v[k] & 0x83ff83ffu) | (pk_sub(B1, (v[k] & 0x0c000c00u) + (dq << 10)) & 0x7c007c00u);
        }
        if (s < 3) {
            const unsigned sh = (used + (used >> 16)) & 31u;
            win.x = __builtin_amdgcn_alignbit(win.y, win.x, sh); win.y = __builtin_amdgcn_alignbit(win.z, win.y, sh);
            win.z = __builtin_amdgcn_alignbit(win.w, win.z, sh); win.w >>= sh;
        }
    }
}
__device__ __forceinline__ void q_patch(unsigned B, const QIn& q, u32x4 (&W)[8]) {
    if (B == 0u) {
#pragma unroll
        for (int j = 0; j < 6; ++j) W[j] = q.P[j];
        W[6] = q.X[0]; W[7] = q.X[1];
    } else if (q.lb >= 128u) {
#pragma unroll
        for (int k = 0; k < 32; ++k) W[k >> 2][k & 3] = __builtin_amdgcn_perm(q.X[k >> 3][(k >> 1) & 3], W[k >> 2][k & 3], (k & 1) ? 0x07020600u : 0x05020400u);
    }
}

constexpr unsigned C_HDR = 26624, C_STR = C_HDR + 128, C_EXC = 29696;
__device__ __forceinline__ unsigned c_encode(const u32x4 (&V)[32], unsigned B, u32x4 (&PK)[26], LAS unsigned char* scr, const gchar* pb, unsigned lane) {
    const unsigned B1 = (B + 1u) * 0x00010001u;
    unsigned seg[8], pos[8], hdr = 0;
#pragma unroll
    for (int g = 0; g < 4; ++g) {
        unsigned e2[16];
#pragma unroll
        for (int v = 0; v < 8; ++v) {
            u32x4 r;
#pragma unroll
            for (int c = 0; c < 4; ++c) r[c] = __builtin_amdgcn_alignbit(V[g * 8 + v][c], V[g * 8 + v][c], 31);
#pragma unroll
            for (int i = 0; i < 3; ++i) PK[g * 6 + ((3 * v + i) >> 2)][(3 * v + i) & 3] = __builtin_amdgcn_perm(r.w, r[i], 0x04020100u + ((unsigned)i << 24));
            e2[2 * v] = __builtin_amdgcn_perm(r.y, r.x, 0x0c070c03u); e2[2 * v + 1] = __builtin_amdgcn_perm(r.w, r.z, 0x0c070c03u);
        }
#pragma unroll
        for (int s = 2 * g; s < 2 * g + 2; ++s) {
            unsigned a = 0, p = 0, cw = 0;
#pragma unroll
            for (int j = 0; j < 8; ++j) {
                const unsigned dq = pk_sub(B1, e2[(s & 1) * 8 + j]), ix = pk_min(dq, 0x00070007u) | 0x0c000c00u;
                const unsigned len = __builtin_amdgcn_perm(0x05040201u, 0x00000003u, ix) + pk_subs(dq, 0x00070007u);
                cw |= __builtin_amdgcn_perm(0u, 0x03020100u, ix) << (2 * j);
                p += len & 0xffffu; a |= 1u << (p & 31u);
                p += len >> 16;     a |= 1u << (p & 31u);
            }
            seg[s] = a >> 1; pos[s] = p; PK[24 + (s >> 2)][s & 3] = cw;
        }
        if (max(pos[2 * g], pos[2 * g + 1]) > 31u) {
            u32x4 E[2];
#pragma unroll
            for (int v = 0; v < 8; ++v) E[v >> 2][v & 3] = __builtin_amdgcn_perm(e2[2 * v + 1], e2[2 * v], 0x06040200u);
            store16_sys(pb + (C_EXC + g * 2048u), lane * 32u, E[0]);
            store16_sys(pb + (C_EXC + g * 2048u + 16u), lane * 32u, E[1]);
            seg[2 * g] = seg[2 * g + 1] = pos[2 * g] = pos[2 * g + 1] = 0u; hdr |= 256u << g;
        }
    }
    const unsigned tot = (pos[0] + pos[1] + pos[2] + pos[3]) + (pos[4] + pos[5] + pos[6] + pos[7]);
    const unsigned incl = wave_scan(tot), total = (unsigned)__builtin_amdgcn_readlane((int)incl, 63);
    *(LAS u32x4*)(scr + 128u + lane * 16u) = (u32x4){0u, 0u, 0u, 0u}; *(LAS u32x4*)(scr + 1152u + lane * 16u) = (u32x4){0u, 0u, 0u, 0u};
    *(LAS unsigned short*)(scr + lane * 2u) = (unsigned short)(hdr | tot);
    unsigned off = incl - tot;
#pragma unroll
    for (int s = 0; s < 8; ++s) {
        const unsigned long long x = (unsigned long long)seg[s] << (off & 31u);
        LAS unsigned* q = (LAS unsigned*)(scr + 128u + (off >> 5) * 4u);
        __hip_atomic_fetch_or(q, (unsigned)x, __ATOMIC_RELAXED, __HIP_MEMORY_SCOPE_WORKGROUP);
        __hip_atomic_fetch_or(q + 1, (unsigned)(x >> 32), __ATOMIC_RELAXED, __HIP_MEMORY_SCOPE_WORKGROUP);
        off += pos[s];
    }
    return 128u + ((total + 7u) >> 3);
}
__device__ __forceinline__ size_t out_off(int q) { return (size_t)((q >> 4) * 128 + ((q >> 2) & 3) * 16 + (q & 3)) * (GN * 4); }
__device__ __forceinline__ void c_decode(__amdgpu_buffer_rsrc_t rs, unsigned B, bool bad, const gchar* ob, unsigned ovo, unsigned lane) {
    const unsigned hdr = __builtin_amdgcn_raw_buffer_load_b16(rs, lane * 2u, C_HDR, 17);
    u32x4 PK[26];
    PK[24] = RLOAD4(rs, lane * 16u, 24 * 1024); PK[25] = RLOAD4(rs, lane * 16u, 25 * 1024);
#pragma unroll
    for (int q = 0; q < 24; ++q) PK[q] = RLOAD4(rs, lane * 16u, q * 1024);
    const unsigned tot = hdr & 255u, off = wave_scan(tot) - tot;
    const u32x4 w0 = RLOAD4(rs, (off >> 5) * 4u, C_STR), w1 = RLOAD4(rs, (off >> 5) * 4u + 16u, C_STR);
    const unsigned w2 = RLOAD1(rs, (off >> 5) * 4u + 32u, C_STR);
    u32x4 X[8];
#pragma unroll
    for (int g = 0; g < 4; ++g) if (hdr & (256u << g)) { X[2 * g] = RLOAD4(rs, lane * 32u, C_EXC + g * 2048u); X[2 * g + 1] = RLOAD4(rs, lane * 32u + 16u, C_EXC + g * 2048u); }
    __builtin_amdgcn_s_waitcnt(0x0F70);
    unsigned win[9] = {w0.x, w0.y, w0.z, w0.w, w1.x, w1.y, w1.z, w1.w, w2};
#pragma unroll
    for (int i = 0; i < 8; ++i) win[i] = __builtin_amdgcn_alignbit(win[i + 1], win[i], off);
    const unsigned B1 = (B + 1u) * 0x00010001u;
#pragma unroll
    for (int g = 0; g < 4; ++g) {
        const bool exc = hdr & (256u << g);
        unsigned e2[16];
#pragma unroll
        for (int s = 2 * g; s < 2 * g + 2; ++s) {
            const unsigned cw = PK[24 + (s >> 2)][s & 3], nz = cw | (cw >> 1);
            unsigned a = win[0] << 1, used = 0;
#pragma unroll
            for (int j = 0; j < 8; ++j) {
                const unsigned l0 = ffbl((a & ~1u) | ((nz >> (2 * j)) & 1u));
                a >>= l0;
                const unsigned l1 = ffbl((a & ~1u) | ((nz >> (2 * j + 16)) & 1u));
                a >>= l1;
                const unsigned len = l0 | (l1 << 16);
                used += len;
                const unsigned dq = __builtin_amdgcn_perm(0x00000006u, 0x00050400u, pk_min(len, 0x00040004u) | 0x0c000c00u) + pk_subs(len, 0x00040004u);
                e2[(s & 1) * 8 + j] = pk_sub(B1, dq + ((cw >> (2 * j)) & 0x00030003u));
            }
            const unsigned sh = exc ? 0u : used + (used >> 16);
#pragma unroll
            for (int i = 0; i < 7 - s; ++i) win[i] = __builtin_amdgcn_alignbit(win[i + 1], win[i], sh);
        }
        if (exc) {
#pragma unroll
            for (int j = 0; j < 16; ++j) e2[j] = __builtin_amdgcn_perm(0u, X[2 * g + (j >> 3)][(j >> 1) & 3], (j & 1) ? 0x0c030c02u : 0x0c010c00u);
        }
#pragma unroll
        for (int v = 0; v < 8; ++v) {
            unsigned P[3];
#pragma unroll
            for (int i = 0; i < 3; ++i) P[i] = PK[g * 6 + ((3 * v + i) >> 2)][(3 * v + i) & 3];
            u32x4 r;
            r.x = __builtin_amdgcn_perm(e2[2 * v], P[0], 0x04020100u); r.y = __builtin_amdgcn_perm(e2[2 * v], P[1], 0x06020100u); r.z = __builtin_amdgcn_perm(e2[2 * v + 1], P[2], 0x04020100u);
            r.w = __builtin_amdgcn_perm(P[1], P[0], 0x0c0c0703u) | __builtin_amdgcn_perm(e2[2 * v + 1], P[2], 0x06030c0cu);
#pragma unroll
            for (int c = 0; c < 4; ++c) r[c] = bad ? 0x7fc00000u : __builtin_amdgcn_alignbit(r[c], r[c], 1);
            store16_sys(ob + out_off(g * 8 + v), ovo, r);
        }
    }
}

__device__ __forceinline__ void item_decode(int i, int d, int& isW, int& j, int& u) {
    if (i < 208) {
        constexpr int N0 = 26 * (8 - DEFER), N1 = N0 + 19 * DEFER;
        int r;
        if (i < N0) { j = i / 26; r = i - j * 26; } else if (i < N1) { j = (i - N0) / 19; r = i - N0 - j * 19; j += 8 - DEFER; } else { j = (i - N1) / 7; r = 19 + i - N1 - j * 7; j += 8 - DEFER; }
        isW = r >= 7 && r < 19; u = isW ? r - 7 : r < 7 ? 2 * r : 2 * r - 37; if (!isW && u >= 2 * d) u += 2;
    }
    else { const int r = i - 208; j = r / 6; const int t = r - j * 6; isW = t < 4; u = isW ? NEC + t : 2 * d + t - 4; }
}

__device__ __forceinline__ void item_load(const Params& p, int i, int d, f32x4 (&L)[16]) {
    const bool valid = i < 256;
    int isW, j, u; item_decode(valid ? i : 0, d, isW, j, u);
    const unsigned t = threadIdx.x;
    const gchar* b = uni(isW ? (const char*)p.w + ((size_t)(j * 64) * GN + d * NL + u * 256) * 4 : (const char*)p.x + ((size_t)(u * 256) * KS + j * 64) * 4);
    unsigned vo = isW ? (t >> 6) * (4u * GN * 4u) + (t & 63u) * 16u : (t >> 3) * (KS * 4u) + ((t & 7u) ^ ((t >> 4) & 7u)) * 16u;
    if (!valid) vo = 0;
    const unsigned s0 = isW ? GN * 4u : 128u, s1 = isW ? 2u * GN * 4u : 64u * KS * 4u, s2 = isW ? 32u * GN * 4u : 128u * KS * 4u, s3 = isW ? 16u * GN * 4u : 32u * KS * 4u;
#pragma unroll
    for (int n = 0; n < 16; ++n)
        L[n] = *(const GAS f32x4*)(b + ((n & 1) * s0 + ((n >> 1) & 1) * s1 + ((n >> 2) & 1) * s2 + (n >> 3) * s3) + vo);
}

__device__ __forceinline__ unsigned pk2(float a, float b, int e) {
    typedef _Float16 h2 __attribute__((ext_vector_type(2)));
    h2 h; h.x = (_Float16)__builtin_amdgcn_ldexpf(a, e); h.y = (_Float16)__builtin_amdgcn_ldexpf(b, e);
    return __builtin_bit_cast(unsigned, h);
}

#define POLL_SLEEP 8
#define SELF_DELAY 6
#define FLAG_SLEEP 16
#define PUSH_REL 1u
#define RAW_FIRST 2u
#define SEND_PRIO 3
#define SELF_SPREAD 20
#define CMAP 2
#define RAW_WAVE 6
#define BEHIND 0
#define OUT_W2 4
#define WARM_OUT 0
#define OUT_W 3
#define REL_AT 23
#define LINK_EXTRA 8u
#define WG_SENDERS 1
#define WG_SENDERS_C 2
#define PUSH_N 7
#define STR2(x) #x
#define STR(x) STR2(x)
#define LBAR do { asm volatile("s_waitcnt lgkmcnt(0)" ::: "memory"); __builtin_amdgcn_s_barrier(); asm volatile("" ::: "memory"); } while (0)

typedef volatile LAS unsigned vl32;
typedef volatile LAS u32x4 vl128;
#define LGKM0 asm volatile("s_waitcnt lgkmcnt(0)" ::: "memory")
__device__ __forceinline__ u32x4 lds_wait4(LAS unsigned char* lds, unsigned off, unsigned v) {
    for (;;) {
        const u32x4 q = *(vl128*)(lds + off);
        if (rfl(min(min(q.x, q.y), min(q.z, q.w))) >= v) { asm volatile("" ::: "memory"); return q; }
        __builtin_amdgcn_s_sleep(1);
    }
}
__device__ __forceinline__ unsigned item_amax(LAS unsigned char* lds, const f32x4 (&L)[16], unsigned k) {
    unsigned mu = 0; float sm = 0.f;
#pragma unroll
    for (int n = 0; n < 16; ++n) {
        const u32x4 q = __builtin_bit_cast(u32x4, L[n]) & 0x7fffffffu; mu = max(max(mu, q.x), max(q.y, max(q.z, q.w)));
        const f32x4 f = __builtin_bit_cast(f32x4, q); sm += (f.x + f.y) + (f.z + f.w);
    }
    const unsigned mb = wave_max(mu >> 23);
    const unsigned em = wave_mean_exp(sm * (2.32f / 4096.f));
    LGKM0;
    if ((threadIdx.x & 63) == 63) *(vl32*)(lds + L_AMAX + (k & 1u) * 16 + (threadIdx.x >> 6) * 4) = ((k + 1u) << 8) | mb;
    return min(em, mb);
}

__device__ __forceinline__ unsigned img_off(int isW, unsigned q, int c) {
    return (isW ? q * 2048u + (c >> 2) * 8192u + (c & 1) * 1024u : q * 1024u + (c >> 2) * 4096u + (c & 1) * 8192u) + ((c >> 1) & 1) * HTB;
}
__device__ __forceinline__ int conv_step(const Params& p, LAS unsigned char* lds, gu32* tick, int d, unsigned k, f32x4 (&L)[16], const f32x4 (&Ln)[16], unsigned& top) {
    unsigned t = threadIdx.x; asm volatile("" : "+v"(t));
    const u32x4 am = lds_wait4(lds, L_AMAX + (k & 1u) * 16, (k + 1u) << 8) & 255u;
    const int i = (int)rfl(*(vl32*)(lds + L_ITEM + (k & 15u) * 4)), i1 = (int)rfl(*(vl32*)(lds + L_ITEM + ((k + 1u) & 15u) * 4)), i2 = (int)rfl(*(vl32*)(lds + L_ITEM + ((k + 2u) & 15u) * 4));
    const bool valid = i < 256;
    const int cls = (int)rfl(max(max(am.x, am.y), max(am.z, am.w)) + 2u) >> 4;
    const int e = 128 - 16 * cls;
    int isW, j, u; item_decode(valid ? i : 0, d, isW, j, u);
    u32x4 H[8];
#pragma unroll
    for (int h = 0; h < 2; ++h) {
        const f32x4* Q = L + 8 * h; u32x4* G = H + 4 * h;
        if (isW) {
#pragma unroll
            for (int c = 0; c < 4; ++c) { G[c].x = pk2(Q[0][c], Q[1][c], e); G[c].y = pk2(Q[2][c], Q[3][c], e); G[c].z = pk2(Q[4][c], Q[5][c], e); G[c].w = pk2(Q[6][c], Q[7][c], e); }
        } else {
#pragma unroll
            for (int c = 0; c < 4; ++c) { G[c].x = pk2(Q[2 * c].x, Q[2 * c].y, e); G[c].y = pk2(Q[2 * c].z, Q[2 * c].w, e); G[c].z = pk2(Q[2 * c + 1].x, Q[2 * c + 1].y, e); G[c].w = pk2(Q[2 * c + 1].z, Q[2 * c + 1].w, e); }
        }
    }
    unsigned tk = 255u;
    gu32* q = tick; asm volatile("" : "+v"(q));
    if (t == 0 && tick && valid) tk = __hip_atomic_fetch_add(q, 1u, __ATOMIC_RELAXED, __HIP_MEMORY_SCOPE_AGENT);
    if (valid) {
        if (k >= 4) while (rfl(*(vl32*)(lds + L_CONS)) < k - 3u) __builtin_amdgcn_s_sleep(2);
        asm volatile("" ::: "memory");
        if (t == 0) *(vl32*)(lds + L_CLS + (k & 15u) * 4) = (unsigned)cls;
        LAS unsigned char* img = lds + (k & 3u) * UNIT;
#pragma unroll
        for (int c = 0; c < 8; ++c) *(LAS u32x4*)(img + (tick ? rfl(t >> 6) * Q_BYTES + c * 1024u : img_off(isW, rfl(t >> 6), c)) + (t & 63u) * 16u) = H[c];
        LGKM0;
        if ((t & 63u) == 0) *(vl32*)(lds + L_QI + (k & 3u) * 16 + (t >> 6) * 4) =
            (tick && k >= RAW_FIRST) ? ((2u * k + 1u) << 20) | ((unsigned)min(max((int)top + e - 112, 2), 31) << 14) : ((2u * k + 2u) << 20) | Q_BYTES;
    }
    item_load(p, i2, d, L);
    __builtin_amdgcn_s_waitcnt(0x4F70);
    asm volatile("" : "+v"(tk));
    if (t == 0) *(vl32*)(lds + L_ITEM + ((k + 3u) & 15u) * 4) = 24u + (tk & 255u);
    top = item_amax(lds, Ln, k + 1u);
    return i1;
}

__device__ __forceinline__ gu32* stage_unit(gchar* myws, int rank, int srel, int i, int& isW, const gchar*& ub) {
    int j, u; item_decode(i, rank, isW, j, u);
    const int idx = (srel == 7) ? j : 8 + j * 7 + srel;
    ub = myws + WS_STAGE + (size_t)isW * OPND_BYTES + (size_t)(u * NKT + idx) * UNIT;
    return (gu32*)(myws + WS_FLAG) + (isW * 16 + u) * NKT + idx;
}
__device__ __forceinline__ void publish(gu32* f, unsigned call, unsigned info, bool remote) {
    if ((threadIdx.x & 63u) == 0) {
        if (remote) __hip_atomic_store((GAS unsigned long long*)f, ((unsigned long long)call << 32) | info, __ATOMIC_RELAXED, __HIP_MEMORY_SCOPE_SYSTEM);
        else __hip_atomic_store(f, (call << 8) | (info & 31u), __ATOMIC_RELAXED, __HIP_MEMORY_SCOPE_SYSTEM);
    }
}
__device__ __forceinline__ void send_fetch(LAS unsigned char* lds, unsigned k, int q, u32x4 (&H)[8]) {
#pragma unroll
    for (int c = 0; c < 8; ++c) H[c] = *(const LAS u32x4*)(lds + (k & 3u) * UNIT + q * 8192u + c * 1024u + (threadIdx.x & 63u) * 16u);
}
template <int Q> __device__ __forceinline__ void send8(const gchar* ub, const u32x4 (&H)[8], unsigned size, gu32* pf, unsigned call, unsigned pv, bool remote) {
#pragma unroll
    for (int c = 0; c < 8; ++c) {
        if (c < 6 || c * 1024u + (threadIdx.x & 63u) * 16u < size) store16_sys(ub + (Q * 8192u + c * 1024u), (threadIdx.x & 63u) * 16u, H[c]);
        asm volatile("s_waitcnt vmcnt(" STR(PUSH_N) ")" ::: "memory");
        if (c < 6 && Q * 6 + c == PUSH_N - 1 && pf) publish(pf, call, pv, remote);
    }
}

__device__ __forceinline__ unsigned push_items(const Params& p, LAS unsigned char* lds, gchar* myws, int rank, int d, int first, int stride, gu32* tick) {
    unsigned call;
#define PSTAMP(i) do {} while (0)
    if (rfl(threadIdx.x >> 6) < 4) {
        f32x4 RA[16], RB[16];
        if (!tick) for (int z = 0; z < SELF_DELAY + ((first + 32) % 200) * SELF_SPREAD / 200; ++z) __builtin_amdgcn_s_sleep(32);
        item_load(p, first, d, RA);
        PSTAMP(12);
        LBAR;
        PSTAMP(13);
        unsigned top = item_amax(lds, RA, 0);
        PSTAMP(14);
        item_load(p, first + stride, d, RB);
        PSTAMP(15);
        int nx; unsigned k = 0;
        do { conv_step(p, lds, tick, d, k, RA, RB, top); nx = conv_step(p, lds, tick, d, k + 1u, RB, RA, top); k += 2; } while (nx < 256);
        LBAR;
        call = rfl(*(vl32*)(lds + L_CALL));
    } else {
        const unsigned ts = threadIdx.x - 256u;
        if (ts < 144) *(vl32*)(lds + L_CTL + ts * 4) = (ts >= (L_ITEM - L_CTL) / 4 && ts < (L_ITEM - L_CTL) / 4 + 3) ? (unsigned)(first + stride * (int)(ts - (L_ITEM - L_CTL) / 4)) : 0u;
        unsigned tk = 0;
        gu32* q = (gu32*)(myws + WS_TICKET); asm volatile("" : "+v"(q));
        if (ts == 0) tk = __hip_atomic_fetch_add(q, 1u, __ATOMIC_RELAXED, __HIP_MEMORY_SCOPE_AGENT);
        gchar* dws = uni(p.ws[d]);
        LBAR;
        if (ts < 64) {
            asm volatile("" : "+v"(tk));
            call = (rfl(tk) / GRID + 1u) & 0xffffffu;
            if (ts == 0) *(vl32*)(lds + L_CALL) = call;
            u32x4 HA[8], HB[8]; gu32* pf = nullptr; unsigned pv = 0;
            const bool remote = tick != nullptr;
            __builtin_amdgcn_s_setprio(SEND_PRIO);
            int i = first; unsigned k = 0;
            lds_wait4(lds, L_QI, 2u << 20); send_fetch(lds, 0, 0, HA);
            do {
                const gchar* ub; gu32* f;
                if (remote) { const unsigned ui = ((call & 1u) * 8 + ((rank - d - 1) & 7)) * 256 + i; ub = dws + WS_PIN + (size_t)ui * UNIT; f = (gu32*)(dws + WS_AFLAG + ui * 8); }
                else { int isW; f = stage_unit(myws, rank, 7, i, isW, ub); }
                ub = uni(ub);
                const u32x4 qv = *(vl128*)(lds + L_QI + (k & 3u) * 16) & 0xfffffu;
                const unsigned q0 = rfl(qv.x), q1 = rfl(qv.y), q2 = rfl(qv.z), q3 = rfl(qv.w);
                const unsigned fv = rfl(*(vl32*)(lds + L_CLS + (k & 15u) * 4)) | (q0 >> 14 << 5) | (q1 >> 14 << 10) | (q2 >> 14 << 15) | (q3 >> 14 << 20);
                i = (int)rfl(*(vl32*)(lds + L_ITEM + ((k + 1u) & 15u) * 4));
                send_fetch(lds, k, 1, HB); send8<0>(ub, HA, q0 & 0x3fffu, pf, call, pv, remote);
                send_fetch(lds, k, 2, HA); send8<1>(ub, HB, q1 & 0x3fffu, pf, call, pv, remote);
                send_fetch(lds, k, 3, HB); send8<2>(ub, HA, q2 & 0x3fffu, pf, call, pv, remote);
                LGKM0;
                if (ts == 0) *(vl32*)(lds + L_CONS) = k + 1u;
                if (i < 256) { lds_wait4(lds, L_QI + ((k + 1u) & 3u) * 16, (2u * k + 4u) << 20);
                    send_fetch(lds, k + 1u, 0, HA); }
                else if (tick && ts == 0) __hip_atomic_fetch_add(tick + (WS_OREL - WS_LINKQ) / 4, PUSH_REL, __ATOMIC_RELAXED, __HIP_MEMORY_SCOPE_AGENT);
                send8<3>(ub, HB, q3 & 0x3fffu, pf, call, pv, remote);
                pf = f; pv = fv; ++k;
            } while (i < 256);
            asm volatile("s_waitcnt vmcnt(0)" ::: "memory");
            publish(pf, call, pv, remote);
            __builtin_amdgcn_s_setprio(0);
        } else if (tick) {
            for (unsigned n = rfl(ts >> 6) - 1u;; n += 3u) {
                const unsigned k = n >> 2, qq = n & 3u;
                if (rfl(*(vl32*)(lds + L_ITEM + (k & 15u) * 4)) >= 256u) break;
                unsigned st;
                while ((st = rfl(*(vl32*)(lds + L_QI + (k & 3u) * 16 + qq * 4))) < ((2u * k + 1u) << 20)) __builtin_amdgcn_s_sleep(1);
                if (st >= ((2u * k + 2u) << 20)) continue;
                asm volatile("" ::: "memory");
                u32x4 H[8];
                send_fetch(lds, k, (int)qq, H);
                const unsigned B = (st >> 14) & 31u, size = q_encode(H, B, lds + (k & 3u) * UNIT + qq * Q_BYTES, ts & 63u);
                LGKM0;
                if ((ts & 63u) == 0) *(vl32*)(lds + L_QI + (k & 3u) * 16 + qq * 4) = ((2u * k + 2u) << 20) | (size < Q_BYTES ? B << 14 : 0u) | size;
            }
        }
        LBAR;
        call = rfl(*(vl32*)(lds + L_CALL));
    }
    return call;
}

__device__ __forceinline__ void unpack_units(LAS unsigned char* lds, gchar* myws, int rank, unsigned call, int g0, unsigned wv) {
    const unsigned lane = threadIdx.x & 63u;
    gu32* tmo = (gu32*)(myws + WS_TMO);
#define UT(i) do {} while (0)
    for (int g = g0; g < 7 * 256; g += 128) {
        const int i = g / 7, srel = g - i * 7;
        const unsigned ui = ((call & 1u) * 8 + srel) * 256 + i;
        const GAS unsigned long long* af = (const GAS unsigned long long*)(myws + WS_AFLAG) + ui;
        unsigned info;
        const unsigned long long t0 = __builtin_amdgcn_s_memrealtime();
        for (;;) {
            const unsigned long long fv = __hip_atomic_load(af, __ATOMIC_RELAXED, __HIP_MEMORY_SCOPE_SYSTEM);
            info = rfl((unsigned)fv);
            if (rfl((unsigned)(fv >> 32)) >= call) break;
            if (rfl(ld_agt(tmo)) == call || __builtin_amdgcn_s_memrealtime() - t0 > DEADLINE) {
                if (lane == 0) __hip_atomic_store(tmo, call, __ATOMIC_RELAXED, __HIP_MEMORY_SCOPE_AGENT);
                return;
            }
            __builtin_amdgcn_s_sleep(FLAG_SLEEP);
        }
        asm volatile("" ::: "memory");
        UT(0);
        const __amdgpu_buffer_rsrc_t rs = __builtin_amdgcn_make_buffer_rsrc((void*)uni(myws + WS_PIN + (size_t)ui * UNIT), 0, UNIT, 0x00020000);
        int isW; const gchar* ub; gu32* f = stage_unit(myws, rank, srel, i, isW, ub); ub = uni(ub);
        QIn Q[2];
        const unsigned q0 = (wv & 1u) * 2u, B[2] = {(info >> (5u + 5u * q0)) & 31u, (info >> (10u + 5u * q0)) & 31u};
#pragma unroll
        for (int q = 0; q < 2; ++q) Q[q].lb = __builtin_amdgcn_raw_buffer_load_b8(rs, lane, (q0 + q) * Q_BYTES + Q_LEN, 17);
#pragma unroll
        for (int q = 0; q < 2; ++q) q_load1(rs, (q0 + q) * Q_BYTES, lane, Q[q]);
#pragma unroll
        for (int q = 0; q < 2; ++q) q_load2(rs, (q0 + q) * Q_BYTES, lane, B[q] == 0u, Q[q]);
        __builtin_amdgcn_s_waitcnt(0x0F70);
        UT(1);
        u32x4 W[2][8];
#pragma unroll
        for (int q = 0; q < 2; ++q) q_finish(B[q], Q[q], W[q]);
#pragma unroll
        for (int q = 0; q < 2; ++q) {
            q_patch(B[q], Q[q], W[q]);
#pragma unroll
            for (int c = 0; c < 8; ++c) store16_sys(ub + img_off(isW, q0 + q, c), lane * 16u, W[q][c]);
        }
        UT(2);
        asm volatile("s_waitcnt vmcnt(0)" ::: "memory");
        unsigned old = 0;
        if (lane == 0) old = __hip_atomic_fetch_add((LAS unsigned*)(lds + L_UCNT + (wv >> 1) * 64u + (unsigned)(g >> 7) * 4u), 1u, __ATOMIC_RELAXED, __HIP_MEMORY_SCOPE_WORKGROUP);
        if (rfl(old) == 1u) publish(f, call, info, false);
        UT(3);
    }
}

__global__ void __launch_bounds__(NTHR, 2) gemm_rs_kernel(const Params p) {
    extern __shared__ __attribute__((aligned(16))) unsigned char smem[];
    LAS unsigned char* lds = (LAS unsigned char*)smem;
    const int tid = threadIdx.x;
    const int rank = (int)rfl(p.rank), bid = (int)rfl(blockIdx.x);
    gchar* myws = uni(p.ws[rank]);
    STAMP(0);
    const int xcd = bid & 7, slot = bid >> 3;
    const int pd = slot < 7 ? (rank + 1 + slot) & 7 : rank, pfirst = slot < 7 ? xcd : (31 - slot) * 8 + xcd, pstride = slot < 7 ? 8 : 200;
    gu32* tmo = (gu32*)(myws + WS_TMO);

    int pm, pn;
    {
        int ar = -1;
        if (slot < 11) {
            if (slot >= 7) { pm = 2 * rank + (slot & 1); pn = 2 * xcd + ((slot - 7) >> 1); }
            else { ar = 2 * slot + (xcd >> 2); pn = NEC + (xcd & 3); }
        } else {
            const int li = slot - 11, q = li / 7;
            pn = 3 * (xcd & 3) + q; ar = 2 * (li - q * 7) + (xcd >> 2);
        }
        if (ar >= 0) pm = ar + (ar >= 2 * rank ? 2 : 0);
    }
    const unsigned call = push_items(p, lds, myws, rank, pd, pfirst, pstride, slot < 7 ? (gu32*)(myws + WS_LINKQ + slot * 128) : nullptr);
    if (slot >= 7 && slot < 11) unpack_units(lds, myws, rank, call, (int)rfl(tid >> 7) * 32 + (slot - 7) * 8 + xcd, rfl(tid >> 6));
    STAMP(1);

    const int wid = rfl(tid >> 6), lane = tid & 63, wr = wid >> 2, wc = wid & 3, fr = lane & 15, fq = lane >> 4;
    const gchar* cA = myws + WS_STAGE + (size_t)pm * NKT * UNIT;
    const gchar* cB = myws + WS_STAGE + OPND_BYTES + (size_t)pn * NKT * UNIT;
    const gu32* fA = (const gu32*)(myws + WS_FLAG) + pm * NKT;
    const gu32* fB = (const gu32*)(myws + WS_FLAG) + (16 + pn) * NKT;
    const unsigned voff[2] = {(unsigned)tid * 16u, (unsigned)tid * 16u + 8192u}; const unsigned ldsw = (unsigned)wid * 1024u;
    const unsigned aoff0 = (unsigned)((wr * 64 + fr) * 128 + ((fq ^ (fr >> 1)) * 16)), aoff1 = aoff0 ^ 64u;
    const unsigned boff = (unsigned)(fq * 2048 + (wc * 16 + fr) * 16);
    int ready = 0; unsigned long long chg = 0; int scur = 0, smax = 0;

#define SA(b, h) (((b) * 2 + (h)) * HTB)
#define SB(b, h) ((4 + (b) * 2 + (h)) * HTB)
#define STAGE(bufoff, gbase) do { _Pragma("unroll") for (int _i = 0; _i < 2; ++_i) \
        __builtin_amdgcn_global_load_lds((const gu32*)((gbase) + voff[_i]), (LAS unsigned*)(lds + (bufoff) + ldsw + _i * 8192), 16, 0, GLDS_AUX); } while (0)
#define LDA(dst, b, h) do { _Pragma("unroll") for (int m = 0; m < 4; ++m) { dst[m][0] = *(const LAS h16x8*)(lds + SA(b, h) + aoff0 + m * 2048); dst[m][1] = *(const LAS h16x8*)(lds + SA(b, h) + aoff1 + m * 2048); } } while (0)
#define LDB(dst, b, h) do { _Pragma("unroll") for (int n = 0; n < 2; ++n) _Pragma("unroll") for (int k = 0; k < 2; ++k) dst[n][k] = *(const LAS h16x8*)(lds + SB(b, h) + boff + n * 1024 + k * 8192); } while (0)
#define MMA(ai, bj, At, Bt) do { __builtin_amdgcn_s_setprio(1); _Pragma("unroll") for (int m = 0; m < 4; ++m) _Pragma("unroll") for (int n = 0; n < 2; ++n) _Pragma("unroll") for (int k = 0; k < 2; ++k) \
        acc[ai][bj][m][n] = __builtin_amdgcn_mfma_f32_16x16x32_f16(At[m][k], Bt[n][k], acc[ai][bj][m][n], 0, 0, 0); __builtin_amdgcn_s_setprio(0); } while (0)
#define WAIT_V(n) asm volatile("s_waitcnt vmcnt(" #n ")" ::: "memory")
#define WAIT_L(n) asm volatile("s_waitcnt lgkmcnt(" #n ")" ::: "memory")
#define BAR __builtin_amdgcn_s_barrier()
#define SCHED __builtin_amdgcn_sched_barrier(0)
#define HOOK(need) do { if (ready < (need)) { \
        if (wid == 0) { \
            const unsigned long long t0 = __builtin_amdgcn_s_memrealtime(); unsigned fa, fb; int rd; \
            unsigned ln = voff[0]; asm volatile("" : "+v"(ln)); ln >>= 4;     \
            for (;;) { \
                fa = ld_sys(fA + ln); fb = ld_sys(fB + ln); \
                const unsigned long long okm = __ballot((fa >> 8) >= call && (fb >> 8) >= call); \
                rd = (okm == ~0ull) ? 64 : (int)__builtin_ctzll(~okm); \
                if (rd >= (need)) break; \
                if (rfl(ld_agt(tmo)) == call || __builtin_amdgcn_s_memrealtime() - t0 > DEADLINE) { \
                    if (ln == 0) __hip_atomic_store(tmo, call, __ATOMIC_RELAXED, __HIP_MEMORY_SCOPE_AGENT); \
                    rd = 64; break; } \
                __builtin_amdgcn_s_sleep(POLL_SLEEP); \
            } \
            const int sv = (int)((fa & 31u) + (fb & 31u)); \
            const int sp = __builtin_amdgcn_update_dpp(-1, sv, 0x138, 0xf, 0xf, false);     \
            const unsigned long long cg = __ballot(sv != sp); \
            *(LAS int*)(lds + L_TAB + ln * 4) = sv; \
            if (ln == 0) { u32x4 h; h.x = (unsigned)rd; h.y = (unsigned)cg; h.z = (unsigned)(cg >> 32); h.w = 0; *(LAS u32x4*)(lds + L_HDR) = h; } \
            DIAG_WAIT(t0); \
            asm volatile("s_waitcnt vmcnt(0) lgkmcnt(0)" ::: "memory"); \
        } \
        asm volatile("" ::: "memory"); BAR; asm volatile("" ::: "memory"); \
        { const u32x4 h = *(const LAS u32x4*)(lds + L_HDR); ready = (int)rfl(h.x); chg = ((unsigned long long)rfl(h.z) << 32) | rfl(h.y); } \
    } } while (0)
#define DIAG_WAIT(t0) do {} while (0)
#define PKMUL4(op) ".set rs_i, 0\n\t.set rs_d, 0\n\t.irpc c, \"" op "\"\n\t.ifc \\c,:\n\t.set rs_d, 1\n\t.endif\n\t" \
        ".if rs_d == 0\n\t.ifnc \\c,v\n\t.ifnc \\c,[\n\t.set rs_i, rs_i*10 + \\c\n\t.endif\n\t.endif\n\t.endif\n\t.endr\n\t" \
        "v_pk_mul_f32 v[rs_i:rs_i+1], v[rs_i:rs_i+1], %16\n\tv_pk_mul_f32 v[rs_i+2:rs_i+3], v[rs_i+2:rs_i+3], %16\n\t"
#define RS16(a) asm volatile("s_cmp_eq_u32 %17, 0\n\ts_cbranch_scc1 .Lrs%=\n\ts_nop 15\n\ts_nop 3\n\t" \
            PKMUL4("%0") PKMUL4("%1") PKMUL4("%2") PKMUL4("%3") PKMUL4("%4") PKMUL4("%5") PKMUL4("%6") PKMUL4("%7") \
            PKMUL4("%8") PKMUL4("%9") PKMUL4("%10") PKMUL4("%11") PKMUL4("%12") PKMUL4("%13") PKMUL4("%14") PKMUL4("%15") \
            "s_nop 4\n.Lrs%=:" \
            : "+v"(acc[a][0][0][0]), "+v"(acc[a][0][0][1]), "+v"(acc[a][0][1][0]), "+v"(acc[a][0][1][1]), "+v"(acc[a][0][2][0]), "+v"(acc[a][0][2][1]), "+v"(acc[a][0][3][0]), "+v"(acc[a][0][3][1]), \
              "+v"(acc[a][1][0][0]), "+v"(acc[a][1][0][1]), "+v"(acc[a][1][1][0]), "+v"(acc[a][1][1][1]), "+v"(acc[a][1][2][0]), "+v"(acc[a][1][2][1]), "+v"(acc[a][1][3][0]), "+v"(acc[a][1][3][1]) \
            : "s"(fac), "s"(de) : "scc")
#define RESCALE(kt) do { int de = 0; \
        if ((chg >> (kt)) & 1ull) { \
            const int s = (int)rfl(*(const LAS unsigned*)(lds + L_TAB + (kt) * 4)); \
            smax = max(smax, s); const int sn = max(s, smax - 4); de = max(16 * (scur - sn), -126); scur = sn; } \
        const unsigned long long fac = (unsigned long long)(unsigned)((de + 127) << 23) * 0x100000001ull; \
        RS16(0); RS16(1); \
    } while (0)
#define TRIP do { \
        const int t2 = (t + 2) & (NKT - 1);             \
        const gchar* a1 = cA + (size_t)(t + 1) * UNIT; \
        const gchar* a2 = cA + (size_t)t2 * UNIT; const gchar* b2 = cB + (size_t)t2 * UNIT; \
        const gchar* a3 = a2 + UNIT; const gchar* b3 = b2 + UNIT; \
        RESCALE(t); \
        LDB(B0, 0, 0); SCHED; LDA(At, 0, 0); STAGE(SA(1, 1), a1 + HTB); \
        WAIT_L(8); BAR; WAIT_L(0); MMA(0, 0, At, B0); BAR; SCHED; \
        LDB(B1, 0, 1); STAGE(SB(0, 0), b2); \
        BAR; WAIT_L(0); MMA(0, 1, At, B1); BAR; \
        LDA(At, 0, 1); STAGE(SA(0, 0), a2); \
        BAR; WAIT_L(0); MMA(1, 0, At, B0); BAR; SCHED; \
        STAGE(SB(0, 1), b2 + HTB); \
        WAIT_V(6); BAR; MMA(1, 1, At, B1); BAR; \
        RESCALE(t + 1); \
        LDB(B0, 1, 0); SCHED; LDA(At, 1, 0); STAGE(SA(0, 1), a2 + HTB); \
        WAIT_L(8); BAR; WAIT_L(0); MMA(0, 0, At, B0); BAR; SCHED; \
        LDB(B1, 1, 1); STAGE(SB(1, 0), b3); \
        BAR; WAIT_L(0); MMA(0, 1, At, B1); BAR; \
        LDA(At, 1, 1); STAGE(SA(1, 0), a3); \
        BAR; WAIT_L(0); MMA(1, 0, At, B0); BAR; SCHED; \
        STAGE(SB(1, 1), b3 + HTB); \
        WAIT_V(6); BAR; MMA(1, 1, At, B1); BAR; \
    } while (0)

    f32x4 acc[2][2][4][2];
#pragma unroll
    for (int a = 0; a < 2; ++a)
#pragma unroll
        for (int b = 0; b < 2; ++b)
#pragma unroll
            for (int m = 0; m < 4; ++m)
#pragma unroll
                for (int n = 0; n < 2; ++n) acc[a][b][m][n] = (f32x4){0.f, 0.f, 0.f, 0.f};
    h16x8 At[4][2], B0[2][2], B1[2][2];

    HOOK(2);
    STAMP(2);
    STAGE(SB(0, 0), cB); STAGE(SA(0, 0), cA); STAGE(SB(0, 1), cB + HTB); STAGE(SA(0, 1), cA + HTB);
    if (wr == 1) BAR;
    WAIT_V(4); BAR;
    STAGE(SB(1, 0), cB + UNIT); STAGE(SA(1, 0), cA + UNIT); STAGE(SB(1, 1), cB + UNIT + HTB);
    WAIT_V(6); BAR;
    for (int t = 0; t < NKT; t += 2) {
        HOOK(min(t + 4, NKT));
        TRIP;
    }
    if (wr == 0) BAR;
    STAMP(3);

    const int dest = pm >> 1;
    const unsigned ovo = (unsigned)(wr * 64 + 4 * fq) * (GN * 4u) + (unsigned)(wc * 16 + fr) * 16u;
    const float qn = __builtin_nanf("");
    u32x4 V[32];
    {
        const bool bad = rfl(ld_agt(tmo)) == call;
        const int oe = 16 * scur - 256;
#pragma unroll
        for (int a = 0; a < 2; ++a)
#pragma unroll
            for (int m = 0; m < 4; ++m)
#pragma unroll
                for (int r = 0; r < 4; ++r) {
                    f32x4 v;
                    v.x = __builtin_amdgcn_ldexpf(acc[a][0][m][0][r], oe); v.y = __builtin_amdgcn_ldexpf(acc[a][0][m][1][r], oe);
                    v.z = __builtin_amdgcn_ldexpf(acc[a][1][m][0][r], oe); v.w = __builtin_amdgcn_ldexpf(acc[a][1][m][1][r], oe);
                    if (bad) v = (f32x4){qn, qn, qn, qn};
                    V[(a * 4 + m) * 4 + r] = __builtin_bit_cast(u32x4, v);
                }
    }
#define OSTAMP(k) do {} while (0)
#define DIAG_END do {} while (0)
    const unsigned cidx = (unsigned)(((pm & 1) * 16 + pn) * 8 + wid);
    const int link = (dest - rank - 1) & 7;
    gu32* rel = (gu32*)(myws + WS_OREL + link * 128);
    gchar* dws = uni(p.ws[dest]);
    bool wide = dest == rank;
#define LINK_ACQUIRE do { \
        while ((int)rfl(*(vl32*)(lds + L_TURN)) + (pn < NEC ? WG_SENDERS : WG_SENDERS_C) <= wid) __builtin_amdgcn_s_sleep(8); \
        unsigned tk = 0; \
        if (lane == 0) tk = __hip_atomic_fetch_add((gu32*)(myws + WS_OTK + link * 128), 1u, __ATOMIC_RELAXED, __HIP_MEMORY_SCOPE_AGENT); \
        tk = rfl(tk) & 255u; \
        wide = (7 - (wid & ~1)) * 8 > (int)(256u - tk) + BEHIND;                                      \
        const unsigned need = (call - 1u) * (256u + 8u * PUSH_REL) + tk + 1u - LINK_EXTRA;         \
        const unsigned long long t0 = __builtin_amdgcn_s_memrealtime(); \
        for (;;) { \
            const int ahead = (int)(need - rfl(ld_agt(rel))); \
            if (ahead <= 0 || __builtin_amdgcn_s_memrealtime() - t0 > DEADLINE) break;         \
            if (ahead > 4) __builtin_amdgcn_s_sleep(32); else __builtin_amdgcn_s_sleep(4); \
        } \
        __builtin_amdgcn_s_setprio(SEND_PRIO); } while (0)
#define LINK_PACE(q, rel_at) do { \
        if (wide) asm volatile("s_waitcnt vmcnt(" STR(OUT_W2) ")" ::: "memory"); else asm volatile("s_waitcnt vmcnt(" STR(OUT_W) ")" ::: "memory"); \
        if ((q) == (rel_at) && dest != rank && lane == 0) { \
            __hip_atomic_fetch_add((LAS unsigned*)(lds + L_TURN), 1u, __ATOMIC_RELAXED, __HIP_MEMORY_SCOPE_WORKGROUP); \
            __hip_atomic_fetch_add(rel, 1u, __ATOMIC_RELAXED, __HIP_MEMORY_SCOPE_AGENT); } \
        OSTAMP((q) < 27 ? (q) : 26); } while (0)
    const bool direct = dest == rank || (pn >= NEC && wid >= RAW_WAVE);
    unsigned myB = 0;
    if (direct) {
        const gchar* ob = uni((const char*)p.out[dest] + ((size_t)((pm & 1) * 256) * GN + rank * NL + pn * 256) * 4);
        if (dest != rank) {
            LINK_ACQUIRE;
        }
#pragma unroll
        for (int q = 0; q < 32; ++q) { store16_sys(ob + out_off(q), ovo, V[q]); LINK_PACE(q, REL_AT + 5); }
    }
    int packed = direct ? 0 : 1;
    asm volatile("" : "+v"(packed)); packed = (int)rfl(packed);
    if (packed) {
        const gchar* pb = uni(dws + WS_PACK + (size_t)(rank * 256 + cidx) * CHUNK);
        unsigned mu = 0; float sm = 0.f;
#pragma unroll
        for (int q = 0; q < 32; ++q) {
            const u32x4 w = V[q] & 0x7fffffffu; mu = max(max(mu, w.x), max(w.y, max(w.z, w.w)));
            const f32x4 f = __builtin_bit_cast(f32x4, w); sm += (f.x + f.y) + (f.z + f.w);
        }
        myB = min(max(min(wave_mean_exp(sm * (2.32f / 8192.f)), wave_max(mu >> 23)), 2u), 254u);
        u32x4 PK[26];
        LAS unsigned char* scr = lds + L_SCR + wid * 3072;
        const unsigned tail = c_encode(V, myB, PK, scr, pb, lane);
        LINK_ACQUIRE;
#pragma unroll
        for (int q = 0; q < 3; ++q) if (q * 1024u + lane * 16u < tail) store16_sys(pb + (C_HDR + q * 1024u), lane * 16u, *(const LAS u32x4*)(scr + q * 1024u + lane * 16u));
#pragma unroll
        for (int q = 0; q < 26; ++q) { store16_sys(pb + q * 1024u, lane * 16u, PK[q]); LINK_PACE(q, REL_AT - 1); }
    }
    if (dest != rank) {
        WAIT_V(0);
        if (lane == 0) __hip_atomic_store((gu32*)(dws + WS_CFLAG) + rank * 256 + cidx, (call << 8) | myB, __ATOMIC_RELAXED, __HIP_MEMORY_SCOPE_SYSTEM);
        __builtin_amdgcn_s_setprio(0);
        OSTAMP(28);
        const gu32* cf = (const gu32*)(myws + WS_CFLAG) + dest * 256 + cidx;
        unsigned fv;
        const unsigned long long t0 = __builtin_amdgcn_s_memrealtime();
        for (;;) {
            fv = rfl(ld_sys(cf));
            if ((fv >> 8) >= call) break;
            if (rfl(ld_agt(tmo)) == call || __builtin_amdgcn_s_memrealtime() - t0 > DEADLINE) {
                if (lane == 0) __hip_atomic_store(tmo, call, __ATOMIC_RELAXED, __HIP_MEMORY_SCOPE_AGENT);
                break;
            }
            __builtin_amdgcn_s_sleep(FLAG_SLEEP);
        }
        const bool bad = rfl(ld_agt(tmo)) == call;
        asm volatile("" ::: "memory");
        OSTAMP(29);
        if (bad || (fv & 255u)) {
            const __amdgpu_buffer_rsrc_t rs = __builtin_amdgcn_make_buffer_rsrc((void*)uni(myws + WS_PACK + (size_t)(dest * 256 + cidx) * CHUNK), 0, bad ? 0 : CHUNK, 0x00020000);
            c_decode(rs, fv & 255u, bad, uni((const char*)p.out[rank] + ((size_t)((pm & 1) * 256) * GN + dest * NL + pn * 256) * 4), ovo, lane);
            if (bad && lane == 0) p.out[rank][(size_t)rank * NL] = qn;
        }
    }
    DIAG_END;
}

extern "C" void kernel_launch(int rank, int nranks, ncclComm_t comm,
                              hipStream_t stream,
                              void* const* d_in, const int* in_sizes, int n_in,
                              void* d_out, int out_size,
                              const int* in_dtypes, int out_dtype,
                              void* const* d_out_all,
                              void* const* d_ws_all, size_t ws_bytes) {
    (void)comm; (void)in_dtypes; (void)out_dtype; (void)d_out;
    if (nranks != NR || (unsigned)rank >= (unsigned)NR || n_in < 2 || ws_bytes < WS_NEED ||
        in_sizes[0] != GM * KS || in_sizes[1] != KS * GN || out_size != (GM / NR) * GN) {
        fprintf(stderr, "gemm_rs: unexpected problem shape\n");
        return;
    }
    HIP_CHECK(hipFuncSetAttribute((const void*)gemm_rs_kernel, hipFuncAttributeMaxDynamicSharedMemorySize, (int)LDS_BYTES));
    Params p;
    p.x = (const float*)d_in[0]; p.w = (const float*)d_in[1];
    for (int i = 0; i < NR; ++i) { p.out[i] = (float*)d_out_all[i]; p.ws[i] = (char*)d_ws_all[i]; }
    p.rank = rank; p.pad = 0;
    hipLaunchKernelGGL(gemm_rs_kernel, dim3(GRID), dim3(NTHR), LDS_BYTES, stream, p);
    HIP_CHECK(hipGetLastError());
}
```

```cpp
#include <hip/hip_runtime.h>
#include <rccl/rccl.h>
#include <cstdio>
#include <cstdint>

#define HIP_CHECK(expr) do { hipError_t _e = (expr); if (_e != hipSuccess) { \
    fprintf(stderr, "HIP error %s at %s:%d\n", hipGetErrorString(_e), __FILE__, __LINE__); \
    return; } } while (0)

#define LAS __attribute__((address_space(3)))
#define GAS __attribute__((address_space(1)))
typedef GAS char gchar;
typedef GAS unsigned gu32;
typedef _Float16 h16x8 __attribute__((ext_vector_type(8)));
typedef float f32x4 __attribute__((ext_vector_type(4)));
typedef float f32x2 __attribute__((ext_vector_type(2)));
typedef unsigned u32x4 __attribute__((ext_vector_type(4)));

constexpr int NR = 8;
constexpr int GM = 4096, GN = 32768, KS = 512, NL = 4096;
constexpr int GRID = 256, NTHR = 512;
constexpr unsigned UNIT = 32768;
constexpr unsigned HTB = 16384;
constexpr int NKT = 64;
constexpr size_t WS_TICKET = 0, WS_TMO = 128, WS_TASKQ = 256, WS_LINKQ = 1024, WS_OTK = 2048, WS_OREL = 3072;
constexpr size_t WS_FLAG = 4096;
constexpr size_t WS_CFLAG = 16384;
constexpr size_t WS_AFLAG = 32768;
constexpr size_t WS_PX = 65536;
constexpr size_t WS_PW = WS_PX + (size_t)256 * UNIT;
constexpr size_t WS_STAGE = WS_PW + (size_t)14 * 128 * UNIT;
constexpr unsigned KT_BYTES = 32 * UNIT;
constexpr unsigned CHUNK = 37888;
constexpr size_t WS_PACK = WS_STAGE + (size_t)NKT * KT_BYTES;
constexpr size_t WS_NEED = WS_PACK + (size_t)NR * 256 * CHUNK;
constexpr size_t WS_DIAG = 65536, WS_DBASE = WS_NEED;
__device__ __forceinline__ size_t pin_off(unsigned par, int isW, int dd, int j, int u) {
    return isW ? WS_PW + (size_t)((par * 7 + dd) * 128 + j * 16 + u) * UNIT : WS_PX + (size_t)((j * 2 + par) * 16 + u) * UNIT;
}
#define NEARLY 10
constexpr int NE = NEARLY;
constexpr unsigned long long DEADLINE = 100000000ull;
constexpr unsigned L_CTL = 8 * HTB;
constexpr unsigned L_AMAX = L_CTL, L_CONS = L_CTL + 32, L_TURN = L_CTL + 48, L_CALL = L_CTL + 52, L_LAND = L_CTL + 56, L_LST = L_CTL + 60, L_ITEM = L_CTL + 64, L_QI = L_CTL + 128;
constexpr unsigned L_ECNT = L_CTL + 192, L_UCNT = L_CTL + 224, L_FREE = L_CTL + 256, L_DESC = L_CTL + 320, L_ZERO_END = L_CTL + 448;
constexpr unsigned L_HDR = L_CTL + 576, L_TAB = L_CTL + 640, L_SCR = L_CTL + 1024, LDS_BYTES = L_SCR + 8 * 3072;

#define GLDS_AUX 17
struct Params {
    const float* x; const float* w;
    float* out[NR]; char* ws[NR];
    int rank; int pad;
};

#define STAMP(i) do {} while (0)

__device__ __forceinline__ void store16_sys(const gchar* sbase, unsigned voff, u32x4 v) {
    asm volatile("global_store_dwordx4 %0, %1, %2 sc0 sc1\n\ts_nop 1" :: "v"(voff), "v"(v), "s"(sbase) : "memory");
}
__device__ __forceinline__ void glds16_sys(unsigned dst, unsigned voff, const gchar* sbase) {
    unsigned keep;
    asm volatile("s_mov_b32 %0, m0\n\ts_mov_b32 m0, %1\n\ts_nop 0\n\tglobal_load_lds_dwordx4 %2, %3 sc0 sc1\n\ts_mov_b32 m0, %0" : "=&s"(keep) : "s"(dst), "v"(voff), "s"(sbase) : "memory");
}
__device__ __forceinline__ unsigned ld_sys(const gu32* q) { return __hip_atomic_load(q, __ATOMIC_RELAXED, __HIP_MEMORY_SCOPE_SYSTEM); }
__device__ __forceinline__ unsigned ld_agt(const gu32* q) { return __hip_atomic_load(q, __ATOMIC_RELAXED, __HIP_MEMORY_SCOPE_AGENT); }
__device__ __forceinline__ unsigned rfl(unsigned v) { return (unsigned)__builtin_amdgcn_readfirstlane((int)v); }

template <class T> __device__ __forceinline__ gchar* uni(T q) {
    unsigned lo = rfl((unsigned)(uintptr_t)q), hi = rfl((unsigned)((uintptr_t)q >> 32));
    asm volatile("s_nop 4" : "+s"(lo), "+s"(hi));
    return (gchar*)(((uintptr_t)hi << 32) | lo);
}

__device__ __forceinline__ unsigned wave_max(unsigned x) {
    int mb = (int)x;
    mb = max(mb, __builtin_amdgcn_update_dpp(0, mb, 0x111, 0xf, 0xf, false));
    mb = max(mb, __builtin_amdgcn_update_dpp(0, mb, 0x112, 0xf, 0xf, false));
    mb = max(mb, __builtin_amdgcn_update_dpp(0, mb, 0x114, 0xf, 0xf, false));
    mb = max(mb, __builtin_amdgcn_update_dpp(0, mb, 0x118, 0xf, 0xf, false));
    mb = max(mb, __builtin_amdgcn_update_dpp(0, mb, 0x142, 0xa, 0xf, false));
    mb = max(mb, __builtin_amdgcn_update_dpp(0, mb, 0x143, 0xc, 0xf, false));
    return (unsigned)__builtin_amdgcn_readlane(mb, 63);
}
__device__ __forceinline__ unsigned wave_mean_exp(float sm) {
#define DPP_ADD(ctl, rm) sm += __builtin_bit_cast(float, __builtin_amdgcn_update_dpp(0, __builtin_bit_cast(int, sm), ctl, rm, 0xf, false))
    DPP_ADD(0x111, 0xf); DPP_ADD(0x112, 0xf); DPP_ADD(0x114, 0xf); DPP_ADD(0x118, 0xf); DPP_ADD(0x142, 0xa); DPP_ADD(0x143, 0xc);
    return ((unsigned)__builtin_amdgcn_readlane(__builtin_bit_cast(int, sm), 63) >> 23) & 255u;
}

typedef unsigned short u16x2 __attribute__((ext_vector_type(2)));
typedef unsigned u32x2 __attribute__((ext_vector_type(2)));
constexpr unsigned Q_P6 = 6144, Q_LEN = 6656, Q_STR = 6720, Q_BYTES = 8192;
__device__ __forceinline__ u16x2 as16(unsigned a) { return __builtin_bit_cast(u16x2, a); }
__device__ __forceinline__ unsigned as32(u16x2 a) { return __builtin_bit_cast(unsigned, a); }
__device__ __forceinline__ unsigned pk_sub(unsigned a, unsigned b) { return as32(as16(a) - as16(b)); }
__device__ __forceinline__ unsigned pk_subs(unsigned a, unsigned b) { return as32(__builtin_elementwise_sub_sat(as16(a), as16(b))); }
__device__ __forceinline__ unsigned pk_min(unsigned a, unsigned b) { return as32(__builtin_elementwise_min(as16(a), as16(b))); }
__device__ __forceinline__ unsigned ffbl(unsigned a) { unsigned r; asm("v_ffbl_b32 %0, %1" : "=v"(r) : "v"(a)); return r; }
__device__ __forceinline__ unsigned wave_scan(unsigned x) {
    int v = (int)x;
    v += __builtin_amdgcn_update_dpp(0, v, 0x111, 0xf, 0xf, false);
    v += __builtin_amdgcn_update_dpp(0, v, 0x112, 0xf, 0xf, false);
    v += __builtin_amdgcn_update_dpp(0, v, 0x114, 0xf, 0xf, false);
    v += __builtin_amdgcn_update_dpp(0, v, 0x118, 0xf, 0xf, false);
    v += __builtin_amdgcn_update_dpp(0, v, 0x142, 0xa, 0xf, false);
    v += __builtin_amdgcn_update_dpp(0, v, 0x143, 0xc, 0xf, false);
    return (unsigned)v;
}
__device__ __forceinline__ unsigned q_encode(const u32x4 (&W)[8], unsigned B, LAS unsigned char* dst, unsigned lane) {
    const unsigned B1 = (B + 1u) * 0x04000400u;
    unsigned v[32], seg[4], pos[4];
    u32x4 X[4];
#pragma unroll
    for (int s = 0; s < 4; ++s) {
        unsigned a = 0, p = 0;
        X[s].x = __builtin_amdgcn_perm(W[2 * s].y, W[2 * s].x, 0x07050301u); X[s].y = __builtin_amdgcn_perm(W[2 * s].w, W[2 * s].z, 0x07050301u);
        X[s].z = __builtin_amdgcn_perm(W[2 * s + 1].y, W[2 * s + 1].x, 0x07050301u); X[s].w = __builtin_amdgcn_perm(W[2 * s + 1].w, W[2 * s + 1].z, 0x07050301u);
#pragma unroll
        for (int k = 8 * s; k < 8 * s + 8; ++k) {
            const unsigned w = W[k >> 2][k & 3];
            const unsigned dq = as32(as16(pk_sub(B1, w & 0x7c007c00u)) >> 10);
            const unsigned ix = pk_min(dq, 0x00070007u) | 0x0c000c00u;
            const unsigned len = __builtin_amdgcn_perm(0x05040201u, 0x00000003u, ix) + pk_subs(dq, 0x00070007u);
            v[k] = (w & 0x83ff83ffu) | (__builtin_amdgcn_perm(0u, 0x03020100u, ix) << 10);
            p += len & 0xffffu; a |= 1u << (p & 31u);
            p += len >> 16;     a |= 1u << (p & 31u);
        }
        seg[s] = a >> 1; pos[s] = p;
        asm volatile("" : "+v"(v[8 * s]), "+v"(v[8 * s + 1]), "+v"(v[8 * s + 2]), "+v"(v[8 * s + 3]), "+v"(v[8 * s + 4]), "+v"(v[8 * s + 5]), "+v"(v[8 * s + 6]), "+v"(v[8 * s + 7]),
                     "+v"(X[s]), "+v"(seg[s]), "+v"(pos[s]));
    }
    const bool ovf = max(max(pos[0], pos[1]), max(pos[2], pos[3])) > 31u;
    const unsigned tot = ovf ? 0u : pos[0] + pos[1] + pos[2] + pos[3];
    const unsigned incl = wave_scan(tot), total = (unsigned)__builtin_amdgcn_readlane((int)incl, 63);
    const unsigned long long om = __ballot(ovf);
    const unsigned xb = Q_STR + (((total + 127u) >> 7) << 4), size = xb + 64u * (unsigned)__popcll(om);
    if (size > Q_BYTES - 64u) return Q_BYTES;
    *(LAS u32x4*)(dst + Q_STR + lane * 16u) = (u32x4){0u, 0u, 0u, 0u};
#pragma unroll
    for (int j = 0; j < 6; ++j)
#pragma unroll
        for (int i = 0; i < 4; ++i) v[4 * j + i] |= (v[26 + j] << (12 - 3 * i)) & 0x70007000u;
#pragma unroll
    for (int j = 0; j < 6; ++j) v[24 + j / 3] |= (v[26 + j] >> (1 + j % 3)) & (0x80008000u >> (1 + j % 3));
#pragma unroll
    for (int j = 0; j < 6; ++j) *(LAS u32x4*)(dst + j * 1024u + lane * 16u) = (u32x4){v[4 * j], v[4 * j + 1], v[4 * j + 2], v[4 * j + 3]};
    *(LAS u32x2*)(dst + Q_P6 + lane * 8u) = (u32x2){v[24], v[25]};
    const unsigned slot = __builtin_amdgcn_mbcnt_hi((unsigned)(om >> 32), __builtin_amdgcn_mbcnt_lo((unsigned)om, 0u));
    *(LAS unsigned char*)(dst + Q_LEN + lane) = (unsigned char)(ovf ? 128u + slot : tot);
    if (!ovf) {
        unsigned off = incl - tot;
#pragma unroll
        for (int s = 0; s < 4; ++s) {
            const unsigned long long x = (unsigned long long)seg[s] << (off & 31u);
            LAS unsigned* q = (LAS unsigned*)(dst + Q_STR + (off >> 5) * 4u);
            __hip_atomic_fetch_or(q, (unsigned)x, __ATOMIC_RELAXED, __HIP_MEMORY_SCOPE_WORKGROUP);
            __hip_atomic_fetch_or(q + 1, (unsigned)(x >> 32), __ATOMIC_RELAXED, __HIP_MEMORY_SCOPE_WORKGROUP);
            off += pos[s];
        }
    } else {
#pragma unroll
        for (int j = 0; j < 4; ++j) *(LAS u32x4*)(dst + xb + slot * 64u + j * 16u) = X[j];
    }
    return size;
}
#define RLOAD4(rs, vo, so) __builtin_bit_cast(u32x4, __builtin_amdgcn_raw_buffer_load_b128(rs, vo, so, 17))
struct QIn { u32x4 P[6]; u32x2 P6; unsigned lb, off, w4; u32x4 win, X[4]; };
__device__ __forceinline__ void q_read(const LAS unsigned char* src, unsigned lane, bool raw, QIn& q) {
    q.lb = raw ? 0u : *(const LAS unsigned char*)(src + Q_LEN + lane);
#pragma unroll
    for (int j = 0; j < 6; ++j) q.P[j] = *(const LAS u32x4*)(src + j * 1024u + lane * 16u);
    q.P6 = *(const LAS u32x2*)(src + Q_P6 + lane * 8u);
    const unsigned tot = q.lb >= 128u ? 0u : q.lb, incl = wave_scan(tot), total = (unsigned)__builtin_amdgcn_readlane((int)incl, 63);
    q.off = incl - tot;
    const LAS unsigned* w = (const LAS unsigned*)(src + Q_STR + (q.off >> 5) * 4u);
    q.win = (u32x4){w[0], w[1], w[2], w[3]}; q.w4 = w[4];
    if (raw || q.lb >= 128u) {
        const unsigned xo = raw ? Q_P6 + lane * 16u : min(Q_STR + (((total + 127u) >> 7) << 4) + (q.lb - 128u) * 64u, Q_BYTES - 64u), xs = raw ? 1024u : 16u;
        q.X[0] = *(const LAS u32x4*)(src + xo); q.X[1] = *(const LAS u32x4*)(src + xo + xs);
        q.X[2] = *(const LAS u32x4*)(src + xo + 32u); q.X[3] = *(const LAS u32x4*)(src + xo + 48u);
    }
}
__device__ __forceinline__ void q_finish(unsigned B, const QIn& q, u32x4 (&W)[8]) {
    unsigned v[32];
#pragma unroll
    for (int k = 0; k < 24; ++k) v[k] = q.P[k >> 2][k & 3];
    v[24] = q.P6.x; v[25] = q.P6.y;
#pragma unroll
    for (int j = 0; j < 6; ++j) {
        unsigned d = (v[24 + j / 3] << (1 + j % 3)) & 0x80008000u;
#pragma unroll
        for (int i = 0; i < 4; ++i) d |= (v[4 * j + i] & 0x70007000u) >> (12 - 3 * i);
        v[26 + j] = d;
    }
    u32x4 win;
    win.x = __builtin_amdgcn_alignbit(q.win.y, q.win.x, q.off); win.y = __builtin_amdgcn_alignbit(q.win.z, q.win.y, q.off);
    win.z = __builtin_amdgcn_alignbit(q.win.w, q.win.z, q.off); win.w = __builtin_amdgcn_alignbit(q.w4, q.win.w, q.off);
    const unsigned B1 = (B + 1u) * 0x04000400u;
#pragma unroll
    for (int s = 0; s < 4; ++s) {
        unsigned a = win.x << 1, used = 0;
#pragma unroll
        for (int k = 8 * s; k < 8 * s + 8; ++k) {
            const unsigned nz = v[k] | (v[k] >> 1);
            const unsigned l0 = ffbl((a & ~1u) | ((nz >> 10) & 1u));
            a >>= l0;
            const unsigned l1 = ffbl((a & ~1u) | ((nz >> 26) & 1u));
            a >>= l1;
            const unsigned len = l0 | (l1 << 16);
            used += len;
            const unsigned dq = __builtin_amdgcn_perm(0x00000006u, 0x00050400u, pk_min(len, 0x00040004u) | 0x0c000c00u) + pk_subs(len, 0x00040004u);
            W[k >> 2][k & 3] = (v[k] & 0x83ff83ffu) | (pk_sub(B1, (v[k] & 0x0c000c00u) + (dq << 10)) & 0x7c007c00u);
        }
        if (s < 3) {
            const unsigned sh = (used + (used >> 16)) & 31u;
            win.x = __builtin_amdgcn_alignbit(win.y, win.x, sh); win.y = __builtin_amdgcn_alignbit(win.z, win.y, sh);
            win.z = __builtin_amdgcn_alignbit(win.w, win.z, sh); win.w >>= sh;
        }
    }
}
__device__ __forceinline__ void q_patch(bool raw, const QIn& q, u32x4 (&W)[8]) {
    if (raw) {
#pragma unroll
        for (int j = 0; j < 6; ++j) W[j] = q.P[j];
        W[6] = q.X[0]; W[7] = q.X[1];
    } else if (q.lb >= 128u) {
#pragma unroll
        for (int k = 0; k < 32; ++k) W[k >> 2][k & 3] = __builtin_amdgcn_perm(q.X[k >> 3][(k >> 1) & 3], W[k >> 2][k & 3], (k & 1) ? 0x07020600u : 0x05020400u);
    }
}

constexpr unsigned C_HDR = 26624, C_STR = C_HDR + 128, C_EXC = 29696;
__device__ __forceinline__ unsigned c_encode(const u32x4 (&V)[32], unsigned B, u32x4 (&PK)[26], LAS unsigned char* scr, const gchar* pb, unsigned lane) {
    const unsigned B1 = (B + 1u) * 0x00010001u;
    unsigned seg[8], pos[8], hdr = 0;
#pragma unroll
    for (int g = 0; g < 4; ++g) {
        unsigned e2[16];
#pragma unroll
        for (int v = 0; v < 8; ++v) {
            u32x4 r;
#pragma unroll
            for (int c = 0; c < 4; ++c) r[c] = __builtin_amdgcn_alignbit(V[g * 8 + v][c], V[g * 8 + v][c], 31);
#pragma unroll
            for (int i = 0; i < 3; ++i) PK[g * 6 + ((3 * v + i) >> 2)][(3 * v + i) & 3] = __builtin_amdgcn_perm(r.w, r[i], 0x04020100u + ((unsigned)i << 24));
            e2[2 * v] = __builtin_amdgcn_perm(r.y, r.x, 0x0c070c03u); e2[2 * v + 1] = __builtin_amdgcn_perm(r.w, r.z, 0x0c070c03u);
        }
#pragma unroll
        for (int s = 2 * g; s < 2 * g + 2; ++s) {
            unsigned a = 0, p = 0, cw = 0;
#pragma unroll
            for (int j = 0; j < 8; ++j) {
                const unsigned dq = pk_sub(B1, e2[(s & 1) * 8 + j]), ix = pk_min(dq, 0x00070007u) | 0x0c000c00u;
                const unsigned len = __builtin_amdgcn_perm(0x05040201u, 0x00000003u, ix) + pk_subs(dq, 0x00070007u);
                cw |= __builtin_amdgcn_perm(0u, 0x03020100u, ix) << (2 * j);
                p += len & 0xffffu; a |= 1u << (p & 31u);
                p += len >> 16;     a |= 1u << (p & 31u);
            }
            seg[s] = a >> 1; pos[s] = p; PK[24 + (s >> 2)][s & 3] = cw;
        }
        if (max(pos[2 * g], pos[2 * g + 1]) > 31u) {
            u32x4 E[2];
#pragma unroll
            for (int v = 0; v < 8; ++v) E[v >> 2][v & 3] = __builtin_amdgcn_perm(e2[2 * v + 1], e2[2 * v], 0x06040200u);
            store16_sys(pb + (C_EXC + g * 2048u), lane * 32u, E[0]);
            store16_sys(pb + (C_EXC + g * 2048u + 16u), lane * 32u, E[1]);
            seg[2 * g] = seg[2 * g + 1] = pos[2 * g] = pos[2 * g + 1] = 0u; hdr |= 256u << g;
        }
    }
    const unsigned tot = (pos[0] + pos[1] + pos[2] + pos[3]) + (pos[4] + pos[5] + pos[6] + pos[7]);
    const unsigned incl = wave_scan(tot), total = (unsigned)__builtin_amdgcn_readlane((int)incl, 63);
    *(LAS u32x4*)(scr + 128u + lane * 16u) = (u32x4){0u, 0u, 0u, 0u}; *(LAS u32x4*)(scr + 1152u + lane * 16u) = (u32x4){0u, 0u, 0u, 0u};
    *(LAS unsigned short*)(scr + lane * 2u) = (unsigned short)(hdr | tot);
    unsigned off = incl - tot;
#pragma unroll
    for (int s = 0; s < 8; ++s) {
        const unsigned long long x = (unsigned long long)seg[s] << (off & 31u);
        LAS unsigned* q = (LAS unsigned*)(scr + 128u + (off >> 5) * 4u);
        __hip_atomic_fetch_or(q, (unsigned)x, __ATOMIC_RELAXED, __HIP_MEMORY_SCOPE_WORKGROUP);
        __hip_atomic_fetch_or(q + 1, (unsigned)(x >> 32), __ATOMIC_RELAXED, __HIP_MEMORY_SCOPE_WORKGROUP);
        off += pos[s];
    }
    return 128u + ((total + 7u) >> 3);
}
__device__ __forceinline__ size_t out_off(int q) { return (size_t)((q >> 4) * 128 + ((q >> 2) & 3) * 16 + (q & 3)) * (GN * 4); }
struct CDec { unsigned win[9], a, used, hdr, B1, e2[64]; };
__device__ __forceinline__ void c_begin(CDec& d, const LAS unsigned char* scr, unsigned B) {
    const unsigned tot = d.hdr & 255u, off = wave_scan(tot) - tot;
#pragma unroll
    for (int i = 0; i < 9; ++i) d.win[i] = *(const LAS unsigned*)(scr + 128u + (off >> 5) * 4u + i * 4u);
#pragma unroll
    for (int i = 0; i < 8; ++i) d.win[i] = __builtin_amdgcn_alignbit(d.win[i + 1], d.win[i], off);
    d.B1 = (B + 1u) * 0x00010001u;
}
__device__ __forceinline__ void c_parse(CDec& d, const u32x4 (&PK)[26], int H) {
    const int s = H >> 1;
    const unsigned cw = PK[24 + (s >> 2)][s & 3], nz = cw | (cw >> 1);
    if (!(H & 1)) { d.a = d.win[0] << 1; d.used = 0; }
#pragma unroll
    for (int j = (H & 1) * 4; j < (H & 1) * 4 + 4; ++j) {
        const unsigned l0 = ffbl((d.a & ~1u) | ((nz >> (2 * j)) & 1u));
        d.a >>= l0;
        const unsigned l1 = ffbl((d.a & ~1u) | ((nz >> (2 * j + 16)) & 1u));
        d.a >>= l1;
        const unsigned len = l0 | (l1 << 16);
        d.used += len;
        const unsigned dq = __builtin_amdgcn_perm(0x00000006u, 0x00050400u, pk_min(len, 0x00040004u) | 0x0c000c00u) + pk_subs(len, 0x00040004u);
        d.e2[s * 8 + j] = pk_sub(d.B1, dq + ((cw >> (2 * j)) & 0x00030003u));
    }
    asm volatile("" : "+v"(d.e2[4 * H]), "+v"(d.e2[4 * H + 1]), "+v"(d.e2[4 * H + 2]), "+v"(d.e2[4 * H + 3]));
    if (H & 1) {
        const unsigned sh = (d.hdr & (256u << (s >> 1))) ? 0u : d.used + (d.used >> 16);
#pragma unroll
        for (int i = 0; i < 7 - s; ++i) d.win[i] = __builtin_amdgcn_alignbit(d.win[i + 1], d.win[i], sh);
    }
}
__device__ __forceinline__ void c_store(CDec& d, const u32x4 (&PK)[26], const u32x4 (&X)[8], bool bad, const gchar* ob, unsigned ovo) {
#pragma unroll
    for (int g = 0; g < 4; ++g) {
        unsigned* e2 = d.e2 + 16 * g;
        if (d.hdr & (256u << g)) {
#pragma unroll
            for (int j = 0; j < 16; ++j) e2[j] = __builtin_amdgcn_perm(0u, X[2 * g + (j >> 3)][(j >> 1) & 3], (j & 1) ? 0x0c030c02u : 0x0c010c00u);
        }
#pragma unroll
        for (int v = 0; v < 8; ++v) {
            unsigned P[3];
#pragma unroll
            for (int i = 0; i < 3; ++i) P[i] = PK[g * 6 + ((3 * v + i) >> 2)][(3 * v + i) & 3];
            u32x4 r;
            r.x = __builtin_amdgcn_perm(e2[2 * v], P[0], 0x04020100u); r.y = __builtin_amdgcn_perm(e2[2 * v], P[1], 0x06020100u); r.z = __builtin_amdgcn_perm(e2[2 * v + 1], P[2], 0x04020100u);
            r.w = __builtin_amdgcn_perm(P[1], P[0], 0x0c0c0703u) | __builtin_amdgcn_perm(e2[2 * v + 1], P[2], 0x06030c0cu);
#pragma unroll
            for (int c = 0; c < 4; ++c) r[c] = bad ? 0x7fc00000u : __builtin_amdgcn_alignbit(r[c], r[c], 1);
            store16_sys(ob + out_off(g * 8 + v), ovo, r);
        }
    }
}

__device__ __forceinline__ void item_decode(int i, int d, int& isW, int& j, int& u) {
    constexpr int R0 = 14 + NE, R1 = 18 - NE;
    if (i < 8 * R0) {
        j = i / R0; const int r = i - j * R0;
        isW = r >= 7 && r < 7 + NE; u = isW ? r - 7 : r < 7 ? 2 * r : 2 * (r - 7 - NE) + 1; if (!isW && u >= 2 * d) u += 2;
    } else { const int r = i - 8 * R0; j = r / R1; const int t = r - j * R1; isW = t < 16 - NE; u = isW ? NE + t : 2 * d + t - (16 - NE); }
}
constexpr int NTASK = 1152, NTASK_DYN0 = 264, NTASK_DYN = 96 + NTASK - NTASK_DYN0;
__device__ __forceinline__ void task_decode(int t, int& isW, int& j, int& u, int& dd) {
    constexpr int R0 = 16 + 8 * NE, R1 = 8 * (16 - NE);
    if (t < 8 * R0) { j = t / R0; int r = t - j * R0; isW = r >= 16; if (isW) { r -= 16; u = r >> 3; dd = r & 7; } else { u = r < 8 ? 2 * r : 2 * r - 15; dd = 7; } }
    else { int r = t - 8 * R0; j = r / R1; r -= j * R1; isW = 1; u = NE + (r >> 3); dd = r & 7; }
}

__device__ __forceinline__ void task_load(const Params& p, int rank, int i, f32x4 (&L)[16]) {
    const bool valid = i < NTASK;
    int isW, j, u, dd; task_decode(valid ? i : 0, isW, j, u, dd);
    const unsigned t = threadIdx.x;
    const gchar* b = uni(isW ? (const char*)p.w + ((size_t)(j * 64) * GN + ((rank + 1 + dd) & 7) * NL + u * 256) * 4 : (const char*)p.x + ((size_t)(u * 256) * KS + j * 64) * 4);
    unsigned vo = isW ? (t >> 6) * (4u * GN * 4u) + (t & 63u) * 16u : (t >> 3) * (KS * 4u) + ((t & 7u) ^ ((t >> 4) & 7u)) * 16u;
    if (!valid) vo = 0;
    const unsigned s0 = isW ? GN * 4u : 128u, s1 = isW ? 2u * GN * 4u : 64u * KS * 4u, s2 = isW ? 32u * GN * 4u : 128u * KS * 4u, s3 = isW ? 16u * GN * 4u : 32u * KS * 4u;
#pragma unroll
    for (int n = 0; n < 16; ++n)
        L[n] = *(const GAS f32x4*)(b + ((n & 1) * s0 + ((n >> 1) & 1) * s1 + ((n >> 2) & 1) * s2 + (n >> 3) * s3) + vo);
}

__device__ __forceinline__ unsigned pk2(float a, float b, int e) {
    typedef _Float16 h2 __attribute__((ext_vector_type(2)));
    h2 h; h.x = (_Float16)__builtin_amdgcn_ldexpf(a, e); h.y = (_Float16)__builtin_amdgcn_ldexpf(b, e);
    return __builtin_bit_cast(unsigned, h);
}

#define POLL_SLEEP 8
#define FLAG_SLEEP 16
#define PUSH_REL 1u
#define SEND_PRIO 3
#define OUT_WAVES 2
#define WIDE_FROM 248u
#define OUT_W2 7
#define OUT_W 3
#define REL_AT 22
#define LINK_EXTRA 8u
#define PULL_N 7
#define RAW_TASKS 8
#define WARM 3
#define START_NOW 64
#define START_DELAY 2
#define START_SPREAD 12
#define STR2(x) #x
#define STR(x) STR2(x)
#define LBAR do { asm volatile("s_waitcnt lgkmcnt(0)" ::: "memory"); __builtin_amdgcn_s_barrier(); asm volatile("" ::: "memory"); } while (0)

typedef volatile LAS unsigned vl32;
typedef volatile LAS u32x4 vl128;
#define LGKM0 asm volatile("s_waitcnt lgkmcnt(0)" ::: "memory")
__device__ __forceinline__ u32x4 lds_wait4(LAS unsigned char* lds, unsigned off, unsigned v) {
    for (;;) {
        const u32x4 q = *(vl128*)(lds + off);
        if (rfl(min(min(q.x, q.y), min(q.z, q.w))) >= v) { asm volatile("" ::: "memory"); return q; }
        __builtin_amdgcn_s_sleep(1);
    }
}
__device__ __forceinline__ unsigned lds_wait1(LAS unsigned char* lds, unsigned off, unsigned v) {
    unsigned x;
    while ((x = rfl(*(vl32*)(lds + off))) < v) __builtin_amdgcn_s_sleep(1);
    asm volatile("" ::: "memory");
    return x;
}
__device__ __forceinline__ void task_stats(LAS unsigned char* lds, const f32x4 (&L)[16], unsigned k) {
    unsigned mu = 0; float sm = 0.f;
#pragma unroll
    for (int n = 0; n < 16; ++n) {
        const u32x4 q = __builtin_bit_cast(u32x4, L[n]) & 0x7fffffffu; mu = max(max(mu, q.x), max(q.y, max(q.z, q.w)));
        const f32x4 f = __builtin_bit_cast(f32x4, q); sm += (f.x + f.y) + (f.z + f.w);
    }
    const unsigned mb = wave_max(mu >> 23), em = wave_mean_exp(sm * (2.32f / 4096.f));
    LGKM0;
    if ((threadIdx.x & 63) == 63) *(vl32*)(lds + L_AMAX + (k & 1u) * 16 + (threadIdx.x >> 6) * 4) = ((k + 1u) << 16) | (em << 8) | mb;
}

__device__ __forceinline__ unsigned img_off(int isW, unsigned q, int c) {
    return (isW ? q * 2048u + (c >> 2) * 8192u + (c & 1) * 1024u : q * 1024u + (c >> 2) * 4096u + (c & 1) * 8192u) + ((c >> 1) & 1) * HTB;
}
__device__ __forceinline__ int conv_step(const Params& p, LAS unsigned char* lds, gu32* tick, int rank, unsigned k, f32x4 (&L)[16], const f32x4 (&Ln)[16]) {
    unsigned t = threadIdx.x; asm volatile("" : "+v"(t));
    const u32x4 am = lds_wait4(lds, L_AMAX + (k & 1u) * 16, (k + 1u) << 16);
    const int i = (int)rfl(*(vl32*)(lds + L_ITEM + (k & 15u) * 4)), i1 = (int)rfl(*(vl32*)(lds + L_ITEM + ((k + 1u) & 15u) * 4)), i2 = (int)rfl(*(vl32*)(lds + L_ITEM + ((k + 2u) & 15u) * 4));
    const bool valid = i < NTASK;
    if (valid) {
        const unsigned mb = rfl(max(max(am.x & 255u, am.y & 255u), max(am.z & 255u, am.w & 255u))), em = (rfl(max(max(am.x, am.y), max(am.z, am.w))) >> 8) & 255u;
        const int cls = (int)(mb + 2u) >> 4;
        const int e = 128 - 16 * cls;
        int isW, j, u, dd; task_decode(i, isW, j, u, dd);
        u32x4 H[8];
#pragma unroll
        for (int h = 0; h < 2; ++h) {
            const f32x4* Q = L + 8 * h; u32x4* G = H + 4 * h;
            if (isW) {
#pragma unroll
                for (int c = 0; c < 4; ++c) { G[c].x = pk2(Q[0][c], Q[1][c], e); G[c].y = pk2(Q[2][c], Q[3][c], e); G[c].z = pk2(Q[4][c], Q[5][c], e); G[c].w = pk2(Q[6][c], Q[7][c], e); }
            } else {
#pragma unroll
                for (int c = 0; c < 4; ++c) { G[c].x = pk2(Q[2 * c].x, Q[2 * c].y, e); G[c].y = pk2(Q[2 * c].z, Q[2 * c].w, e); G[c].z = pk2(Q[2 * c + 1].x, Q[2 * c + 1].y, e); G[c].w = pk2(Q[2 * c + 1].z, Q[2 * c + 1].w, e); }
            }
        }
        const unsigned q = rfl(t >> 6);
        if (k >= 4) lds_wait1(lds, L_CONS + q * 4, k - 3u);
#pragma unroll
        for (int c = 0; c < 8; ++c) *(LAS u32x4*)(lds + (k & 3u) * UNIT + q * Q_BYTES + c * 1024u + (t & 63u) * 16u) = H[c];
        LGKM0;
        if ((t & 63u) == 0) *(vl32*)(lds + L_QI + (k & 3u) * 16 + q * 4) = ((k + 1u) << 16) | ((unsigned)min(max((int)min(em, mb) + e - 112, 2), 31) << 5) | (unsigned)cls;
    }
    unsigned tk = NTASK_DYN - 1;
    gu32* q = tick; asm volatile("" : "+v"(q));
    if (t == 0 && tick && valid) tk = __hip_atomic_fetch_add(q, 1u, __ATOMIC_RELAXED, __HIP_MEMORY_SCOPE_AGENT);
    task_load(p, rank, i2, L);
    __builtin_amdgcn_s_waitcnt(0x4F70);
    asm volatile("" : "+v"(tk));
    if (t == 0) *(vl32*)(lds + L_ITEM + ((k + 3u) & 15u) * 4) = (unsigned)NTASK_DYN0 + tk % (unsigned)NTASK_DYN;
    task_stats(lds, Ln, k + 1u);
    return i1;
}

__device__ __forceinline__ gu32* stage_unit(gchar* myws, int srel, int isW, int j, int u, const gchar*& ub) {
    const int idx = j * 8 + ((srel + 1) & 7);
    ub = myws + WS_STAGE + (size_t)(idx * 32 + isW * 16 + u) * UNIT;
    return (gu32*)(myws + WS_FLAG) + (isW * 16 + u) * NKT + idx;
}
__device__ __forceinline__ void lds_fetch8(const LAS unsigned char* src, unsigned lane, u32x4 (&H)[8]) {
#pragma unroll
    for (int c = 0; c < 8; ++c) H[c] = *(const LAS u32x4*)(src + c * 1024u + lane * 16u);
}

__device__ __forceinline__ unsigned produce(const Params& p, LAS unsigned char* lds, gchar* myws, int rank, int first, int second, gu32* tick) {
    unsigned call;
    if (rfl(threadIdx.x >> 6) < 4) {
        f32x4 RA[16], RB[16];
        if (first >= START_NOW && first < NTASK) for (int z = 0; z < START_DELAY + (first - START_NOW) * START_SPREAD / 128; ++z) __builtin_amdgcn_s_sleep(32);
        if (first < NTASK) task_load(p, rank, first, RA);
        LBAR;
        if (first < NTASK) {
            task_stats(lds, RA, 0);
            task_load(p, rank, second, RB);
            int nx; unsigned k = 0;
            do { conv_step(p, lds, tick, rank, k, RA, RB); nx = conv_step(p, lds, tick, rank, k + 1u, RB, RA); k += 2; } while (nx < NTASK);
        }
        LBAR;
        call = rfl(*(vl32*)(lds + L_CALL));
    } else {
        const unsigned ts = threadIdx.x - 256u, lane = ts & 63u, q = rfl(ts >> 6);
        constexpr unsigned I0 = (L_ITEM - L_CTL) / 4;
        if (ts < (L_ZERO_END - L_CTL) / 4) *(vl32*)(lds + L_CTL + ts * 4) = ts == I0 ? (unsigned)first : ts == I0 + 1 ? (unsigned)second : ts == I0 + 2 ? (unsigned)second + 32u : 0u;
        unsigned tk = 0;
        gu32* tq = (gu32*)(myws + WS_TICKET); asm volatile("" : "+v"(tq));
        if (ts == 0) tk = __hip_atomic_fetch_add(tq, 1u, __ATOMIC_RELAXED, __HIP_MEMORY_SCOPE_AGENT);
        const gchar* peer = (const gchar*)p.ws[(rank + 1 + min(lane, 6u)) & 7];
        LBAR;
        asm volatile("" : "+v"(tk));
        if (ts == 0) *(vl32*)(lds + L_CALL) = (tk / GRID + 1u) & 0xffffffu;
        call = lds_wait1(lds, L_CALL, 1u);
        for (unsigned k = 0;; ++k) {
            const int i = (int)rfl(*(vl32*)(lds + L_ITEM + (k & 15u) * 4));
            if (i >= NTASK) break;
            const unsigned st = lds_wait1(lds, L_QI + (k & 3u) * 16 + q * 4, (k + 1u) << 16);
            int isW, j, u, dd; task_decode(i, isW, j, u, dd);
            LAS unsigned char* src = lds + (k & 3u) * UNIT + q * Q_BYTES;
            u32x4 H[8];
            lds_fetch8(src, lane, H);
            const gchar* ub; gu32* sf = stage_unit(myws, 7, isW, j, u, ub);
            const bool peers = !isW || dd != 7, rawt = i < RAW_TASKS;
            const gchar* pb = uni(myws + pin_off(call & 1u, isW, min(dd, 6), j, u) + q * Q_BYTES);
            if (dd == 7 && !rawt) {
                ub = uni(ub);
#pragma unroll
                for (int c = 0; c < 8; ++c) store16_sys(ub + img_off(isW, q, c), lane * 16u, H[c]);
            }
            unsigned size = Q_BYTES;
            if (peers) {
                if (!rawt) { size = q_encode(H, (st >> 5) & 31u, src, lane); asm volatile("" ::: "memory"); lds_fetch8(src, lane, H); }
#pragma unroll
                for (int c = 0; c < 8; ++c) if (c < 6 || c * 1024u + lane * 16u < size) store16_sys(pb + c * 1024u, lane * 16u, H[c]);
            }
            LGKM0;
            if (lane == 0) *(vl32*)(lds + L_CONS + q * 4) = k + 1u;
            asm volatile("s_waitcnt vmcnt(0)" ::: "memory");
            const unsigned add = (1u << 30) | (((size - Q_LEN + 63u) >> 6) << (10u + 5u * q));
            unsigned old = 0;
            if (lane == 0) old = __hip_atomic_fetch_add((LAS unsigned*)(lds + L_ECNT + (k & 7u) * 4), add, __ATOMIC_RELAXED, __HIP_MEMORY_SCOPE_WORKGROUP);
            old = rfl(old);
            if ((old >> 30) == 3u) {
                if (lane == 0) *(vl32*)(lds + L_ECNT + (k & 7u) * 4) = 0u;
                if (peers && (isW ? lane == (unsigned)dd : lane < 7u))
                    __hip_atomic_store((GAS unsigned long long*)(peer + WS_AFLAG) + ((((call & 1u) * 8 + (6u - lane)) * 2 + isW) * 128 + j * 16 + u),
                                       ((unsigned long long)call << 32) | (old + add) | (st & 1023u), __ATOMIC_RELAXED, __HIP_MEMORY_SCOPE_SYSTEM);
                if (dd == 7 && !rawt && lane == 0) __hip_atomic_store(sf, (call << 8) | (st & 31u), __ATOMIC_RELAXED, __HIP_MEMORY_SCOPE_SYSTEM);
            }
            if (dd == 7 && rawt) {
                ub = uni(ub);
#pragma unroll
                for (int c = 0; c < 8; ++c) store16_sys(ub + img_off(isW, q, c), lane * 16u, H[c]);
                asm volatile("s_waitcnt vmcnt(0)" ::: "memory");
                if (lane == 0 && (__hip_atomic_fetch_add((LAS unsigned*)(lds + L_UCNT), 1u, __ATOMIC_RELAXED, __HIP_MEMORY_SCOPE_WORKGROUP) & 3u) == 3u)
                    __hip_atomic_store(sf, (call << 8) | (st & 31u), __ATOMIC_RELAXED, __HIP_MEMORY_SCOPE_SYSTEM);
            }
        }
        LBAR;
    }
    return call;
}

__device__ __forceinline__ void fetch_inputs(const Params& p, LAS unsigned char* lds, gchar* myws, int rank, unsigned call, int srel, int first) {
    const unsigned lane = threadIdx.x & 63u, wv = rfl(threadIdx.x >> 6);
    if (wv == 1) {
        gu32* tmo = (gu32*)(myws + WS_TMO);
        unsigned k = 0;
        for (int i = first;;) {
            unsigned info = 0; bool ok = true;
            if (i < 256) {
                int isW, j, u; item_decode(i, rank, isW, j, u);
                const GAS unsigned long long* af = (const GAS unsigned long long*)(myws + WS_AFLAG) + ((((call & 1u) * 8 + srel) * 2 + isW) * 128 + j * 16 + u);
                const unsigned long long t0 = __builtin_amdgcn_s_memrealtime();
                for (;;) {
                    const unsigned long long fv = __hip_atomic_load(af, __ATOMIC_RELAXED, __HIP_MEMORY_SCOPE_SYSTEM);
                    info = rfl((unsigned)fv);
                    if (rfl((unsigned)(fv >> 32)) >= call) break;
                    if (rfl(ld_agt(tmo)) == call || __builtin_amdgcn_s_memrealtime() - t0 > DEADLINE) {
                        if (lane == 0) __hip_atomic_store(tmo, call, __ATOMIC_RELAXED, __HIP_MEMORY_SCOPE_AGENT);
                        ok = false; break;
                    }
                    __builtin_amdgcn_s_sleep(2);
                }
            }
            if (ok) { if (lane == 0) *(vl128*)(lds + L_DESC + (k & 7u) * 16) = (u32x4){k + 1u, (unsigned)i, info, 0u}; ++k; }
            if (i >= 256) break;
            if (ok) lds_wait1(lds, L_LST, k);
            unsigned tk = 0;
            if (lane == 0) tk = __hip_atomic_fetch_add((gu32*)(myws + WS_LINKQ + srel * 128), 1u, __ATOMIC_RELAXED, __HIP_MEMORY_SCOPE_AGENT);
            i = 8 + (int)(rfl(tk) & 255u);
        }
    } else if (wv == 0) {
        const gchar* sws = uni(p.ws[(rank + 1 + srel) & 7]);
        for (int z = 0; z < WARM; ++z) __builtin_amdgcn_s_sleep(16);
        { const unsigned v = ld_sys((const gu32*)(sws + WS_AFLAG)); asm volatile("" :: "v"(v)); }
        __builtin_amdgcn_s_setprio(SEND_PRIO);
#define LWAIT(off, v, n) do { if (rfl(*(vl32*)(lds + (off))) < (v)) { asm volatile("s_waitcnt vmcnt(0)" ::: "memory"); if (lane == 0) *(vl32*)(lds + L_LAND) = (n); lds_wait1(lds, off, v); } } while (0)
        unsigned k = 0;
        for (;; ++k) {
            LWAIT(L_DESC + (k & 7u) * 16, k + 1u, 4u * k);
            const u32x4 de = *(vl128*)(lds + L_DESC + (k & 7u) * 16);
            const int i = (int)rfl(de.y); const unsigned info = rfl(de.z);
            if (i >= 256) break;
            if (lane == 0) *(vl32*)(lds + L_LST) = k + 1u;
            int isW, j, u; item_decode(i, rank, isW, j, u);
            const gchar* ub = uni(sws + pin_off(call & 1u, isW, 6 - srel, j, u));
#pragma unroll
            for (unsigned q = 0; q < 4; ++q) {
                if (k >= 4) LWAIT(L_FREE + ((k & 3u) * 4 + q) * 4, k - 3u, 4u * k + q);
                const unsigned size = Q_LEN + ((info >> (10u + 5u * q)) & 31u) * 64u, dst = (unsigned)(uintptr_t)lds + (k & 3u) * UNIT + q * Q_BYTES;
#pragma unroll
                for (int c = 0; c < 8; ++c) {
                    if (c < 6 || c * 1024u + lane * 16u < size) glds16_sys(dst + c * 1024u, lane * 16u, ub + (q * Q_BYTES + c * 1024u));
                    asm volatile("s_waitcnt vmcnt(" STR(PULL_N) ")" ::: "memory");
                    if (c == PULL_N - 1 && lane == 0) *(vl32*)(lds + L_LAND) = 4u * k + q;
                }
            }
        }
        if (lane == 0) __hip_atomic_fetch_add((gu32*)(myws + WS_OREL + srel * 128), PUSH_REL, __ATOMIC_RELAXED, __HIP_MEMORY_SCOPE_AGENT);
        asm volatile("s_waitcnt vmcnt(0)" ::: "memory");
        if (lane == 0) *(vl32*)(lds + L_LAND) = 4u * k;
        __builtin_amdgcn_s_setprio(0);
    } else {
        for (unsigned n = wv - 2u;; n += 6u) {
            const unsigned k = n >> 2, q = n & 3u;
            unsigned kd = k;
            for (;;) {
                const u32x4 d0 = *(vl128*)(lds + L_DESC + (k & 7u) * 16), d1 = *(vl128*)(lds + L_DESC + ((k + 7u) & 7u) * 16);
                if (rfl(d0.x) >= k + 1u) break;
                if (k > 0 && rfl(d1.x) == k && rfl(d1.y) >= 256u) { kd = k - 1u; break; }
                __builtin_amdgcn_s_sleep(1);
            }
            const u32x4 de = *(vl128*)(lds + L_DESC + (kd & 7u) * 16);
            const int i = (int)rfl(de.y); const unsigned info = rfl(de.z);
            if (i >= 256) break;
            lds_wait1(lds, L_LAND, n + 1u);
            const bool raw = ((info >> (10u + 5u * q)) & 31u) == 24u;
            QIn Q;
            q_read(lds + (k & 3u) * UNIT + q * Q_BYTES, lane, raw, Q);
            LGKM0;
            if (lane == 0) *(vl32*)(lds + L_FREE + ((k & 3u) * 4 + q) * 4) = k + 1u;
            u32x4 W[8];
            if (!raw) q_finish((info >> 5) & 31u, Q, W);
            q_patch(raw, Q, W);
            int isW, j, u; item_decode(i, rank, isW, j, u);
            const gchar* ub; gu32* f = stage_unit(myws, srel, isW, j, u, ub); ub = uni(ub);
#pragma unroll
            for (int c = 0; c < 8; ++c) store16_sys(ub + img_off(isW, q, c), lane * 16u, W[c]);
            asm volatile("s_waitcnt vmcnt(0)" ::: "memory");
            if (lane == 0) {
                const unsigned old = __hip_atomic_fetch_add((LAS unsigned*)(lds + L_UCNT + (k & 7u) * 4), 1u, __ATOMIC_RELAXED, __HIP_MEMORY_SCOPE_WORKGROUP);
                if ((old & 3u) == 3u) __hip_atomic_store(f, (call << 8) | (info & 31u), __ATOMIC_RELAXED, __HIP_MEMORY_SCOPE_SYSTEM);
            }
        }
    }
    LBAR;
}

__global__ void __launch_bounds__(NTHR, 2) gemm_rs_kernel(const Params p) {
    extern __shared__ __attribute__((aligned(16))) unsigned char smem[];
    LAS unsigned char* lds = (LAS unsigned char*)smem;
    const int tid = threadIdx.x;
    const int rank = (int)rfl(p.rank), bid = (int)rfl(blockIdx.x);
    gchar* myws = uni(p.ws[rank]);
    STAMP(0);
    const int xcd = bid & 7, slot = bid >> 3;
    gu32* tmo = (gu32*)(myws + WS_TMO);

    int pm, pn;
    {
        int ar = -1;
        if (slot < 11) {
            const int c = slot < 7 ? xcd * 7 + slot : xcd * 4 + slot - 7, o = slot < 7 ? c - 28 : c;
            if (c < 28) { ar = c >> 1; pn = (slot < 7 ? 14 : 12) + (c & 1); } else { pm = 2 * rank + (o & 1); pn = o >> 1; }
        } else {
            const int li = slot - 11, q = li / 7;
            pn = 3 * (xcd & 3) + q; ar = 2 * (li - q * 7) + (xcd >> 2);
        }
        if (ar >= 0) pm = ar + (ar >= 2 * rank ? 2 : 0);
    }
    const bool encoder = slot >= 7 && slot < 11;
    const unsigned call = produce(p, lds, myws, rank, slot < 7 ? NTASK : encoder ? bid - 24 : slot < 15 ? bid - 88 : bid - 56, encoder ? 144 + bid : NTASK, encoder ? (gu32*)(myws + WS_TASKQ) : nullptr);
    STAMP(5);
    if (slot < 7) fetch_inputs(p, lds, myws, rank, call, slot, xcd);
    STAMP(1);

    const int wid = rfl(tid >> 6), lane = tid & 63, wr = wid >> 2, wc = wid & 3, fr = lane & 15, fq = lane >> 4;
    const gchar* cA = myws + WS_STAGE + (size_t)pm * UNIT;
    const gchar* cB = myws + WS_STAGE + (size_t)(16 + pn) * UNIT;
    const gu32* fA = (const gu32*)(myws + WS_FLAG) + pm * NKT;
    const gu32* fB = (const gu32*)(myws + WS_FLAG) + (16 + pn) * NKT;
    const unsigned voff[2] = {(unsigned)tid * 16u, (unsigned)tid * 16u + 8192u}; const unsigned ldsw = (unsigned)wid * 1024u;
    const unsigned aoff0 = (unsigned)((wr * 64 + fr) * 128 + ((fq ^ (fr >> 1)) * 16)), aoff1 = aoff0 ^ 64u;
    const unsigned boff = (unsigned)(fq * 2048 + (wc * 16 + fr) * 16);
    int ready = 0; unsigned long long chg = 0; int scur = 0, smax = 0;

#define SA(b, h) (((b) * 2 + (h)) * HTB)
#define SB(b, h) ((4 + (b) * 2 + (h)) * HTB)
#define STAGE(bufoff, gbase) do { _Pragma("unroll") for (int _i = 0; _i < 2; ++_i) \
        __builtin_amdgcn_global_load_lds((const gu32*)((gbase) + voff[_i]), (LAS unsigned*)(lds + (bufoff) + ldsw + _i * 8192), 16, 0, GLDS_AUX); } while (0)
#define LDA(dst, b, h) do { _Pragma("unroll") for (int m = 0; m < 4; ++m) { dst[m][0] = *(const LAS h16x8*)(lds + SA(b, h) + aoff0 + m * 2048); dst[m][1] = *(const LAS h16x8*)(lds + SA(b, h) + aoff1 + m * 2048); } } while (0)
#define LDB(dst, b, h) do { _Pragma("unroll") for (int n = 0; n < 2; ++n) _Pragma("unroll") for (int k = 0; k < 2; ++k) dst[n][k] = *(const LAS h16x8*)(lds + SB(b, h) + boff + n * 1024 + k * 8192); } while (0)
#define MMA(ai, bj, At, Bt) do { __builtin_amdgcn_s_setprio(1); _Pragma("unroll") for (int m = 0; m < 4; ++m) _Pragma("unroll") for (int n = 0; n < 2; ++n) _Pragma("unroll") for (int k = 0; k < 2; ++k) \
        acc[ai][bj][m][n] = __builtin_amdgcn_mfma_f32_16x16x32_f16(At[m][k], Bt[n][k], acc[ai][bj][m][n], 0, 0, 0); __builtin_amdgcn_s_setprio(0); } while (0)
#define WAIT_V(n) asm volatile("s_waitcnt vmcnt(" #n ")" ::: "memory")
#define WAIT_L(n) asm volatile("s_waitcnt lgkmcnt(" #n ")" ::: "memory")
#define BAR __builtin_amdgcn_s_barrier()
#define SCHED __builtin_amdgcn_sched_barrier(0)
#define HOOK(need) do { if (ready < (need)) { \
        if (wid == 0) { \
            const unsigned long long t0 = __builtin_amdgcn_s_memrealtime(); unsigned fa, fb; int rd; \
            unsigned ln = voff[0]; asm volatile("" : "+v"(ln)); ln >>= 4;     \
            for (;;) { \
                fa = ld_sys(fA + ln); fb = ld_sys(fB + ln); \
                const unsigned long long okm = __ballot((fa >> 8) >= call && (fb >> 8) >= call); \
                rd = (okm == ~0ull) ? 64 : (int)__builtin_ctzll(~okm); \
                if (rd >= (need)) break; \
                if (rfl(ld_agt(tmo)) == call || __builtin_amdgcn_s_memrealtime() - t0 > DEADLINE) { \
                    if (ln == 0) __hip_atomic_store(tmo, call, __ATOMIC_RELAXED, __HIP_MEMORY_SCOPE_AGENT); \
                    rd = 64; break; } \
                __builtin_amdgcn_s_sleep(POLL_SLEEP); \
            } \
            const int sv = (int)((fa & 31u) + (fb & 31u)); \
            const int sp = __builtin_amdgcn_update_dpp(-1, sv, 0x138, 0xf, 0xf, false);     \
            const unsigned long long cg = __ballot(sv != sp); \
            *(LAS int*)(lds + L_TAB + ln * 4) = sv; \
            if (ln == 0) { u32x4 h; h.x = (unsigned)rd; h.y = (unsigned)cg; h.z = (unsigned)(cg >> 32); h.w = 0; *(LAS u32x4*)(lds + L_HDR) = h; } \
            DIAG_WAIT(t0); \
            asm volatile("s_waitcnt vmcnt(0) lgkmcnt(0)" ::: "memory"); \
        } \
        asm volatile("" ::: "memory"); BAR; asm volatile("" ::: "memory"); \
        { const u32x4 h = *(const LAS u32x4*)(lds + L_HDR); ready = (int)rfl(h.x); chg = ((unsigned long long)rfl(h.z) << 32) | rfl(h.y); } \
    } } while (0)
#define DIAG_WAIT(t0) do {} while (0)
#define PKMUL4(op) ".set rs_i, 0\n\t.set rs_d, 0\n\t.irpc c, \"" op "\"\n\t.ifc \\c,:\n\t.set rs_d, 1\n\t.endif\n\t" \
        ".if rs_d == 0\n\t.ifnc \\c,v\n\t.ifnc \\c,[\n\t.set rs_i, rs_i*10 + \\c\n\t.endif\n\t.endif\n\t.endif\n\t.endr\n\t" \
        "v_pk_mul_f32 v[rs_i:rs_i+1], v[rs_i:rs_i+1], %16\n\tv_pk_mul_f32 v[rs_i+2:rs_i+3], v[rs_i+2:rs_i+3], %16\n\t"
#define RS16(a) asm volatile("s_cmp_eq_u32 %17, 0\n\ts_cbranch_scc1 .Lrs%=\n\ts_nop 15\n\ts_nop 3\n\t" \
            PKMUL4("%0") PKMUL4("%1") PKMUL4("%2") PKMUL4("%3") PKMUL4("%4") PKMUL4("%5") PKMUL4("%6") PKMUL4("%7") \
            PKMUL4("%8") PKMUL4("%9") PKMUL4("%10") PKMUL4("%11") PKMUL4("%12") PKMUL4("%13") PKMUL4("%14") PKMUL4("%15") \
            "s_nop 4\n.Lrs%=:" \
            : "+v"(acc[a][0][0][0]), "+v"(acc[a][0][0][1]), "+v"(acc[a][0][1][0]), "+v"(acc[a][0][1][1]), "+v"(acc[a][0][2][0]), "+v"(acc[a][0][2][1]), "+v"(acc[a][0][3][0]), "+v"(acc[a][0][3][1]), \
              "+v"(acc[a][1][0][0]), "+v"(acc[a][1][0][1]), "+v"(acc[a][1][1][0]), "+v"(acc[a][1][1][1]), "+v"(acc[a][1][2][0]), "+v"(acc[a][1][2][1]), "+v"(acc[a][1][3][0]), "+v"(acc[a][1][3][1]) \
            : "s"(fac), "s"(de) : "scc")
#define RESCALE(kt) do { int de = 0; \
        if ((chg >> (kt)) & 1ull) { \
            const int s = (int)rfl(*(const LAS unsigned*)(lds + L_TAB + (kt) * 4)); \
            smax = max(smax, s); const int sn = max(s, smax - 4); de = max(16 * (scur - sn), -126); scur = sn; } \
        const unsigned long long fac = (unsigned long long)(unsigned)((de + 127) << 23) * 0x100000001ull; \
        RS16(0); RS16(1); \
    } while (0)
#define TRIP do { \
        const int t2 = (t + 2) & (NKT - 1);             \
        const gchar* a1 = cA + (size_t)(t + 1) * KT_BYTES; \
        const gchar* a2 = cA + (size_t)t2 * KT_BYTES; const gchar* b2 = cB + (size_t)t2 * KT_BYTES; \
        const gchar* a3 = a2 + KT_BYTES; const gchar* b3 = b2 + KT_BYTES; \
        RESCALE(t); \
        LDB(B0, 0, 0); SCHED; LDA(At, 0, 0); STAGE(SA(1, 1), a1 + HTB); \
        WAIT_L(8); BAR; WAIT_L(0); MMA(0, 0, At, B0); BAR; SCHED; \
        LDB(B1, 0, 1); STAGE(SB(0, 0), b2); \
        BAR; WAIT_L(0); MMA(0, 1, At, B1); BAR; \
        LDA(At, 0, 1); STAGE(SA(0, 0), a2); \
        BAR; WAIT_L(0); MMA(1, 0, At, B0); BAR; SCHED; \
        STAGE(SB(0, 1), b2 + HTB); \
        WAIT_V(6); BAR; MMA(1, 1, At, B1); BAR; \
        RESCALE(t + 1); \
        LDB(B0, 1, 0); SCHED; LDA(At, 1, 0); STAGE(SA(0, 1), a2 + HTB); \
        WAIT_L(8); BAR; WAIT_L(0); MMA(0, 0, At, B0); BAR; SCHED; \
        LDB(B1, 1, 1); STAGE(SB(1, 0), b3); \
        BAR; WAIT_L(0); MMA(0, 1, At, B1); BAR; \
        LDA(At, 1, 1); STAGE(SA(1, 0), a3); \
        BAR; WAIT_L(0); MMA(1, 0, At, B0); BAR; SCHED; \
        STAGE(SB(1, 1), b3 + HTB); \
        WAIT_V(6); BAR; MMA(1, 1, At, B1); BAR; \
    } while (0)

    f32x4 acc[2][2][4][2];
#pragma unroll
    for (int a = 0; a < 2; ++a)
#pragma unroll
        for (int b = 0; b < 2; ++b)
#pragma unroll
            for (int m = 0; m < 4; ++m)
#pragma unroll
                for (int n = 0; n < 2; ++n) acc[a][b][m][n] = (f32x4){0.f, 0.f, 0.f, 0.f};
    h16x8 At[4][2], B0[2][2], B1[2][2];

    HOOK(2);
    STAMP(2);
    STAGE(SB(0, 0), cB); STAGE(SA(0, 0), cA); STAGE(SB(0, 1), cB + HTB); STAGE(SA(0, 1), cA + HTB);
    if (wr == 1) BAR;
    WAIT_V(4); BAR;
    STAGE(SB(1, 0), cB + KT_BYTES); STAGE(SA(1, 0), cA + KT_BYTES); STAGE(SB(1, 1), cB + KT_BYTES + HTB);
    WAIT_V(6); BAR;
    for (int t = 0; t < NKT; t += 2) {
        HOOK(min(t + 4, NKT));
        TRIP;
    }
    if (wr == 0) BAR;
    STAMP(3);

    const int dest = pm >> 1;
    const unsigned ovo = (unsigned)(wr * 64 + 4 * fq) * (GN * 4u) + (unsigned)(wc * 16 + fr) * 16u;
    const float qn = __builtin_nanf("");
    u32x4 V[32];
    {
        const bool bad = rfl(ld_agt(tmo)) == call;
        const int oe = 16 * scur - 256;
#pragma unroll
        for (int a = 0; a < 2; ++a)
#pragma unroll
            for (int m = 0; m < 4; ++m)
#pragma unroll
                for (int r = 0; r < 4; ++r) {
                    f32x4 v;
                    v.x = __builtin_amdgcn_ldexpf(acc[a][0][m][0][r], oe); v.y = __builtin_amdgcn_ldexpf(acc[a][0][m][1][r], oe);
                    v.z = __builtin_amdgcn_ldexpf(acc[a][1][m][0][r], oe); v.w = __builtin_amdgcn_ldexpf(acc[a][1][m][1][r], oe);
                    if (bad) v = (f32x4){qn, qn, qn, qn};
                    V[(a * 4 + m) * 4 + r] = __builtin_bit_cast(u32x4, v);
                }
    }
#define OSTAMP(k) do {} while (0)
#define DIAG_END do {} while (0)
    gchar* dws = uni(p.ws[dest]);
    if (dest == rank) {
        const gchar* ob = uni((const char*)p.out[rank] + ((size_t)((pm & 1) * 256) * GN + rank * NL + pn * 256) * 4);
#pragma unroll
        for (int q = 0; q < 32; ++q) { store16_sys(ob + out_off(q), ovo, V[q]); WAIT_V(4); }
    }
    int packed = dest - rank;
    asm volatile("" : "+v"(packed)); packed = (int)rfl(packed);
    if (packed) {
        const unsigned cidx = (unsigned)(((pm & 1) * 16 + pn) * 8 + wid);
        const gchar* pb = uni(myws + WS_PACK + (size_t)(dest * 256 + cidx) * CHUNK);
        unsigned mu = 0; float sm = 0.f;
#pragma unroll
        for (int q = 0; q < 32; ++q) {
            const u32x4 w = V[q] & 0x7fffffffu; mu = max(max(mu, w.x), max(w.y, max(w.z, w.w)));
            const f32x4 f = __builtin_bit_cast(f32x4, w); sm += (f.x + f.y) + (f.z + f.w);
        }
        const unsigned myB = min(max(min(wave_mean_exp(sm * (2.32f / 8192.f)), wave_max(mu >> 23)), 2u), 254u);
        u32x4 PK[26];
        LAS unsigned char* scr = lds + L_SCR + wid * 3072;
        const unsigned tail = c_encode(V, myB, PK, scr, pb, lane);
        asm volatile("" ::: "memory");
#pragma unroll
        for (int q = 0; q < 3; ++q) if (q * 1024u + lane * 16u < tail) store16_sys(pb + (C_HDR + q * 1024u), lane * 16u, *(const LAS u32x4*)(scr + q * 1024u + lane * 16u));
#pragma unroll
        for (int q = 0; q < 26; ++q) { store16_sys(pb + q * 1024u, lane * 16u, PK[q]); WAIT_V(4); }
        WAIT_V(0);
        if (lane == 0) __hip_atomic_store((GAS unsigned long long*)(dws + WS_CFLAG) + rank * 256 + cidx, ((unsigned long long)call << 32) | (tail << 8) | myB, __ATOMIC_RELAXED, __HIP_MEMORY_SCOPE_SYSTEM);
        OSTAMP(31);
        if (pn < NE && wid < OUT_WAVES) {
            const int link = (dest - rank - 1) & 7;
            gu32* rel = (gu32*)(myws + WS_OREL + link * 128);
            for (;;) {
                unsigned ct = 0;
                if (lane == 0) ct = __hip_atomic_fetch_add((gu32*)(myws + WS_OTK + link * 128 + 64), 1u, __ATOMIC_RELAXED, __HIP_MEMORY_SCOPE_AGENT);
                ct = rfl(ct) % (256u + 2u * NE * OUT_WAVES);
                if (ct >= 256u) break;
                const unsigned T = ct >> 3, cw_ = ct & 7u, crow = T < 2u * NE ? T / NE : T & 1u, ccol = T < 2u * NE ? T % NE : T >> 1, ci = (crow * 16u + ccol) * 8u + cw_;
                const GAS unsigned long long* cf = (const GAS unsigned long long*)(myws + WS_CFLAG) + dest * 256 + ci;
                unsigned fv;
                unsigned long long t0 = __builtin_amdgcn_s_memrealtime();
                for (;;) {
                    const unsigned long long f2 = __hip_atomic_load(cf, __ATOMIC_RELAXED, __HIP_MEMORY_SCOPE_SYSTEM);
                    fv = rfl((unsigned)f2);
                    if (rfl((unsigned)(f2 >> 32)) >= call) break;
                    if (rfl(ld_agt(tmo)) == call || __builtin_amdgcn_s_memrealtime() - t0 > DEADLINE) {
                        if (lane == 0) __hip_atomic_store(tmo, call, __ATOMIC_RELAXED, __HIP_MEMORY_SCOPE_AGENT);
                        break;
                    }
                    __builtin_amdgcn_s_sleep(FLAG_SLEEP);
                }
                const bool bad = rfl(ld_agt(tmo)) == call;
                unsigned tk = 0;
                if (lane == 0) tk = __hip_atomic_fetch_add((gu32*)(myws + WS_OTK + link * 128), 1u, __ATOMIC_RELAXED, __HIP_MEMORY_SCOPE_AGENT);
                tk = rfl(tk) & 255u;
                const bool wide = tk >= WIDE_FROM;
                const unsigned need = (call - 1u) * (256u + 8u * PUSH_REL) + tk + 1u - LINK_EXTRA;
                t0 = __builtin_amdgcn_s_memrealtime();
                for (;;) {
                    const int ahead = (int)(need - rfl(ld_agt(rel)));
                    if (ahead <= 0 || __builtin_amdgcn_s_memrealtime() - t0 > DEADLINE) break;
                    if (ahead > 4) __builtin_amdgcn_s_sleep(32); else __builtin_amdgcn_s_sleep(4);
                }
                __builtin_amdgcn_s_setprio(SEND_PRIO);
                asm volatile("" ::: "memory");
                const gchar* rb = uni(dws + WS_PACK + (size_t)(rank * 256 + ci) * CHUNK);
                const __amdgpu_buffer_rsrc_t rs = __builtin_amdgcn_make_buffer_rsrc((void*)rb, 0, bad ? 0 : CHUNK, 0x00020000);
                const unsigned rtail = bad ? 0u : min(fv >> 8, 3072u);
#pragma unroll
                for (int q = 0; q < 3; ++q) if (q * 1024u + lane * 16u < rtail) glds16_sys((unsigned)(uintptr_t)scr + q * 1024u, lane * 16u, rb + (C_HDR + q * 1024u));
                u32x4 X[8]; CDec D;
#pragma unroll
                for (int n = 0; n < 26; ++n) {
                    const int q = n < 2 ? 24 + n : n - 2;
                    PK[q] = RLOAD4(rs, lane * 16u, q * 1024);
                    if (n >= 10) { SCHED; c_parse(D, PK, n - 10); SCHED; }
                    if (wide) asm volatile("s_waitcnt vmcnt(" STR(OUT_W2) ")" ::: "memory"); else asm volatile("s_waitcnt vmcnt(" STR(OUT_W) ")" ::: "memory");
                    if (n == REL_AT && lane == 0) __hip_atomic_fetch_add(rel, 1u, __ATOMIC_RELAXED, __HIP_MEMORY_SCOPE_AGENT);
                    if (n == 8) {
                        D.hdr = *(volatile LAS unsigned short*)(scr + lane * 2u);
#pragma unroll
                        for (int g = 0; g < 4; ++g) if (D.hdr & (256u << g)) { X[2 * g] = RLOAD4(rs, lane * 32u, C_EXC + g * 2048u); X[2 * g + 1] = RLOAD4(rs, lane * 32u + 16u, C_EXC + g * 2048u); }
                        c_begin(D, scr, fv & 255u);
                    }
                }
                __builtin_amdgcn_s_waitcnt(0x0F70);
                __builtin_amdgcn_s_setprio(0);
                c_store(D, PK, X, bad, uni((const char*)p.out[rank] + ((size_t)(crow * 256) * GN + dest * NL + ccol * 256) * 4),
                        ((cw_ >> 2) * 64u + 4u * fq) * (GN * 4u) + ((cw_ & 3u) * 16u + fr) * 16u);
            }
        }
    }
    DIAG_END;
}

extern "C" void kernel_launch(int rank, int nranks, ncclComm_t comm,
                              hipStream_t stream,
                              void* const* d_in, const int* in_sizes, int n_in,
                              void* d_out, int out_size,
                              const int* in_dtypes, int out_dtype,
                              void* const* d_out_all,
                              void* const* d_ws_all, size_t ws_bytes) {
    (void)comm; (void)in_dtypes; (void)out_dtype; (void)d_out;
    if (nranks != NR || (unsigned)rank >= (unsigned)NR || n_in < 2 || ws_bytes < WS_NEED ||
        in_sizes[0] != GM * KS || in_sizes[1] != KS * GN || out_size != (GM / NR) * GN) {
        fprintf(stderr, "gemm_rs: unexpected problem shape\n");
        return;
    }
    HIP_CHECK(hipFuncSetAttribute((const void*)gemm_rs_kernel, hipFuncAttributeMaxDynamicSharedMemorySize, (int)LDS_BYTES));
    Params p;
    p.x = (const float*)d_in[0]; p.w = (const float*)d_in[1];
    for (int i = 0; i < NR; ++i) { p.out[i] = (float*)d_out_all[i]; p.ws[i] = (char*)d_ws_all[i]; }
    p.rank = rank; p.pad = 0;
    hipLaunchKernelGGL(gemm_rs_kernel, dim3(GRID), dim3(NTHR), LDS_BYTES, stream, p);
    HIP_CHECK(hipGetLastError());
}
```

```cpp
#include <hip/hip_runtime.h>
#include <rccl/rccl.h>
#include <cstdio>
#include <cstdint>

#define HIP_CHECK(expr) do { hipError_t _e = (expr); if (_e != hipSuccess) { \
    fprintf(stderr, "HIP error %s at %s:%d\n", hipGetErrorString(_e), __FILE__, __LINE__); \
    return; } } while (0)

#define LAS __attribute__((address_space(3)))
#define GAS __attribute__((address_space(1)))
typedef GAS char gchar;
typedef GAS unsigned gu32;
typedef _Float16 h16x8 __attribute__((ext_vector_type(8)));
typedef float f32x4 __attribute__((ext_vector_type(4)));
typedef float f32x2 __attribute__((ext_vector_type(2)));
typedef unsigned u32x4 __attribute__((ext_vector_type(4)));

constexpr int NR = 8;
constexpr int GM = 4096, GN = 32768, KS = 512, NL = 4096;
constexpr int GRID = 256, NTHR = 512;
constexpr unsigned UNIT = 32768;
constexpr unsigned HTB = 16384;
constexpr int NKT = 64;
constexpr size_t WS_TICKET = 0, WS_TMO = 128, WS_TASKQ = 256, WS_LINKQ = 1024, WS_OTK = 2048, WS_OREL = 3072;
constexpr size_t WS_FLAG = 4096;
constexpr size_t WS_CFLAG = 16384;
constexpr size_t WS_AFLAG = 32768;
constexpr size_t WS_IFLAG = 12288;
constexpr size_t WS_INBOX = 65536;
constexpr size_t WS_PX = WS_INBOX + (size_t)224 * UNIT;
constexpr size_t WS_PW = WS_PX + (size_t)256 * UNIT;
constexpr size_t WS_STAGE = WS_PW + (size_t)14 * 128 * UNIT;
constexpr unsigned KT_BYTES = 32 * UNIT;
constexpr unsigned CHUNK = 37888;
constexpr size_t WS_PACK = WS_STAGE + (size_t)NKT * KT_BYTES;
constexpr size_t WS_NEED = WS_PACK + (size_t)NR * 256 * CHUNK;
constexpr size_t WS_DIAG = 65536, WS_DBASE = WS_NEED;
__device__ __forceinline__ size_t pin_off(unsigned par, int isW, int dd, int j, int u) {
    return isW ? WS_PW + (size_t)((par * 7 + dd) * 128 + j * 16 + u) * UNIT : WS_PX + (size_t)(par * 128 + j * 16 + u) * UNIT;
}
#define NEARLY 11
constexpr int NE = NEARLY;
constexpr unsigned long long DEADLINE = 100000000ull;
constexpr unsigned L_CTL = 8 * HTB;
constexpr unsigned L_AMAX = L_CTL, L_CONS = L_CTL + 32, L_TURN = L_CTL + 48, L_CALL = L_CTL + 52, L_LAND = L_CTL + 56, L_LST = L_CTL + 60, L_ITEM = L_CTL + 64, L_QI = L_CTL + 128;
constexpr unsigned L_ECNT = L_CTL + 192, L_UCNT = L_CTL + 224, L_FREE = L_CTL + 256, L_DESC = L_CTL + 320, L_ZERO_END = L_CTL + 448;
constexpr unsigned L_HDR = L_CTL + 576, L_TAB = L_CTL + 640, L_SCR = L_CTL + 1024, LDS_BYTES = L_SCR + 8 * 3072;

#define GLDS_AUX 17
struct Params {
    const float* x; const float* w;
    float* out[NR]; char* ws[NR];
    int rank; int pad;
};

#define STAMP(i) do {} while (0)

__device__ __forceinline__ void store16_sys(const gchar* sbase, unsigned voff, u32x4 v) {
    asm volatile("global_store_dwordx4 %0, %1, %2 sc0 sc1\n\ts_nop 1" :: "v"(voff), "v"(v), "s"(sbase) : "memory");
}
__device__ __forceinline__ void glds16_sys(unsigned dst, unsigned voff, const gchar* sbase) {
    unsigned keep;
    asm volatile("s_mov_b32 %0, m0\n\ts_mov_b32 m0, %1\n\ts_nop 0\n\tglobal_load_lds_dwordx4 %2, %3 sc0 sc1\n\ts_mov_b32 m0, %0" : "=&s"(keep) : "s"(dst), "v"(voff), "s"(sbase) : "memory");
}
__device__ __forceinline__ unsigned ld_sys(const gu32* q) { return __hip_atomic_load(q, __ATOMIC_RELAXED, __HIP_MEMORY_SCOPE_SYSTEM); }
__device__ __forceinline__ unsigned ld_agt(const gu32* q) { return __hip_atomic_load(q, __ATOMIC_RELAXED, __HIP_MEMORY_SCOPE_AGENT); }
__device__ __forceinline__ unsigned rfl(unsigned v) { return (unsigned)__builtin_amdgcn_readfirstlane((int)v); }

template <class T> __device__ __forceinline__ gchar* uni(T q) {
    unsigned lo = rfl((unsigned)(uintptr_t)q), hi = rfl((unsigned)((uintptr_t)q >> 32));
    asm volatile("s_nop 4" : "+s"(lo), "+s"(hi));
    return (gchar*)(((uintptr_t)hi << 32) | lo);
}

__device__ __forceinline__ unsigned wave_max(unsigned x) {
    int mb = (int)x;
    mb = max(mb, __builtin_amdgcn_update_dpp(0, mb, 0x111, 0xf, 0xf, false));
    mb = max(mb, __builtin_amdgcn_update_dpp(0, mb, 0x112, 0xf, 0xf, false));
    mb = max(mb, __builtin_amdgcn_update_dpp(0, mb, 0x114, 0xf, 0xf, false));
    mb = max(mb, __builtin_amdgcn_update_dpp(0, mb, 0x118, 0xf, 0xf, false));
    mb = max(mb, __builtin_amdgcn_update_dpp(0, mb, 0x142, 0xa, 0xf, false));
    mb = max(mb, __builtin_amdgcn_update_dpp(0, mb, 0x143, 0xc, 0xf, false));
    return (unsigned)__builtin_amdgcn_readlane(mb, 63);
}
__device__ __forceinline__ unsigned wave_mean_exp(float sm) {
#define DPP_ADD(ctl, rm) sm += __builtin_bit_cast(float, __builtin_amdgcn_update_dpp(0, __builtin_bit_cast(int, sm), ctl, rm, 0xf, false))
    DPP_ADD(0x111, 0xf); DPP_ADD(0x112, 0xf); DPP_ADD(0x114, 0xf); DPP_ADD(0x118, 0xf); DPP_ADD(0x142, 0xa); DPP_ADD(0x143, 0xc);
    return ((unsigned)__builtin_amdgcn_readlane(__builtin_bit_cast(int, sm), 63) >> 23) & 255u;
}

typedef unsigned short u16x2 __attribute__((ext_vector_type(2)));
typedef unsigned u32x2 __attribute__((ext_vector_type(2)));
constexpr unsigned Q_P6 = 6144, Q_LEN = 6656, Q_STR = 6720, Q_BYTES = 8192;
__device__ __forceinline__ u16x2 as16(unsigned a) { return __builtin_bit_cast(u16x2, a); }
__device__ __forceinline__ unsigned as32(u16x2 a) { return __builtin_bit_cast(unsigned, a); }
__device__ __forceinline__ unsigned pk_sub(unsigned a, unsigned b) { return as32(as16(a) - as16(b)); }
__device__ __forceinline__ unsigned pk_subs(unsigned a, unsigned b) { return as32(__builtin_elementwise_sub_sat(as16(a), as16(b))); }
__device__ __forceinline__ unsigned pk_min(unsigned a, unsigned b) { return as32(__builtin_elementwise_min(as16(a), as16(b))); }
__device__ __forceinline__ unsigned ffbl(unsigned a) { unsigned r; asm("v_ffbl_b32 %0, %1" : "=v"(r) : "v"(a)); return r; }
__device__ __forceinline__ unsigned wave_scan(unsigned x) {
    int v = (int)x;
    v += __builtin_amdgcn_update_dpp(0, v, 0x111, 0xf, 0xf, false);
    v += __builtin_amdgcn_update_dpp(0, v, 0x112, 0xf, 0xf, false);
    v += __builtin_amdgcn_update_dpp(0, v, 0x114, 0xf, 0xf, false);
    v += __builtin_amdgcn_update_dpp(0, v, 0x118, 0xf, 0xf, false);
    v += __builtin_amdgcn_update_dpp(0, v, 0x142, 0xa, 0xf, false);
    v += __builtin_amdgcn_update_dpp(0, v, 0x143, 0xc, 0xf, false);
    return (unsigned)v;
}
__device__ __forceinline__ unsigned q_encode(const u32x4 (&W)[8], unsigned B, LAS unsigned char* dst, unsigned lane) {
    const unsigned B1 = (B + 1u) * 0x04000400u;
    unsigned v[32], seg[4], pos[4];
    u32x4 X[4];
#pragma unroll
    for (int s = 0; s < 4; ++s) {
        unsigned a = 0, p = 0;
        X[s].x = __builtin_amdgcn_perm(W[2 * s].y, W[2 * s].x, 0x07050301u); X[s].y = __builtin_amdgcn_perm(W[2 * s].w, W[2 * s].z, 0x07050301u);
        X[s].z = __builtin_amdgcn_perm(W[2 * s + 1].y, W[2 * s + 1].x, 0x07050301u); X[s].w = __builtin_amdgcn_perm(W[2 * s + 1].w, W[2 * s + 1].z, 0x07050301u);
#pragma unroll
        for (int k = 8 * s; k < 8 * s + 8; ++k) {
            const unsigned w = W[k >> 2][k & 3];
            const unsigned dq = as32(as16(pk_sub(B1, w & 0x7c007c00u)) >> 10);
            const unsigned ix = pk_min(dq, 0x00070007u) | 0x0c000c00u;
            const unsigned len = __builtin_amdgcn_perm(0x05040201u, 0x00000003u, ix) + pk_subs(dq, 0x00070007u);
            v[k] = (w & 0x83ff83ffu) | (__builtin_amdgcn_perm(0u, 0x03020100u, ix) << 10);
            p += len & 0xffffu; a |= 1u << (p & 31u);
            p += len >> 16;     a |= 1u << (p & 31u);
        }
        seg[s] = a >> 1; pos[s] = p;
        asm volatile("" : "+v"(v[8 * s]), "+v"(v[8 * s + 1]), "+v"(v[8 * s + 2]), "+v"(v[8 * s + 3]), "+v"(v[8 * s + 4]), "+v"(v[8 * s + 5]), "+v"(v[8 * s + 6]), "+v"(v[8 * s + 7]),
                     "+v"(X[s]), "+v"(seg[s]), "+v"(pos[s]));
    }
    const bool ovf = max(max(pos[0], pos[1]), max(pos[2], pos[3])) > 31u;
    const unsigned tot = ovf ? 0u : pos[0] + pos[1] + pos[2] + pos[3];
    const unsigned incl = wave_scan(tot), total = (unsigned)__builtin_amdgcn_readlane((int)incl, 63);
    const unsigned long long om = __ballot(ovf);
    const unsigned xb = Q_STR + (((total + 127u) >> 7) << 4), size = xb + 64u * (unsigned)__popcll(om);
    if (size > Q_BYTES - 64u) return Q_BYTES;
    *(LAS u32x4*)(dst + Q_STR + lane * 16u) = (u32x4){0u, 0u, 0u, 0u};
#pragma unroll
    for (int j = 0; j < 6; ++j)
#pragma unroll
        for (int i = 0; i < 4; ++i) v[4 * j + i] |= (v[26 + j] << (12 - 3 * i)) & 0x70007000u;
#pragma unroll
    for (int j = 0; j < 6; ++j) v[24 + j / 3] |= (v[26 + j] >> (1 + j % 3)) & (0x80008000u >> (1 + j % 3));
#pragma unroll
    for (int j = 0; j < 6; ++j) *(LAS u32x4*)(dst + j * 1024u + lane * 16u) = (u32x4){v[4 * j], v[4 * j + 1], v[4 * j + 2], v[4 * j + 3]};
    *(LAS u32x2*)(dst + Q_P6 + lane * 8u) = (u32x2){v[24], v[25]};
    const unsigned slot = __builtin_amdgcn_mbcnt_hi((unsigned)(om >> 32), __builtin_amdgcn_mbcnt_lo((unsigned)om, 0u));
    *(LAS unsigned char*)(dst + Q_LEN + lane) = (unsigned char)(ovf ? 128u + slot : tot);
    if (!ovf) {
        unsigned off = incl - tot;
#pragma unroll
        for (int s = 0; s < 4; ++s) {
            const unsigned long long x = (unsigned long long)seg[s] << (off & 31u);
            LAS unsigned* q = (LAS unsigned*)(dst + Q_STR + (off >> 5) * 4u);
            __hip_atomic_fetch_or(q, (unsigned)x, __ATOMIC_RELAXED, __HIP_MEMORY_SCOPE_WORKGROUP);
            __hip_atomic_fetch_or(q + 1, (unsigned)(x >> 32), __ATOMIC_RELAXED, __HIP_MEMORY_SCOPE_WORKGROUP);
            off += pos[s];
        }
    } else {
#pragma unroll
        for (int j = 0; j < 4; ++j) *(LAS u32x4*)(dst + xb + slot * 64u + j * 16u) = X[j];
    }
    return size;
}
#define RLOAD4(rs, vo, so) __builtin_bit_cast(u32x4, __builtin_amdgcn_raw_buffer_load_b128(rs, vo, so, 17))
struct QIn { u32x4 P[6]; u32x2 P6; unsigned lb, off, w4; u32x4 win, X[4]; };
__device__ __forceinline__ void q_read(const LAS unsigned char* src, unsigned lane, bool raw, QIn& q) {
    q.lb = raw ? 0u : *(const LAS unsigned char*)(src + Q_LEN + lane);
#pragma unroll
    for (int j = 0; j < 6; ++j) q.P[j] = *(const LAS u32x4*)(src + j * 1024u + lane * 16u);
    q.P6 = *(const LAS u32x2*)(src + Q_P6 + lane * 8u);
    const unsigned tot = q.lb >= 128u ? 0u : q.lb, incl = wave_scan(tot), total = (unsigned)__builtin_amdgcn_readlane((int)incl, 63);
    q.off = incl - tot;
    const LAS unsigned* w = (const LAS unsigned*)(src + Q_STR + (q.off >> 5) * 4u);
    q.win = (u32x4){w[0], w[1], w[2], w[3]}; q.w4 = w[4];
    if (raw || q.lb >= 128u) {
        const unsigned xo = raw ? Q_P6 + lane * 16u : min(Q_STR + (((total + 127u) >> 7) << 4) + (q.lb - 128u) * 64u, Q_BYTES - 64u), xs = raw ? 1024u : 16u;
        q.X[0] = *(const LAS u32x4*)(src + xo); q.X[1] = *(const LAS u32x4*)(src + xo + xs);
        q.X[2] = *(const LAS u32x4*)(src + xo + 32u); q.X[3] = *(const LAS u32x4*)(src + xo + 48u);
    }
}
__device__ __forceinline__ void q_finish(unsigned B, const QIn& q, u32x4 (&W)[8]) {
    unsigned v[32];
#pragma unroll
    for (int k = 0; k < 24; ++k) v[k] = q.P[k >> 2][k & 3];
    v[24] = q.P6.x; v[25] = q.P6.y;
#pragma unroll
    for (int j = 0; j < 6; ++j) {
        unsigned d = (v[24 + j / 3] << (1 + j % 3)) & 0x80008000u;
#pragma unroll
        for (int i = 0; i < 4; ++i) d |= (v[4 * j + i] & 0x70007000u) >> (12 - 3 * i);
        v[26 + j] = d;
    }
    u32x4 win;
    win.x = __builtin_amdgcn_alignbit(q.win.y, q.win.x, q.off); win.y = __builtin_amdgcn_alignbit(q.win.z, q.win.y, q.off);
    win.z = __builtin_amdgcn_alignbit(q.win.w, q.win.z, q.off); win.w = __builtin_amdgcn_alignbit(q.w4, q.win.w, q.off);
    const unsigned B1 = (B + 1u) * 0x04000400u;
#pragma unroll
    for (int s = 0; s < 4; ++s) {
        unsigned a = win.x << 1, used = 0;
#pragma unroll
        for (int k = 8 * s; k < 8 * s + 8; ++k) {
            const unsigned nz = v[k] | (v[k] >> 1);
            const unsigned l0 = ffbl((a & ~1u) | ((nz >> 10) & 1u));
            a >>= l0;
            const unsigned l1 = ffbl((a & ~1u) | ((nz >> 26) & 1u));
            a >>= l1;
            const unsigned len = l0 | (l1 << 16);
            used += len;
            const unsigned dq = __builtin_amdgcn_perm(0x00000006u, 0x00050400u, pk_min(len, 0x00040004u) | 0x0c000c00u) + pk_subs(len, 0x00040004u);
            W[k >> 2][k & 3] = (v[k] & 0x83ff83ffu) | (pk_sub(B1, (v[k] & 0x0c000c00u) + (dq << 10)) & 0x7c007c00u);
        }
        if (s < 3) {
            const unsigned sh = (used + (used >> 16)) & 31u;
            win.x = __builtin_amdgcn_alignbit(win.y, win.x, sh); win.y = __builtin_amdgcn_alignbit(win.z, win.y, sh);
            win.z = __builtin_amdgcn_alignbit(win.w, win.z, sh); win.w >>= sh;
        }
    }
}
__device__ __forceinline__ void q_patch(bool raw, const QIn& q, u32x4 (&W)[8]) {
    if (raw) {
#pragma unroll
        for (int j = 0; j < 6; ++j) W[j] = q.P[j];
        W[6] = q.X[0]; W[7] = q.X[1];
    } else if (q.lb >= 128u) {
#pragma unroll
        for (int k = 0; k < 32; ++k) W[k >> 2][k & 3] = __builtin_amdgcn_perm(q.X[k >> 3][(k >> 1) & 3], W[k >> 2][k & 3], (k & 1) ? 0x07020600u : 0x05020400u);
    }
}

constexpr unsigned C_HDR = 26624, C_STR = C_HDR + 128, C_EXC = 29696;
__device__ __forceinline__ unsigned c_encode(const u32x4 (&V)[32], unsigned B, u32x4 (&PK)[26], LAS unsigned char* scr, const gchar* pb, unsigned lane) {
    const unsigned B1 = (B + 1u) * 0x00010001u;
    unsigned seg[8], pos[8], hdr = 0;
#pragma unroll
    for (int g = 0; g < 4; ++g) {
        unsigned e2[16];
#pragma unroll
        for (int v = 0; v < 8; ++v) {
            u32x4 r;
#pragma unroll
            for (int c = 0; c < 4; ++c) r[c] = __builtin_amdgcn_alignbit(V[g * 8 + v][c], V[g * 8 + v][c], 31);
#pragma unroll
            for (int i = 0; i < 3; ++i) PK[g * 6 + ((3 * v + i) >> 2)][(3 * v + i) & 3] = __builtin_amdgcn_perm(r.w, r[i], 0x04020100u + ((unsigned)i << 24));
            e2[2 * v] = __builtin_amdgcn_perm(r.y, r.x, 0x0c070c03u); e2[2 * v + 1] = __builtin_amdgcn_perm(r.w, r.z, 0x0c070c03u);
        }
#pragma unroll
        for (int s = 2 * g; s < 2 * g + 2; ++s) {
            unsigned a = 0, p = 0, cw = 0;
#pragma unroll
            for (int j = 0; j < 8; ++j) {
                const unsigned dq = pk_sub(B1, e2[(s & 1) * 8 + j]), ix = pk_min(dq, 0x00070007u) | 0x0c000c00u;
                const unsigned len = __builtin_amdgcn_perm(0x05040201u, 0x00000003u, ix) + pk_subs(dq, 0x00070007u);
                cw |= __builtin_amdgcn_perm(0u, 0x03020100u, ix) << (2 * j);
                p += len & 0xffffu; a |= 1u << (p & 31u);
                p += len >> 16;     a |= 1u << (p & 31u);
            }
            seg[s] = a >> 1; pos[s] = p; PK[24 + (s >> 2)][s & 3] = cw;
        }
        if (max(pos[2 * g], pos[2 * g + 1]) > 31u) {
            u32x4 E[2];
#pragma unroll
            for (int v = 0; v < 8; ++v) E[v >> 2][v & 3] = __builtin_amdgcn_perm(e2[2 * v + 1], e2[2 * v], 0x06040200u);
            store16_sys(pb + (C_EXC + g * 2048u), lane * 32u, E[0]);
            store16_sys(pb + (C_EXC + g * 2048u + 16u), lane * 32u, E[1]);
            seg[2 * g] = seg[2 * g + 1] = pos[2 * g] = pos[2 * g + 1] = 0u; hdr |= 256u << g;
        }
    }
    const unsigned tot = (pos[0] + pos[1] + pos[2] + pos[3]) + (pos[4] + pos[5] + pos[6] + pos[7]);
    const unsigned incl = wave_scan(tot), total = (unsigned)__builtin_amdgcn_readlane((int)incl, 63);
    *(LAS u32x4*)(scr + 128u + lane * 16u) = (u32x4){0u, 0u, 0u, 0u}; *(LAS u32x4*)(scr + 1152u + lane * 16u) = (u32x4){0u, 0u, 0u, 0u};
    *(LAS unsigned short*)(scr + lane * 2u) = (unsigned short)(hdr | tot);
    unsigned off = incl - tot;
#pragma unroll
    for (int s = 0; s < 8; ++s) {
        const unsigned long long x = (unsigned long long)seg[s] << (off & 31u);
        LAS unsigned* q = (LAS unsigned*)(scr + 128u + (off >> 5) * 4u);
        __hip_atomic_fetch_or(q, (unsigned)x, __ATOMIC_RELAXED, __HIP_MEMORY_SCOPE_WORKGROUP);
        __hip_atomic_fetch_or(q + 1, (unsigned)(x >> 32), __ATOMIC_RELAXED, __HIP_MEMORY_SCOPE_WORKGROUP);
        off += pos[s];
    }
    return 128u + ((total + 7u) >> 3);
}
__device__ __forceinline__ size_t out_off(int q) { return (size_t)((q >> 4) * 128 + ((q >> 2) & 3) * 16 + (q & 3)) * (GN * 4); }
struct CDec { unsigned win[9], a, used, hdr, B1, e2[64]; };
__device__ __forceinline__ void c_begin(CDec& d, const LAS unsigned char* scr, unsigned B) {
    const unsigned tot = d.hdr & 255u, off = wave_scan(tot) - tot;
#pragma unroll
    for (int i = 0; i < 9; ++i) d.win[i] = *(const LAS unsigned*)(scr + 128u + (off >> 5) * 4u + i * 4u);
#pragma unroll
    for (int i = 0; i < 8; ++i) d.win[i] = __builtin_amdgcn_alignbit(d.win[i + 1], d.win[i], off);
    d.B1 = (B + 1u) * 0x00010001u;
}
__device__ __forceinline__ void c_parse(CDec& d, const u32x4 (&PK)[26], int H) {
    const int s = H >> 1;
    const unsigned cw = PK[24 + (s >> 2)][s & 3], nz = cw | (cw >> 1);
    if (!(H & 1)) { d.a = d.win[0] << 1; d.used = 0; }
#pragma unroll
    for (int j = (H & 1) * 4; j < (H & 1) * 4 + 4; ++j) {
        const unsigned l0 = ffbl((d.a & ~1u) | ((nz >> (2 * j)) & 1u));
        d.a >>= l0;
        const unsigned l1 = ffbl((d.a & ~1u) | ((nz >> (2 * j + 16)) & 1u));
        d.a >>= l1;
        const unsigned len = l0 | (l1 << 16);
        d.used += len;
        const unsigned dq = __builtin_amdgcn_perm(0x00000006u, 0x00050400u, pk_min(len, 0x00040004u) | 0x0c000c00u) + pk_subs(len, 0x00040004u);
        d.e2[s * 8 + j] = pk_sub(d.B1, dq + ((cw >> (2 * j)) & 0x00030003u));
    }
    asm volatile("" : "+v"(d.e2[4 * H]), "+v"(d.e2[4 * H + 1]), "+v"(d.e2[4 * H + 2]), "+v"(d.e2[4 * H + 3]));
    if (H & 1) {
        const unsigned sh = (d.hdr & (256u << (s >> 1))) ? 0u : d.used + (d.used >> 16);
#pragma unroll
        for (int i = 0; i < 7 - s; ++i) d.win[i] = __builtin_amdgcn_alignbit(d.win[i + 1], d.win[i], sh);
    }
}
__device__ __forceinline__ void c_store(CDec& d, const u32x4 (&PK)[26], const u32x4 (&X)[8], bool bad, const gchar* ob, unsigned ovo) {
#pragma unroll
    for (int g = 0; g < 4; ++g) {
        unsigned* e2 = d.e2 + 16 * g;
        if (d.hdr & (256u << g)) {
#pragma unroll
            for (int j = 0; j < 16; ++j) e2[j] = __builtin_amdgcn_perm(0u, X[2 * g + (j >> 3)][(j >> 1) & 3], (j & 1) ? 0x0c030c02u : 0x0c010c00u);
        }
#pragma unroll
        for (int v = 0; v < 8; ++v) {
            unsigned P[3];
#pragma unroll
            for (int i = 0; i < 3; ++i) P[i] = PK[g * 6 + ((3 * v + i) >> 2)][(3 * v + i) & 3];
            u32x4 r;
            r.x = __builtin_amdgcn_perm(e2[2 * v], P[0], 0x04020100u); r.y = __builtin_amdgcn_perm(e2[2 * v], P[1], 0x06020100u); r.z = __builtin_amdgcn_perm(e2[2 * v + 1], P[2], 0x04020100u);
            r.w = __builtin_amdgcn_perm(P[1], P[0], 0x0c0c0703u) | __builtin_amdgcn_perm(e2[2 * v + 1], P[2], 0x06030c0cu);
#pragma unroll
            for (int c = 0; c < 4; ++c) r[c] = bad ? 0x7fc00000u : __builtin_amdgcn_alignbit(r[c], r[c], 1);
            store16_sys(ob + out_off(g * 8 + v), ovo, r);
        }
    }
}

__device__ __forceinline__ void item_decode(int i, int d, int& isW, int& j, int& u) {
    constexpr int R0 = 14 + NE, R1 = 18 - NE;
    if (i < 8 * R0) {
        j = i / R0; int r = i - j * R0;
        if (i >= 8 && i < R0) r = i < 15 ? i - 1 + NE : i - 7;
        isW = r >= 7 && r < 7 + NE; u = isW ? r - 7 : r < 7 ? 2 * r : 2 * (r - 7 - NE) + 1; if (!isW && u >= 2 * d) u += 2;
    } else { const int r = i - 8 * R0; j = r / R1; const int t = r - j * R1; isW = t < 16 - NE; u = isW ? NE + t : 2 * d + t - (16 - NE); }
}
constexpr int NTASK = 1152, NTASK_DYN0 = 264, NTASK_DYN = 96 + NTASK - NTASK_DYN0;
__device__ __forceinline__ void task_decode(int t, int& isW, int& j, int& u, int& dd) {
    constexpr int R0 = 16 + 8 * NE, R1 = 8 * (16 - NE);
    if (t < 8 * R0) { j = t / R0; int r = t - j * R0; isW = r >= 16; if (isW) { r -= 16; u = r >> 3; dd = r & 7; } else { u = r < 8 ? 2 * r : 2 * r - 15; dd = 7; } }
    else { int r = t - 8 * R0; j = r / R1; r -= j * R1; isW = 1; u = NE + (r >> 3); dd = r & 7; }
}

__device__ __forceinline__ void task_load(const Params& p, int rank, int i, f32x4 (&L)[16]) {
    const bool valid = i < NTASK;
    int isW, j, u, dd; task_decode(valid ? i : 0, isW, j, u, dd);
    const unsigned t = threadIdx.x;
    const gchar* b = uni(isW ? (const char*)p.w + ((size_t)(j * 64) * GN + ((rank + 1 + dd) & 7) * NL + u * 256) * 4 : (const char*)p.x + ((size_t)(u * 256) * KS + j * 64) * 4);
    unsigned vo = isW ? (t >> 6) * (4u * GN * 4u) + (t & 63u) * 16u : (t >> 3) * (KS * 4u) + ((t & 7u) ^ ((t >> 4) & 7u)) * 16u;
    if (!valid) vo = 0;
    const unsigned s0 = isW ? GN * 4u : 128u, s1 = isW ? 2u * GN * 4u : 64u * KS * 4u, s2 = isW ? 32u * GN * 4u : 128u * KS * 4u, s3 = isW ? 16u * GN * 4u : 32u * KS * 4u;
#pragma unroll
    for (int n = 0; n < 16; ++n)
        L[n] = *(const GAS f32x4*)(b + ((n & 1) * s0 + ((n >> 1) & 1) * s1 + ((n >> 2) & 1) * s2 + (n >> 3) * s3) + vo);
}

__device__ __forceinline__ unsigned pk2(float a, float b, int e) {
    typedef _Float16 h2 __attribute__((ext_vector_type(2)));
    h2 h; h.x = (_Float16)__builtin_amdgcn_ldexpf(a, e); h.y = (_Float16)__builtin_amdgcn_ldexpf(b, e);
    return __builtin_bit_cast(unsigned, h);
}

#define POLL_SLEEP 8
#define FLAG_SLEEP 16
#define PUSH_REL 1u
#define SEND_PRIO 3
#define WMAJOR 1
#define STAGGER 6
#define OUT_WAVES 2
#define WIDE_FROM 236u
#define OUT_W2 3
#define OUT_W 2
#define REL_AT 24
#define LINK_EXTRA 12u
#define PULL_N 7
#define SKIP_PUSHED 1
#define NPUSH 1
#define PUSH_W 0
#define START_D1 0
#define START_NOW 112
#define START_DELAY 4
#define START_SPREAD 12
#define STR2(x) #x
#define STR(x) STR2(x)
#define LBAR do { asm volatile("s_waitcnt lgkmcnt(0)" ::: "memory"); __builtin_amdgcn_s_barrier(); asm volatile("" ::: "memory"); } while (0)

typedef volatile LAS unsigned vl32;
typedef volatile LAS u32x4 vl128;
#define LGKM0 asm volatile("s_waitcnt lgkmcnt(0)" ::: "memory")
__device__ __forceinline__ u32x4 lds_wait4(LAS unsigned char* lds, unsigned off, unsigned v) {
    for (;;) {
        const u32x4 q = *(vl128*)(lds + off);
        if (rfl(min(min(q.x, q.y), min(q.z, q.w))) >= v) { asm volatile("" ::: "memory"); return q; }
        __builtin_amdgcn_s_sleep(1);
    }
}
__device__ __forceinline__ unsigned lds_wait1(LAS unsigned char* lds, unsigned off, unsigned v) {
    unsigned x;
    while ((x = rfl(*(vl32*)(lds + off))) < v) __builtin_amdgcn_s_sleep(1);
    asm volatile("" ::: "memory");
    return x;
}
__device__ __forceinline__ void task_stats(LAS unsigned char* lds, const f32x4 (&L)[16], unsigned k) {
    unsigned mu = 0; float sm = 0.f;
#pragma unroll
    for (int n = 0; n < 16; ++n) {
        const u32x4 q = __builtin_bit_cast(u32x4, L[n]) & 0x7fffffffu; mu = max(max(mu, q.x), max(q.y, max(q.z, q.w)));
        const f32x4 f = __builtin_bit_cast(f32x4, q); sm += (f.x + f.y) + (f.z + f.w);
    }
    const unsigned mb = wave_max(mu >> 23), em = wave_mean_exp(sm * (2.32f / 4096.f));
    LGKM0;
    if ((threadIdx.x & 63) == 63) *(vl32*)(lds + L_AMAX + (k & 1u) * 16 + (threadIdx.x >> 6) * 4) = ((k + 1u) << 16) | (em << 8) | mb;
}

__device__ __forceinline__ unsigned img_off(int isW, unsigned q, int c) {
    return (isW ? q * 2048u + (c >> 2) * 8192u + (c & 1) * 1024u : q * 1024u + (c >> 2) * 4096u + (c & 1) * 8192u) + ((c >> 1) & 1) * HTB;
}
__device__ __forceinline__ int conv_step(const Params& p, LAS unsigned char* lds, gu32* tick, int rank, unsigned k, f32x4 (&L)[16], const f32x4 (&Ln)[16]) {
    unsigned t = threadIdx.x; asm volatile("" : "+v"(t));
    const u32x4 am = lds_wait4(lds, L_AMAX + (k & 1u) * 16, (k + 1u) << 16);
    const int i = (int)rfl(*(vl32*)(lds + L_ITEM + (k & 15u) * 4)), i1 = (int)rfl(*(vl32*)(lds + L_ITEM + ((k + 1u) & 15u) * 4)), i2 = (int)rfl(*(vl32*)(lds + L_ITEM + ((k + 2u) & 15u) * 4));
    const bool valid = i < NTASK;
    if (valid) {
        const unsigned mb = rfl(max(max(am.x & 255u, am.y & 255u), max(am.z & 255u, am.w & 255u))), em = (rfl(max(max(am.x, am.y), max(am.z, am.w))) >> 8) & 255u;
        const int cls = (int)(mb + 2u) >> 4;
        const int e = 128 - 16 * cls;
        int isW, j, u, dd; task_decode(i, isW, j, u, dd);
        u32x4 H[8];
#pragma unroll
        for (int h = 0; h < 2; ++h) {
            const f32x4* Q = L + 8 * h; u32x4* G = H + 4 * h;
            if (isW) {
#pragma unroll
                for (int c = 0; c < 4; ++c) { G[c].x = pk2(Q[0][c], Q[1][c], e); G[c].y = pk2(Q[2][c], Q[3][c], e); G[c].z = pk2(Q[4][c], Q[5][c], e); G[c].w = pk2(Q[6][c], Q[7][c], e); }
            } else {
#pragma unroll
                for (int c = 0; c < 4; ++c) { G[c].x = pk2(Q[2 * c].x, Q[2 * c].y, e); G[c].y = pk2(Q[2 * c].z, Q[2 * c].w, e); G[c].z = pk2(Q[2 * c + 1].x, Q[2 * c + 1].y, e); G[c].w = pk2(Q[2 * c + 1].z, Q[2 * c + 1].w, e); }
            }
        }
        const unsigned q = rfl(t >> 6);
        if (k >= 4) lds_wait1(lds, L_CONS + q * 4, k - 3u);
#pragma unroll
        for (int c = 0; c < 8; ++c) *(LAS u32x4*)(lds + (k & 3u) * UNIT + q * Q_BYTES + c * 1024u + (t & 63u) * 16u) = H[c];
        LGKM0;
        if ((t & 63u) == 0) *(vl32*)(lds + L_QI + (k & 3u) * 16 + q * 4) = ((k + 1u) << 16) | ((unsigned)min(max((int)min(em, mb) + e - 112, 2), 31) << 5) | (unsigned)cls;
    }
    unsigned tk = NTASK_DYN - 1;
    gu32* q = tick; asm volatile("" : "+v"(q));
    if (t == 0 && tick && valid) tk = __hip_atomic_fetch_add(q, 1u, __ATOMIC_RELAXED, __HIP_MEMORY_SCOPE_AGENT);
    task_load(p, rank, i2, L);
    __builtin_amdgcn_s_waitcnt(0x4F70);
    asm volatile("" : "+v"(tk));
    if (t == 0) *(vl32*)(lds + L_ITEM + ((k + 3u) & 15u) * 4) = (unsigned)NTASK_DYN0 + tk % (unsigned)NTASK_DYN;
    task_stats(lds, Ln, k + 1u);
    return i1;
}

__device__ __forceinline__ gu32* stage_unit(gchar* myws, int srel, int isW, int j, int u, const gchar*& ub) {
    const int idx = j * 8 + ((srel + 1) & 7);
    ub = myws + WS_STAGE + (size_t)(idx * 32 + isW * 16 + u) * UNIT;
    return (gu32*)(myws + WS_FLAG) + (isW * 16 + u) * NKT + idx;
}
__device__ __forceinline__ void lds_fetch8(const LAS unsigned char* src, unsigned lane, u32x4 (&H)[8]) {
#pragma unroll
    for (int c = 0; c < 8; ++c) H[c] = *(const LAS u32x4*)(src + c * 1024u + lane * 16u);
}

__device__ __forceinline__ unsigned produce(const Params& p, LAS unsigned char* lds, gchar* myws, int rank, int first, int second, gu32* tick) {
    unsigned call;
    if (rfl(threadIdx.x >> 6) < 4) {
        f32x4 RA[16], RB[16];
        const int dly = first >= NTASK ? 0 : first >= START_NOW ? START_DELAY + (first - START_NOW) * START_SPREAD / 128 : (first < 8 || first >= 32) ? START_D1 : 0;
        for (int z = 0; z < dly; ++z) __builtin_amdgcn_s_sleep(32);
        task_load(p, rank, first, RA);
        LBAR;
        task_stats(lds, RA, 0);
        task_load(p, rank, second, RB);
        int nx; unsigned k = 0;
        do { conv_step(p, lds, tick, rank, k, RA, RB); nx = conv_step(p, lds, tick, rank, k + 1u, RB, RA); k += 2; } while (nx < NTASK);
        LBAR;
        call = rfl(*(vl32*)(lds + L_CALL));
    } else {
        const unsigned ts = threadIdx.x - 256u, lane = ts & 63u, q = rfl(ts >> 6);
        constexpr unsigned I0 = (L_ITEM - L_CTL) / 4;
        if (ts < (L_ZERO_END - L_CTL) / 4) *(vl32*)(lds + L_CTL + ts * 4) = ts == I0 ? (unsigned)first : ts == I0 + 1 ? (unsigned)second : ts == I0 + 2 ? (unsigned)second + 32u : 0u;
        unsigned tk = 0;
        gu32* tq = (gu32*)(myws + WS_TICKET); asm volatile("" : "+v"(tq));
        if (ts == 0) tk = __hip_atomic_fetch_add(tq, 1u, __ATOMIC_RELAXED, __HIP_MEMORY_SCOPE_AGENT);
        const gchar* peer = (const gchar*)p.ws[(rank + 1 + min(lane, 6u)) & 7];
        LBAR;
        asm volatile("" : "+v"(tk));
        if (ts == 0) *(vl32*)(lds + L_CALL) = (tk / GRID + 1u) & 0xffffffu;
        call = lds_wait1(lds, L_CALL, 1u);
        for (unsigned k = 0;; ++k) {
            const int i = (int)rfl(*(vl32*)(lds + L_ITEM + (k & 15u) * 4));
            if (i >= NTASK) break;
            const unsigned st = lds_wait1(lds, L_QI + (k & 3u) * 16 + q * 4, (k + 1u) << 16);
            int isW, j, u, dd; task_decode(i, isW, j, u, dd);
            LAS unsigned char* src = lds + (k & 3u) * UNIT + q * Q_BYTES;
            u32x4 H[8];
            lds_fetch8(src, lane, H);
            const gchar* ub; gu32* sf = stage_unit(myws, 7, isW, j, u, ub);
            const bool own = dd == 7, peers = !isW || dd != 7;
            if (own) {
                ub = uni(ub);
#pragma unroll
                for (int c = 0; c < 8; ++c) store16_sys(ub + img_off(isW, q, c), lane * 16u, H[c]);
            }
            unsigned size = Q_BYTES;
            if (peers) {
                const gchar* pb = uni(myws + pin_off(call & 1u, isW, dd, j, u) + q * Q_BYTES);
                size = q_encode(H, (st >> 5) & 31u, src, lane); asm volatile("" ::: "memory"); lds_fetch8(src, lane, H);
#pragma unroll
                for (int c = 0; c < 8; ++c) if (c < 6 || c * 1024u + lane * 16u < size) store16_sys(pb + c * 1024u, lane * 16u, H[c]);
            }
            LGKM0;
            if (lane == 0) *(vl32*)(lds + L_CONS + q * 4) = k + 1u;
            asm volatile("s_waitcnt vmcnt(0)" ::: "memory");
            const unsigned add = (1u << 30) | (((size - Q_LEN + 63u) >> 6) << (10u + 5u * q));
            unsigned old = 0;
            if (lane == 0) old = __hip_atomic_fetch_add((LAS unsigned*)(lds + L_ECNT + (k & 7u) * 4), add, __ATOMIC_RELAXED, __HIP_MEMORY_SCOPE_WORKGROUP);
            old = rfl(old);
            if ((old >> 30) == 3u) {
                if (lane == 0) *(vl32*)(lds + L_ECNT + (k & 7u) * 4) = 0u;
                const unsigned fi = (call & 1u) * 8 + (6u - lane);
                if (peers && (isW ? lane == (unsigned)dd : lane < 7u))
                    __hip_atomic_store((GAS unsigned long long*)(peer + WS_AFLAG + ((fi * 2 + isW) * 128 + j * 16 + u) * 8),
                                       ((unsigned long long)call << 32) | (old + add) | (st & 1023u), __ATOMIC_RELAXED, __HIP_MEMORY_SCOPE_SYSTEM);
                if (own && lane == 0) __hip_atomic_store(sf, (call << 8) | (st & 31u), __ATOMIC_RELAXED, __HIP_MEMORY_SCOPE_SYSTEM);
            }
        }
        LBAR;
    }
    return call;
}

__device__ __forceinline__ unsigned push_first(const Params& p, LAS unsigned char* lds, gchar* myws, int rank, int link, int item) {
    const unsigned t = threadIdx.x;
    const int d = (rank + 1 + link) & 7;
    unsigned tk = 0;
    gu32* tq = (gu32*)(myws + WS_TICKET); asm volatile("" : "+v"(tq));
    if (t == 0) tk = __hip_atomic_fetch_add(tq, 1u, __ATOMIC_RELAXED, __HIP_MEMORY_SCOPE_AGENT);
    f32x4 L[NPUSH][8]; int isW[NPUSH];
#pragma unroll
    for (int r = 0; r < NPUSH; ++r) {
        int j, u; item_decode(item + 8 * r, d, isW[r], j, u);
        const gchar* b = uni(isW[r] ? (const char*)p.w + (size_t)(d * NL + u * 256) * 4 : (const char*)p.x + (size_t)(u * 256) * KS * 4);
        const unsigned vo = isW[r] ? (t >> 6) * (4u * GN * 4u) + (t & 63u) * 16u : (t >> 3) * (KS * 4u) + ((t & 7u) ^ ((t >> 4) & 7u)) * 16u;
        const unsigned s0 = isW[r] ? GN * 4u : 128u, s1 = isW[r] ? 2u * GN * 4u : 64u * KS * 4u, s2 = isW[r] ? 32u * GN * 4u : 128u * KS * 4u;
        if (r) __builtin_amdgcn_s_waitcnt(0x0F70);
#pragma unroll
        for (int n = 0; n < 8; ++n) L[r][n] = *(const GAS f32x4*)(b + ((n & 1) * s0 + ((n >> 1) & 1) * s1 + (n >> 2) * s2) + vo);
    }
    gchar* dws = uni(p.ws[d]);
    if (t >= 8 && t < (L_ZERO_END - L_CTL) / 4 && t != (L_CALL - L_CTL) / 4) *(vl32*)(lds + L_CTL + t * 4) = 0u;
    asm volatile("" : "+v"(tk));
    if (t == 0) *(vl32*)(lds + L_CALL) = (tk / GRID + 1u) & 0xffffffu;
    unsigned call = 0, cl[NPUSH];
#pragma unroll
    for (int r = 0; r < NPUSH; ++r) {
        unsigned mu = 0;
#pragma unroll
        for (int n = 0; n < 8; ++n) { const u32x4 q = __builtin_bit_cast(u32x4, L[r][n]) & 0x7fffffffu; mu = max(max(mu, q.x), max(q.y, max(q.z, q.w))); }
        mu = wave_max(mu >> 23);
        const unsigned am = r ? L_HDR : L_AMAX;
        if ((t & 63u) == 0) *(vl32*)(lds + am + (t >> 6) * 4) = mu;
        LBAR;
        const u32x4 a0 = *(vl128*)(lds + am), a1 = *(vl128*)(lds + am + 16);
        call = rfl(*(vl32*)(lds + L_CALL));
        cl[r] = (rfl(max(max(max(a0.x, a0.y), max(a0.z, a0.w)), max(max(a1.x, a1.y), max(a1.z, a1.w)))) + 2u) >> 4;
        const int e = 128 - 16 * (int)cl[r];
        const gchar* ib = uni(dws + WS_INBOX + (size_t)(((call & 1u) * 7 + (6 - link)) * 16 + item + 8 * r) * UNIT);
        const unsigned so = isW[r] ? (t >> 6) * 2048u + (t & 63u) * 16u : t * 16u, c1 = isW[r] ? 1024u : 8192u;
#pragma unroll
        for (int c = 0; c < 4; ++c) {
            const f32x4* Q = L[r];
            u32x4 H;
            if (isW[r]) { H.x = pk2(Q[0][c], Q[1][c], e); H.y = pk2(Q[2][c], Q[3][c], e); H.z = pk2(Q[4][c], Q[5][c], e); H.w = pk2(Q[6][c], Q[7][c], e); }
            else { H.x = pk2(Q[2 * c].x, Q[2 * c].y, e); H.y = pk2(Q[2 * c].z, Q[2 * c].w, e); H.z = pk2(Q[2 * c + 1].x, Q[2 * c + 1].y, e); H.w = pk2(Q[2 * c + 1].z, Q[2 * c + 1].w, e); }
            if ((r || c > 1) && (r < NPUSH - 1 || rfl(t >> 6) != 1u)) asm volatile("s_waitcnt vmcnt(" STR(PUSH_W) ")" ::: "memory");
            store16_sys(ib + ((c >> 1) * HTB + (c & 1) * c1), so, H);
        }
    }
    asm volatile("s_waitcnt vmcnt(0)" ::: "memory");
    unsigned old = 0;
    if ((t & 63u) == 0) old = __hip_atomic_fetch_add((LAS unsigned*)(lds + L_ECNT), 1u, __ATOMIC_RELAXED, __HIP_MEMORY_SCOPE_WORKGROUP);
    if (rfl(old) == 7u && (t & 63u) < NPUSH) {
        __hip_atomic_store((GAS unsigned long long*)(dws + WS_IFLAG) + ((call & 1u) * 8 + (6 - link)) * 16 + item + 8 * (t & 63u), ((unsigned long long)call << 32) | ((t & 63u) ? cl[NPUSH - 1] : cl[0]),
                           __ATOMIC_RELAXED, __HIP_MEMORY_SCOPE_SYSTEM);
    }
    return call;
}

template <int SLEEP> __device__ __forceinline__ bool flag_wait(const gchar* f, gu32* tmo, unsigned call, unsigned& info) {
    const unsigned long long t0 = __builtin_amdgcn_s_memrealtime();
    for (;;) {
        const unsigned long long fv = __hip_atomic_load((const GAS unsigned long long*)f, __ATOMIC_RELAXED, __HIP_MEMORY_SCOPE_SYSTEM);
        info = rfl((unsigned)fv);
        if (rfl((unsigned)(fv >> 32)) >= call) break;
        if (rfl(ld_agt(tmo)) == call || __builtin_amdgcn_s_memrealtime() - t0 > DEADLINE) {
            if ((threadIdx.x & 63u) == 0) __hip_atomic_store(tmo, call, __ATOMIC_RELAXED, __HIP_MEMORY_SCOPE_AGENT);
            return false;
        }
        __builtin_amdgcn_s_sleep(SLEEP);
    }
    asm volatile("" ::: "memory");
    return true;
}

__device__ __forceinline__ void fetch_inputs(const Params& p, LAS unsigned char* lds, gchar* myws, int rank, unsigned call, int srel, int first) {
    const unsigned lane = threadIdx.x & 63u, wv = rfl(threadIdx.x >> 6);
    gu32* tmo = (gu32*)(myws + WS_TMO);
    const unsigned fi = (call & 1u) * 8 + srel;
    if (wv == 1) {
        unsigned k = 0;
        for (;;) {
            unsigned tk = 0;
            if (lane == 0) tk = __hip_atomic_fetch_add((gu32*)(myws + WS_LINKQ + srel * 128), 1u, __ATOMIC_RELAXED, __HIP_MEMORY_SCOPE_AGENT);
            const int i = 8 * NPUSH + (int)(rfl(tk) % (264u - 8u * NPUSH));
            unsigned info = 0; bool ok = true;
            if (i < 256) { int isW, j, u; item_decode(i, rank, isW, j, u); ok = flag_wait<2>(myws + WS_AFLAG + ((fi * 2 + isW) * 128 + j * 16 + u) * 8, tmo, call, info); }
            if (ok) { if (lane == 0) *(vl128*)(lds + L_DESC + (k & 7u) * 16) = (u32x4){k + 1u, (unsigned)i, info, 0u}; ++k; }
            if (i >= 256) break;
            if (ok) lds_wait1(lds, L_LST, k);
        }
    } else if (wv == 0) {
        const gchar* sws = uni(p.ws[(rank + 1 + srel) & 7]);
        __builtin_amdgcn_s_setprio(SEND_PRIO);
#define LWAIT(off, v, n) do { if (rfl(*(vl32*)(lds + (off))) < (v)) { asm volatile("s_waitcnt vmcnt(0)" ::: "memory"); land = (n); if (lane == 0) *(vl32*)(lds + L_LAND) = land; lds_wait1(lds, off, v); } } while (0)
        unsigned k = 0, land = 0;
        for (;; ++k) {
            LWAIT(L_DESC + (k & 7u) * 16, k + 1u, 4u * k);
            const u32x4 de = *(vl128*)(lds + L_DESC + (k & 7u) * 16);
            const int i = (int)rfl(de.y); const unsigned info = rfl(de.z);
            if (i >= 256) break;
            if (lane == 0) *(vl32*)(lds + L_LST) = k + 1u;
            int isW, j, u; item_decode(i, rank, isW, j, u);
            const gchar* ub = uni(sws + pin_off(call & 1u, isW, 6 - srel, j, u));
#pragma unroll
            for (unsigned q = 0; q < 4; ++q) {
                if (k >= 4) LWAIT(L_FREE + ((k & 3u) * 4 + q) * 4, k - 3u, 4u * k + q);
                const unsigned size = Q_LEN + ((info >> (10u + 5u * q)) & 31u) * 64u, dst = (unsigned)(uintptr_t)lds + (k & 3u) * UNIT + q * Q_BYTES;
#pragma unroll
                for (int c = 0; c < 8; ++c) {
                    if (c < 6 || c * 1024u + lane * 16u < size) glds16_sys(dst + c * 1024u, lane * 16u, ub + (q * Q_BYTES + c * 1024u));
                    asm volatile("s_waitcnt vmcnt(" STR(PULL_N) ")" ::: "memory");
                    if (c == (PULL_N - 1) % 7 && 4u * k + q > land + (PULL_N > 7)) { land = 4u * k + q - (PULL_N > 7); if (lane == 0) *(vl32*)(lds + L_LAND) = land; }
                }
            }
        }
        if (lane == 0) __hip_atomic_fetch_add((gu32*)(myws + WS_OREL + srel * 128), PUSH_REL, __ATOMIC_RELAXED, __HIP_MEMORY_SCOPE_AGENT);
        asm volatile("s_waitcnt vmcnt(0)" ::: "memory");
        if (lane == 0) *(vl32*)(lds + L_LAND) = 4u * k;
        __builtin_amdgcn_s_setprio(0);
    } else {
        unsigned inf0;
        for (int it = first; it < 8 * NPUSH; it += 8)
            if (wv >= 4 && flag_wait<2>(myws + WS_IFLAG + (fi * 16 + it) * 8, tmo, call, inf0)) {
                const __amdgpu_buffer_rsrc_t rs = __builtin_amdgcn_make_buffer_rsrc((void*)uni(myws + WS_INBOX + (size_t)(((call & 1u) * 7 + srel) * 16 + it) * UNIT), 0, UNIT, 0x00020000);
                u32x4 W[8];
#pragma unroll
                for (int c = 0; c < 8; ++c) W[c] = RLOAD4(rs, lane * 16u, (wv - 4u) * Q_BYTES + c * 1024u);
                int isW, j, u; item_decode(it, rank, isW, j, u);
                const gchar* ub; gu32* f = stage_unit(myws, srel, isW, j, u, ub); ub = uni(ub);
                __builtin_amdgcn_s_waitcnt(0x0F70);
#pragma unroll
                for (int c = 0; c < 8; ++c) store16_sys(ub + ((wv - 4u) * Q_BYTES + c * 1024u), lane * 16u, W[c]);
                asm volatile("s_waitcnt vmcnt(0)" ::: "memory");
                if (lane == 0 && __hip_atomic_fetch_add((LAS unsigned*)(lds + L_QI + (it >> 3) * 4), 1u, __ATOMIC_RELAXED, __HIP_MEMORY_SCOPE_WORKGROUP) == 3u)
                    __hip_atomic_store(f, (call << 8) | (inf0 & 31u), __ATOMIC_RELAXED, __HIP_MEMORY_SCOPE_SYSTEM);
            }
        for (unsigned n = wv - 2u;; n += 6u) {
            const unsigned k = n >> 2, q = n & 3u;
            unsigned kd = k;
            for (;;) {
                const u32x4 d0 = *(vl128*)(lds + L_DESC + (k & 7u) * 16), d1 = *(vl128*)(lds + L_DESC + ((k + 7u) & 7u) * 16);
                if (rfl(d0.x) >= k + 1u) break;
                if (k > 0 && rfl(d1.x) == k && rfl(d1.y) >= 256u) { kd = k - 1u; break; }
                __builtin_amdgcn_s_sleep(1);
            }
            const u32x4 de = *(vl128*)(lds + L_DESC + (kd & 7u) * 16);
            const int i = (int)rfl(de.y); const unsigned info = rfl(de.z);
            if (i >= 256) break;
            lds_wait1(lds, L_LAND, n + 1u);
            const bool raw = ((info >> (10u + 5u * q)) & 31u) == 24u;
            QIn Q;
            q_read(lds + (k & 3u) * UNIT + q * Q_BYTES, lane, raw, Q);
            LGKM0;
            if (lane == 0) *(vl32*)(lds + L_FREE + ((k & 3u) * 4 + q) * 4) = k + 1u;
            u32x4 W[8];
            if (!raw) q_finish((info >> 5) & 31u, Q, W);
            q_patch(raw, Q, W);
            int isW, j, u; item_decode(i, rank, isW, j, u);
            const gchar* ub; gu32* f = stage_unit(myws, srel, isW, j, u, ub); ub = uni(ub);
#pragma unroll
            for (int c = 0; c < 8; ++c) store16_sys(ub + img_off(isW, q, c), lane * 16u, W[c]);
            asm volatile("s_waitcnt vmcnt(0)" ::: "memory");
            if (lane == 0) {
                const unsigned old = __hip_atomic_fetch_add((LAS unsigned*)(lds + L_UCNT + (k & 7u) * 4), 1u, __ATOMIC_RELAXED, __HIP_MEMORY_SCOPE_WORKGROUP);
                if ((old & 3u) == 3u) __hip_atomic_store(f, (call << 8) | (info & 31u), __ATOMIC_RELAXED, __HIP_MEMORY_SCOPE_SYSTEM);
            }
        }
    }
    LBAR;
}

__global__ void __launch_bounds__(NTHR, 2) gemm_rs_kernel(const Params p) {
    extern __shared__ __attribute__((aligned(16))) unsigned char smem[];
    LAS unsigned char* lds = (LAS unsigned char*)smem;
    const int tid = threadIdx.x;
    const int rank = (int)rfl(p.rank), bid = (int)rfl(blockIdx.x);
    gchar* myws = uni(p.ws[rank]);
    STAMP(0);
    const int xcd = bid & 7, slot = bid >> 3;
    gu32* tmo = (gu32*)(myws + WS_TMO);

    int pm, pn;
    {
        int ar = -1;
        if (slot < 11) {
            const int c = slot < 7 ? xcd * 7 + slot : xcd * 4 + slot - 7, o = slot < 7 ? c - 28 : c;
            if (c < 28) { ar = c >> 1; pn = (slot < 7 ? 14 : 12) + (c & 1); } else { pm = 2 * rank + (o & 1); pn = o >> 1; }
        } else {
            const int li = slot - 11, q = li / 7;
            pn = 3 * (xcd & 3) + q; ar = 2 * (li - q * 7) + (xcd >> 2);
        }
        if (ar >= 0) pm = ar + (ar >= 2 * rank ? 2 : 0);
    }
    const bool encoder = slot >= 7 && slot < 11;
    unsigned call;
    if (slot < 7) { call = push_first(p, lds, myws, rank, slot, xcd); STAMP(5); fetch_inputs(p, lds, myws, rank, call, slot, xcd); }
    else {
        int first = encoder ? bid + 24 : slot < 21 ? bid - 88 : bid - 56;
        if (SKIP_PUSHED && first >= 16 && first < min(64 * NPUSH - 40, 80) && (first & 7) != 7) first = NTASK;
        call = produce(p, lds, myws, rank, first, encoder ? 144 + bid : NTASK, encoder ? (gu32*)(myws + WS_TASKQ) : nullptr); STAMP(5);
    }
    STAMP(1);

    const int wid = rfl(tid >> 6), lane = tid & 63, wr = wid >> 2, wc = wid & 3, fr = lane & 15, fq = lane >> 4;
    const gchar* cA = myws + WS_STAGE + (size_t)pm * UNIT;
    const gchar* cB = myws + WS_STAGE + (size_t)(16 + pn) * UNIT;
    const gu32* fA = (const gu32*)(myws + WS_FLAG) + pm * NKT;
    const gu32* fB = (const gu32*)(myws + WS_FLAG) + (16 + pn) * NKT;
    const unsigned voff[2] = {(unsigned)tid * 16u, (unsigned)tid * 16u + 8192u}; const unsigned ldsw = (unsigned)wid * 1024u;
    const unsigned aoff0 = (unsigned)((wr * 64 + fr) * 128 + ((fq ^ (fr >> 1)) * 16)), aoff1 = aoff0 ^ 64u;
    const unsigned boff = (unsigned)(fq * 2048 + (wc * 16 + fr) * 16);
    int ready = 0; unsigned long long chg = 0; int scur = 0, smax = 0;

#define SA(b, h) (((b) * 2 + (h)) * HTB)
#define SB(b, h) ((4 + (b) * 2 + (h)) * HTB)
#define STAGE(bufoff, gbase) do { _Pragma("unroll") for (int _i = 0; _i < 2; ++_i) \
        __builtin_amdgcn_global_load_lds((const gu32*)((gbase) + voff[_i]), (LAS unsigned*)(lds + (bufoff) + ldsw + _i * 8192), 16, 0, GLDS_AUX); } while (0)
#define LDA(dst, b, h) do { _Pragma("unroll") for (int m = 0; m < 4; ++m) { dst[m][0] = *(const LAS h16x8*)(lds + SA(b, h) + aoff0 + m * 2048); dst[m][1] = *(const LAS h16x8*)(lds + SA(b, h) + aoff1 + m * 2048); } } while (0)
#define LDB(dst, b, h) do { _Pragma("unroll") for (int n = 0; n < 2; ++n) _Pragma("unroll") for (int k = 0; k < 2; ++k) dst[n][k] = *(const LAS h16x8*)(lds + SB(b, h) + boff + n * 1024 + k * 8192); } while (0)
#define MMA(ai, bj, At, Bt) do { __builtin_amdgcn_s_setprio(1); _Pragma("unroll") for (int m = 0; m < 4; ++m) _Pragma("unroll") for (int n = 0; n < 2; ++n) _Pragma("unroll") for (int k = 0; k < 2; ++k) \
        acc[ai][bj][m][n] = __builtin_amdgcn_mfma_f32_16x16x32_f16(At[m][k], Bt[n][k], acc[ai][bj][m][n], 0, 0, 0); __builtin_amdgcn_s_setprio(0); } while (0)
#define WAIT_V(n) asm volatile("s_waitcnt vmcnt(" #n ")" ::: "memory")
#define WAIT_L(n) asm volatile("s_waitcnt lgkmcnt(" #n ")" ::: "memory")
#define BAR __builtin_amdgcn_s_barrier()
#define SCHED __builtin_amdgcn_sched_barrier(0)
#define HOOK(need) do { if (ready < (need)) { \
        if (wid == 0) { \
            const unsigned long long t0 = __builtin_amdgcn_s_memrealtime(); unsigned fa, fb; int rd; \
            unsigned ln = voff[0]; asm volatile("" : "+v"(ln)); ln >>= 4;     \
            for (;;) { \
                fa = ld_sys(fA + ln); fb = ld_sys(fB + ln); \
                const unsigned long long okm = __ballot((fa >> 8) >= call && (fb >> 8) >= call); \
                rd = (okm == ~0ull) ? 64 : (int)__builtin_ctzll(~okm); \
                if (rd >= (need)) break; \
                if (rfl(ld_agt(tmo)) == call || __builtin_amdgcn_s_memrealtime() - t0 > DEADLINE) { \
                    if (ln == 0) __hip_atomic_store(tmo, call, __ATOMIC_RELAXED, __HIP_MEMORY_SCOPE_AGENT); \
                    rd = 64; break; } \
                __builtin_amdgcn_s_sleep(POLL_SLEEP); \
            } \
            const int sv = (int)((fa & 31u) + (fb & 31u)); \
            const int sp = __builtin_amdgcn_update_dpp(-1, sv, 0x138, 0xf, 0xf, false);     \
            const unsigned long long cg = __ballot(sv != sp); \
            *(LAS int*)(lds + L_TAB + ln * 4) = sv; \
            if (ln == 0) { u32x4 h; h.x = (unsigned)rd; h.y = (unsigned)cg; h.z = (unsigned)(cg >> 32); h.w = 0; *(LAS u32x4*)(lds + L_HDR) = h; } \
            DIAG_WAIT(t0); \
            asm volatile("s_waitcnt vmcnt(0) lgkmcnt(0)" ::: "memory"); \
        } \
        asm volatile("" ::: "memory"); BAR; asm volatile("" ::: "memory"); \
        { const u32x4 h = *(const LAS u32x4*)(lds + L_HDR); ready = (int)rfl(h.x); chg = ((unsigned long long)rfl(h.z) << 32) | rfl(h.y); } \
    } } while (0)
#define DIAG_WAIT(t0) do {} while (0)
#define PKMUL4(op) ".set rs_i, 0\n\t.set rs_d, 0\n\t.irpc c, \"" op "\"\n\t.ifc \\c,:\n\t.set rs_d, 1\n\t.endif\n\t" \
        ".if rs_d == 0\n\t.ifnc \\c,v\n\t.ifnc \\c,[\n\t.set rs_i, rs_i*10 + \\c\n\t.endif\n\t.endif\n\t.endif\n\t.endr\n\t" \
        "v_pk_mul_f32 v[rs_i:rs_i+1], v[rs_i:rs_i+1], %16\n\tv_pk_mul_f32 v[rs_i+2:rs_i+3], v[rs_i+2:rs_i+3], %16\n\t"
#define RS16(a) asm volatile("s_cmp_eq_u32 %17, 0\n\ts_cbranch_scc1 .Lrs%=\n\ts_nop 15\n\ts_nop 3\n\t" \
            PKMUL4("%0") PKMUL4("%1") PKMUL4("%2") PKMUL4("%3") PKMUL4("%4") PKMUL4("%5") PKMUL4("%6") PKMUL4("%7") \
            PKMUL4("%8") PKMUL4("%9") PKMUL4("%10") PKMUL4("%11") PKMUL4("%12") PKMUL4("%13") PKMUL4("%14") PKMUL4("%15") \
            "s_nop 4\n.Lrs%=:" \
            : "+v"(acc[a][0][0][0]), "+v"(acc[a][0][0][1]), "+v"(acc[a][0][1][0]), "+v"(acc[a][0][1][1]), "+v"(acc[a][0][2][0]), "+v"(acc[a][0][2][1]), "+v"(acc[a][0][3][0]), "+v"(acc[a][0][3][1]), \
              "+v"(acc[a][1][0][0]), "+v"(acc[a][1][0][1]), "+v"(acc[a][1][1][0]), "+v"(acc[a][1][1][1]), "+v"(acc[a][1][2][0]), "+v"(acc[a][1][2][1]), "+v"(acc[a][1][3][0]), "+v"(acc[a][1][3][1]) \
            : "s"(fac), "s"(de) : "scc")
#define RESCALE(kt) do { int de = 0; \
        if ((chg >> (kt)) & 1ull) { \
            const int s = (int)rfl(*(const LAS unsigned*)(lds + L_TAB + (kt) * 4)); \
            smax = max(smax, s); const int sn = max(s, smax - 4); de = max(16 * (scur - sn), -126); scur = sn; } \
        const unsigned long long fac = (unsigned long long)(unsigned)((de + 127) << 23) * 0x100000001ull; \
        RS16(0); RS16(1); \
    } while (0)
#define TRIP do { \
        const int t2 = (t + 2) & (NKT - 1);             \
        const gchar* a1 = cA + (size_t)(t + 1) * KT_BYTES; \
        const gchar* a2 = cA + (size_t)t2 * KT_BYTES; const gchar* b2 = cB + (size_t)t2 * KT_BYTES; \
        const gchar* a3 = a2 + KT_BYTES; const gchar* b3 = b2 + KT_BYTES; \
        RESCALE(t); \
        LDB(B0, 0, 0); SCHED; LDA(At, 0, 0); STAGE(SA(1, 1), a1 + HTB); \
        WAIT_L(8); BAR; WAIT_L(0); MMA(0, 0, At, B0); BAR; SCHED; \
        LDB(B1, 0, 1); STAGE(SB(0, 0), b2); \
        BAR; WAIT_L(0); MMA(0, 1, At, B1); BAR; \
        LDA(At, 0, 1); STAGE(SA(0, 0), a2); \
        BAR; WAIT_L(0); MMA(1, 0, At, B0); BAR; SCHED; \
        STAGE(SB(0, 1), b2 + HTB); \
        WAIT_V(6); BAR; MMA(1, 1, At, B1); BAR; \
        RESCALE(t + 1); \
        LDB(B0, 1, 0); SCHED; LDA(At, 1, 0); STAGE(SA(0, 1), a2 + HTB); \
        WAIT_L(8); BAR; WAIT_L(0); MMA(0, 0, At, B0); BAR; SCHED; \
        LDB(B1, 1, 1); STAGE(SB(1, 0), b3); \
        BAR; WAIT_L(0); MMA(0, 1, At, B1); BAR; \
        LDA(At, 1, 1); STAGE(SA(1, 0), a3); \
        BAR; WAIT_L(0); MMA(1, 0, At, B0); BAR; SCHED; \
        STAGE(SB(1, 1), b3 + HTB); \
        WAIT_V(6); BAR; MMA(1, 1, At, B1); BAR; \
    } while (0)

    f32x4 acc[2][2][4][2];
#pragma unroll
    for (int a = 0; a < 2; ++a)
#pragma unroll
        for (int b = 0; b < 2; ++b)
#pragma unroll
            for (int m = 0; m < 4; ++m)
#pragma unroll
                for (int n = 0; n < 2; ++n) acc[a][b][m][n] = (f32x4){0.f, 0.f, 0.f, 0.f};
    h16x8 At[4][2], B0[2][2], B1[2][2];

    HOOK(2);
    STAMP(2);
    STAGE(SB(0, 0), cB); STAGE(SA(0, 0), cA); STAGE(SB(0, 1), cB + HTB); STAGE(SA(0, 1), cA + HTB);
    if (wr == 1) BAR;
    WAIT_V(4); BAR;
    STAGE(SB(1, 0), cB + KT_BYTES); STAGE(SA(1, 0), cA + KT_BYTES); STAGE(SB(1, 1), cB + KT_BYTES + HTB);
    WAIT_V(6); BAR;
    for (int t = 0; t < NKT; t += 2) {
        HOOK(min(t + 4, NKT));
        TRIP;
    }
    if (wr == 0) BAR;
    STAMP(3);

    const int dest = pm >> 1;
    const unsigned ovo = (unsigned)(wr * 64 + 4 * fq) * (GN * 4u) + (unsigned)(wc * 16 + fr) * 16u;
    const float qn = __builtin_nanf("");
    u32x4 V[32];
    {
        const bool bad = rfl(ld_agt(tmo)) == call;
        const int oe = 16 * scur - 256;
#pragma unroll
        for (int a = 0; a < 2; ++a)
#pragma unroll
            for (int m = 0; m < 4; ++m)
#pragma unroll
                for (int r = 0; r < 4; ++r) {
                    f32x4 v;
                    v.x = __builtin_amdgcn_ldexpf(acc[a][0][m][0][r], oe); v.y = __builtin_amdgcn_ldexpf(acc[a][0][m][1][r], oe);
                    v.z = __builtin_amdgcn_ldexpf(acc[a][1][m][0][r], oe); v.w = __builtin_amdgcn_ldexpf(acc[a][1][m][1][r], oe);
                    if (bad) v = (f32x4){qn, qn, qn, qn};
                    V[(a * 4 + m) * 4 + r] = __builtin_bit_cast(u32x4, v);
                }
    }
#define OSTAMP(k) do {} while (0)
#define DIAG_END do {} while (0)
    gchar* dws = uni(p.ws[dest]);
    if (dest == rank) {
        const gchar* ob = uni((const char*)p.out[rank] + ((size_t)((pm & 1) * 256) * GN + rank * NL + pn * 256) * 4);
#pragma unroll
        for (int q = 0; q < 32; ++q) { store16_sys(ob + out_off(q), ovo, V[q]); WAIT_V(4); }
    }
    int packed = dest - rank;
    asm volatile("" : "+v"(packed)); packed = (int)rfl(packed);
    if (packed) {
        const unsigned cidx = (unsigned)(((pm & 1) * 16 + pn) * 8 + wid);
        const gchar* pb = uni(myws + WS_PACK + (size_t)(dest * 256 + cidx) * CHUNK);
        if (pn < NE) for (int z = 0; z < (wid >> 1) * STAGGER; ++z) __builtin_amdgcn_s_sleep(32);
        unsigned mu = 0; float sm = 0.f;
#pragma unroll
        for (int q = 0; q < 32; ++q) {
            const u32x4 w = V[q] & 0x7fffffffu; mu = max(max(mu, w.x), max(w.y, max(w.z, w.w)));
            const f32x4 f = __builtin_bit_cast(f32x4, w); sm += (f.x + f.y) + (f.z + f.w);
        }
        const unsigned myB = min(max(min(wave_mean_exp(sm * (2.32f / 8192.f)), wave_max(mu >> 23)), 2u), 254u);
        u32x4 PK[26];
        LAS unsigned char* scr = lds + L_SCR + wid * 3072;
        const unsigned tail = c_encode(V, myB, PK, scr, pb, lane);
        asm volatile("" ::: "memory");
#pragma unroll
        for (int q = 0; q < 3; ++q) if (q * 1024u + lane * 16u < tail) store16_sys(pb + (C_HDR + q * 1024u), lane * 16u, *(const LAS u32x4*)(scr + q * 1024u + lane * 16u));
#pragma unroll
        for (int q = 0; q < 26; ++q) { store16_sys(pb + q * 1024u, lane * 16u, PK[q]); WAIT_V(4); }
        WAIT_V(0);
        if (lane == 0) __hip_atomic_store((GAS unsigned long long*)(dws + WS_CFLAG) + rank * 256 + cidx, ((unsigned long long)call << 32) | (tail << 8) | myB, __ATOMIC_RELAXED, __HIP_MEMORY_SCOPE_SYSTEM);
        OSTAMP(31);
        if (pn < NE && wid < OUT_WAVES) {
            const int link = (dest - rank - 1) & 7;
            gu32* rel = (gu32*)(myws + WS_OREL + link * 128);
            for (;;) {
                unsigned ct = 0;
                if (lane == 0) ct = __hip_atomic_fetch_add((gu32*)(myws + WS_OTK + link * 128 + 64), 1u, __ATOMIC_RELAXED, __HIP_MEMORY_SCOPE_AGENT);
                ct = rfl(ct) % (256u + 2u * NE * OUT_WAVES);
                if (ct >= 256u) break;
                const unsigned T = WMAJOR && ct < 16u * NE ? ct % (2u * NE) : ct >> 3, cw_ = WMAJOR && ct < 16u * NE ? ct / (2u * NE) : ct & 7u, crow = T < 2u * NE ? T / NE : T & 1u, ccol = T < 2u * NE ? T % NE : T >> 1, ci = (crow * 16u + ccol) * 8u + cw_;
                unsigned fv;
                flag_wait<FLAG_SLEEP>(myws + WS_CFLAG + (dest * 256 + ci) * 8, tmo, call, fv);
                const bool bad = rfl(ld_agt(tmo)) == call;
                unsigned tk = 0;
                if (lane == 0) tk = __hip_atomic_fetch_add((gu32*)(myws + WS_OTK + link * 128), 1u, __ATOMIC_RELAXED, __HIP_MEMORY_SCOPE_AGENT);
                tk = rfl(tk) & 255u;
                const bool wide = tk >= WIDE_FROM;
                const unsigned need = (call - 1u) * (256u + 8u * PUSH_REL) + tk + 1u - LINK_EXTRA;
                const unsigned long long t0 = __builtin_amdgcn_s_memrealtime();
                for (;;) {
                    const int ahead = (int)(need - rfl(ld_agt(rel)));
                    if (ahead <= 0 || __builtin_amdgcn_s_memrealtime() - t0 > DEADLINE) break;
                    if (ahead > 4) __builtin_amdgcn_s_sleep(32); else __builtin_amdgcn_s_sleep(4);
                }
                __builtin_amdgcn_s_setprio(SEND_PRIO);
                asm volatile("" ::: "memory");
                const gchar* rb = uni(dws + WS_PACK + (size_t)(rank * 256 + ci) * CHUNK);
                const __amdgpu_buffer_rsrc_t rs = __builtin_amdgcn_make_buffer_rsrc((void*)rb, 0, bad ? 0 : CHUNK, 0x00020000);
                const unsigned rtail = bad ? 0u : min(fv >> 8, 3072u);
#pragma unroll
                for (int q = 0; q < 3; ++q) if (q * 1024u + lane * 16u < rtail) glds16_sys((unsigned)(uintptr_t)scr + q * 1024u, lane * 16u, rb + (C_HDR + q * 1024u));
                u32x4 X[8]; CDec D;
#pragma unroll
                for (int n = 0; n < 26; ++n) {
                    const int q = n < 2 ? 24 + n : n - 2;
                    PK[q] = RLOAD4(rs, lane * 16u, q * 1024);
                    if (n >= 10) { SCHED; c_parse(D, PK, n - 10); SCHED; }
                    if (wide) asm volatile("s_waitcnt vmcnt(" STR(OUT_W2) ")" ::: "memory"); else asm volatile("s_waitcnt vmcnt(" STR(OUT_W) ")" ::: "memory");
                    if (n == REL_AT && lane == 0) __hip_atomic_fetch_add(rel, 1u, __ATOMIC_RELAXED, __HIP_MEMORY_SCOPE_AGENT);
                    if (n == 8) {
                        D.hdr = *(volatile LAS unsigned short*)(scr + lane * 2u);
#pragma unroll
                        for (int g = 0; g < 4; ++g) if (D.hdr & (256u << g)) { X[2 * g] = RLOAD4(rs, lane * 32u, C_EXC + g * 2048u); X[2 * g + 1] = RLOAD4(rs, lane * 32u + 16u, C_EXC + g * 2048u); }
                        c_begin(D, scr, fv & 255u);
                    }
                }
                __builtin_amdgcn_s_waitcnt(0x0F70);
                __builtin_amdgcn_s_setprio(0);
                c_store(D, PK, X, bad, uni((const char*)p.out[rank] + ((size_t)(crow * 256) * GN + dest * NL + ccol * 256) * 4),
                        ((cw_ >> 2) * 64u + 4u * fq) * (GN * 4u) + ((cw_ & 3u) * 16u + fr) * 16u);
            }
        }
    }
    DIAG_END;
}

extern "C" void kernel_launch(int rank, int nranks, ncclComm_t comm,
                              hipStream_t stream,
                              void* const* d_in, const int* in_sizes, int n_in,
                              void* d_out, int out_size,
                              const int* in_dtypes, int out_dtype,
                              void* const* d_out_all,
                              void* const* d_ws_all, size_t ws_bytes) {
    (void)comm; (void)in_dtypes; (void)out_dtype; (void)d_out;
    if (nranks != NR || (unsigned)rank >= (unsigned)NR || n_in < 2 || ws_bytes < WS_NEED ||
        in_sizes[0] != GM * KS || in_sizes[1] != KS * GN || out_size != (GM / NR) * GN) {
        fprintf(stderr, "gemm_rs: unexpected problem shape\n");
        return;
    }
    HIP_CHECK(hipFuncSetAttribute((const void*)gemm_rs_kernel, hipFuncAttributeMaxDynamicSharedMemorySize, (int)LDS_BYTES));
    Params p;
    p.x = (const float*)d_in[0]; p.w = (const float*)d_in[1];
    for (int i = 0; i < NR; ++i) { p.out[i] = (float*)d_out_all[i]; p.ws[i] = (char*)d_ws_all[i]; }
    p.rank = rank; p.pad = 0;
    hipLaunchKernelGGL(gemm_rs_kernel, dim3(GRID), dim3(NTHR), LDS_BYTES, stream, p);
    HIP_CHECK(hipGetLastError());
}
```

```cpp
#include <hip/hip_runtime.h>
#include <rccl/rccl.h>
#include <cstdio>
#include <cstdint>

#define HIP_CHECK(expr) do { hipError_t _e = (expr); if (_e != hipSuccess) { \
    fprintf(stderr, "HIP error %s at %s:%d\n", hipGetErrorString(_e), __FILE__, __LINE__); \
    return; } } while (0)

#define LAS __attribute__((address_space(3)))
#define GAS __attribute__((address_space(1)))
typedef GAS char gchar;
typedef GAS unsigned gu32;
typedef _Float16 h16x8 __attribute__((ext_vector_type(8)));
typedef float f32x4 __attribute__((ext_vector_type(4)));
typedef float f32x2 __attribute__((ext_vector_type(2)));
typedef unsigned u32x4 __attribute__((ext_vector_type(4)));

constexpr int NR = 8;
constexpr int GM = 4096, GN = 32768, KS = 512, NL = 4096;
constexpr int GRID = 256, NTHR = 512;
constexpr unsigned UNIT = 32768;
constexpr unsigned HTB = 16384;
constexpr int NKT = 64;
constexpr size_t WS_TICKET = 0, WS_TMO = 128, WS_TASKQ = 256, WS_LINKQ = 1024, WS_OTK = 2048, WS_OREL = 3072;
constexpr size_t WS_FLAG = 4096;
constexpr size_t WS_CFLAG = 16384;
constexpr size_t WS_AFLAG = 32768;
constexpr size_t WS_IFLAG = 12288;
constexpr size_t WS_FLAG2 = 65536;
constexpr size_t WS_PFLAG = 73728;
constexpr size_t WS_INBOX = 131072;
constexpr size_t WS_PX = WS_INBOX + (size_t)224 * UNIT;
constexpr size_t WS_PW = WS_PX + (size_t)256 * UNIT;
constexpr size_t WS_STAGE = WS_PW + (size_t)14 * 128 * UNIT;
constexpr unsigned KT_BYTES = 32 * UNIT;
constexpr unsigned CHUNK = 37888;
constexpr size_t WS_STAGE2 = WS_STAGE + (size_t)NKT * KT_BYTES;
constexpr size_t WS_PACK = WS_STAGE2 + (size_t)114 * 8 * UNIT;
constexpr size_t WS_NEED = WS_PACK + (size_t)NR * 256 * CHUNK;
constexpr size_t WS_DIAG = 65536, WS_DBASE = WS_NEED;
__device__ __forceinline__ size_t pin_off(unsigned par, int isW, int dd, int j, int u) {
    return isW ? WS_PW + (size_t)((par * 7 + dd) * 128 + j * 16 + u) * UNIT : WS_PX + (size_t)(par * 128 + j * 16 + u) * UNIT;
}
#define OUT_AUX 19
#define NEARLY 11
constexpr int NE = NEARLY;
constexpr unsigned long long DEADLINE = 100000000ull;
constexpr unsigned L_CTL = 8 * HTB;
constexpr unsigned L_AMAX = L_CTL, L_CONS = L_CTL + 32, L_TURN = L_CTL + 48, L_CALL = L_CTL + 52, L_LAND = L_CTL + 56, L_LST = L_CTL + 60, L_ITEM = L_CTL + 64, L_QI = L_CTL + 128;
constexpr unsigned L_ECNT = L_CTL + 192, L_UCNT = L_CTL + 224, L_FREE = L_CTL + 256, L_DESC = L_CTL + 320, L_ZERO_END = L_CTL + 448;
constexpr unsigned L_HDR = L_CTL + 576, L_TAB = L_CTL + 640, L_SCR = L_CTL + 1024, LDS_BYTES = L_SCR + 8 * 3072;

#define GLDS_AUX 17
struct Params {
    const float* x; const float* w;
    float* out[NR]; char* ws[NR];
    int rank; int pad;
};

#define STAMP(i) do {} while (0)

__device__ __forceinline__ void store16_sys(const gchar* sbase, unsigned voff, u32x4 v) {
    asm volatile("global_store_dwordx4 %0, %1, %2 sc0 sc1\n\ts_nop 1" :: "v"(voff), "v"(v), "s"(sbase) : "memory");
}
__device__ __forceinline__ void glds16_sys(unsigned dst, unsigned voff, const gchar* sbase) {
    unsigned keep;
    asm volatile("s_mov_b32 %0, m0\n\ts_mov_b32 m0, %1\n\ts_nop 0\n\tglobal_load_lds_dwordx4 %2, %3 sc0 sc1\n\ts_mov_b32 m0, %0" : "=&s"(keep) : "s"(dst), "v"(voff), "s"(sbase) : "memory");
}
__device__ __forceinline__ unsigned ld_sys(const gu32* q) { return __hip_atomic_load(q, __ATOMIC_RELAXED, __HIP_MEMORY_SCOPE_SYSTEM); }
__device__ __forceinline__ unsigned ld_agt(const gu32* q) { return __hip_atomic_load(q, __ATOMIC_RELAXED, __HIP_MEMORY_SCOPE_AGENT); }
__device__ __forceinline__ unsigned rfl(unsigned v) { return (unsigned)__builtin_amdgcn_readfirstlane((int)v); }

template <class T> __device__ __forceinline__ gchar* uni(T q) {
    unsigned lo = rfl((unsigned)(uintptr_t)q), hi = rfl((unsigned)((uintptr_t)q >> 32));
    asm volatile("s_nop 4" : "+s"(lo), "+s"(hi));
    return (gchar*)(((uintptr_t)hi << 32) | lo);
}

__device__ __forceinline__ unsigned wave_max(unsigned x) {
    int mb = (int)x;
    mb = max(mb, __builtin_amdgcn_update_dpp(0, mb, 0x111, 0xf, 0xf, false));
    mb = max(mb, __builtin_amdgcn_update_dpp(0, mb, 0x112, 0xf, 0xf, false));
    mb = max(mb, __builtin_amdgcn_update_dpp(0, mb, 0x114, 0xf, 0xf, false));
    mb = max(mb, __builtin_amdgcn_update_dpp(0, mb, 0x118, 0xf, 0xf, false));
    mb = max(mb, __builtin_amdgcn_update_dpp(0, mb, 0x142, 0xa, 0xf, false));
    mb = max(mb, __builtin_amdgcn_update_dpp(0, mb, 0x143, 0xc, 0xf, false));
    return (unsigned)__builtin_amdgcn_readlane(mb, 63);
}
__device__ __forceinline__ unsigned wave_mean_exp(float sm) {
#define DPP_ADD(ctl, rm) sm += __builtin_bit_cast(float, __builtin_amdgcn_update_dpp(0, __builtin_bit_cast(int, sm), ctl, rm, 0xf, false))
    DPP_ADD(0x111, 0xf); DPP_ADD(0x112, 0xf); DPP_ADD(0x114, 0xf); DPP_ADD(0x118, 0xf); DPP_ADD(0x142, 0xa); DPP_ADD(0x143, 0xc);
    return ((unsigned)__builtin_amdgcn_readlane(__builtin_bit_cast(int, sm), 63) >> 23) & 255u;
}

typedef unsigned short u16x2 __attribute__((ext_vector_type(2)));
typedef unsigned u32x2 __attribute__((ext_vector_type(2)));
constexpr unsigned Q_P6 = 6144, Q_LEN = 6656, Q_STR = 6720, Q_BYTES = 8192;
__device__ __forceinline__ u16x2 as16(unsigned a) { return __builtin_bit_cast(u16x2, a); }
__device__ __forceinline__ unsigned as32(u16x2 a) { return __builtin_bit_cast(unsigned, a); }
__device__ __forceinline__ unsigned pk_sub(unsigned a, unsigned b) { return as32(as16(a) - as16(b)); }
__device__ __forceinline__ unsigned pk_subs(unsigned a, unsigned b) { return as32(__builtin_elementwise_sub_sat(as16(a), as16(b))); }
__device__ __forceinline__ unsigned pk_min(unsigned a, unsigned b) { return as32(__builtin_elementwise_min(as16(a), as16(b))); }
__device__ __forceinline__ unsigned ffbl(unsigned a) { unsigned r; asm("v_ffbl_b32 %0, %1" : "=v"(r) : "v"(a)); return r; }
__device__ __forceinline__ unsigned wave_scan(unsigned x) {
    int v = (int)x;
    v += __builtin_amdgcn_update_dpp(0, v, 0x111, 0xf, 0xf, false);
    v += __builtin_amdgcn_update_dpp(0, v, 0x112, 0xf, 0xf, false);
    v += __builtin_amdgcn_update_dpp(0, v, 0x114, 0xf, 0xf, false);
    v += __builtin_amdgcn_update_dpp(0, v, 0x118, 0xf, 0xf, false);
    v += __builtin_amdgcn_update_dpp(0, v, 0x142, 0xa, 0xf, false);
    v += __builtin_amdgcn_update_dpp(0, v, 0x143, 0xc, 0xf, false);
    return (unsigned)v;
}
__device__ __forceinline__ unsigned q_encode(const u32x4 (&W)[8], unsigned B, LAS unsigned char* dst, unsigned lane) {
    const unsigned B1 = (B + 1u) * 0x04000400u;
    unsigned v[32], seg[4], pos[4];
    u32x4 X[4];
#pragma unroll
    for (int s = 0; s < 4; ++s) {
        unsigned a = 0, p = 0;
        X[s].x = __builtin_amdgcn_perm(W[2 * s].y, W[2 * s].x, 0x07050301u); X[s].y = __builtin_amdgcn_perm(W[2 * s].w, W[2 * s].z, 0x07050301u);
        X[s].z = __builtin_amdgcn_perm(W[2 * s + 1].y, W[2 * s + 1].x, 0x07050301u); X[s].w = __builtin_amdgcn_perm(W[2 * s + 1].w, W[2 * s + 1].z, 0x07050301u);
#pragma unroll
        for (int k = 8 * s; k < 8 * s + 8; ++k) {
            const unsigned w = W[k >> 2][k & 3];
            const unsigned dq = as32(as16(pk_sub(B1, w & 0x7c007c00u)) >> 10);
            const unsigned ix = pk_min(dq, 0x00070007u) | 0x0c000c00u;
            const unsigned len = __builtin_amdgcn_perm(0x05040201u, 0x00000003u, ix) + pk_subs(dq, 0x00070007u);
            v[k] = (w & 0x83ff83ffu) | (__builtin_amdgcn_perm(0u, 0x03020100u, ix) << 10);
            p += len & 0xffffu; a |= 1u << (p & 31u);
            p += len >> 16;     a |= 1u << (p & 31u);
        }
        seg[s] = a >> 1; pos[s] = p;
        asm volatile("" : "+v"(v[8 * s]), "+v"(v[8 * s + 1]), "+v"(v[8 * s + 2]), "+v"(v[8 * s + 3]), "+v"(v[8 * s + 4]), "+v"(v[8 * s + 5]), "+v"(v[8 * s + 6]), "+v"(v[8 * s + 7]),
                     "+v"(X[s]), "+v"(seg[s]), "+v"(pos[s]));
    }
    const bool ovf = max(max(pos[0], pos[1]), max(pos[2], pos[3])) > 31u;
    const unsigned tot = ovf ? 0u : pos[0] + pos[1] + pos[2] + pos[3];
    const unsigned incl = wave_scan(tot), total = (unsigned)__builtin_amdgcn_readlane((int)incl, 63);
    const unsigned long long om = __ballot(ovf);
    const unsigned xb = Q_STR + (((total + 127u) >> 7) << 4), size = xb + 64u * (unsigned)__popcll(om);
    if (size > Q_BYTES - 64u) return Q_BYTES;
    *(LAS u32x4*)(dst + Q_STR + lane * 16u) = (u32x4){0u, 0u, 0u, 0u};
#pragma unroll
    for (int j = 0; j < 6; ++j)
#pragma unroll
        for (int i = 0; i < 4; ++i) v[4 * j + i] |= (v[26 + j] << (12 - 3 * i)) & 0x70007000u;
#pragma unroll
    for (int j = 0; j < 6; ++j) v[24 + j / 3] |= (v[26 + j] >> (1 + j % 3)) & (0x80008000u >> (1 + j % 3));
#pragma unroll
    for (int j = 0; j < 6; ++j) *(LAS u32x4*)(dst + j * 1024u + lane * 16u) = (u32x4){v[4 * j], v[4 * j + 1], v[4 * j + 2], v[4 * j + 3]};
    *(LAS u32x2*)(dst + Q_P6 + lane * 8u) = (u32x2){v[24], v[25]};
    const unsigned slot = __builtin_amdgcn_mbcnt_hi((unsigned)(om >> 32), __builtin_amdgcn_mbcnt_lo((unsigned)om, 0u));
    *(LAS unsigned char*)(dst + Q_LEN + lane) = (unsigned char)(ovf ? 128u + slot : tot);
    if (!ovf) {
        unsigned off = incl - tot;
#pragma unroll
        for (int s = 0; s < 4; ++s) {
            const unsigned long long x = (unsigned long long)seg[s] << (off & 31u);
            LAS unsigned* q = (LAS unsigned*)(dst + Q_STR + (off >> 5) * 4u);
            __hip_atomic_fetch_or(q, (unsigned)x, __ATOMIC_RELAXED, __HIP_MEMORY_SCOPE_WORKGROUP);
            __hip_atomic_fetch_or(q + 1, (unsigned)(x >> 32), __ATOMIC_RELAXED, __HIP_MEMORY_SCOPE_WORKGROUP);
            off += pos[s];
        }
    } else {
#pragma unroll
        for (int j = 0; j < 4; ++j) *(LAS u32x4*)(dst + xb + slot * 64u + j * 16u) = X[j];
    }
    return size;
}
#define RLOAD4(rs, vo, so) __builtin_bit_cast(u32x4, __builtin_amdgcn_raw_buffer_load_b128(rs, vo, so, 17))
struct QIn { u32x4 P[6]; u32x2 P6; unsigned lb, off, w4; u32x4 win, X[4]; };
__device__ __forceinline__ void q_read(const LAS unsigned char* src, unsigned lane, bool raw, QIn& q) {
    q.lb = raw ? 0u : *(const LAS unsigned char*)(src + Q_LEN + lane);
#pragma unroll
    for (int j = 0; j < 6; ++j) q.P[j] = *(const LAS u32x4*)(src + j * 1024u + lane * 16u);
    q.P6 = *(const LAS u32x2*)(src + Q_P6 + lane * 8u);
    const unsigned tot = q.lb >= 128u ? 0u : q.lb, incl = wave_scan(tot), total = (unsigned)__builtin_amdgcn_readlane((int)incl, 63);
    q.off = incl - tot;
    const LAS unsigned* w = (const LAS unsigned*)(src + Q_STR + (q.off >> 5) * 4u);
    q.win = (u32x4){w[0], w[1], w[2], w[3]}; q.w4 = w[4];
    if (raw || q.lb >= 128u) {
        const unsigned xo = raw ? Q_P6 + lane * 16u : min(Q_STR + (((total + 127u) >> 7) << 4) + (q.lb - 128u) * 64u, Q_BYTES - 64u), xs = raw ? 1024u : 16u;
        q.X[0] = *(const LAS u32x4*)(src + xo); q.X[1] = *(const LAS u32x4*)(src + xo + xs);
        q.X[2] = *(const LAS u32x4*)(src + xo + 32u); q.X[3] = *(const LAS u32x4*)(src + xo + 48u);
    }
}
__device__ __forceinline__ void q_finish(unsigned B, const QIn& q, u32x4 (&W)[8]) {
    unsigned v[32];
#pragma unroll
    for (int k = 0; k < 24; ++k) v[k] = q.P[k >> 2][k & 3];
    v[24] = q.P6.x; v[25] = q.P6.y;
#pragma unroll
    for (int j = 0; j < 6; ++j) {
        unsigned d = (v[24 + j / 3] << (1 + j % 3)) & 0x80008000u;
#pragma unroll
        for (int i = 0; i < 4; ++i) d |= (v[4 * j + i] & 0x70007000u) >> (12 - 3 * i);
        v[26 + j] = d;
    }
    u32x4 win;
    win.x = __builtin_amdgcn_alignbit(q.win.y, q.win.x, q.off); win.y = __builtin_amdgcn_alignbit(q.win.z, q.win.y, q.off);
    win.z = __builtin_amdgcn_alignbit(q.win.w, q.win.z, q.off); win.w = __builtin_amdgcn_alignbit(q.w4, q.win.w, q.off);
    const unsigned B1 = (B + 1u) * 0x04000400u;
#pragma unroll
    for (int s = 0; s < 4; ++s) {
        unsigned a = win.x << 1, used = 0;
#pragma unroll
        for (int k = 8 * s; k < 8 * s + 8; ++k) {
            const unsigned nz = v[k] | (v[k] >> 1);
            const unsigned l0 = ffbl((a & ~1u) | ((nz >> 10) & 1u));
            a >>= l0;
            const unsigned l1 = ffbl((a & ~1u) | ((nz >> 26) & 1u));
            a >>= l1;
            const unsigned len = l0 | (l1 << 16);
            used += len;
            const unsigned dq = __builtin_amdgcn_perm(0x00000006u, 0x00050400u, pk_min(len, 0x00040004u) | 0x0c000c00u) + pk_subs(len, 0x00040004u);
            W[k >> 2][k & 3] = (v[k] & 0x83ff83ffu) | (pk_sub(B1, (v[k] & 0x0c000c00u) + (dq << 10)) & 0x7c007c00u);
        }
        if (s < 3) {
            const unsigned sh = (used + (used >> 16)) & 31u;
            win.x = __builtin_amdgcn_alignbit(win.y, win.x, sh); win.y = __builtin_amdgcn_alignbit(win.z, win.y, sh);
            win.z = __builtin_amdgcn_alignbit(win.w, win.z, sh); win.w >>= sh;
        }
    }
}
__device__ __forceinline__ void q_patch(bool raw, const QIn& q, u32x4 (&W)[8]) {
    if (raw) {
#pragma unroll
        for (int j = 0; j < 6; ++j) W[j] = q.P[j];
        W[6] = q.X[0]; W[7] = q.X[1];
    } else if (q.lb >= 128u) {
#pragma unroll
        for (int k = 0; k < 32; ++k) W[k >> 2][k & 3] = __builtin_amdgcn_perm(q.X[k >> 3][(k >> 1) & 3], W[k >> 2][k & 3], (k & 1) ? 0x07020600u : 0x05020400u);
    }
}

constexpr unsigned C_HDR = 26624, C_STR = C_HDR + 128, C_EXC = 29696;
__device__ __forceinline__ unsigned c_encode(const u32x4 (&V)[32], unsigned B, u32x4 (&PK)[26], LAS unsigned char* scr, const gchar* pb, unsigned lane) {
    const unsigned B1 = (B + 1u) * 0x00010001u;
    unsigned seg[8], pos[8], hdr = 0;
#pragma unroll
    for (int g = 0; g < 4; ++g) {
        unsigned e2[16];
#pragma unroll
        for (int v = 0; v < 8; ++v) {
            u32x4 r;
#pragma unroll
            for (int c = 0; c < 4; ++c) r[c] = __builtin_amdgcn_alignbit(V[g * 8 + v][c], V[g * 8 + v][c], 31);
#pragma unroll
            for (int i = 0; i < 3; ++i) PK[g * 6 + ((3 * v + i) >> 2)][(3 * v + i) & 3] = __builtin_amdgcn_perm(r.w, r[i], 0x04020100u + ((unsigned)i << 24));
            e2[2 * v] = __builtin_amdgcn_perm(r.y, r.x, 0x0c070c03u); e2[2 * v + 1] = __builtin_amdgcn_perm(r.w, r.z, 0x0c070c03u);
        }
#pragma unroll
        for (int s = 2 * g; s < 2 * g + 2; ++s) {
            unsigned a = 0, p = 0, cw = 0;
#pragma unroll
            for (int j = 0; j < 8; ++j) {
                const unsigned dq = pk_sub(B1, e2[(s & 1) * 8 + j]), ix = pk_min(dq, 0x00070007u) | 0x0c000c00u;
                const unsigned len = __builtin_amdgcn_perm(0x05040201u, 0x00000003u, ix) + pk_subs(dq, 0x00070007u);
                cw |= __builtin_amdgcn_perm(0u, 0x03020100u, ix) << (2 * j);
                p += len & 0xffffu; a |= 1u << (p & 31u);
                p += len >> 16;     a |= 1u << (p & 31u);
            }
            seg[s] = a >> 1; pos[s] = p; PK[24 + (s >> 2)][s & 3] = cw;
        }
        if (max(pos[2 * g], pos[2 * g + 1]) > 31u) {
            u32x4 E[2];
#pragma unroll
            for (int v = 0; v < 8; ++v) E[v >> 2][v & 3] = __builtin_amdgcn_perm(e2[2 * v + 1], e2[2 * v], 0x06040200u);
            store16_sys(pb + (C_EXC + g * 2048u), lane * 32u, E[0]);
            store16_sys(pb + (C_EXC + g * 2048u + 16u), lane * 32u, E[1]);
            seg[2 * g] = seg[2 * g + 1] = pos[2 * g] = pos[2 * g + 1] = 0u; hdr |= 256u << g;
        }
    }
    const unsigned tot = (pos[0] + pos[1] + pos[2] + pos[3]) + (pos[4] + pos[5] + pos[6] + pos[7]);
    const unsigned incl = wave_scan(tot), total = (unsigned)__builtin_amdgcn_readlane((int)incl, 63);
    *(LAS u32x4*)(scr + 128u + lane * 16u) = (u32x4){0u, 0u, 0u, 0u}; *(LAS u32x4*)(scr + 1152u + lane * 16u) = (u32x4){0u, 0u, 0u, 0u};
    *(LAS unsigned short*)(scr + lane * 2u) = (unsigned short)(hdr | tot);
    unsigned off = incl - tot;
#pragma unroll
    for (int s = 0; s < 8; ++s) {
        const unsigned long long x = (unsigned long long)seg[s] << (off & 31u);
        LAS unsigned* q = (LAS unsigned*)(scr + 128u + (off >> 5) * 4u);
        __hip_atomic_fetch_or(q, (unsigned)x, __ATOMIC_RELAXED, __HIP_MEMORY_SCOPE_WORKGROUP);
        __hip_atomic_fetch_or(q + 1, (unsigned)(x >> 32), __ATOMIC_RELAXED, __HIP_MEMORY_SCOPE_WORKGROUP);
        off += pos[s];
    }
    return 128u + ((total + 7u) >> 3);
}
__device__ __forceinline__ size_t out_off(int q) { return (size_t)((q >> 4) * 128 + ((q >> 2) & 3) * 16 + (q & 3)) * (GN * 4); }
struct CDec { unsigned win[9], a, used, hdr, B1, E[32]; };
__device__ __forceinline__ void c_begin(CDec& d, const LAS unsigned char* scr, unsigned B) {
    const unsigned tot = d.hdr & 255u, off = wave_scan(tot) - tot;
#pragma unroll
    for (int i = 0; i < 9; ++i) d.win[i] = *(const LAS unsigned*)(scr + 128u + (off >> 5) * 4u + i * 4u);
#pragma unroll
    for (int i = 0; i < 8; ++i) d.win[i] = __builtin_amdgcn_alignbit(d.win[i + 1], d.win[i], off);
    d.B1 = (B + 1u) * 0x00010001u;
}
__device__ __forceinline__ void c_parse(CDec& d, const u32x4 (&PK)[26], const u32x4 (&X)[8], int H) {
    const int s = H >> 1;
    const unsigned cw = PK[24 + (s >> 2)][s & 3], nz = cw | (cw >> 1);
    const bool exc = d.hdr & (256u << (s >> 1));
    if (!(H & 1)) { d.a = d.win[0] << 1; d.used = 0; }
    unsigned e2[4];
#pragma unroll
    for (int j = (H & 1) * 4; j < (H & 1) * 4 + 4; ++j) {
        const unsigned l0 = ffbl((d.a & ~1u) | ((nz >> (2 * j)) & 1u));
        d.a >>= l0;
        const unsigned l1 = ffbl((d.a & ~1u) | ((nz >> (2 * j + 16)) & 1u));
        d.a >>= l1;
        const unsigned len = l0 | (l1 << 16);
        d.used += len;
        const unsigned dq = __builtin_amdgcn_perm(0x00000006u, 0x00050400u, pk_min(len, 0x00040004u) | 0x0c000c00u) + pk_subs(len, 0x00040004u);
        e2[j & 3] = pk_sub(d.B1, dq + ((cw >> (2 * j)) & 0x00030003u));
    }
#pragma unroll
    for (int i = 0; i < 2; ++i) d.E[2 * H + i] = exc ? X[s][(H & 1) * 2 + i] : __builtin_amdgcn_perm(e2[2 * i + 1], e2[2 * i], 0x06040200u);
    asm volatile("" : "+v"(d.E[2 * H]), "+v"(d.E[2 * H + 1]));
    if (H & 1) {
        const unsigned sh = exc ? 0u : d.used + (d.used >> 16);
#pragma unroll
        for (int i = 0; i < 7 - s; ++i) d.win[i] = __builtin_amdgcn_alignbit(d.win[i + 1], d.win[i], sh);
    }
}
__device__ __forceinline__ void c_store(const CDec& d, const u32x4 (&PK)[26], __amdgpu_buffer_rsrc_t ro, bool bad, unsigned ovo) {
    u32x4 A[32];
#pragma unroll
    for (int q = 0; q < 12; ++q) A[q] = RLOAD4(ro, ovo, (unsigned)out_off(q));
#pragma unroll
    for (int q = 0; q < 32; ++q) {
        if (q + 12 < 32) A[q + 12] = RLOAD4(ro, ovo, (unsigned)out_off(q + 12));
        unsigned P[3];
#pragma unroll
        for (int i = 0; i < 3; ++i) P[i] = PK[(3 * q + i) >> 2][(3 * q + i) & 3];
        u32x4 r;
        r.x = __builtin_amdgcn_perm(d.E[q], P[0], 0x04020100u); r.y = __builtin_amdgcn_perm(d.E[q], P[1], 0x05020100u); r.z = __builtin_amdgcn_perm(d.E[q], P[2], 0x06020100u);
        r.w = __builtin_amdgcn_perm(P[1], P[0], 0x0c0c0703u) | __builtin_amdgcn_perm(d.E[q], P[2], 0x07030c0cu);
#pragma unroll
        for (int c = 0; c < 4; ++c) r[c] = bad ? 0x7fc00000u : __builtin_amdgcn_alignbit(r[c], r[c], 1);
        const f32x4 o = __builtin_bit_cast(f32x4, r) + __builtin_bit_cast(f32x4, A[q]);
        __builtin_amdgcn_raw_buffer_store_b128(__builtin_bit_cast(u32x4, o), ro, ovo, (unsigned)out_off(q), OUT_AUX);
        __builtin_amdgcn_sched_barrier(0);
    }
}

constexpr int NITEM = 240;
__device__ __forceinline__ void item_decode(int i, int d, int s, int& isW, int& j, int& u) {
    constexpr int R0 = 12 + NE, R1 = 18 - NE;
    if (i < 8 * R0) {
        j = i / R0; int r = i - j * R0;
        if (i >= 6 && i < R0) r = i < 12 ? i + NE : i - 6;
        isW = r >= 6 && r < 6 + NE; u = isW ? r - 6 : r < 6 ? 2 * r : 2 * (r - 6 - NE) + 1;
        if (!isW) { if (u >= 2 * min(d, s)) u += 2; if (u >= 2 * max(d, s)) u += 2; }
    } else { const int r = i - 8 * R0; j = r / R1; const int t = r - j * R1; isW = t < 16 - NE; u = isW ? NE + t : 2 * d + t - (16 - NE); }
}
constexpr int NENC = 32 + 14 * (12 - NE);
constexpr int NTASK = 1152, NTASK_DYN0 = 200 + 2 * NENC, NTASK_DYN = 3 * NENC + NTASK - NTASK_DYN0;
__device__ __forceinline__ void task_decode(int t, int& isW, int& j, int& u, int& dd) {
    constexpr int R0 = 16 + 8 * NE, R1 = 8 * (16 - NE);
    if (t < 8 * R0) { j = t / R0; int r = t - j * R0; isW = r >= 16; if (isW) { r -= 16; u = r >> 3; dd = r & 7; } else { u = r < 8 ? 2 * r : 2 * r - 15; dd = 7; } }
    else { int r = t - 8 * R0; j = r / R1; r -= j * R1; isW = 1; u = NE + (r >> 3); dd = r & 7; }
}

__device__ __forceinline__ void task_load(const Params& p, int rank, int i, f32x4 (&L)[16]) {
    const bool valid = i < NTASK;
    int isW, j, u, dd; task_decode(valid ? i : 0, isW, j, u, dd);
    const unsigned t = threadIdx.x;
    const gchar* b = uni(isW ? (const char*)p.w + ((size_t)(j * 64) * GN + ((rank + 1 + dd) & 7) * NL + u * 256) * 4 : (const char*)p.x + ((size_t)(u * 256) * KS + j * 64) * 4);
    unsigned vo = isW ? (t >> 6) * (4u * GN * 4u) + (t & 63u) * 16u : (t >> 3) * (KS * 4u) + ((t & 7u) ^ ((t >> 4) & 7u)) * 16u;
    if (!valid) vo = 0;
    const unsigned s0 = isW ? GN * 4u : 128u, s1 = isW ? 2u * GN * 4u : 64u * KS * 4u, s2 = isW ? 32u * GN * 4u : 128u * KS * 4u, s3 = isW ? 16u * GN * 4u : 32u * KS * 4u;
#pragma unroll
    for (int n = 0; n < 16; ++n)
        L[n] = __builtin_nontemporal_load((const GAS f32x4*)(b + ((n & 1) * s0 + ((n >> 1) & 1) * s1 + ((n >> 2) & 1) * s2 + (n >> 3) * s3) + vo));
}

__device__ __forceinline__ unsigned pk2(float a, float b, int e) {
    typedef _Float16 h2 __attribute__((ext_vector_type(2)));
    h2 h; h.x = (_Float16)__builtin_amdgcn_ldexpf(a, e); h.y = (_Float16)__builtin_amdgcn_ldexpf(b, e);
    return __builtin_bit_cast(unsigned, h);
}

#define POLL_SLEEP 8
#define FLAG_SLEEP 16
#define PUSH_REL 1u
#define SEND_PRIO 3
#define MINI_LEAD 48
#define LOC_W 8
#define OUT_WAVES 4
#define WIDE_FROM 236u
#define OUT_W2 3
#define OUT_W 2
#define REL_AT 24
#define LINK_EXTRA 12u
#define PULL_N 7
#define NPUSH 1
#define PUSH_W 0
#define START_NOW 112
#define START_DELAY 4
#define START_SPREAD 12
#define STR2(x) #x
#define STR(x) STR2(x)
#define LBAR do { asm volatile("s_waitcnt lgkmcnt(0)" ::: "memory"); __builtin_amdgcn_s_barrier(); asm volatile("" ::: "memory"); } while (0)

typedef volatile LAS unsigned vl32;
typedef volatile LAS u32x4 vl128;
#define LGKM0 asm volatile("s_waitcnt lgkmcnt(0)" ::: "memory")
__device__ __forceinline__ u32x4 lds_wait4(LAS unsigned char* lds, unsigned off, unsigned v) {
    for (;;) {
        const u32x4 q = *(vl128*)(lds + off);
        if (rfl(min(min(q.x, q.y), min(q.z, q.w))) >= v) { asm volatile("" ::: "memory"); return q; }
        __builtin_amdgcn_s_sleep(1);
    }
}
__device__ __forceinline__ unsigned lds_wait1(LAS unsigned char* lds, unsigned off, unsigned v) {
    unsigned x;
    while ((x = rfl(*(vl32*)(lds + off))) < v) __builtin_amdgcn_s_sleep(1);
    asm volatile("" ::: "memory");
    return x;
}
__device__ __forceinline__ void task_stats(LAS unsigned char* lds, const f32x4 (&L)[16], unsigned k) {
    unsigned mu = 0; float sm = 0.f;
#pragma unroll
    for (int n = 0; n < 16; ++n) {
        const u32x4 q = __builtin_bit_cast(u32x4, L[n]) & 0x7fffffffu; mu = max(max(mu, q.x), max(q.y, max(q.z, q.w)));
        const f32x4 f = __builtin_bit_cast(f32x4, q); sm += (f.x + f.y) + (f.z + f.w);
    }
    const unsigned mb = wave_max(mu >> 23), em = wave_mean_exp(sm * (2.32f / 4096.f));
    LGKM0;
    if ((threadIdx.x & 63) == 63) *(vl32*)(lds + L_AMAX + (k & 1u) * 16 + (threadIdx.x >> 6) * 4) = ((k + 1u) << 16) | (em << 8) | mb;
}

__device__ __forceinline__ unsigned img_off(int isW, unsigned q, int c) {
    return (isW ? q * 2048u + (c >> 2) * 8192u + (c & 1) * 1024u : q * 1024u + (c >> 2) * 4096u + (c & 1) * 8192u) + ((c >> 1) & 1) * HTB;
}
__device__ __forceinline__ int conv_step(const Params& p, LAS unsigned char* lds, gu32* tick, int rank, unsigned k, f32x4 (&L)[16], const f32x4 (&Ln)[16]) {
    unsigned t = threadIdx.x; asm volatile("" : "+v"(t));
    const u32x4 am = lds_wait4(lds, L_AMAX + (k & 1u) * 16, (k + 1u) << 16);
    const int i = (int)rfl(*(vl32*)(lds + L_ITEM + (k & 15u) * 4)), i1 = (int)rfl(*(vl32*)(lds + L_ITEM + ((k + 1u) & 15u) * 4)), i2 = (int)rfl(*(vl32*)(lds + L_ITEM + ((k + 2u) & 15u) * 4));
    const bool valid = i < NTASK;
    if (valid) {
        const unsigned mb = rfl(max(max(am.x & 255u, am.y & 255u), max(am.z & 255u, am.w & 255u))), em = (rfl(max(max(am.x, am.y), max(am.z, am.w))) >> 8) & 255u;
        const int cls = (int)(mb + 2u) >> 4;
        const int e = 128 - 16 * cls;
        int isW, j, u, dd; task_decode(i, isW, j, u, dd);
        u32x4 H[8];
#pragma unroll
        for (int h = 0; h < 2; ++h) {
            const f32x4* Q = L + 8 * h; u32x4* G = H + 4 * h;
            if (isW) {
#pragma unroll
                for (int c = 0; c < 4; ++c) { G[c].x = pk2(Q[0][c], Q[1][c], e); G[c].y = pk2(Q[2][c], Q[3][c], e); G[c].z = pk2(Q[4][c], Q[5][c], e); G[c].w = pk2(Q[6][c], Q[7][c], e); }
            } else {
#pragma unroll
                for (int c = 0; c < 4; ++c) { G[c].x = pk2(Q[2 * c].x, Q[2 * c].y, e); G[c].y = pk2(Q[2 * c].z, Q[2 * c].w, e); G[c].z = pk2(Q[2 * c + 1].x, Q[2 * c + 1].y, e); G[c].w = pk2(Q[2 * c + 1].z, Q[2 * c + 1].w, e); }
            }
        }
        const unsigned q = rfl(t >> 6);
        if (k >= 4) lds_wait1(lds, L_CONS + q * 4, k - 3u);
#pragma unroll
        for (int c = 0; c < 8; ++c) *(LAS u32x4*)(lds + (k & 3u) * UNIT + q * Q_BYTES + c * 1024u + (t & 63u) * 16u) = H[c];
        LGKM0;
        if ((t & 63u) == 0) *(vl32*)(lds + L_QI + (k & 3u) * 16 + q * 4) = ((k + 1u) << 16) | ((unsigned)min(max((int)min(em, mb) + e - 112, 2), 31) << 5) | (unsigned)cls;
    }
    unsigned tk = NTASK_DYN - 1;
    gu32* q = tick; asm volatile("" : "+v"(q));
    if (t == 0 && tick && valid) tk = __hip_atomic_fetch_add(q, 1u, __ATOMIC_RELAXED, __HIP_MEMORY_SCOPE_AGENT);
    task_load(p, rank, i2, L);
    __builtin_amdgcn_s_waitcnt(0x4F70);
    asm volatile("" : "+v"(tk));
    if (t == 0) *(vl32*)(lds + L_ITEM + ((k + 3u) & 15u) * 4) = (unsigned)NTASK_DYN0 + tk % (unsigned)NTASK_DYN;
    task_stats(lds, Ln, k + 1u);
    return i1;
}

__device__ __forceinline__ gu32* stage_unit(gchar* myws, int srel, int isW, int j, int u, const gchar*& ub) {
    const int idx = j * 8 + ((srel + 1) & 7);
    ub = myws + WS_STAGE + (size_t)(idx * 32 + isW * 16 + u) * UNIT;
    return (gu32*)(myws + WS_FLAG) + (isW * 16 + u) * NKT + idx;
}
__device__ __forceinline__ void lds_fetch8(const LAS unsigned char* src, unsigned lane, u32x4 (&H)[8]) {
#pragma unroll
    for (int c = 0; c < 8; ++c) H[c] = *(const LAS u32x4*)(src + c * 1024u + lane * 16u);
}

__device__ __forceinline__ unsigned produce(const Params& p, LAS unsigned char* lds, gchar* myws, int rank, int first, int second, gu32* tick) {
    unsigned call;
    if (rfl(threadIdx.x >> 6) < 4) {
        f32x4 RA[16], RB[16];
        const int dly = first >= NTASK ? 0 : first >= START_NOW ? START_DELAY + (first - START_NOW) * START_SPREAD / 128 : 0;
        for (int z = 0; z < dly; ++z) __builtin_amdgcn_s_sleep(32);
        task_load(p, rank, first, RA);
        LBAR;
        task_stats(lds, RA, 0);
        task_load(p, rank, second, RB);
        int nx; unsigned k = 0;
        do { conv_step(p, lds, tick, rank, k, RA, RB); nx = conv_step(p, lds, tick, rank, k + 1u, RB, RA); k += 2; } while (nx < NTASK);
        LBAR;
        call = rfl(*(vl32*)(lds + L_CALL));
    } else {
        const unsigned ts = threadIdx.x - 256u, lane = ts & 63u, q = rfl(ts >> 6);
        constexpr unsigned I0 = (L_ITEM - L_CTL) / 4;
        if (ts < (L_ZERO_END - L_CTL) / 4) *(vl32*)(lds + L_CTL + ts * 4) = ts == I0 ? (unsigned)first : ts == I0 + 1 ? (unsigned)second : ts == I0 + 2 ? (unsigned)second + NENC : 0u;
        unsigned tk = 0;
        gu32* tq = (gu32*)(myws + WS_TICKET); asm volatile("" : "+v"(tq));
        if (ts == 0) tk = __hip_atomic_fetch_add(tq, 1u, __ATOMIC_RELAXED, __HIP_MEMORY_SCOPE_AGENT);
        const gchar* peer = (const gchar*)p.ws[(rank + 1 + min(lane, 6u)) & 7];
        LBAR;
        asm volatile("" : "+v"(tk));
        if (ts == 0) *(vl32*)(lds + L_CALL) = (tk / GRID + 1u) & 0xffffffu;
        call = lds_wait1(lds, L_CALL, 1u);
        for (unsigned k = 0;; ++k) {
            const int i = (int)rfl(*(vl32*)(lds + L_ITEM + (k & 15u) * 4));
            if (i >= NTASK) break;
            const unsigned st = lds_wait1(lds, L_QI + (k & 3u) * 16 + q * 4, (k + 1u) << 16);
            int isW, j, u, dd; task_decode(i, isW, j, u, dd);
            LAS unsigned char* src = lds + (k & 3u) * UNIT + q * Q_BYTES;
            u32x4 H[8];
            lds_fetch8(src, lane, H);
            const gchar* ub; gu32* sf = stage_unit(myws, 7, isW, j, u, ub);
            const bool peers = isW ? dd != 7 : (u >> 1) != rank, slice = isW == peers;
            const int s2 = (isW ? dd * 16 + u : 112 + (u & 1)) * 8 + j;
            if (dd == 7) {
                ub = uni(ub);
#pragma unroll
                for (int c = 0; c < 8; ++c) store16_sys(ub + img_off(isW, q, c), lane * 16u, H[c]);
            }
            if (slice) {
                ub = uni(myws + WS_STAGE2 + (size_t)s2 * UNIT);
#pragma unroll
                for (int c = 0; c < 8; ++c) store16_sys(ub + img_off(isW, q, c), lane * 16u, H[c]);
            }
            unsigned size = Q_BYTES;
            if (peers) {
                const gchar* pb = uni(myws + pin_off(call & 1u, isW, dd, j, u) + q * Q_BYTES);
                size = q_encode(H, (st >> 5) & 31u, src, lane); asm volatile("" ::: "memory"); lds_fetch8(src, lane, H);
#pragma unroll
                for (int c = 0; c < 8; ++c) if (c < 6 || c * 1024u + lane * 16u < size) store16_sys(pb + c * 1024u, lane * 16u, H[c]);
            }
            LGKM0;
            if (lane == 0) *(vl32*)(lds + L_CONS + q * 4) = k + 1u;
            asm volatile("s_waitcnt vmcnt(0)" ::: "memory");
            const unsigned add = (1u << 30) | (((size - Q_LEN + 63u) >> 6) << (10u + 5u * q));
            unsigned old = 0;
            if (lane == 0) old = __hip_atomic_fetch_add((LAS unsigned*)(lds + L_ECNT + (k & 7u) * 4), add, __ATOMIC_RELAXED, __HIP_MEMORY_SCOPE_WORKGROUP);
            old = rfl(old);
            if ((old >> 30) == 3u) {
                if (lane == 0) *(vl32*)(lds + L_ECNT + (k & 7u) * 4) = 0u;
                const unsigned fi = (call & 1u) * 8 + (6u - lane);
                if (peers && (isW ? lane == (unsigned)dd : lane < 7u))
                    __hip_atomic_store((GAS unsigned long long*)(peer + WS_AFLAG + ((fi * 2 + isW) * 128 + j * 16 + u) * 8),
                                       ((unsigned long long)call << 32) | (old + add) | (st & 1023u), __ATOMIC_RELAXED, __HIP_MEMORY_SCOPE_SYSTEM);
                if (lane == 0 ? dd == 7 : lane == 1 && slice) __hip_atomic_store(lane ? (gu32*)(myws + WS_FLAG2) + s2 : sf, (call << 8) | (st & 31u), __ATOMIC_RELAXED, __HIP_MEMORY_SCOPE_SYSTEM);
            }
        }
        LBAR;
    }
    return call;
}

__device__ __forceinline__ unsigned push_first(const Params& p, LAS unsigned char* lds, gchar* myws, int rank, int link, int item) {
    const unsigned t = threadIdx.x;
    const int d = (rank + 1 + link) & 7;
    unsigned tk = 0;
    gu32* tq = (gu32*)(myws + WS_TICKET); asm volatile("" : "+v"(tq));
    if (t == 0) tk = __hip_atomic_fetch_add(tq, 1u, __ATOMIC_RELAXED, __HIP_MEMORY_SCOPE_AGENT);
    f32x4 L[NPUSH][8]; int isW[NPUSH];
#pragma unroll
    for (int r = 0; r < NPUSH; ++r) {
        int j, u; item_decode(item + 8 * r, d, rank, isW[r], j, u);
        const gchar* b = uni(isW[r] ? (const char*)p.w + (size_t)(d * NL + u * 256) * 4 : (const char*)p.x + (size_t)(u * 256) * KS * 4);
        const unsigned vo = isW[r] ? (t >> 6) * (4u * GN * 4u) + (t & 63u) * 16u : (t >> 3) * (KS * 4u) + ((t & 7u) ^ ((t >> 4) & 7u)) * 16u;
        const unsigned s0 = isW[r] ? GN * 4u : 128u, s1 = isW[r] ? 2u * GN * 4u : 64u * KS * 4u, s2 = isW[r] ? 32u * GN * 4u : 128u * KS * 4u;
        if (r) __builtin_amdgcn_s_waitcnt(0x0F70);
#pragma unroll
        for (int n = 0; n < 8; ++n) L[r][n] = *(const GAS f32x4*)(b + ((n & 1) * s0 + ((n >> 1) & 1) * s1 + (n >> 2) * s2) + vo);
    }
    gchar* dws = uni(p.ws[d]);
    if (t >= 8 && t < (L_ZERO_END - L_CTL) / 4 && t != (L_CALL - L_CTL) / 4) *(vl32*)(lds + L_CTL + t * 4) = 0u;
    asm volatile("" : "+v"(tk));
    if (t == 0) *(vl32*)(lds + L_CALL) = (tk / GRID + 1u) & 0xffffffu;
    unsigned call = 0, cl[NPUSH];
#pragma unroll
    for (int r = 0; r < NPUSH; ++r) {
        unsigned mu = 0;
#pragma unroll
        for (int n = 0; n < 8; ++n) { const u32x4 q = __builtin_bit_cast(u32x4, L[r][n]) & 0x7fffffffu; mu = max(max(mu, q.x), max(q.y, max(q.z, q.w))); }
        mu = wave_max(mu >> 23);
        const unsigned am = r ? L_HDR : L_AMAX;
        if ((t & 63u) == 0) *(vl32*)(lds + am + (t >> 6) * 4) = mu;
        LBAR;
        const u32x4 a0 = *(vl128*)(lds + am), a1 = *(vl128*)(lds + am + 16);
        call = rfl(*(vl32*)(lds + L_CALL));
        cl[r] = (rfl(max(max(max(a0.x, a0.y), max(a0.z, a0.w)), max(max(a1.x, a1.y), max(a1.z, a1.w)))) + 2u) >> 4;
        const int e = 128 - 16 * (int)cl[r];
        const gchar* ib = uni(dws + WS_INBOX + (size_t)(((call & 1u) * 7 + (6 - link)) * 16 + item + 8 * r) * UNIT);
        const unsigned so = isW[r] ? (t >> 6) * 2048u + (t & 63u) * 16u : t * 16u, c1 = isW[r] ? 1024u : 8192u;
#pragma unroll
        for (int c = 0; c < 4; ++c) {
            const f32x4* Q = L[r];
            u32x4 H;
            if (isW[r]) { H.x = pk2(Q[0][c], Q[1][c], e); H.y = pk2(Q[2][c], Q[3][c], e); H.z = pk2(Q[4][c], Q[5][c], e); H.w = pk2(Q[6][c], Q[7][c], e); }
            else { H.x = pk2(Q[2 * c].x, Q[2 * c].y, e); H.y = pk2(Q[2 * c].z, Q[2 * c].w, e); H.z = pk2(Q[2 * c + 1].x, Q[2 * c + 1].y, e); H.w = pk2(Q[2 * c + 1].z, Q[2 * c + 1].w, e); }
            if ((r || c > 1) && (r < NPUSH - 1 || rfl(t >> 6) != 1u)) asm volatile("s_waitcnt vmcnt(" STR(PUSH_W) ")" ::: "memory");
            store16_sys(ib + ((c >> 1) * HTB + (c & 1) * c1), so, H);
        }
    }
    asm volatile("s_waitcnt vmcnt(0)" ::: "memory");
    unsigned old = 0;
    if ((t & 63u) == 0) old = __hip_atomic_fetch_add((LAS unsigned*)(lds + L_ECNT), 1u, __ATOMIC_RELAXED, __HIP_MEMORY_SCOPE_WORKGROUP);
    if (rfl(old) == 7u && (t & 63u) < NPUSH) {
        __hip_atomic_store((GAS unsigned long long*)(dws + WS_IFLAG) + ((call & 1u) * 8 + (6 - link)) * 16 + item + 8 * (t & 63u), ((unsigned long long)call << 32) | ((t & 63u) ? cl[NPUSH - 1] : cl[0]),
                           __ATOMIC_RELAXED, __HIP_MEMORY_SCOPE_SYSTEM);
    }
    return call;
}

template <int SLEEP> __device__ __forceinline__ bool flag_wait(const gchar* f, gu32* tmo, unsigned call, unsigned& info) {
    const unsigned long long t0 = __builtin_amdgcn_s_memrealtime();
    for (;;) {
        const unsigned long long fv = __hip_atomic_load((const GAS unsigned long long*)f, __ATOMIC_RELAXED, __HIP_MEMORY_SCOPE_SYSTEM);
        info = rfl((unsigned)fv);
        if (rfl((unsigned)(fv >> 32)) >= call) break;
        if (rfl(ld_agt(tmo)) == call || __builtin_amdgcn_s_memrealtime() - t0 > DEADLINE) {
            if ((threadIdx.x & 63u) == 0) __hip_atomic_store(tmo, call, __ATOMIC_RELAXED, __HIP_MEMORY_SCOPE_AGENT);
            return false;
        }
        __builtin_amdgcn_s_sleep(SLEEP);
    }
    asm volatile("" ::: "memory");
    return true;
}

__device__ __forceinline__ void fetch_inputs(const Params& p, LAS unsigned char* lds, gchar* myws, int rank, unsigned call, int srel, int first) {
    const unsigned lane = threadIdx.x & 63u, wv = rfl(threadIdx.x >> 6);
    gu32* tmo = (gu32*)(myws + WS_TMO);
    const unsigned fi = (call & 1u) * 8 + srel;
    const int src = (rank + 1 + srel) & 7;
    if (wv == 1) {
        unsigned k = 0;
        for (;;) {
            unsigned tk = 0;
            if (lane == 0) tk = __hip_atomic_fetch_add((gu32*)(myws + WS_LINKQ + srel * 128), 1u, __ATOMIC_RELAXED, __HIP_MEMORY_SCOPE_AGENT);
            const int i = 8 * NPUSH + (int)(rfl(tk) % (NITEM + 8u - 8u * NPUSH));
            unsigned info = 0; bool ok = true;
            if (i < NITEM) { int isW, j, u; item_decode(i, rank, src, isW, j, u); ok = flag_wait<2>(myws + WS_AFLAG + ((fi * 2 + isW) * 128 + j * 16 + u) * 8, tmo, call, info); }
            if (ok) { if (lane == 0) *(vl128*)(lds + L_DESC + (k & 7u) * 16) = (u32x4){k + 1u, (unsigned)i, info, 0u}; ++k; }
            if (i >= NITEM) break;
            if (ok) lds_wait1(lds, L_LST, k);
        }
    } else if (wv == 0) {
        const gchar* sws = uni(p.ws[src]);
        __builtin_amdgcn_s_setprio(SEND_PRIO);
#define LWAIT(off, v, n) do { if (rfl(*(vl32*)(lds + (off))) < (v)) { asm volatile("s_waitcnt vmcnt(0)" ::: "memory"); land = (n); if (lane == 0) *(vl32*)(lds + L_LAND) = land; lds_wait1(lds, off, v); } } while (0)
        unsigned k = 0, land = 0;
        for (;; ++k) {
            LWAIT(L_DESC + (k & 7u) * 16, k + 1u, 4u * k);
            const u32x4 de = *(vl128*)(lds + L_DESC + (k & 7u) * 16);
            const int i = (int)rfl(de.y); const unsigned info = rfl(de.z);
            if (i >= NITEM) break;
            if (lane == 0) *(vl32*)(lds + L_LST) = k + 1u;
            int isW, j, u; item_decode(i, rank, src, isW, j, u);
            const gchar* ub = uni(sws + pin_off(call & 1u, isW, 6 - srel, j, u));
#pragma unroll
            for (unsigned q = 0; q < 4; ++q) {
                if (k >= 4) LWAIT(L_FREE + ((k & 3u) * 4 + q) * 4, k - 3u, 4u * k + q);
                const unsigned size = Q_LEN + ((info >> (10u + 5u * q)) & 31u) * 64u, dst = (unsigned)(uintptr_t)lds + (k & 3u) * UNIT + q * Q_BYTES;
#pragma unroll
                for (int c = 0; c < 8; ++c) {
                    if (c < 6 || c * 1024u + lane * 16u < size) glds16_sys(dst + c * 1024u, lane * 16u, ub + (q * Q_BYTES + c * 1024u));
                    asm volatile("s_waitcnt vmcnt(" STR(PULL_N) ")" ::: "memory");
                    if (c == (PULL_N - 1) % 7 && 4u * k + q > land + (PULL_N > 7)) { land = 4u * k + q - (PULL_N > 7); if (lane == 0) *(vl32*)(lds + L_LAND) = land; }
                }
            }
        }
        if (lane == 0) __hip_atomic_fetch_add((gu32*)(myws + WS_OREL + srel * 128), PUSH_REL, __ATOMIC_RELAXED, __HIP_MEMORY_SCOPE_AGENT);
        asm volatile("s_waitcnt vmcnt(0)" ::: "memory");
        if (lane == 0) *(vl32*)(lds + L_LAND) = 4u * k;
        __builtin_amdgcn_s_setprio(0);
    } else {
        unsigned inf0;
        for (int it = first; it < 8 * NPUSH; it += 8)
            if (wv >= 4 && flag_wait<2>(myws + WS_IFLAG + (fi * 16 + it) * 8, tmo, call, inf0)) {
                const __amdgpu_buffer_rsrc_t rs = __builtin_amdgcn_make_buffer_rsrc((void*)uni(myws + WS_INBOX + (size_t)(((call & 1u) * 7 + srel) * 16 + it) * UNIT), 0, UNIT, 0x00020000);
                u32x4 W[8];
#pragma unroll
                for (int c = 0; c < 8; ++c) W[c] = RLOAD4(rs, lane * 16u, (wv - 4u) * Q_BYTES + c * 1024u);
                int isW, j, u; item_decode(it, rank, src, isW, j, u);
                const gchar* ub; gu32* f = stage_unit(myws, srel, isW, j, u, ub); ub = uni(ub);
                __builtin_amdgcn_s_waitcnt(0x0F70);
#pragma unroll
                for (int c = 0; c < 8; ++c) store16_sys(ub + ((wv - 4u) * Q_BYTES + c * 1024u), lane * 16u, W[c]);
                asm volatile("s_waitcnt vmcnt(0)" ::: "memory");
                if (lane == 0 && __hip_atomic_fetch_add((LAS unsigned*)(lds + L_QI + (it >> 3) * 4), 1u, __ATOMIC_RELAXED, __HIP_MEMORY_SCOPE_WORKGROUP) == 3u)
                    __hip_atomic_store(f, (call << 8) | (inf0 & 31u), __ATOMIC_RELAXED, __HIP_MEMORY_SCOPE_SYSTEM);
            }
        for (unsigned n = wv - 2u;; n += 6u) {
            const unsigned k = n >> 2, q = n & 3u;
            unsigned kd = k;
            for (;;) {
                const u32x4 d0 = *(vl128*)(lds + L_DESC + (k & 7u) * 16), d1 = *(vl128*)(lds + L_DESC + ((k + 7u) & 7u) * 16);
                if (rfl(d0.x) >= k + 1u) break;
                if (k > 0 && rfl(d1.x) == k && rfl(d1.y) >= (unsigned)NITEM) { kd = k - 1u; break; }
                __builtin_amdgcn_s_sleep(1);
            }
            const u32x4 de = *(vl128*)(lds + L_DESC + (kd & 7u) * 16);
            const int i = (int)rfl(de.y); const unsigned info = rfl(de.z);
            if (i >= NITEM) break;
            lds_wait1(lds, L_LAND, n + 1u);
            const bool raw = ((info >> (10u + 5u * q)) & 31u) == 24u;
            QIn Q;
            q_read(lds + (k & 3u) * UNIT + q * Q_BYTES, lane, raw, Q);
            LGKM0;
            if (lane == 0) *(vl32*)(lds + L_FREE + ((k & 3u) * 4 + q) * 4) = k + 1u;
            u32x4 W[8];
            if (!raw) q_finish((info >> 5) & 31u, Q, W);
            q_patch(raw, Q, W);
            int isW, j, u; item_decode(i, rank, src, isW, j, u);
            const gchar* ub; gu32* f = stage_unit(myws, srel, isW, j, u, ub); ub = uni(ub);
#pragma unroll
            for (int c = 0; c < 8; ++c) store16_sys(ub + img_off(isW, q, c), lane * 16u, W[c]);
            asm volatile("s_waitcnt vmcnt(0)" ::: "memory");
            if (lane == 0) {
                const unsigned old = __hip_atomic_fetch_add((LAS unsigned*)(lds + L_UCNT + (k & 7u) * 4), 1u, __ATOMIC_RELAXED, __HIP_MEMORY_SCOPE_WORKGROUP);
                if ((old & 3u) == 3u) __hip_atomic_store(f, (call << 8) | (info & 31u), __ATOMIC_RELAXED, __HIP_MEMORY_SCOPE_SYSTEM);
            }
        }
    }
    LBAR;
}

__global__ void __launch_bounds__(NTHR, 2) gemm_rs_kernel(const Params p) {
    extern __shared__ __attribute__((aligned(16))) unsigned char smem[];
    LAS unsigned char* lds = (LAS unsigned char*)smem;
    const int tid = threadIdx.x;
    const int rank = (int)rfl(p.rank), bid = (int)rfl(blockIdx.x);
    gchar* myws = uni(p.ws[rank]);
    STAMP(0);
    const int xcd = bid & 7, slot = bid >> 3;
    gu32* tmo = (gu32*)(myws + WS_TMO);

    int pm, pn;
    {
        int ar = -1;
        if (slot < 11) {
            const int c = slot < 7 ? xcd * 7 + slot : xcd * 4 + slot - 7, o = slot < 7 ? c - 28 : c;
            if (c < 28) { ar = c >> 1; pn = (slot < 7 ? 14 : 12) + (c & 1); } else { pm = 2 * rank + (o & 1); pn = o >> 1; }
        } else {
            const int li = slot - 11, q = li / 7;
            pn = 3 * (xcd & 3) + q; ar = 2 * (li - q * 7) + (xcd >> 2);
        }
        if (ar >= 0) pm = ar + (ar >= 2 * rank ? 2 : 0);
    }
    const int enc = slot < 7 ? -1 : slot < 11 ? bid - 56 : pn >= NE ? 32 + (pn - NE) * 14 + pm - (pm > 2 * rank ? 2 : 0) : -1;
    unsigned call;
    if (slot < 7) { call = push_first(p, lds, myws, rank, slot, xcd); STAMP(5); fetch_inputs(p, lds, myws, rank, call, slot, xcd); }
    else { call = produce(p, lds, myws, rank, slot < 11 ? bid + 24 : slot < 21 ? bid - 88 : bid - 56, enc >= 0 ? 200 + enc : NTASK, enc >= 0 ? (gu32*)(myws + WS_TASKQ) : nullptr); STAMP(5); }
    STAMP(1);

    const int wid = rfl(tid >> 6), wr = wid >> 2, wc = wid & 3;
#define LANE_VALS int lane = tid; asm volatile("" : "+v"(lane)); lane &= 63; const int fr = lane & 15, fq = lane >> 4
    const int dest = pm >> 1, link = (dest - rank - 1) & 7;
    const bool early = slot >= 11 && pn < NE, worker = early && ((pm ^ pn) & 1);
    const int mini0 = enc >= 0 ? enc : early && !worker ? NENC + pn * 7 + link : 224, ministep = enc >= 0 ? 224 : 7 * NE;
    const int njob = 1 + (mini0 < 224) + (mini0 + ministep < 224) + (mini0 + 2 * ministep < 224), jmain = enc >= 0 ? 1 : 0;
    static_assert(NENC + 21 * NE >= 224, "every own-slice tile has a workgroup");
    gchar* dws = uni(p.ws[dest]);
    LAS unsigned char* scr = lds + L_SCR + wid * 3072;

#define KIDX(t) ((t) + ((((t) + 6 - link) * kmag) >> 8))
#define SA(b, h) (((b) * 2 + (h)) * HTB)
#define SB(b, h) ((4 + (b) * 2 + (h)) * HTB)
#define STAGE(bufoff, gbase) do { _Pragma("unroll") for (int _i = 0; _i < 2; ++_i) \
        __builtin_amdgcn_global_load_lds((const gu32*)((gbase) + voff[_i]), (LAS unsigned*)(lds + (bufoff) + ldsw + _i * 8192), 16, 0, GLDS_AUX); } while (0)
#define LDA(dst, b, h) do { _Pragma("unroll") for (int m = 0; m < 4; ++m) { dst[m][0] = *(const LAS h16x8*)(lds + SA(b, h) + aoff0 + m * 2048); dst[m][1] = *(const LAS h16x8*)(lds + SA(b, h) + aoff1 + m * 2048); } } while (0)
#define LDB(dst, b, h) do { _Pragma("unroll") for (int n = 0; n < 2; ++n) _Pragma("unroll") for (int k = 0; k < 2; ++k) dst[n][k] = *(const LAS h16x8*)(lds + SB(b, h) + boff + n * 1024 + k * 8192); } while (0)
#define MMA(ai, bj, At, Bt) do { __builtin_amdgcn_s_setprio(1); _Pragma("unroll") for (int m = 0; m < 4; ++m) _Pragma("unroll") for (int n = 0; n < 2; ++n) _Pragma("unroll") for (int k = 0; k < 2; ++k) \
        acc[ai][bj][m][n] = __builtin_amdgcn_mfma_f32_16x16x32_f16(At[m][k], Bt[n][k], acc[ai][bj][m][n], 0, 0, 0); __builtin_amdgcn_s_setprio(0); } while (0)
#define WAIT_V(n) asm volatile("s_waitcnt vmcnt(" #n ")" ::: "memory")
#define WAIT_L(n) asm volatile("s_waitcnt lgkmcnt(" #n ")" ::: "memory")
#define BAR __builtin_amdgcn_s_barrier()
#define SCHED __builtin_amdgcn_sched_barrier(0)
#define HOOK(need) do { if (ready < (need)) { \
        if (wid == 0) { \
            const unsigned long long t0 = __builtin_amdgcn_s_memrealtime(); unsigned fa, fb; int rd; \
            unsigned ln = voff[0]; asm volatile("" : "+v"(ln)); ln >>= 4;     \
            for (;;) { \
                { const int li_ = KIDX(min((int)ln, nk - 1)); fa = ld_sys(fA + li_); fb = ld_sys(fB + li_); } \
                const unsigned long long okm = __ballot((fa >> 8) >= call && (fb >> 8) >= call); \
                rd = (okm == ~0ull) ? 64 : (int)__builtin_ctzll(~okm); \
                if (rd >= (need)) break; \
                if (rfl(ld_agt(tmo)) == call || __builtin_amdgcn_s_memrealtime() - t0 > DEADLINE) { \
                    if (ln == 0) __hip_atomic_store(tmo, call, __ATOMIC_RELAXED, __HIP_MEMORY_SCOPE_AGENT); \
                    rd = 64; break; } \
                __builtin_amdgcn_s_sleep(POLL_SLEEP); \
            } \
            const int sv = (int)((fa & 31u) + (fb & 31u)); \
            const int sp = __builtin_amdgcn_update_dpp(-1, sv, 0x138, 0xf, 0xf, false);     \
            const unsigned long long cg = __ballot(sv != sp); \
            *(LAS int*)(lds + L_TAB + ln * 4) = sv; \
            if (ln == 0) { u32x4 h; h.x = (unsigned)rd; h.y = (unsigned)cg; h.z = (unsigned)(cg >> 32); h.w = 0; *(LAS u32x4*)(lds + L_HDR) = h; } \
            DIAG_WAIT(t0); \
            asm volatile("s_waitcnt vmcnt(0) lgkmcnt(0)" ::: "memory"); \
        } \
        asm volatile("" ::: "memory"); BAR; asm volatile("" ::: "memory"); \
        { const u32x4 h = *(const LAS u32x4*)(lds + L_HDR); ready = (int)rfl(h.x); chg = ((unsigned long long)rfl(h.z) << 32) | rfl(h.y); } \
    } } while (0)
#define DIAG_WAIT(t0) do {} while (0)
#define PKMUL4(op) ".set rs_i, 0\n\t.set rs_d, 0\n\t.irpc c, \"" op "\"\n\t.ifc \\c,:\n\t.set rs_d, 1\n\t.endif\n\t" \
        ".if rs_d == 0\n\t.ifnc \\c,v\n\t.ifnc \\c,[\n\t.set rs_i, rs_i*10 + \\c\n\t.endif\n\t.endif\n\t.endif\n\t.endr\n\t" \
        "v_pk_mul_f32 v[rs_i:rs_i+1], v[rs_i:rs_i+1], %16\n\tv_pk_mul_f32 v[rs_i+2:rs_i+3], v[rs_i+2:rs_i+3], %16\n\t"
#define RS16(a) asm volatile("s_cmp_eq_u32 %17, 0\n\ts_cbranch_scc1 .Lrs%=\n\ts_nop 15\n\ts_nop 3\n\t" \
            PKMUL4("%0") PKMUL4("%1") PKMUL4("%2") PKMUL4("%3") PKMUL4("%4") PKMUL4("%5") PKMUL4("%6") PKMUL4("%7") \
            PKMUL4("%8") PKMUL4("%9") PKMUL4("%10") PKMUL4("%11") PKMUL4("%12") PKMUL4("%13") PKMUL4("%14") PKMUL4("%15") \
            "s_nop 4\n.Lrs%=:" \
            : "+v"(acc[a][0][0][0]), "+v"(acc[a][0][0][1]), "+v"(acc[a][0][1][0]), "+v"(acc[a][0][1][1]), "+v"(acc[a][0][2][0]), "+v"(acc[a][0][2][1]), "+v"(acc[a][0][3][0]), "+v"(acc[a][0][3][1]), \
              "+v"(acc[a][1][0][0]), "+v"(acc[a][1][0][1]), "+v"(acc[a][1][1][0]), "+v"(acc[a][1][1][1]), "+v"(acc[a][1][2][0]), "+v"(acc[a][1][2][1]), "+v"(acc[a][1][3][0]), "+v"(acc[a][1][3][1]) \
            : "s"(fac), "s"(de) : "scc")
#define RESCALE(kt) do { int de = 0; \
        if ((chg >> (kt)) & 1ull) { \
            const int s = (int)rfl(*(const LAS unsigned*)(lds + L_TAB + (kt) * 4)); \
            smax = max(smax, s); const int sn = max(s, smax - 4); de = max(16 * (scur - sn), -126); scur = sn; } \
        const unsigned long long fac = (unsigned long long)(unsigned)((de + 127) << 23) * 0x100000001ull; \
        RS16(0); RS16(1); \
    } while (0)
#define TRIP do { \
        const int t2 = t + 2 < nk ? t + 2 : 0;          \
        const unsigned o1 = KIDX(t + 1) * kstr, o2 = KIDX(t2) * kstr, o3 = KIDX(t2 + 1) * kstr; \
        const gchar* a1 = cA + o1; const gchar* a2 = cA + o2; const gchar* b2 = cB + o2; const gchar* a3 = cA + o3; const gchar* b3 = cB + o3; \
        RESCALE(t); \
        LDB(B0, 0, 0); SCHED; LDA(At, 0, 0); STAGE(SA(1, 1), a1 + HTB); \
        WAIT_L(8); BAR; WAIT_L(0); MMA(0, 0, At, B0); BAR; SCHED; \
        LDB(B1, 0, 1); STAGE(SB(0, 0), b2); \
        BAR; WAIT_L(0); MMA(0, 1, At, B1); BAR; \
        LDA(At, 0, 1); STAGE(SA(0, 0), a2); \
        BAR; WAIT_L(0); MMA(1, 0, At, B0); BAR; SCHED; \
        STAGE(SB(0, 1), b2 + HTB); \
        WAIT_V(6); BAR; MMA(1, 1, At, B1); BAR; \
        RESCALE(t + 1); \
        LDB(B0, 1, 0); SCHED; LDA(At, 1, 0); STAGE(SA(0, 1), a2 + HTB); \
        WAIT_L(8); BAR; WAIT_L(0); MMA(0, 0, At, B0); BAR; SCHED; \
        LDB(B1, 1, 1); STAGE(SB(1, 0), b3); \
        BAR; WAIT_L(0); MMA(0, 1, At, B1); BAR; \
        LDA(At, 1, 1); STAGE(SA(1, 0), a3); \
        BAR; WAIT_L(0); MMA(1, 0, At, B0); BAR; SCHED; \
        STAGE(SB(1, 1), b3 + HTB); \
        WAIT_V(6); BAR; MMA(1, 1, At, B1); BAR; \
    } while (0)

    for (int job = 0; job < njob; ++job) {
    const bool mini = job != jmain;
    const int mi = mini0 + (job - (job > jmain)) * ministep, mT = mi / 7, ml = mi - mT * 7;
    const int mrow = mT < 2 * NE ? mT / NE : mT & 1, mcol = mT < 2 * NE ? mT % NE : mT >> 1;
    const int ua = mini ? (112 + mrow) * 8 : pm, ub_ = mini ? (ml * 16 + mcol) * 8 : 16 + pn;
    const int nk = mini ? 8 : dest == rank ? NKT : NKT - 8, kmag = nk == NKT - 8 ? 37 : 0;
    const unsigned kstr = mini ? UNIT : KT_BYTES;
    const gchar* cA = myws + (mini ? WS_STAGE2 : WS_STAGE) + (size_t)ua * UNIT;
    const gchar* cB = myws + (mini ? WS_STAGE2 : WS_STAGE) + (size_t)ub_ * UNIT;
    const gu32* fA = (const gu32*)(myws + (mini ? WS_FLAG2 : WS_FLAG)) + ua * (mini ? 1 : NKT);
    const gu32* fB = (const gu32*)(myws + (mini ? WS_FLAG2 : WS_FLAG)) + ub_ * (mini ? 1 : NKT);
    int ready = 0; unsigned long long chg = 0; int scur = 0, smax = 0;
    u32x4 V[32];
    {
    LANE_VALS;
    const unsigned voff[2] = {(unsigned)(wid * 64 + lane) * 16u, (unsigned)(wid * 64 + lane) * 16u + 8192u}; const unsigned ldsw = (unsigned)wid * 1024u;
    const unsigned aoff0 = (unsigned)((wr * 64 + fr) * 128 + ((fq ^ (fr >> 1)) * 16)), aoff1 = aoff0 ^ 64u;
    const unsigned boff = (unsigned)(fq * 2048 + (wc * 16 + fr) * 16);
    f32x4 acc[2][2][4][2];
#pragma unroll
    for (int a = 0; a < 2; ++a)
#pragma unroll
        for (int b = 0; b < 2; ++b)
#pragma unroll
            for (int m = 0; m < 4; ++m)
#pragma unroll
                for (int n = 0; n < 2; ++n) acc[a][b][m][n] = (f32x4){0.f, 0.f, 0.f, 0.f};
    h16x8 At[4][2], B0[2][2], B1[2][2];

    WAIT_V(0);
    if (mini && job > jmain && wid == 0) {
        const unsigned long long t0 = __builtin_amdgcn_s_memrealtime();
        while ((int)(rfl(ld_agt((const gu32*)(myws + WS_OTK + ml * 128 + 64))) % (256u + NE * OUT_WAVES)) < mT * 8 - MINI_LEAD && __builtin_amdgcn_s_memrealtime() - t0 < 100000ull)
            __builtin_amdgcn_s_sleep(64);
    }
    HOOK(2);
    if (!mini) STAMP(2); else STAMP(20 + 4 * (job & 1));
    const unsigned ok1 = KIDX(1) * kstr;
    STAGE(SB(0, 0), cB); STAGE(SA(0, 0), cA); STAGE(SB(0, 1), cB + HTB); STAGE(SA(0, 1), cA + HTB);
    if (wr == 1) BAR;
    WAIT_V(4); BAR;
    STAGE(SB(1, 0), cB + ok1); STAGE(SA(1, 0), cA + ok1); STAGE(SB(1, 1), cB + ok1 + HTB);
    WAIT_V(6); BAR;
    if (mini) STAMP(21 + 4 * (job & 1));
    for (int t = 0; t < nk; t += 2) {
        HOOK(min(t + 4, nk));
        TRIP;
    }
    if (wr == 0) BAR;
    if (!mini) STAMP(3); else STAMP(22 + 4 * (job & 1));

    const float qn = __builtin_nanf("");
    {
        const bool bad = rfl(ld_agt(tmo)) == call;
        const int oe = 16 * scur - 256;
#pragma unroll
        for (int a = 0; a < 2; ++a)
#pragma unroll
            for (int m = 0; m < 4; ++m)
#pragma unroll
                for (int r = 0; r < 4; ++r) {
                    f32x4 v;
                    v.x = __builtin_amdgcn_ldexpf(acc[a][0][m][0][r], oe); v.y = __builtin_amdgcn_ldexpf(acc[a][0][m][1][r], oe);
                    v.z = __builtin_amdgcn_ldexpf(acc[a][1][m][0][r], oe); v.w = __builtin_amdgcn_ldexpf(acc[a][1][m][1][r], oe);
                    if (bad) v = (f32x4){qn, qn, qn, qn};
                    V[(a * 4 + m) * 4 + r] = __builtin_bit_cast(u32x4, v);
                }
    }
    }
    LANE_VALS;
    const unsigned ovo = (unsigned)(wr * 64 + 4 * fq) * (GN * 4u) + (unsigned)(wc * 16 + fr) * 16u;
#define OSTAMP(k) do {} while (0)
#define DIAG_END do {} while (0)
    if (mini || dest == rank) {
        const gchar* ob = uni((const char*)p.out[rank] + ((size_t)((mini ? mrow : pm & 1) * 256) * GN + (mini ? ((rank + 1 + ml) & 7) * NL + mcol * 256 : rank * NL + pn * 256)) * 4);
#pragma unroll
        for (int q = 0; q < 32; ++q) { store16_sys(ob + out_off(q), ovo, V[q]); asm volatile("s_waitcnt vmcnt(" STR(LOC_W) ")" ::: "memory"); }
        WAIT_V(0);
        if (mini && lane == 0) __hip_atomic_store((GAS unsigned long long*)(myws + WS_PFLAG) + ml * 256 + (mrow * 16 + mcol) * 8 + wid, (unsigned long long)call << 32, __ATOMIC_RELAXED, __HIP_MEMORY_SCOPE_SYSTEM);
    }
    if (mini) STAMP(23 + 4 * (job & 1));
    int packed = mini ? 0 : dest - rank;
    asm volatile("" : "+v"(packed)); packed = (int)rfl(packed);
    if (packed) {
        const unsigned cidx = (unsigned)(((pm & 1) * 16 + pn) * 8 + wid);
        const gchar* pb = uni(myws + WS_PACK + (size_t)(dest * 256 + cidx) * CHUNK);
        unsigned mu = 0; float sm = 0.f;
#pragma unroll
        for (int q = 0; q < 32; ++q) {
            const u32x4 w = V[q] & 0x7fffffffu; mu = max(max(mu, w.x), max(w.y, max(w.z, w.w)));
            const f32x4 f = __builtin_bit_cast(f32x4, w); sm += (f.x + f.y) + (f.z + f.w);
        }
        const unsigned myB = min(max(min(wave_mean_exp(sm * (2.32f / 8192.f)), wave_max(mu >> 23)), 2u), 254u);
        u32x4 PK[26];
        const unsigned tail = c_encode(V, myB, PK, scr, pb, lane);
        asm volatile("" ::: "memory");
#pragma unroll
        for (int q = 0; q < 3; ++q) if (q * 1024u + lane * 16u < tail) store16_sys(pb + (C_HDR + q * 1024u), lane * 16u, *(const LAS u32x4*)(scr + q * 1024u + lane * 16u));
#pragma unroll
        for (int q = 0; q < 26; ++q) { store16_sys(pb + q * 1024u, lane * 16u, PK[q]); WAIT_V(4); }
        WAIT_V(0);
        if (lane == 0) __hip_atomic_store((GAS unsigned long long*)(dws + WS_CFLAG) + rank * 256 + cidx, ((unsigned long long)call << 32) | (tail << 8) | myB, __ATOMIC_RELAXED, __HIP_MEMORY_SCOPE_SYSTEM);
        OSTAMP(31);
    }
    }
    LANE_VALS;
    {
        if (worker && wid < OUT_WAVES) {
            u32x4 PK[26];
            gu32* rel = (gu32*)(myws + WS_OREL + link * 128);
            for (;;) {
                unsigned ct = 0;
                if (lane == 0) ct = __hip_atomic_fetch_add((gu32*)(myws + WS_OTK + link * 128 + 64), 1u, __ATOMIC_RELAXED, __HIP_MEMORY_SCOPE_AGENT);
                ct = rfl(ct) % (256u + NE * OUT_WAVES);
                if (ct >= 256u) break;
                const unsigned T = ct >> 3, cw_ = ct & 7u, crow = T < 2u * NE ? T / NE : T & 1u, ccol = T < 2u * NE ? T % NE : T >> 1, ci = (crow * 16u + ccol) * 8u + cw_;
                unsigned fv;
                flag_wait<FLAG_SLEEP>(myws + WS_PFLAG + (link * 256 + ci) * 8, tmo, call, fv);
                flag_wait<FLAG_SLEEP>(myws + WS_CFLAG + (dest * 256 + ci) * 8, tmo, call, fv);
                const bool bad = rfl(ld_agt(tmo)) == call;
                unsigned tk = 0;
                if (lane == 0) tk = __hip_atomic_fetch_add((gu32*)(myws + WS_OTK + link * 128), 1u, __ATOMIC_RELAXED, __HIP_MEMORY_SCOPE_AGENT);
                tk = rfl(tk) & 255u;
                const bool wide = tk >= WIDE_FROM;
                const unsigned need = (call - 1u) * (256u + 8u * PUSH_REL) + tk + 1u - LINK_EXTRA;
                const unsigned long long t0 = __builtin_amdgcn_s_memrealtime();
                for (;;) {
                    const int ahead = (int)(need - rfl(ld_agt(rel)));
                    if (ahead <= 0 || __builtin_amdgcn_s_memrealtime() - t0 > DEADLINE) break;
                    if (ahead > 4) __builtin_amdgcn_s_sleep(32); else __builtin_amdgcn_s_sleep(4);
                }
                __builtin_amdgcn_s_setprio(SEND_PRIO);
                asm volatile("" ::: "memory");
                const gchar* rb = uni(dws + WS_PACK + (size_t)(rank * 256 + ci) * CHUNK);
                const __amdgpu_buffer_rsrc_t rs = __builtin_amdgcn_make_buffer_rsrc((void*)rb, 0, bad ? 0 : CHUNK, 0x00020000);
                const unsigned rtail = bad ? 0u : min(fv >> 8, 3072u);
#pragma unroll
                for (int q = 0; q < 3; ++q) if (q * 1024u + lane * 16u < rtail) glds16_sys((unsigned)(uintptr_t)scr + q * 1024u, lane * 16u, rb + (C_HDR + q * 1024u));
                u32x4 X[8]; CDec D;
#pragma unroll
                for (int n = 0; n < 26; ++n) {
                    const int q = n < 2 ? 24 + n : n - 2;
                    PK[q] = RLOAD4(rs, lane * 16u, q * 1024);
                    if (n >= 10) { SCHED; c_parse(D, PK, X, n - 10); SCHED; }
                    if (wide) asm volatile("s_waitcnt vmcnt(" STR(OUT_W2) ")" ::: "memory"); else asm volatile("s_waitcnt vmcnt(" STR(OUT_W) ")" ::: "memory");
                    if (n == REL_AT && lane == 0) __hip_atomic_fetch_add(rel, 1u, __ATOMIC_RELAXED, __HIP_MEMORY_SCOPE_AGENT);
                    if (n == 8) {
                        D.hdr = *(volatile LAS unsigned short*)(scr + lane * 2u);
#pragma unroll
                        for (int g = 0; g < 4; ++g) if (D.hdr & (256u << g)) { X[2 * g] = RLOAD4(rs, lane * 32u, C_EXC + g * 2048u); X[2 * g + 1] = RLOAD4(rs, lane * 32u + 16u, C_EXC + g * 2048u); }
                        c_begin(D, scr, fv & 255u);
                    }
                }
                __builtin_amdgcn_s_setprio(0);
                c_store(D, PK, __builtin_amdgcn_make_buffer_rsrc((void*)uni((const char*)p.out[rank] + ((size_t)(crow * 256) * GN + dest * NL + ccol * 256) * 4), 0, 255u * GN * 4u + 1024u, 0x00020000),
                        bad, ((cw_ >> 2) * 64u + 4u * fq) * (GN * 4u) + ((cw_ & 3u) * 16u + fr) * 16u);
            }
        }
    }
    DIAG_END;
}

extern "C" void kernel_launch(int rank, int nranks, ncclComm_t comm,
                              hipStream_t stream,
                              void* const* d_in, const int* in_sizes, int n_in,
                              void* d_out, int out_size,
                              const int* in_dtypes, int out_dtype,
                              void* const* d_out_all,
                              void* const* d_ws_all, size_t ws_bytes) {
    (void)comm; (void)in_dtypes; (void)out_dtype; (void)d_out;
    if (nranks != NR || (unsigned)rank >= (unsigned)NR || n_in < 2 || ws_bytes < WS_NEED ||
        in_sizes[0] != GM * KS || in_sizes[1] != KS * GN || out_size != (GM / NR) * GN) {
        fprintf(stderr, "gemm_rs: unexpected problem shape\n");
        return;
    }
    HIP_CHECK(hipFuncSetAttribute((const void*)gemm_rs_kernel, hipFuncAttributeMaxDynamicSharedMemorySize, (int)LDS_BYTES));
    Params p;
    p.x = (const float*)d_in[0]; p.w = (const float*)d_in[1];
    for (int i = 0; i < NR; ++i) { p.out[i] = (float*)d_out_all[i]; p.ws[i] = (char*)d_ws_all[i]; }
    p.rank = rank; p.pad = 0;
    hipLaunchKernelGGL(gemm_rs_kernel, dim3(GRID), dim3(NTHR), LDS_BYTES, stream, p);
    HIP_CHECK(hipGetLastError());
}
```

```cpp
#include <hip/hip_runtime.h>
#include <rccl/rccl.h>
#include <cstdio>
#include <cstdint>

#define HIP_CHECK(expr) do { hipError_t _e = (expr); if (_e != hipSuccess) { \
    fprintf(stderr, "HIP error %s at %s:%d\n", hipGetErrorString(_e), __FILE__, __LINE__); \
    return; } } while (0)

#define LAS __attribute__((address_space(3)))
#define GAS __attribute__((address_space(1)))
typedef GAS char gchar;
typedef GAS unsigned gu32;
typedef _Float16 h16x8 __attribute__((ext_vector_type(8)));
typedef float f32x4 __attribute__((ext_vector_type(4)));
typedef float f32x2 __attribute__((ext_vector_type(2)));
typedef unsigned u32x4 __attribute__((ext_vector_type(4)));

constexpr int NR = 8;
constexpr int GM = 4096, GN = 32768, KS = 512, NL = 4096;
constexpr int GRID = 256, NTHR = 512;
constexpr unsigned UNIT = 32768;
constexpr unsigned HTB = 16384;
constexpr int NKT = 64;
constexpr size_t WS_TICKET = 0, WS_TMO = 128, WS_TASKQ = 256, WS_LINKQ = 1024, WS_OTK = 2048, WS_OREL = 3072;
constexpr size_t WS_FLAG = 4096;
constexpr size_t WS_CFLAG = 16384;
constexpr size_t WS_AFLAG = 32768;
constexpr size_t WS_IFLAG = 12288;
constexpr size_t WS_FLAG2 = 65536;
constexpr size_t WS_PFLAG = 73728;
constexpr size_t WS_INBOX = 131072;
constexpr size_t WS_PX = WS_INBOX + (size_t)224 * UNIT;
constexpr size_t WS_PW = WS_PX + (size_t)256 * UNIT;
constexpr size_t WS_STAGE = WS_PW + (size_t)14 * 128 * UNIT;
constexpr unsigned KT_BYTES = 32 * UNIT;
constexpr unsigned CHUNK = 37888;
constexpr size_t WS_STAGE2 = WS_STAGE + (size_t)NKT * KT_BYTES;
constexpr size_t WS_PACK = WS_STAGE2 + (size_t)114 * 8 * UNIT;
constexpr size_t WS_NEED = WS_PACK + (size_t)NR * 256 * CHUNK;
constexpr size_t WS_DIAG = 65536, WS_DBASE = WS_NEED;
__device__ __forceinline__ size_t pin_off(unsigned par, int isW, int dd, int j, int u) {
    return isW ? WS_PW + (size_t)((par * 7 + dd) * 128 + j * 16 + u) * UNIT : WS_PX + (size_t)(par * 128 + j * 16 + u) * UNIT;
}
#define STR2(x) #x
#define STR(x) STR2(x)
#define DEC_W 12
#define NOW 12
#define OPULL_N 4
#define NEARLY 10
constexpr int NE = NEARLY;
constexpr unsigned long long DEADLINE = 100000000ull;
constexpr unsigned L_CTL = 8 * HTB;
constexpr unsigned L_AMAX = L_CTL, L_CONS = L_CTL + 32, L_TURN = L_CTL + 48, L_CALL = L_CTL + 52, L_LAND = L_CTL + 56, L_LST = L_CTL + 60, L_ITEM = L_CTL + 64, L_QI = L_CTL + 128;
constexpr unsigned L_ECNT = L_CTL + 192, L_UCNT = L_CTL + 224, L_FREE = L_CTL + 256, L_DESC = L_CTL + 320, L_ZERO_END = L_CTL + 448;
constexpr unsigned L_HDR = L_CTL + 576, L_TAB = L_CTL + 640, L_SCR = L_CTL + 1024, LDS_BYTES = L_SCR + 8 * 3072;

#define GLDS_AUX 17
struct Params {
    const float* x; const float* w;
    float* out[NR]; char* ws[NR];
    int rank; int pad;
};

#define STAMP(i) do {} while (0)

__device__ __forceinline__ void store16_sys(const gchar* sbase, unsigned voff, u32x4 v) {
    asm volatile("global_store_dwordx4 %0, %1, %2 sc0 sc1\n\ts_nop 1" :: "v"(voff), "v"(v), "s"(sbase) : "memory");
}
__device__ __forceinline__ void glds16_sys(unsigned dst, unsigned voff, const gchar* sbase) {
    unsigned keep;
    asm volatile("s_mov_b32 %0, m0\n\ts_mov_b32 m0, %1\n\ts_nop 0\n\tglobal_load_lds_dwordx4 %2, %3 sc0 sc1\n\ts_mov_b32 m0, %0" : "=&s"(keep) : "s"(dst), "v"(voff), "s"(sbase) : "memory");
}
__device__ __forceinline__ unsigned ld_sys(const gu32* q) { return __hip_atomic_load(q, __ATOMIC_RELAXED, __HIP_MEMORY_SCOPE_SYSTEM); }
__device__ __forceinline__ unsigned ld_agt(const gu32* q) { return __hip_atomic_load(q, __ATOMIC_RELAXED, __HIP_MEMORY_SCOPE_AGENT); }
__device__ __forceinline__ unsigned rfl(unsigned v) { return (unsigned)__builtin_amdgcn_readfirstlane((int)v); }

template <class T> __device__ __forceinline__ gchar* uni(T q) {
    unsigned lo = rfl((unsigned)(uintptr_t)q), hi = rfl((unsigned)((uintptr_t)q >> 32));
    asm volatile("s_nop 4" : "+s"(lo), "+s"(hi));
    return (gchar*)(((uintptr_t)hi << 32) | lo);
}

__device__ __forceinline__ unsigned wave_max(unsigned x) {
    int mb = (int)x;
    mb = max(mb, __builtin_amdgcn_update_dpp(0, mb, 0x111, 0xf, 0xf, false));
    mb = max(mb, __builtin_amdgcn_update_dpp(0, mb, 0x112, 0xf, 0xf, false));
    mb = max(mb, __builtin_amdgcn_update_dpp(0, mb, 0x114, 0xf, 0xf, false));
    mb = max(mb, __builtin_amdgcn_update_dpp(0, mb, 0x118, 0xf, 0xf, false));
    mb = max(mb, __builtin_amdgcn_update_dpp(0, mb, 0x142, 0xa, 0xf, false));
    mb = max(mb, __builtin_amdgcn_update_dpp(0, mb, 0x143, 0xc, 0xf, false));
    return (unsigned)__builtin_amdgcn_readlane(mb, 63);
}
__device__ __forceinline__ unsigned wave_mean_exp(float sm) {
#define DPP_ADD(ctl, rm) sm += __builtin_bit_cast(float, __builtin_amdgcn_update_dpp(0, __builtin_bit_cast(int, sm), ctl, rm, 0xf, false))
    DPP_ADD(0x111, 0xf); DPP_ADD(0x112, 0xf); DPP_ADD(0x114, 0xf); DPP_ADD(0x118, 0xf); DPP_ADD(0x142, 0xa); DPP_ADD(0x143, 0xc);
    return ((unsigned)__builtin_amdgcn_readlane(__builtin_bit_cast(int, sm), 63) >> 23) & 255u;
}

typedef unsigned short u16x2 __attribute__((ext_vector_type(2)));
typedef unsigned u32x2 __attribute__((ext_vector_type(2)));
constexpr unsigned Q_P6 = 6144, Q_LEN = 6656, Q_STR = 6720, Q_BYTES = 8192;
__device__ __forceinline__ u16x2 as16(unsigned a) { return __builtin_bit_cast(u16x2, a); }
__device__ __forceinline__ unsigned as32(u16x2 a) { return __builtin_bit_cast(unsigned, a); }
__device__ __forceinline__ unsigned pk_sub(unsigned a, unsigned b) { return as32(as16(a) - as16(b)); }
__device__ __forceinline__ unsigned pk_subs(unsigned a, unsigned b) { return as32(__builtin_elementwise_sub_sat(as16(a), as16(b))); }
__device__ __forceinline__ unsigned pk_min(unsigned a, unsigned b) { return as32(__builtin_elementwise_min(as16(a), as16(b))); }
__device__ __forceinline__ unsigned ffbl(unsigned a) { unsigned r; asm("v_ffbl_b32 %0, %1" : "=v"(r) : "v"(a)); return r; }
__device__ __forceinline__ unsigned wave_scan(unsigned x) {
    int v = (int)x;
    v += __builtin_amdgcn_update_dpp(0, v, 0x111, 0xf, 0xf, false);
    v += __builtin_amdgcn_update_dpp(0, v, 0x112, 0xf, 0xf, false);
    v += __builtin_amdgcn_update_dpp(0, v, 0x114, 0xf, 0xf, false);
    v += __builtin_amdgcn_update_dpp(0, v, 0x118, 0xf, 0xf, false);
    v += __builtin_amdgcn_update_dpp(0, v, 0x142, 0xa, 0xf, false);
    v += __builtin_amdgcn_update_dpp(0, v, 0x143, 0xc, 0xf, false);
    return (unsigned)v;
}
__device__ __forceinline__ unsigned q_encode(const u32x4 (&W)[8], unsigned B, LAS unsigned char* dst, unsigned lane) {
    const unsigned B1 = (B + 1u) * 0x04000400u;
    unsigned v[32], seg[4], pos[4];
    u32x4 X[4];
#pragma unroll
    for (int s = 0; s < 4; ++s) {
        unsigned a = 0, p = 0;
        X[s].x = __builtin_amdgcn_perm(W[2 * s].y, W[2 * s].x, 0x07050301u); X[s].y = __builtin_amdgcn_perm(W[2 * s].w, W[2 * s].z, 0x07050301u);
        X[s].z = __builtin_amdgcn_perm(W[2 * s + 1].y, W[2 * s + 1].x, 0x07050301u); X[s].w = __builtin_amdgcn_perm(W[2 * s + 1].w, W[2 * s + 1].z, 0x07050301u);
#pragma unroll
        for (int k = 8 * s; k < 8 * s + 8; ++k) {
            const unsigned w = W[k >> 2][k & 3];
            const unsigned dq = as32(as16(pk_sub(B1, w & 0x7c007c00u)) >> 10);
            const unsigned ix = pk_min(dq, 0x00070007u) | 0x0c000c00u;
            const unsigned len = __builtin_amdgcn_perm(0x05040201u, 0x00000003u, ix) + pk_subs(dq, 0x00070007u);
            v[k] = (w & 0x83ff83ffu) | (__builtin_amdgcn_perm(0u, 0x03020100u, ix) << 10);
            p += len & 0xffffu; a |= 1u << (p & 31u);
            p += len >> 16;     a |= 1u << (p & 31u);
        }
        seg[s] = a >> 1; pos[s] = p;
        asm volatile("" : "+v"(v[8 * s]), "+v"(v[8 * s + 1]), "+v"(v[8 * s + 2]), "+v"(v[8 * s + 3]), "+v"(v[8 * s + 4]), "+v"(v[8 * s + 5]), "+v"(v[8 * s + 6]), "+v"(v[8 * s + 7]),
                     "+v"(X[s]), "+v"(seg[s]), "+v"(pos[s]));
    }
    const bool ovf = max(max(pos[0], pos[1]), max(pos[2], pos[3])) > 31u;
    const unsigned tot = ovf ? 0u : pos[0] + pos[1] + pos[2] + pos[3];
    const unsigned incl = wave_scan(tot), total = (unsigned)__builtin_amdgcn_readlane((int)incl, 63);
    const unsigned long long om = __ballot(ovf);
    const unsigned xb = Q_STR + (((total + 127u) >> 7) << 4), size = xb + 64u * (unsigned)__popcll(om);
    if (size > Q_BYTES - 64u) return Q_BYTES;
    *(LAS u32x4*)(dst + Q_STR + lane * 16u) = (u32x4){0u, 0u, 0u, 0u};
#pragma unroll
    for (int j = 0; j < 6; ++j)
#pragma unroll
        for (int i = 0; i < 4; ++i) v[4 * j + i] |= (v[26 + j] << (12 - 3 * i)) & 0x70007000u;
#pragma unroll
    for (int j = 0; j < 6; ++j) v[24 + j / 3] |= (v[26 + j] >> (1 + j % 3)) & (0x80008000u >> (1 + j % 3));
#pragma unroll
    for (int j = 0; j < 6; ++j) *(LAS u32x4*)(dst + j * 1024u + lane * 16u) = (u32x4){v[4 * j], v[4 * j + 1], v[4 * j + 2], v[4 * j + 3]};
    *(LAS u32x2*)(dst + Q_P6 + lane * 8u) = (u32x2){v[24], v[25]};
    const unsigned slot = __builtin_amdgcn_mbcnt_hi((unsigned)(om >> 32), __builtin_amdgcn_mbcnt_lo((unsigned)om, 0u));
    *(LAS unsigned char*)(dst + Q_LEN + lane) = (unsigned char)(ovf ? 128u + slot : tot);
    if (!ovf) {
        unsigned off = incl - tot;
#pragma unroll
        for (int s = 0; s < 4; ++s) {
            const unsigned long long x = (unsigned long long)seg[s] << (off & 31u);
            LAS unsigned* q = (LAS unsigned*)(dst + Q_STR + (off >> 5) * 4u);
            __hip_atomic_fetch_or(q, (unsigned)x, __ATOMIC_RELAXED, __HIP_MEMORY_SCOPE_WORKGROUP);
            __hip_atomic_fetch_or(q + 1, (unsigned)(x >> 32), __ATOMIC_RELAXED, __HIP_MEMORY_SCOPE_WORKGROUP);
            off += pos[s];
        }
    } else {
#pragma unroll
        for (int j = 0; j < 4; ++j) *(LAS u32x4*)(dst + xb + slot * 64u + j * 16u) = X[j];
    }
    return size;
}
#define RLOAD4(rs, vo, so) __builtin_bit_cast(u32x4, __builtin_amdgcn_raw_buffer_load_b128(rs, vo, so, 17))
struct QIn { u32x4 P[6]; u32x2 P6; unsigned lb, off, w4; u32x4 win, X[4]; };
__device__ __forceinline__ void q_read(const LAS unsigned char* src, unsigned lane, bool raw, QIn& q) {
    q.lb = raw ? 0u : *(const LAS unsigned char*)(src + Q_LEN + lane);
#pragma unroll
    for (int j = 0; j < 6; ++j) q.P[j] = *(const LAS u32x4*)(src + j * 1024u + lane * 16u);
    q.P6 = *(const LAS u32x2*)(src + Q_P6 + lane * 8u);
    const unsigned tot = q.lb >= 128u ? 0u : q.lb, incl = wave_scan(tot), total = (unsigned)__builtin_amdgcn_readlane((int)incl, 63);
    q.off = incl - tot;
    const LAS unsigned* w = (const LAS unsigned*)(src + Q_STR + (q.off >> 5) * 4u);
    q.win = (u32x4){w[0], w[1], w[2], w[3]}; q.w4 = w[4];
    if (raw || q.lb >= 128u) {
        const unsigned xo = raw ? Q_P6 + lane * 16u : min(Q_STR + (((total + 127u) >> 7) << 4) + (q.lb - 128u) * 64u, Q_BYTES - 64u), xs = raw ? 1024u : 16u;
        q.X[0] = *(const LAS u32x4*)(src + xo); q.X[1] = *(const LAS u32x4*)(src + xo + xs);
        q.X[2] = *(const LAS u32x4*)(src + xo + 32u); q.X[3] = *(const LAS u32x4*)(src + xo + 48u);
    }
}
__device__ __forceinline__ void q_finish(unsigned B, const QIn& q, u32x4 (&W)[8]) {
    unsigned v[32];
#pragma unroll
    for (int k = 0; k < 24; ++k) v[k] = q.P[k >> 2][k & 3];
    v[24] = q.P6.x; v[25] = q.P6.y;
#pragma unroll
    for (int j = 0; j < 6; ++j) {
        unsigned d = (v[24 + j / 3] << (1 + j % 3)) & 0x80008000u;
#pragma unroll
        for (int i = 0; i < 4; ++i) d |= (v[4 * j + i] & 0x70007000u) >> (12 - 3 * i);
        v[26 + j] = d;
    }
    u32x4 win;
    win.x = __builtin_amdgcn_alignbit(q.win.y, q.win.x, q.off); win.y = __builtin_amdgcn_alignbit(q.win.z, q.win.y, q.off);
    win.z = __builtin_amdgcn_alignbit(q.win.w, q.win.z, q.off); win.w = __builtin_amdgcn_alignbit(q.w4, q.win.w, q.off);
    const unsigned B1 = (B + 1u) * 0x04000400u;
#pragma unroll
    for (int s = 0; s < 4; ++s) {
        unsigned a = win.x << 1, used = 0;
#pragma unroll
        for (int k = 8 * s; k < 8 * s + 8; ++k) {
            const unsigned nz = v[k] | (v[k] >> 1);
            const unsigned l0 = ffbl((a & ~1u) | ((nz >> 10) & 1u));
            a >>= l0;
            const unsigned l1 = ffbl((a & ~1u) | ((nz >> 26) & 1u));
            a >>= l1;
            const unsigned len = l0 | (l1 << 16);
            used += len;
            const unsigned dq = __builtin_amdgcn_perm(0x00000006u, 0x00050400u, pk_min(len, 0x00040004u) | 0x0c000c00u) + pk_subs(len, 0x00040004u);
            W[k >> 2][k & 3] = (v[k] & 0x83ff83ffu) | (pk_sub(B1, (v[k] & 0x0c000c00u) + (dq << 10)) & 0x7c007c00u);
        }
        if (s < 3) {
            const unsigned sh = (used + (used >> 16)) & 31u;
            win.x = __builtin_amdgcn_alignbit(win.y, win.x, sh); win.y = __builtin_amdgcn_alignbit(win.z, win.y, sh);
            win.z = __builtin_amdgcn_alignbit(win.w, win.z, sh); win.w >>= sh;
        }
    }
}
__device__ __forceinline__ void q_patch(bool raw, const QIn& q, u32x4 (&W)[8]) {
    if (raw) {
#pragma unroll
        for (int j = 0; j < 6; ++j) W[j] = q.P[j];
        W[6] = q.X[0]; W[7] = q.X[1];
    } else if (q.lb >= 128u) {
#pragma unroll
        for (int k = 0; k < 32; ++k) W[k >> 2][k & 3] = __builtin_amdgcn_perm(q.X[k >> 3][(k >> 1) & 3], W[k >> 2][k & 3], (k & 1) ? 0x07020600u : 0x05020400u);
    }
}

constexpr unsigned C_HDR = 26624, C_STR = C_HDR + 128, C_EXC = 29696;
__device__ __forceinline__ unsigned c_encode(const u32x4 (&V)[32], unsigned B, u32x4 (&PK)[26], LAS unsigned char* scr, const gchar* pb, unsigned lane) {
    const unsigned B1 = (B + 1u) * 0x00010001u;
    unsigned seg[8], pos[8], hdr = 0;
#pragma unroll
    for (int g = 0; g < 4; ++g) {
        unsigned e2[16];
#pragma unroll
        for (int v = 0; v < 8; ++v) {
            u32x4 r;
#pragma unroll
            for (int c = 0; c < 4; ++c) r[c] = __builtin_amdgcn_alignbit(V[g * 8 + v][c], V[g * 8 + v][c], 31);
#pragma unroll
            for (int i = 0; i < 3; ++i) PK[g * 6 + ((3 * v + i) >> 2)][(3 * v + i) & 3] = __builtin_amdgcn_perm(r.w, r[i], 0x04020100u + ((unsigned)i << 24));
            e2[2 * v] = __builtin_amdgcn_perm(r.y, r.x, 0x0c070c03u); e2[2 * v + 1] = __builtin_amdgcn_perm(r.w, r.z, 0x0c070c03u);
        }
#pragma unroll
        for (int s = 2 * g; s < 2 * g + 2; ++s) {
            unsigned a = 0, p = 0, cw = 0;
#pragma unroll
            for (int j = 0; j < 8; ++j) {
                const unsigned dq = pk_sub(B1, e2[(s & 1) * 8 + j]), ix = pk_min(dq, 0x00070007u) | 0x0c000c00u;
                const unsigned len = __builtin_amdgcn_perm(0x05040201u, 0x00000003u, ix) + pk_subs(dq, 0x00070007u);
                cw |= __builtin_amdgcn_perm(0u, 0x03020100u, ix) << (2 * j);
                p += len & 0xffffu; a |= 1u << (p & 31u);
                p += len >> 16;     a |= 1u << (p & 31u);
            }
            seg[s] = a >> 1; pos[s] = p; PK[24 + (s >> 2)][s & 3] = cw;
        }
        if (max(pos[2 * g], pos[2 * g + 1]) > 31u) {
            u32x4 E[2];
#pragma unroll
            for (int v = 0; v < 8; ++v) E[v >> 2][v & 3] = __builtin_amdgcn_perm(e2[2 * v + 1], e2[2 * v], 0x06040200u);
            store16_sys(pb + (C_EXC + g * 2048u), lane * 32u, E[0]);
            store16_sys(pb + (C_EXC + g * 2048u + 16u), lane * 32u, E[1]);
            seg[2 * g] = seg[2 * g + 1] = pos[2 * g] = pos[2 * g + 1] = 0u; hdr |= 256u << g;
        }
    }
    const unsigned tot = (pos[0] + pos[1] + pos[2] + pos[3]) + (pos[4] + pos[5] + pos[6] + pos[7]);
    const unsigned incl = wave_scan(tot), total = (unsigned)__builtin_amdgcn_readlane((int)incl, 63);
    *(LAS u32x4*)(scr + 128u + lane * 16u) = (u32x4){0u, 0u, 0u, 0u}; *(LAS u32x4*)(scr + 1152u + lane * 16u) = (u32x4){0u, 0u, 0u, 0u};
    *(LAS unsigned short*)(scr + lane * 2u) = (unsigned short)(hdr | tot);
    unsigned off = incl - tot;
#pragma unroll
    for (int s = 0; s < 8; ++s) {
        const unsigned long long x = (unsigned long long)seg[s] << (off & 31u);
        LAS unsigned* q = (LAS unsigned*)(scr + 128u + (off >> 5) * 4u);
        __hip_atomic_fetch_or(q, (unsigned)x, __ATOMIC_RELAXED, __HIP_MEMORY_SCOPE_WORKGROUP);
        __hip_atomic_fetch_or(q + 1, (unsigned)(x >> 32), __ATOMIC_RELAXED, __HIP_MEMORY_SCOPE_WORKGROUP);
        off += pos[s];
    }
    return 128u + ((total + 7u) >> 3);
}
__device__ __forceinline__ size_t out_off(int q) { return (size_t)((q >> 4) * 128 + ((q >> 2) & 3) * 16 + (q & 3)) * (GN * 4); }
struct CDec { unsigned win[9], a, used, hdr, B1, E[32]; };
__device__ __forceinline__ void c_begin(CDec& d, const LAS unsigned char* scr, unsigned B) {
    const unsigned tot = d.hdr & 255u, off = wave_scan(tot) - tot;
#pragma unroll
    for (int i = 0; i < 9; ++i) d.win[i] = *(const LAS unsigned*)(scr + 128u + (off >> 5) * 4u + i * 4u);
#pragma unroll
    for (int i = 0; i < 8; ++i) d.win[i] = __builtin_amdgcn_alignbit(d.win[i + 1], d.win[i], off);
    d.B1 = (B + 1u) * 0x00010001u;
}
__device__ __forceinline__ void c_parse(CDec& d, const u32x4 (&CW)[2], int H) {
    const int s = H >> 1;
    const unsigned cw = CW[s >> 2][s & 3], nz = cw | (cw >> 1);
    const bool exc = d.hdr & (256u << (s >> 1));
    if (!(H & 1)) { d.a = d.win[0] << 1; d.used = 0; }
    unsigned e2[4];
#pragma unroll
    for (int j = (H & 1) * 4; j < (H & 1) * 4 + 4; ++j) {
        const unsigned l0 = ffbl((d.a & ~1u) | ((nz >> (2 * j)) & 1u));
        d.a >>= l0;
        const unsigned l1 = ffbl((d.a & ~1u) | ((nz >> (2 * j + 16)) & 1u));
        d.a >>= l1;
        const unsigned len = l0 | (l1 << 16);
        d.used += len;
        const unsigned dq = __builtin_amdgcn_perm(0x00000006u, 0x00050400u, pk_min(len, 0x00040004u) | 0x0c000c00u) + pk_subs(len, 0x00040004u);
        e2[j & 3] = pk_sub(d.B1, dq + ((cw >> (2 * j)) & 0x00030003u));
    }
#pragma unroll
    for (int i = 0; i < 2; ++i) d.E[2 * H + i] = __builtin_amdgcn_perm(e2[2 * i + 1], e2[2 * i], 0x06040200u);
    asm volatile("" : "+v"(d.E[2 * H]), "+v"(d.E[2 * H + 1]));
    if (H & 1) {
        const unsigned sh = exc ? 0u : d.used + (d.used >> 16);
#pragma unroll
        for (int i = 0; i < 7 - s; ++i) d.win[i] = __builtin_amdgcn_alignbit(d.win[i + 1], d.win[i], sh);
    }
}
__device__ __forceinline__ void c_patch(CDec& d, const u32x4 (&X)[8]) {
#pragma unroll
    for (int g = 0; g < 4; ++g) if (d.hdr & (256u << g)) {
#pragma unroll
        for (int v = 0; v < 8; ++v) d.E[8 * g + v] = X[2 * g + (v >> 2)][v & 3];
    }
}
__device__ __forceinline__ void c_store(const CDec& d, const LAS unsigned char* src, const u32x4 (&A)[32], bool bad, const gchar* ob, unsigned ovo, unsigned lane) {
#pragma unroll
    for (int h = 0; h < 8; ++h) {
        u32x4 PK[3];
#pragma unroll
        for (int i = 0; i < 3; ++i) PK[i] = *(const LAS u32x4*)(src + (3 * h + i) * 1024u + lane * 16u);
#pragma unroll
        for (int v = 0; v < 4; ++v) {
            const int q = 4 * h + v;
            unsigned P[3];
#pragma unroll
            for (int i = 0; i < 3; ++i) P[i] = PK[(3 * v + i) >> 2][(3 * v + i) & 3];
            u32x4 r;
            r.x = __builtin_amdgcn_perm(d.E[q], P[0], 0x04020100u); r.y = __builtin_amdgcn_perm(d.E[q], P[1], 0x05020100u); r.z = __builtin_amdgcn_perm(d.E[q], P[2], 0x06020100u);
            r.w = __builtin_amdgcn_perm(P[1], P[0], 0x0c0c0703u) | __builtin_amdgcn_perm(d.E[q], P[2], 0x07030c0cu);
#pragma unroll
            for (int c = 0; c < 4; ++c) r[c] = bad ? 0x7fc00000u : __builtin_amdgcn_alignbit(r[c], r[c], 1);
            const f32x4 o = __builtin_bit_cast(f32x4, r) + __builtin_bit_cast(f32x4, A[q]);
            asm volatile("global_store_dwordx4 %0, %1, %2 sc0 sc1 nt\n\ts_nop 1" :: "v"(ovo), "v"(o), "s"(ob + out_off(q)) : "memory");
            asm volatile("s_waitcnt vmcnt(" STR(DEC_W) ")" ::: "memory");
        }
    }
}

constexpr int NITEM = 240;
__device__ __forceinline__ void item_decode(int i, int d, int s, int& isW, int& j, int& u) {
    constexpr int R0 = 12 + NE, R1 = 18 - NE;
    if (i < 8 * R0) {
        j = i / R0; int r = i - j * R0;
        if (i >= 6 && i < R0) r = i < 12 ? i + NE : i - 6;
        isW = r >= 6 && r < 6 + NE; u = isW ? r - 6 : r < 6 ? 2 * r : 2 * (r - 6 - NE) + 1;
        if (!isW) { if (u >= 2 * min(d, s)) u += 2; if (u >= 2 * max(d, s)) u += 2; }
    } else { const int r = i - 8 * R0; j = r / R1; const int t = r - j * R1; isW = t < 16 - NE; u = isW ? NE + t : 2 * d + t - (16 - NE); }
}
constexpr int NENC = 32 + 14 * (12 - NE);
constexpr int NTASK = 1152, NTASK_DYN0 = 200 + 2 * NENC, NTASK_DYN = 3 * NENC + NTASK - NTASK_DYN0;
__device__ __forceinline__ void task_decode(int t, int& isW, int& j, int& u, int& dd) {
    constexpr int R0 = 16 + 8 * NE, R1 = 8 * (16 - NE);
    if (t < 8 * R0) { j = t / R0; int r = t - j * R0; isW = r >= 16; if (isW) { r -= 16; u = r >> 3; dd = r & 7; } else { u = r < 8 ? 2 * r : 2 * r - 15; dd = 7; } }
    else { int r = t - 8 * R0; j = r / R1; r -= j * R1; isW = 1; u = NE + (r >> 3); dd = r & 7; }
}

__device__ __forceinline__ void task_load(const Params& p, int rank, int i, f32x4 (&L)[16]) {
    const bool valid = i < NTASK;
    int isW, j, u, dd; task_decode(valid ? i : 0, isW, j, u, dd);
    const unsigned t = threadIdx.x;
    const gchar* b = uni(isW ? (const char*)p.w + ((size_t)(j * 64) * GN + ((rank + 1 + dd) & 7) * NL + u * 256) * 4 : (const char*)p.x + ((size_t)(u * 256) * KS + j * 64) * 4);
    unsigned vo = isW ? (t >> 6) * (4u * GN * 4u) + (t & 63u) * 16u : (t >> 3) * (KS * 4u) + ((t & 7u) ^ ((t >> 4) & 7u)) * 16u;
    if (!valid) vo = 0;
    const unsigned s0 = isW ? GN * 4u : 128u, s1 = isW ? 2u * GN * 4u : 64u * KS * 4u, s2 = isW ? 32u * GN * 4u : 128u * KS * 4u, s3 = isW ? 16u * GN * 4u : 32u * KS * 4u;
#pragma unroll
    for (int n = 0; n < 16; ++n)
        L[n] = __builtin_nontemporal_load((const GAS f32x4*)(b + ((n & 1) * s0 + ((n >> 1) & 1) * s1 + ((n >> 2) & 1) * s2 + (n >> 3) * s3) + vo));
}

__device__ __forceinline__ unsigned pk2(float a, float b, int e) {
    typedef _Float16 h2 __attribute__((ext_vector_type(2)));
    h2 h; h.x = (_Float16)__builtin_amdgcn_ldexpf(a, e); h.y = (_Float16)__builtin_amdgcn_ldexpf(b, e);
    return __builtin_bit_cast(unsigned, h);
}

#define POLL_SLEEP 8
#define FLAG_SLEEP 16
#define SEND_PRIO 3
#define MINI_LEAD 64
#define HAND_Q 3
#define LOC_W 8
#define LINK_EXTRA 0u
#define PULL_N 7
#define NPUSH 1
#define PUSH_W 0
#define START_NOW 112
#define START_DELAY 4
#define START_SPREAD 12
#define LBAR do { asm volatile("s_waitcnt lgkmcnt(0)" ::: "memory"); __builtin_amdgcn_s_barrier(); asm volatile("" ::: "memory"); } while (0)

typedef volatile LAS unsigned vl32;
typedef volatile LAS u32x4 vl128;
#define LGKM0 asm volatile("s_waitcnt lgkmcnt(0)" ::: "memory")
__device__ __forceinline__ u32x4 lds_wait4(LAS unsigned char* lds, unsigned off, unsigned v) {
    for (;;) {
        const u32x4 q = *(vl128*)(lds + off);
        if (rfl(min(min(q.x, q.y), min(q.z, q.w))) >= v) { asm volatile("" ::: "memory"); return q; }
        __builtin_amdgcn_s_sleep(1);
    }
}
__device__ __forceinline__ unsigned lds_wait1(LAS unsigned char* lds, unsigned off, unsigned v) {
    unsigned x;
    while ((x = rfl(*(vl32*)(lds + off))) < v) __builtin_amdgcn_s_sleep(1);
    asm volatile("" ::: "memory");
    return x;
}
__device__ __forceinline__ void task_stats(LAS unsigned char* lds, const f32x4 (&L)[16], unsigned k) {
    unsigned mu = 0; float sm = 0.f;
#pragma unroll
    for (int n = 0; n < 16; ++n) {
        const u32x4 q = __builtin_bit_cast(u32x4, L[n]) & 0x7fffffffu; mu = max(max(mu, q.x), max(q.y, max(q.z, q.w)));
        const f32x4 f = __builtin_bit_cast(f32x4, q); sm += (f.x + f.y) + (f.z + f.w);
    }
    const unsigned mb = wave_max(mu >> 23), em = wave_mean_exp(sm * (2.32f / 4096.f));
    LGKM0;
    if ((threadIdx.x & 63) == 63) *(vl32*)(lds + L_AMAX + (k & 1u) * 16 + (threadIdx.x >> 6) * 4) = ((k + 1u) << 16) | (em << 8) | mb;
}

__device__ __forceinline__ unsigned img_off(int isW, unsigned q, int c) {
    return (isW ? q * 2048u + (c >> 2) * 8192u + (c & 1) * 1024u : q * 1024u + (c >> 2) * 4096u + (c & 1) * 8192u) + ((c >> 1) & 1) * HTB;
}
__device__ __forceinline__ int conv_step(const Params& p, LAS unsigned char* lds, gu32* tick, int rank, unsigned k, f32x4 (&L)[16], const f32x4 (&Ln)[16]) {
    unsigned t = threadIdx.x; asm volatile("" : "+v"(t));
    const u32x4 am = lds_wait4(lds, L_AMAX + (k & 1u) * 16, (k + 1u) << 16);
    const int i = (int)rfl(*(vl32*)(lds + L_ITEM + (k & 15u) * 4)), i1 = (int)rfl(*(vl32*)(lds + L_ITEM + ((k + 1u) & 15u) * 4)), i2 = (int)rfl(*(vl32*)(lds + L_ITEM + ((k + 2u) & 15u) * 4));
    const bool valid = i < NTASK;
    if (valid) {
        const unsigned mb = rfl(max(max(am.x & 255u, am.y & 255u), max(am.z & 255u, am.w & 255u))), em = (rfl(max(max(am.x, am.y), max(am.z, am.w))) >> 8) & 255u;
        const int cls = (int)(mb + 2u) >> 4;
        const int e = 128 - 16 * cls;
        int isW, j, u, dd; task_decode(i, isW, j, u, dd);
        u32x4 H[8];
#pragma unroll
        for (int h = 0; h < 2; ++h) {
            const f32x4* Q = L + 8 * h; u32x4* G = H + 4 * h;
            if (isW) {
#pragma unroll
                for (int c = 0; c < 4; ++c) { G[c].x = pk2(Q[0][c], Q[1][c], e); G[c].y = pk2(Q[2][c], Q[3][c], e); G[c].z = pk2(Q[4][c], Q[5][c], e); G[c].w = pk2(Q[6][c], Q[7][c], e); }
            } else {
#pragma unroll
                for (int c = 0; c < 4; ++c) { G[c].x = pk2(Q[2 * c].x, Q[2 * c].y, e); G[c].y = pk2(Q[2 * c].z, Q[2 * c].w, e); G[c].z = pk2(Q[2 * c + 1].x, Q[2 * c + 1].y, e); G[c].w = pk2(Q[2 * c + 1].z, Q[2 * c + 1].w, e); }
            }
        }
        const unsigned q = rfl(t >> 6);
        if (k >= 4) lds_wait1(lds, L_CONS + q * 4, k - 3u);
#pragma unroll
        for (int c = 0; c < 8; ++c) *(LAS u32x4*)(lds + (k & 3u) * UNIT + q * Q_BYTES + c * 1024u + (t & 63u) * 16u) = H[c];
        LGKM0;
        if ((t & 63u) == 0) *(vl32*)(lds + L_QI + (k & 3u) * 16 + q * 4) = ((k + 1u) << 16) | ((unsigned)min(max((int)min(em, mb) + e - 112, 2), 31) << 5) | (unsigned)cls;
    }
    unsigned tk = NTASK_DYN - 1;
    gu32* q = tick; asm volatile("" : "+v"(q));
    if (t == 0 && tick && valid) tk = __hip_atomic_fetch_add(q, 1u, __ATOMIC_RELAXED, __HIP_MEMORY_SCOPE_AGENT);
    task_load(p, rank, i2, L);
    __builtin_amdgcn_s_waitcnt(0x4F70);
    asm volatile("" : "+v"(tk));
    if (t == 0) *(vl32*)(lds + L_ITEM + ((k + 3u) & 15u) * 4) = (unsigned)NTASK_DYN0 + tk % (unsigned)NTASK_DYN;
    task_stats(lds, Ln, k + 1u);
    return i1;
}

__device__ __forceinline__ gu32* stage_unit(gchar* myws, int srel, int isW, int j, int u, const gchar*& ub) {
    const int idx = j * 8 + ((srel + 1) & 7);
    ub = myws + WS_STAGE + (size_t)(idx * 32 + isW * 16 + u) * UNIT;
    return (gu32*)(myws + WS_FLAG) + (isW * 16 + u) * NKT + idx;
}
__device__ __forceinline__ void lds_fetch8(const LAS unsigned char* src, unsigned lane, u32x4 (&H)[8]) {
#pragma unroll
    for (int c = 0; c < 8; ++c) H[c] = *(const LAS u32x4*)(src + c * 1024u + lane * 16u);
}

__device__ __forceinline__ unsigned produce(const Params& p, LAS unsigned char* lds, gchar* myws, int rank, int first, int second, gu32* tick) {
    unsigned call;
    if (rfl(threadIdx.x >> 6) < 4) {
        f32x4 RA[16], RB[16];
        const int dly = first >= NTASK ? 0 : first >= START_NOW ? START_DELAY + (first - START_NOW) * START_SPREAD / 128 : 0;
        for (int z = 0; z < dly; ++z) __builtin_amdgcn_s_sleep(32);
        task_load(p, rank, first, RA);
        LBAR;
        task_stats(lds, RA, 0);
        task_load(p, rank, second, RB);
        int nx; unsigned k = 0;
        do { conv_step(p, lds, tick, rank, k, RA, RB); nx = conv_step(p, lds, tick, rank, k + 1u, RB, RA); k += 2; } while (nx < NTASK);
        LBAR;
        call = rfl(*(vl32*)(lds + L_CALL));
    } else {
        const unsigned ts = threadIdx.x - 256u, lane = ts & 63u, q = rfl(ts >> 6);
        constexpr unsigned I0 = (L_ITEM - L_CTL) / 4;
        if (ts < (L_ZERO_END - L_CTL) / 4) *(vl32*)(lds + L_CTL + ts * 4) = ts == I0 ? (unsigned)first : ts == I0 + 1 ? (unsigned)second : ts == I0 + 2 ? (unsigned)second + NENC : 0u;
        unsigned tk = 0;
        gu32* tq = (gu32*)(myws + WS_TICKET); asm volatile("" : "+v"(tq));
        if (ts == 0) tk = __hip_atomic_fetch_add(tq, 1u, __ATOMIC_RELAXED, __HIP_MEMORY_SCOPE_AGENT);
        const gchar* peer = (const gchar*)p.ws[(rank + 1 + min(lane, 6u)) & 7];
        LBAR;
        asm volatile("" : "+v"(tk));
        if (ts == 0) *(vl32*)(lds + L_CALL) = (tk / GRID + 1u) & 0xffffffu;
        call = lds_wait1(lds, L_CALL, 1u);
        for (unsigned k = 0;; ++k) {
            const int i = (int)rfl(*(vl32*)(lds + L_ITEM + (k & 15u) * 4));
            if (i >= NTASK) break;
            const unsigned st = lds_wait1(lds, L_QI + (k & 3u) * 16 + q * 4, (k + 1u) << 16);
            int isW, j, u, dd; task_decode(i, isW, j, u, dd);
            LAS unsigned char* src = lds + (k & 3u) * UNIT + q * Q_BYTES;
            u32x4 H[8];
            lds_fetch8(src, lane, H);
            const gchar* ub; gu32* sf = stage_unit(myws, 7, isW, j, u, ub);
            const bool peers = isW ? dd != 7 : (u >> 1) != rank, slice = isW == peers;
            const int s2 = (isW ? dd * 16 + u : 112 + (u & 1)) * 8 + j;
            if (dd == 7) {
                ub = uni(ub);
#pragma unroll
                for (int c = 0; c < 8; ++c) store16_sys(ub + img_off(isW, q, c), lane * 16u, H[c]);
            }
            if (slice) {
                ub = uni(myws + WS_STAGE2 + (size_t)s2 * UNIT);
#pragma unroll
                for (int c = 0; c < 8; ++c) store16_sys(ub + img_off(isW, q, c), lane * 16u, H[c]);
            }
            unsigned size = Q_BYTES;
            if (peers) {
                const gchar* pb = uni(myws + pin_off(call & 1u, isW, dd, j, u) + q * Q_BYTES);
                size = q_encode(H, (st >> 5) & 31u, src, lane); asm volatile("" ::: "memory"); lds_fetch8(src, lane, H);
#pragma unroll
                for (int c = 0; c < 8; ++c) if (c < 6 || c * 1024u + lane * 16u < size) store16_sys(pb + c * 1024u, lane * 16u, H[c]);
            }
            LGKM0;
            if (lane == 0) *(vl32*)(lds + L_CONS + q * 4) = k + 1u;
            asm volatile("s_waitcnt vmcnt(0)" ::: "memory");
            const unsigned add = (1u << 30) | (((size - Q_LEN + 63u) >> 6) << (10u + 5u * q));
            unsigned old = 0;
            if (lane == 0) old = __hip_atomic_fetch_add((LAS unsigned*)(lds + L_ECNT + (k & 7u) * 4), add, __ATOMIC_RELAXED, __HIP_MEMORY_SCOPE_WORKGROUP);
            old = rfl(old);
            if ((old >> 30) == 3u) {
                if (lane == 0) *(vl32*)(lds + L_ECNT + (k & 7u) * 4) = 0u;
                const unsigned fi = (call & 1u) * 8 + (6u - lane);
                if (peers && (isW ? lane == (unsigned)dd : lane < 7u))
                    __hip_atomic_store((GAS unsigned long long*)(peer + WS_AFLAG + ((fi * 2 + isW) * 128 + j * 16 + u) * 8),
                                       ((unsigned long long)call << 32) | (old + add) | (st & 1023u), __ATOMIC_RELAXED, __HIP_MEMORY_SCOPE_SYSTEM);
                if (lane == 0 ? dd == 7 : lane == 1 && slice) __hip_atomic_store(lane ? (gu32*)(myws + WS_FLAG2) + s2 : sf, (call << 8) | (st & 31u), __ATOMIC_RELAXED, __HIP_MEMORY_SCOPE_SYSTEM);
            }
        }
        LBAR;
    }
    return call;
}

__device__ __forceinline__ unsigned push_first(const Params& p, LAS unsigned char* lds, gchar* myws, int rank, int link, int item) {
    const unsigned t = threadIdx.x;
    const int d = (rank + 1 + link) & 7;
    unsigned tk = 0;
    gu32* tq = (gu32*)(myws + WS_TICKET); asm volatile("" : "+v"(tq));
    if (t == 0) tk = __hip_atomic_fetch_add(tq, 1u, __ATOMIC_RELAXED, __HIP_MEMORY_SCOPE_AGENT);
    f32x4 L[NPUSH][8]; int isW[NPUSH];
#pragma unroll
    for (int r = 0; r < NPUSH; ++r) {
        int j, u; item_decode(item + 8 * r, d, rank, isW[r], j, u);
        const gchar* b = uni(isW[r] ? (const char*)p.w + (size_t)(d * NL + u * 256) * 4 : (const char*)p.x + (size_t)(u * 256) * KS * 4);
        const unsigned vo = isW[r] ? (t >> 6) * (4u * GN * 4u) + (t & 63u) * 16u : (t >> 3) * (KS * 4u) + ((t & 7u) ^ ((t >> 4) & 7u)) * 16u;
        const unsigned s0 = isW[r] ? GN * 4u : 128u, s1 = isW[r] ? 2u * GN * 4u : 64u * KS * 4u, s2 = isW[r] ? 32u * GN * 4u : 128u * KS * 4u;
        if (r) __builtin_amdgcn_s_waitcnt(0x0F70);
#pragma unroll
        for (int n = 0; n < 8; ++n) L[r][n] = *(const GAS f32x4*)(b + ((n & 1) * s0 + ((n >> 1) & 1) * s1 + (n >> 2) * s2) + vo);
    }
    gchar* dws = uni(p.ws[d]);
    if (t >= 8 && t < (L_ZERO_END - L_CTL) / 4 && t != (L_CALL - L_CTL) / 4) *(vl32*)(lds + L_CTL + t * 4) = 0u;
    asm volatile("" : "+v"(tk));
    if (t == 0) *(vl32*)(lds + L_CALL) = (tk / GRID + 1u) & 0xffffffu;
    unsigned call = 0, cl[NPUSH];
#pragma unroll
    for (int r = 0; r < NPUSH; ++r) {
        unsigned mu = 0;
#pragma unroll
        for (int n = 0; n < 8; ++n) { const u32x4 q = __builtin_bit_cast(u32x4, L[r][n]) & 0x7fffffffu; mu = max(max(mu, q.x), max(q.y, max(q.z, q.w))); }
        mu = wave_max(mu >> 23);
        const unsigned am = r ? L_HDR : L_AMAX;
        if ((t & 63u) == 0) *(vl32*)(lds + am + (t >> 6) * 4) = mu;
        LBAR;
        const u32x4 a0 = *(vl128*)(lds + am), a1 = *(vl128*)(lds + am + 16);
        call = rfl(*(vl32*)(lds + L_CALL));
        cl[r] = (rfl(max(max(max(a0.x, a0.y), max(a0.z, a0.w)), max(max(a1.x, a1.y), max(a1.z, a1.w)))) + 2u) >> 4;
        const int e = 128 - 16 * (int)cl[r];
        const gchar* ib = uni(dws + WS_INBOX + (size_t)(((call & 1u) * 7 + (6 - link)) * 16 + item + 8 * r) * UNIT);
        const unsigned so = isW[r] ? (t >> 6) * 2048u + (t & 63u) * 16u : t * 16u, c1 = isW[r] ? 1024u : 8192u;
#pragma unroll
        for (int c = 0; c < 4; ++c) {
            const f32x4* Q = L[r];
            u32x4 H;
            if (isW[r]) { H.x = pk2(Q[0][c], Q[1][c], e); H.y = pk2(Q[2][c], Q[3][c], e); H.z = pk2(Q[4][c], Q[5][c], e); H.w = pk2(Q[6][c], Q[7][c], e); }
            else { H.x = pk2(Q[2 * c].x, Q[2 * c].y, e); H.y = pk2(Q[2 * c].z, Q[2 * c].w, e); H.z = pk2(Q[2 * c + 1].x, Q[2 * c + 1].y, e); H.w = pk2(Q[2 * c + 1].z, Q[2 * c + 1].w, e); }
            if ((r || c > 1) && (r < NPUSH - 1 || rfl(t >> 6) != 1u)) asm volatile("s_waitcnt vmcnt(" STR(PUSH_W) ")" ::: "memory");
            store16_sys(ib + ((c >> 1) * HTB + (c & 1) * c1), so, H);
        }
    }
    asm volatile("s_waitcnt vmcnt(0)" ::: "memory");
    unsigned old = 0;
    if ((t & 63u) == 0) old = __hip_atomic_fetch_add((LAS unsigned*)(lds + L_ECNT), 1u, __ATOMIC_RELAXED, __HIP_MEMORY_SCOPE_WORKGROUP);
    if (rfl(old) == 7u && (t & 63u) < NPUSH) {
        __hip_atomic_store((GAS unsigned long long*)(dws + WS_IFLAG) + ((call & 1u) * 8 + (6 - link)) * 16 + item + 8 * (t & 63u), ((unsigned long long)call << 32) | ((t & 63u) ? cl[NPUSH - 1] : cl[0]),
                           __ATOMIC_RELAXED, __HIP_MEMORY_SCOPE_SYSTEM);
    }
    return call;
}

template <int SLEEP> __device__ __forceinline__ bool flag_wait(const gchar* f, gu32* tmo, unsigned call, unsigned& info) {
    const unsigned long long t0 = __builtin_amdgcn_s_memrealtime();
    for (;;) {
        const unsigned long long fv = __hip_atomic_load((const GAS unsigned long long*)f, __ATOMIC_RELAXED, __HIP_MEMORY_SCOPE_SYSTEM);
        info = rfl((unsigned)fv);
        if (rfl((unsigned)(fv >> 32)) >= call) break;
        if (rfl(ld_agt(tmo)) == call || __builtin_amdgcn_s_memrealtime() - t0 > DEADLINE) {
            if ((threadIdx.x & 63u) == 0) __hip_atomic_store(tmo, call, __ATOMIC_RELAXED, __HIP_MEMORY_SCOPE_AGENT);
            return false;
        }
        __builtin_amdgcn_s_sleep(SLEEP);
    }
    asm volatile("" ::: "memory");
    return true;
}

__device__ __forceinline__ void fetch_inputs(const Params& p, LAS unsigned char* lds, gchar* myws, int rank, unsigned call, int srel, int first) {
    const unsigned lane = threadIdx.x & 63u, wv = rfl(threadIdx.x >> 6);
    gu32* tmo = (gu32*)(myws + WS_TMO);
    const unsigned fi = (call & 1u) * 8 + srel;
    const int src = (rank + 1 + srel) & 7;
    if (wv == 1) {
        unsigned k = 0;
        for (;;) {
            unsigned tk = 0;
            if (lane == 0) tk = __hip_atomic_fetch_add((gu32*)(myws + WS_LINKQ + srel * 128), 1u, __ATOMIC_RELAXED, __HIP_MEMORY_SCOPE_AGENT);
            const int i = 8 * NPUSH + (int)(rfl(tk) % (NITEM + 8u - 8u * NPUSH));
            unsigned info = 0; bool ok = true;
            if (i < NITEM) { int isW, j, u; item_decode(i, rank, src, isW, j, u); ok = flag_wait<2>(myws + WS_AFLAG + ((fi * 2 + isW) * 128 + j * 16 + u) * 8, tmo, call, info); }
            if (ok) { if (lane == 0) *(vl128*)(lds + L_DESC + (k & 7u) * 16) = (u32x4){k + 1u, (unsigned)i, info, 0u}; ++k; }
            if (i >= NITEM) break;
            if (ok) lds_wait1(lds, L_LST, k);
        }
    } else if (wv == 0) {
        const gchar* sws = uni(p.ws[src]);
        __builtin_amdgcn_s_setprio(SEND_PRIO);
#define LWAIT(off, v, n) do { if (rfl(*(vl32*)(lds + (off))) < (v)) { asm volatile("s_waitcnt vmcnt(0)" ::: "memory"); land = (n); if (lane == 0) *(vl32*)(lds + L_LAND) = land; lds_wait1(lds, off, v); } } while (0)
        unsigned k = 0, land = 0; bool handed = false;
        for (;; ++k) {
            LWAIT(L_DESC + (k & 7u) * 16, k + 1u, 4u * k);
            const u32x4 de = *(vl128*)(lds + L_DESC + (k & 7u) * 16);
            const int i = (int)rfl(de.y); const unsigned info = rfl(de.z);
            if (i >= NITEM) break;
            if (lane == 0) *(vl32*)(lds + L_LST) = k + 1u;
            int isW, j, u; item_decode(i, rank, src, isW, j, u);
            const gchar* ub = uni(sws + pin_off(call & 1u, isW, 6 - srel, j, u));
#pragma unroll
            for (unsigned q = 0; q < 4; ++q) {
                if (k >= 4) LWAIT(L_FREE + ((k & 3u) * 4 + q) * 4, k - 3u, 4u * k + q);
                if (q == HAND_Q) {
                    const u32x4 dn = *(vl128*)(lds + L_DESC + ((k + 1u) & 7u) * 16);
                    handed = rfl(dn.x) == k + 2u && rfl(dn.y) >= (unsigned)NITEM;
                    if (handed && lane == 0) __hip_atomic_fetch_add((gu32*)(myws + WS_OREL + srel * 128), 1u, __ATOMIC_RELAXED, __HIP_MEMORY_SCOPE_AGENT);
                }
                const unsigned size = Q_LEN + ((info >> (10u + 5u * q)) & 31u) * 64u, dst = (unsigned)(uintptr_t)lds + (k & 3u) * UNIT + q * Q_BYTES;
#pragma unroll
                for (int c = 0; c < 8; ++c) {
                    if (c < 6 || c * 1024u + lane * 16u < size) glds16_sys(dst + c * 1024u, lane * 16u, ub + (q * Q_BYTES + c * 1024u));
                    asm volatile("s_waitcnt vmcnt(" STR(PULL_N) ")" ::: "memory");
                    if (c == (PULL_N - 1) % 7 && 4u * k + q > land + (PULL_N > 7)) { land = 4u * k + q - (PULL_N > 7); if (lane == 0) *(vl32*)(lds + L_LAND) = land; }
                }
            }
        }
        if (!handed && lane == 0) __hip_atomic_fetch_add((gu32*)(myws + WS_OREL + srel * 128), 1u, __ATOMIC_RELAXED, __HIP_MEMORY_SCOPE_AGENT);
        asm volatile("s_waitcnt vmcnt(0)" ::: "memory");
        if (lane == 0) *(vl32*)(lds + L_LAND) = 4u * k;
        __builtin_amdgcn_s_setprio(0);
    } else {
        unsigned inf0;
        for (int it = first; it < 8 * NPUSH; it += 8)
            if (wv >= 4 && flag_wait<2>(myws + WS_IFLAG + (fi * 16 + it) * 8, tmo, call, inf0)) {
                const __amdgpu_buffer_rsrc_t rs = __builtin_amdgcn_make_buffer_rsrc((void*)uni(myws + WS_INBOX + (size_t)(((call & 1u) * 7 + srel) * 16 + it) * UNIT), 0, UNIT, 0x00020000);
                u32x4 W[8];
#pragma unroll
                for (int c = 0; c < 8; ++c) W[c] = RLOAD4(rs, lane * 16u, (wv - 4u) * Q_BYTES + c * 1024u);
                int isW, j, u; item_decode(it, rank, src, isW, j, u);
                const gchar* ub; gu32* f = stage_unit(myws, srel, isW, j, u, ub); ub = uni(ub);
                __builtin_amdgcn_s_waitcnt(0x0F70);
#pragma unroll
                for (int c = 0; c < 8; ++c) store16_sys(ub + ((wv - 4u) * Q_BYTES + c * 1024u), lane * 16u, W[c]);
                asm volatile("s_waitcnt vmcnt(0)" ::: "memory");
                if (lane == 0 && __hip_atomic_fetch_add((LAS unsigned*)(lds + L_QI + (it >> 3) * 4), 1u, __ATOMIC_RELAXED, __HIP_MEMORY_SCOPE_WORKGROUP) == 3u)
                    __hip_atomic_store(f, (call << 8) | (inf0 & 31u), __ATOMIC_RELAXED, __HIP_MEMORY_SCOPE_SYSTEM);
            }
        for (unsigned n = wv - 2u;; n += 6u) {
            const unsigned k = n >> 2, q = n & 3u;
            unsigned kd = k;
            for (;;) {
                const u32x4 d0 = *(vl128*)(lds + L_DESC + (k & 7u) * 16), d1 = *(vl128*)(lds + L_DESC + ((k + 7u) & 7u) * 16);
                if (rfl(d0.x) >= k + 1u) break;
                if (k > 0 && rfl(d1.x) == k && rfl(d1.y) >= (unsigned)NITEM) { kd = k - 1u; break; }
                __builtin_amdgcn_s_sleep(1);
            }
            const u32x4 de = *(vl128*)(lds + L_DESC + (kd & 7u) * 16);
            const int i = (int)rfl(de.y); const unsigned info = rfl(de.z);
            if (i >= NITEM) break;
            lds_wait1(lds, L_LAND, n + 1u);
            const bool raw = ((info >> (10u + 5u * q)) & 31u) == 24u;
            QIn Q;
            q_read(lds + (k & 3u) * UNIT + q * Q_BYTES, lane, raw, Q);
            LGKM0;
            if (lane == 0) *(vl32*)(lds + L_FREE + ((k & 3u) * 4 + q) * 4) = k + 1u;
            u32x4 W[8];
            if (!raw) q_finish((info >> 5) & 31u, Q, W);
            q_patch(raw, Q, W);
            int isW, j, u; item_decode(i, rank, src, isW, j, u);
            const gchar* ub; gu32* f = stage_unit(myws, srel, isW, j, u, ub); ub = uni(ub);
#pragma unroll
            for (int c = 0; c < 8; ++c) store16_sys(ub + img_off(isW, q, c), lane * 16u, W[c]);
            asm volatile("s_waitcnt vmcnt(0)" ::: "memory");
            if (lane == 0) {
                const unsigned old = __hip_atomic_fetch_add((LAS unsigned*)(lds + L_UCNT + (k & 7u) * 4), 1u, __ATOMIC_RELAXED, __HIP_MEMORY_SCOPE_WORKGROUP);
                if ((old & 3u) == 3u) __hip_atomic_store(f, (call << 8) | (info & 31u), __ATOMIC_RELAXED, __HIP_MEMORY_SCOPE_SYSTEM);
            }
        }
    }
    LBAR;
}

__device__ __forceinline__ void tile_order(unsigned T, unsigned& row, unsigned& col) { row = T < 2u * NE ? T / NE : T & 1u; col = T < 2u * NE ? T % NE : T >> 1; }
constexpr unsigned NCT = 256u + NOW;

__device__ __forceinline__ void fetch_outputs(const Params& p, LAS unsigned char* lds, gchar* myws, int rank, unsigned call, int link) {
    unsigned lane = threadIdx.x; asm volatile("" : "+v"(lane)); lane &= 63u;
    const unsigned wv = rfl(threadIdx.x >> 6);
    gu32* tmo = (gu32*)(myws + WS_TMO);
    const int src = (rank + 1 + link) & 7;
    const gchar* sws = uni(p.ws[src]);
    asm volatile("s_waitcnt vmcnt(0)" ::: "memory");
    LBAR;
    if (wv == 1) {
        for (unsigned k = 0;; ++k) {
            unsigned ct = 0;
            if (lane == 0) ct = __hip_atomic_fetch_add((gu32*)(myws + WS_OTK + link * 128 + 64), 1u, __ATOMIC_RELAXED, __HIP_MEMORY_SCOPE_AGENT);
            ct = rfl(ct) % NCT;
            unsigned info = 0, ci = 256u; bool ok = true;
            if (ct < 256u) { unsigned row, col; tile_order(ct >> 3, row, col); ci = (row * 16u + col) * 8u + (ct & 7u); ok = flag_wait<2>(myws + WS_CFLAG + (src * 256 + ci) * 8, tmo, call, info); }
            else if (lane == 0) *(vl32*)(lds + L_TURN) = k + 1u;
            if (lane == 0) *(vl128*)(lds + L_DESC + (k & 7u) * 16) = (u32x4){k + 1u, ci, ok ? info : 0u, ct};
            if (ct >= 256u) break;
            lds_wait1(lds, L_LST, k + 1u);
        }
    } else if (wv == 0) {
        {
            unsigned tk = 0;
            if (lane == 0) tk = __hip_atomic_fetch_add((gu32*)(myws + WS_OTK + link * 128), 1u, __ATOMIC_RELAXED, __HIP_MEMORY_SCOPE_AGENT);
            const unsigned need = (call - 1u) * 8u + (rfl(tk) % NOW) * 8u / NOW + 1u - LINK_EXTRA;
            const unsigned long long t0 = __builtin_amdgcn_s_memrealtime();
            while ((int)(need - rfl(ld_agt((const gu32*)(myws + WS_OREL + link * 128)))) > 0 && __builtin_amdgcn_s_memrealtime() - t0 < DEADLINE) __builtin_amdgcn_s_sleep(4);
        }
        __builtin_amdgcn_s_setprio(SEND_PRIO);
        unsigned k = 0, land = 0;
        for (;; ++k) {
            LWAIT(L_DESC + (k & 7u) * 16, k + 1u, 2u * k);
            const u32x4 de = *(vl128*)(lds + L_DESC + (k & 7u) * 16);
            const unsigned ci = rfl(de.y), tail = min(max(rfl(de.z) >> 8, 128u), 3072u);
            if (ci >= 256u) break;
            if (lane == 0) *(vl32*)(lds + L_LST) = k + 1u;
            if (k >= 4) LWAIT(L_FREE + (k & 3u) * 4, k - 3u, 2u * k);
            const gchar* rb = uni(sws + WS_PACK + (size_t)(rank * 256 + ci) * CHUNK);
            const unsigned dst = (unsigned)(uintptr_t)lds + (k & 3u) * UNIT;
#pragma unroll
            for (int n = 0; n < 29; ++n) {
                const unsigned o = n < 3 ? C_HDR + (2 - n) * 1024u : n < 5 ? (21 + n) * 1024u : (n - 5) * 1024u;
                if (n > 2 || o - C_HDR + lane * 16u < tail) glds16_sys(dst + o, lane * 16u, rb + o);
                if (n >= 2) asm volatile("s_waitcnt vmcnt(" STR(OPULL_N) ")" ::: "memory");
                if ((n == 1 + OPULL_N || n == 4 + OPULL_N) && 2u * k + (n == 4 + OPULL_N) > land) { land = 2u * k + (n == 4 + OPULL_N); if (lane == 0) *(vl32*)(lds + L_LAND) = land; }
            }
        }
        asm volatile("s_waitcnt vmcnt(0)" ::: "memory");
        if (lane == 0) *(vl32*)(lds + L_LAND) = 2u * k;
    } else if (wv >= 4) {
        for (unsigned k = wv - 4u;; k += 4u) {
            bool end = false;
            while (rfl(*(vl32*)(lds + L_DESC + (k & 7u) * 16)) < k + 1u && !(end = rfl(*(vl32*)(lds + L_TURN)) - 1u <= k)) __builtin_amdgcn_s_sleep(2);
            if (end) break;
            const u32x4 de = *(vl128*)(lds + L_DESC + (k & 7u) * 16);
            const unsigned ci = rfl(de.y), info = rfl(de.z);
            if (ci >= 256u) break;
            unsigned pv;
            const bool bad = !flag_wait<FLAG_SLEEP>(myws + WS_PFLAG + (link * 256 + ci) * 8, tmo, call, pv) || info == 0u || rfl(ld_agt(tmo)) == call;
            const gchar* ob = uni((const char*)p.out[rank] + ((size_t)((ci >> 7) * 256) * GN + src * NL + ((ci >> 3) & 15u) * 256) * 4);
            const unsigned ovo = (((ci >> 2) & 1u) * 64u + 4u * (lane >> 4)) * (GN * 4u) + ((ci & 3u) * 16u + (lane & 15u)) * 16u;
            const __amdgpu_buffer_rsrc_t ro = __builtin_amdgcn_make_buffer_rsrc((void*)ob, 0, 255u * GN * 4u + 1024u, 0x00020000);
            u32x4 A[32];
#pragma unroll
            for (int q = 0; q < 32; ++q) { A[q] = RLOAD4(ro, ovo, (unsigned)out_off(q)); asm volatile("s_waitcnt vmcnt(" STR(DEC_W) ")" ::: "memory"); }
            const LAS unsigned char* slot = lds + (k & 3u) * UNIT;
            lds_wait1(lds, L_LAND, 2u * k + 1u);
            CDec D; u32x4 X[8], CW[2];
            D.hdr = bad ? 0u : *(const LAS unsigned short*)(slot + C_HDR + lane * 2u);
            CW[0] = *(const LAS u32x4*)(slot + 24 * 1024u + lane * 16u); CW[1] = *(const LAS u32x4*)(slot + 25 * 1024u + lane * 16u);
            const __amdgpu_buffer_rsrc_t rs = __builtin_amdgcn_make_buffer_rsrc((void*)uni(sws + WS_PACK + (size_t)(rank * 256 + ci) * CHUNK), 0, CHUNK, 0x00020000);
#pragma unroll
            for (int g = 0; g < 4; ++g) if (D.hdr & (256u << g)) { X[2 * g] = RLOAD4(rs, lane * 32u, C_EXC + g * 2048u); X[2 * g + 1] = RLOAD4(rs, lane * 32u + 16u, C_EXC + g * 2048u); }
            c_begin(D, slot + C_HDR, info & 255u);
#pragma unroll
            for (int h = 0; h < 16; ++h) c_parse(D, CW, h);
            c_patch(D, X);
            __builtin_amdgcn_s_waitcnt(0x0F70);
            lds_wait1(lds, L_LAND, 2u * k + 2u);
            c_store(D, slot, A, bad, ob, ovo, lane);
            LGKM0;
            if (lane == 0) *(vl32*)(lds + L_FREE + (k & 3u) * 4) = k + 1u;
        }
    }
}

__global__ void __launch_bounds__(NTHR, 2) gemm_rs_kernel(const Params p) {
    extern __shared__ __attribute__((aligned(16))) unsigned char smem[];
    LAS unsigned char* lds = (LAS unsigned char*)smem;
    const int tid = threadIdx.x;
    const int rank = (int)rfl(p.rank), bid = (int)rfl(blockIdx.x);
    gchar* myws = uni(p.ws[rank]);
    STAMP(0);
    const int xcd = bid & 7, slot = bid >> 3;
    gu32* tmo = (gu32*)(myws + WS_TMO);

    int pm, pn;
    {
        int ar = -1;
        if (slot < 11) {
            const int c = slot < 7 ? xcd * 7 + slot : xcd * 4 + slot - 7, o = slot < 7 ? c - 28 : c;
            if (c < 28) { ar = c >> 1; pn = (slot < 7 ? 14 : 12) + (c & 1); } else { pm = 2 * rank + (o & 1); pn = o >> 1; }
        } else {
            const int li = slot - 11, q = li / 7;
            pn = 3 * (xcd & 3) + q; ar = 2 * (li - q * 7) + (xcd >> 2);
        }
        if (ar >= 0) pm = ar + (ar >= 2 * rank ? 2 : 0);
    }
    const int enc = slot < 7 ? -1 : slot < 11 ? bid - 56 : pn >= NE ? 32 + (pn - NE) * 14 + pm - (pm > 2 * rank ? 2 : 0) : -1;
    unsigned call;
    if (slot < 7) { call = push_first(p, lds, myws, rank, slot, xcd); STAMP(5); fetch_inputs(p, lds, myws, rank, call, slot, xcd); }
    else { call = produce(p, lds, myws, rank, slot < 11 ? bid + 24 : slot < 21 ? bid - 88 : bid - 56, enc >= 0 ? 200 + enc : NTASK, enc >= 0 ? (gu32*)(myws + WS_TASKQ) : nullptr); STAMP(5); }
    STAMP(1);

    const int wid = rfl(tid >> 6), wr = wid >> 2, wc = wid & 3;
#define LANE_VALS int lane = tid; asm volatile("" : "+v"(lane)); lane &= 63; const int fr = lane & 15, fq = lane >> 4
    const int dest = pm >> 1, link = (dest - rank - 1) & 7;
    constexpr int NMW = NOW > NE ? 2 * NE - NOW : NE;
    const bool early = slot >= 11 && pn < NE, odd = (pm ^ pn) & 1, worker = early && (odd ? pn < NOW : pn >= NMW);
    const int mini0 = enc >= 0 ? enc : early && !odd && !worker ? NENC + pn * 7 + link : 224, ministep = enc >= 0 ? 224 : 7 * NMW;
    const int njob = 1 + (224 - mini0 + ministep - 1) / ministep, jmain = enc >= 0 ? 1 : 0;
    gchar* dws = uni(p.ws[dest]);
    LAS unsigned char* scr = lds + L_SCR + wid * 3072;

#define KIDX(t) ((t) + ((((t) + 6 - link) * kmag) >> 8))
#define SA(b, h) (((b) * 2 + (h)) * HTB)
#define SB(b, h) ((4 + (b) * 2 + (h)) * HTB)
#define STAGE(bufoff, gbase) do { _Pragma("unroll") for (int _i = 0; _i < 2; ++_i) \
        __builtin_amdgcn_global_load_lds((const gu32*)((gbase) + voff[_i]), (LAS unsigned*)(lds + (bufoff) + ldsw + _i * 8192), 16, 0, GLDS_AUX); } while (0)
#define LDA(dst, b, h) do { _Pragma("unroll") for (int m = 0; m < 4; ++m) { dst[m][0] = *(const LAS h16x8*)(lds + SA(b, h) + aoff0 + m * 2048); dst[m][1] = *(const LAS h16x8*)(lds + SA(b, h) + aoff1 + m * 2048); } } while (0)
#define LDB(dst, b, h) do { _Pragma("unroll") for (int n = 0; n < 2; ++n) _Pragma("unroll") for (int k = 0; k < 2; ++k) dst[n][k] = *(const LAS h16x8*)(lds + SB(b, h) + boff + n * 1024 + k * 8192); } while (0)
#define MMA(ai, bj, At, Bt) do { __builtin_amdgcn_s_setprio(1); _Pragma("unroll") for (int m = 0; m < 4; ++m) _Pragma("unroll") for (int n = 0; n < 2; ++n) _Pragma("unroll") for (int k = 0; k < 2; ++k) \
        acc[ai][bj][m][n] = __builtin_amdgcn_mfma_f32_16x16x32_f16(At[m][k], Bt[n][k], acc[ai][bj][m][n], 0, 0, 0); __builtin_amdgcn_s_setprio(0); } while (0)
#define WAIT_V(n) asm volatile("s_waitcnt vmcnt(" #n ")" ::: "memory")
#define WAIT_L(n) asm volatile("s_waitcnt lgkmcnt(" #n ")" ::: "memory")
#define BAR __builtin_amdgcn_s_barrier()
#define SCHED __builtin_amdgcn_sched_barrier(0)
#define HOOK(need) do { if (ready < (need)) { \
        if (wid == 0) { \
            const unsigned long long t0 = __builtin_amdgcn_s_memrealtime(); unsigned fa, fb; int rd; \
            unsigned ln = voff[0]; asm volatile("" : "+v"(ln)); ln >>= 4;     \
            for (;;) { \
                { const int li_ = KIDX(min((int)ln, nk - 1)); fa = ld_sys(fA + li_); fb = ld_sys(fB + li_); } \
                const unsigned long long okm = __ballot((fa >> 8) >= call && (fb >> 8) >= call); \
                rd = (okm == ~0ull) ? 64 : (int)__builtin_ctzll(~okm); \
                if (rd >= (need)) break; \
                if (rfl(ld_agt(tmo)) == call || __builtin_amdgcn_s_memrealtime() - t0 > DEADLINE) { \
                    if (ln == 0) __hip_atomic_store(tmo, call, __ATOMIC_RELAXED, __HIP_MEMORY_SCOPE_AGENT); \
                    rd = 64; break; } \
                __builtin_amdgcn_s_sleep(POLL_SLEEP); \
            } \
            const int sv = (int)((fa & 31u) + (fb & 31u)); \
            const int sp = __builtin_amdgcn_update_dpp(-1, sv, 0x138, 0xf, 0xf, false);     \
            const unsigned long long cg = __ballot(sv != sp); \
            *(LAS int*)(lds + L_TAB + ln * 4) = sv; \
            if (ln == 0) { u32x4 h; h.x = (unsigned)rd; h.y = (unsigned)cg; h.z = (unsigned)(cg >> 32); h.w = 0; *(LAS u32x4*)(lds + L_HDR) = h; } \
            DIAG_WAIT(t0); \
            asm volatile("s_waitcnt vmcnt(0) lgkmcnt(0)" ::: "memory"); \
        } \
        asm volatile("" ::: "memory"); BAR; asm volatile("" ::: "memory"); \
        { const u32x4 h = *(const LAS u32x4*)(lds + L_HDR); ready = (int)rfl(h.x); chg = ((unsigned long long)rfl(h.z) << 32) | rfl(h.y); } \
    } } while (0)
#define DIAG_WAIT(t0) do {} while (0)
#define PKMUL4(op) ".set rs_i, 0\n\t.set rs_d, 0\n\t.irpc c, \"" op "\"\n\t.ifc \\c,:\n\t.set rs_d, 1\n\t.endif\n\t" \
        ".if rs_d == 0\n\t.ifnc \\c,v\n\t.ifnc \\c,[\n\t.set rs_i, rs_i*10 + \\c\n\t.endif\n\t.endif\n\t.endif\n\t.endr\n\t" \
        "v_pk_mul_f32 v[rs_i:rs_i+1], v[rs_i:rs_i+1], %16\n\tv_pk_mul_f32 v[rs_i+2:rs_i+3], v[rs_i+2:rs_i+3], %16\n\t"
#define RS16(a) asm volatile("s_cmp_eq_u32 %17, 0\n\ts_cbranch_scc1 .Lrs%=\n\ts_nop 15\n\ts_nop 3\n\t" \
            PKMUL4("%0") PKMUL4("%1") PKMUL4("%2") PKMUL4("%3") PKMUL4("%4") PKMUL4("%5") PKMUL4("%6") PKMUL4("%7") \
            PKMUL4("%8") PKMUL4("%9") PKMUL4("%10") PKMUL4("%11") PKMUL4("%12") PKMUL4("%13") PKMUL4("%14") PKMUL4("%15") \
            "s_nop 4\n.Lrs%=:" \
            : "+v"(acc[a][0][0][0]), "+v"(acc[a][0][0][1]), "+v"(acc[a][0][1][0]), "+v"(acc[a][0][1][1]), "+v"(acc[a][0][2][0]), "+v"(acc[a][0][2][1]), "+v"(acc[a][0][3][0]), "+v"(acc[a][0][3][1]), \
              "+v"(acc[a][1][0][0]), "+v"(acc[a][1][0][1]), "+v"(acc[a][1][1][0]), "+v"(acc[a][1][1][1]), "+v"(acc[a][1][2][0]), "+v"(acc[a][1][2][1]), "+v"(acc[a][1][3][0]), "+v"(acc[a][1][3][1]) \
            : "s"(fac), "s"(de) : "scc")
#define RESCALE(kt) do { int de = 0; \
        if ((chg >> (kt)) & 1ull) { \
            const int s = (int)rfl(*(const LAS unsigned*)(lds + L_TAB + (kt) * 4)); \
            smax = max(smax, s); const int sn = max(s, smax - 4); de = max(16 * (scur - sn), -126); scur = sn; } \
        const unsigned long long fac = (unsigned long long)(unsigned)((de + 127) << 23) * 0x100000001ull; \
        RS16(0); RS16(1); \
    } while (0)
#define TRIP do { \
        const int t2 = t + 2 < nk ? t + 2 : 0;          \
        const unsigned o1 = KIDX(t + 1) * kstr, o2 = KIDX(t2) * kstr, o3 = KIDX(t2 + 1) * kstr; \
        const gchar* a1 = cA + o1; const gchar* a2 = cA + o2; const gchar* b2 = cB + o2; const gchar* a3 = cA + o3; const gchar* b3 = cB + o3; \
        RESCALE(t); \
        LDB(B0, 0, 0); SCHED; LDA(At, 0, 0); STAGE(SA(1, 1), a1 + HTB); \
        WAIT_L(8); BAR; WAIT_L(0); MMA(0, 0, At, B0); BAR; SCHED; \
        LDB(B1, 0, 1); STAGE(SB(0, 0), b2); \
        BAR; WAIT_L(0); MMA(0, 1, At, B1); BAR; \
        LDA(At, 0, 1); STAGE(SA(0, 0), a2); \
        BAR; WAIT_L(0); MMA(1, 0, At, B0); BAR; SCHED; \
        STAGE(SB(0, 1), b2 + HTB); \
        WAIT_V(6); BAR; MMA(1, 1, At, B1); BAR; \
        RESCALE(t + 1); \
        LDB(B0, 1, 0); SCHED; LDA(At, 1, 0); STAGE(SA(0, 1), a2 + HTB); \
        WAIT_L(8); BAR; WAIT_L(0); MMA(0, 0, At, B0); BAR; SCHED; \
        LDB(B1, 1, 1); STAGE(SB(1, 0), b3); \
        BAR; WAIT_L(0); MMA(0, 1, At, B1); BAR; \
        LDA(At, 1, 1); STAGE(SA(1, 0), a3); \
        BAR; WAIT_L(0); MMA(1, 0, At, B0); BAR; SCHED; \
        STAGE(SB(1, 1), b3 + HTB); \
        WAIT_V(6); BAR; MMA(1, 1, At, B1); BAR; \
    } while (0)

    for (int job = 0; job < njob; ++job) {
    const bool mini = job != jmain;
    const int mi = mini0 + (job - (job > jmain)) * ministep, mT = mi / 7, ml = mi - mT * 7;
    const int mrow = mT < 2 * NE ? mT / NE : mT & 1, mcol = mT < 2 * NE ? mT % NE : mT >> 1;
    const int ua = mini ? (112 + mrow) * 8 : pm, ub_ = mini ? (ml * 16 + mcol) * 8 : 16 + pn;
    const int nk = mini ? 8 : dest == rank ? NKT : NKT - 8, kmag = nk == NKT - 8 ? 37 : 0;
    const unsigned kstr = mini ? UNIT : KT_BYTES;
    const gchar* cA = myws + (mini ? WS_STAGE2 : WS_STAGE) + (size_t)ua * UNIT;
    const gchar* cB = myws + (mini ? WS_STAGE2 : WS_STAGE) + (size_t)ub_ * UNIT;
    const gu32* fA = (const gu32*)(myws + (mini ? WS_FLAG2 : WS_FLAG)) + ua * (mini ? 1 : NKT);
    const gu32* fB = (const gu32*)(myws + (mini ? WS_FLAG2 : WS_FLAG)) + ub_ * (mini ? 1 : NKT);
    int ready = 0; unsigned long long chg = 0; int scur = 0, smax = 0;
    u32x4 V[32];
    {
    LANE_VALS;
    const unsigned voff[2] = {(unsigned)(wid * 64 + lane) * 16u, (unsigned)(wid * 64 + lane) * 16u + 8192u}; const unsigned ldsw = (unsigned)wid * 1024u;
    const unsigned aoff0 = (unsigned)((wr * 64 + fr) * 128 + ((fq ^ (fr >> 1)) * 16)), aoff1 = aoff0 ^ 64u;
    const unsigned boff = (unsigned)(fq * 2048 + (wc * 16 + fr) * 16);
    f32x4 acc[2][2][4][2];
#pragma unroll
    for (int a = 0; a < 2; ++a)
#pragma unroll
        for (int b = 0; b < 2; ++b)
#pragma unroll
            for (int m = 0; m < 4; ++m)
#pragma unroll
                for (int n = 0; n < 2; ++n) acc[a][b][m][n] = (f32x4){0.f, 0.f, 0.f, 0.f};
    h16x8 At[4][2], B0[2][2], B1[2][2];

    WAIT_V(0);
    if (mini && job > jmain && wid == 0) {
        const unsigned long long t0 = __builtin_amdgcn_s_memrealtime();
        while ((int)(rfl(ld_agt((const gu32*)(myws + WS_OTK + ml * 128 + 64))) % NCT) < mT * 8 - MINI_LEAD && __builtin_amdgcn_s_memrealtime() - t0 < 100000ull)
            __builtin_amdgcn_s_sleep(64);
    }
    HOOK(2);
    if (!mini) STAMP(2); else STAMP(20 + 4 * (job & 1));
    const unsigned ok1 = KIDX(1) * kstr;
    STAGE(SB(0, 0), cB); STAGE(SA(0, 0), cA); STAGE(SB(0, 1), cB + HTB); STAGE(SA(0, 1), cA + HTB);
    if (wr == 1) BAR;
    WAIT_V(4); BAR;
    STAGE(SB(1, 0), cB + ok1); STAGE(SA(1, 0), cA + ok1); STAGE(SB(1, 1), cB + ok1 + HTB);
    WAIT_V(6); BAR;
    if (mini) STAMP(21 + 4 * (job & 1));
    for (int t = 0; t < nk; t += 2) {
        HOOK(min(t + 4, nk));
        TRIP;
    }
    if (wr == 0) BAR;
    if (!mini) STAMP(3); else STAMP(22 + 4 * (job & 1));

    const float qn = __builtin_nanf("");
    {
        const bool bad = rfl(ld_agt(tmo)) == call;
        const int oe = 16 * scur - 256;
#pragma unroll
        for (int a = 0; a < 2; ++a)
#pragma unroll
            for (int m = 0; m < 4; ++m)
#pragma unroll
                for (int r = 0; r < 4; ++r) {
                    f32x4 v;
                    v.x = __builtin_amdgcn_ldexpf(acc[a][0][m][0][r], oe); v.y = __builtin_amdgcn_ldexpf(acc[a][0][m][1][r], oe);
                    v.z = __builtin_amdgcn_ldexpf(acc[a][1][m][0][r], oe); v.w = __builtin_amdgcn_ldexpf(acc[a][1][m][1][r], oe);
                    if (bad) v = (f32x4){qn, qn, qn, qn};
                    V[(a * 4 + m) * 4 + r] = __builtin_bit_cast(u32x4, v);
                }
    }
    }
    LANE_VALS;
    const unsigned ovo = (unsigned)(wr * 64 + 4 * fq) * (GN * 4u) + (unsigned)(wc * 16 + fr) * 16u;
#define OSTAMP(k) do {} while (0)
#define DIAG_END do {} while (0)
    if (mini || dest == rank) {
        const gchar* ob = uni((const char*)p.out[rank] + ((size_t)((mini ? mrow : pm & 1) * 256) * GN + (mini ? ((rank + 1 + ml) & 7) * NL + mcol * 256 : rank * NL + pn * 256)) * 4);
#pragma unroll
        for (int q = 0; q < 32; ++q) { store16_sys(ob + out_off(q), ovo, V[q]); asm volatile("s_waitcnt vmcnt(" STR(LOC_W) ")" ::: "memory"); }
        WAIT_V(0);
        if (mini && lane == 0) __hip_atomic_store((GAS unsigned long long*)(myws + WS_PFLAG) + ml * 256 + (mrow * 16 + mcol) * 8 + wid, (unsigned long long)call << 32, __ATOMIC_RELAXED, __HIP_MEMORY_SCOPE_SYSTEM);
    }
    if (mini) STAMP(23 + 4 * (job & 1));
    int packed = mini ? 0 : dest - rank;
    asm volatile("" : "+v"(packed)); packed = (int)rfl(packed);
    if (packed) {
        const unsigned cidx = (unsigned)(((pm & 1) * 16 + pn) * 8 + wid);
        const gchar* pb = uni(myws + WS_PACK + (size_t)(dest * 256 + cidx) * CHUNK);
        unsigned mu = 0; float sm = 0.f;
#pragma unroll
        for (int q = 0; q < 32; ++q) {
            const u32x4 w = V[q] & 0x7fffffffu; mu = max(max(mu, w.x), max(w.y, max(w.z, w.w)));
            const f32x4 f = __builtin_bit_cast(f32x4, w); sm += (f.x + f.y) + (f.z + f.w);
        }
        const unsigned myB = min(max(min(wave_mean_exp(sm * (2.32f / 8192.f)), wave_max(mu >> 23)), 2u), 254u);
        u32x4 PK[26];
        const unsigned tail = c_encode(V, myB, PK, scr, pb, lane);
        asm volatile("" ::: "memory");
#pragma unroll
        for (int q = 0; q < 3; ++q) if (q * 1024u + lane * 16u < tail) store16_sys(pb + (C_HDR + q * 1024u), lane * 16u, *(const LAS u32x4*)(scr + q * 1024u + lane * 16u));
#pragma unroll
        for (int q = 0; q < 26; ++q) { store16_sys(pb + q * 1024u, lane * 16u, PK[q]); WAIT_V(4); }
        WAIT_V(0);
        if (lane == 0) __hip_atomic_store((GAS unsigned long long*)(dws + WS_CFLAG) + rank * 256 + cidx, ((unsigned long long)call << 32) | (tail << 8) | myB, __ATOMIC_RELAXED, __HIP_MEMORY_SCOPE_SYSTEM);
        OSTAMP(31);
    }
    }
    if (worker) fetch_outputs(p, lds, myws, rank, call, link);
    DIAG_END;
}

extern "C" void kernel_launch(int rank, int nranks, ncclComm_t comm,
                              hipStream_t stream,
                              void* const* d_in, const int* in_sizes, int n_in,
                              void* d_out, int out_size,
                              const int* in_dtypes, int out_dtype,
                              void* const* d_out_all,
                              void* const* d_ws_all, size_t ws_bytes) {
    (void)comm; (void)in_dtypes; (void)out_dtype; (void)d_out;
    if (nranks != NR || (unsigned)rank >= (unsigned)NR || n_in < 2 || ws_bytes < WS_NEED ||
        in_sizes[0] != GM * KS || in_sizes[1] != KS * GN || out_size != (GM / NR) * GN) {
        fprintf(stderr, "gemm_rs: unexpected problem shape\n");
        return;
    }
    HIP_CHECK(hipFuncSetAttribute((const void*)gemm_rs_kernel, hipFuncAttributeMaxDynamicSharedMemorySize, (int)LDS_BYTES));
    Params p;
    p.x = (const float*)d_in[0]; p.w = (const float*)d_in[1];
    for (int i = 0; i < NR; ++i) { p.out[i] = (float*)d_out_all[i]; p.ws[i] = (char*)d_ws_all[i]; }
    p.rank = rank; p.pad = 0;
    hipLaunchKernelGGL(gemm_rs_kernel, dim3(GRID), dim3(NTHR), LDS_BYTES, stream, p);
    HIP_CHECK(hipGetLastError());
}
```

```cpp
#include <hip/hip_runtime.h>
#include <rccl/rccl.h>
#include <cstdio>
#include <cstdint>

#define HIP_CHECK(expr) do { hipError_t _e = (expr); if (_e != hipSuccess) { \
    fprintf(stderr, "HIP error %s at %s:%d\n", hipGetErrorString(_e), __FILE__, __LINE__); \
    return; } } while (0)

#define LAS __attribute__((address_space(3)))
#define GAS __attribute__((address_space(1)))
typedef GAS char gchar;
typedef GAS unsigned gu32;
typedef _Float16 h16x8 __attribute__((ext_vector_type(8)));
typedef float f32x4 __attribute__((ext_vector_type(4)));
typedef float f32x2 __attribute__((ext_vector_type(2)));
typedef unsigned u32x4 __attribute__((ext_vector_type(4)));

constexpr int NR = 8;
constexpr int GM = 4096, GN = 32768, KS = 512, NL = 4096;
constexpr int GRID = 256, NTHR = 512;
constexpr unsigned UNIT = 32768;
constexpr unsigned HTB = 16384;
constexpr int NKT = 64;
constexpr size_t WS_TICKET = 0, WS_TMO = 128, WS_TASKQ = 256, WS_LINKQ = 1024, WS_OTK = 2048, WS_OREL = 3072;
constexpr size_t WS_FLAG = 4096;
constexpr size_t WS_CFLAG = 16384;
constexpr size_t WS_AFLAG = 32768;
constexpr size_t WS_IFLAG = 12288;
constexpr size_t WS_FLAG2 = 65536;
constexpr size_t WS_PFLAG = 73728;
constexpr size_t WS_INBOX = 131072;
constexpr size_t WS_PX = WS_INBOX + (size_t)112 * UNIT;
constexpr size_t WS_PW = WS_PX + (size_t)256 * UNIT;
constexpr size_t WS_STAGE = WS_PW + (size_t)14 * 128 * UNIT;
constexpr unsigned KT_BYTES = 32 * UNIT;
constexpr unsigned CHUNK = 34816;
constexpr size_t WS_STAGE2 = WS_STAGE + (size_t)NKT * KT_BYTES;
constexpr size_t WS_PACK = WS_STAGE2 + (size_t)114 * 8 * UNIT;
constexpr size_t WS_NEED = WS_PACK + (size_t)NR * 256 * CHUNK;
constexpr size_t WS_DIAG = 65536, WS_DBASE = WS_NEED;
__device__ __forceinline__ size_t pin_off(unsigned par, int isW, int dd, int j, int u) {
    return isW ? WS_PW + (size_t)((par * 7 + dd) * 128 + j * 16 + u) * UNIT : WS_PX + (size_t)(par * 128 + j * 16 + u) * UNIT;
}
#define STR2(x) #x
#define STR(x) STR2(x)
#define DEC_W 12
#define WSEL 0
#define DSLEEP 2
#define NOW 12
#define OPULL_N 4
#define NEARLY 10
constexpr int NE = NEARLY;
constexpr unsigned long long DEADLINE = 100000000ull;
constexpr unsigned L_CTL = 4 * CHUNK + 4096;
constexpr unsigned L_AMAX = L_CTL, L_CONS = L_CTL + 32, L_TURN = L_CTL + 48, L_CALL = L_CTL + 52, L_LAND = L_CTL + 56, L_LST = L_CTL + 60, L_ITEM = L_CTL + 64, L_QI = L_CTL + 128;
constexpr unsigned L_ECNT = L_CTL + 192, L_UCNT = L_CTL + 224, L_FREE = L_CTL + 256, L_DESC = L_CTL + 320, L_SIZE = L_CTL + 448, L_ZERO_END = L_CTL + 576;
constexpr unsigned L_HDR = L_CTL + 576, L_TAB = L_CTL + 640, LDS_BYTES = L_CTL + 1024;

#define GLDS_AUX 17
struct Params {
    const float* x; const float* w;
    float* out[NR]; char* ws[NR];
    int rank; int pad;
};

#define STAMP(i) do {} while (0)

__device__ __forceinline__ void store16_sys(const gchar* sbase, unsigned voff, u32x4 v) {
    asm volatile("s_nop 4\n\tglobal_store_dwordx4 %0, %1, %2 sc0 sc1\n\ts_nop 1" :: "v"(voff), "v"(v), "s"(sbase) : "memory");
}
__device__ __forceinline__ void glds16_sys(unsigned dst, unsigned voff, const gchar* sbase) {
    unsigned keep;
    asm volatile("s_mov_b32 %0, m0\n\ts_mov_b32 m0, %1\n\ts_nop 2\n\tglobal_load_lds_dwordx4 %2, %3 sc0 sc1\n\ts_mov_b32 m0, %0" : "=&s"(keep) : "s"(dst), "v"(voff), "s"(sbase) : "memory");
}
__device__ __forceinline__ unsigned ld_sys(const gu32* q) { return __hip_atomic_load(q, __ATOMIC_RELAXED, __HIP_MEMORY_SCOPE_SYSTEM); }
__device__ __forceinline__ unsigned ld_agt(const gu32* q) { return __hip_atomic_load(q, __ATOMIC_RELAXED, __HIP_MEMORY_SCOPE_AGENT); }
__device__ __forceinline__ unsigned rfl(unsigned v) { return (unsigned)__builtin_amdgcn_readfirstlane((int)v); }

template <class T> __device__ __forceinline__ gchar* uni(T q) {
    unsigned lo = rfl((unsigned)(uintptr_t)q), hi = rfl((unsigned)((uintptr_t)q >> 32));
    asm volatile("" : "+s"(lo), "+s"(hi));
    return (gchar*)(((uintptr_t)hi << 32) | lo);
}

__device__ __forceinline__ unsigned wave_max(unsigned x) {
    int mb = (int)x;
    mb = max(mb, __builtin_amdgcn_update_dpp(0, mb, 0x111, 0xf, 0xf, false));
    mb = max(mb, __builtin_amdgcn_update_dpp(0, mb, 0x112, 0xf, 0xf, false));
    mb = max(mb, __builtin_amdgcn_update_dpp(0, mb, 0x114, 0xf, 0xf, false));
    mb = max(mb, __builtin_amdgcn_update_dpp(0, mb, 0x118, 0xf, 0xf, false));
    mb = max(mb, __builtin_amdgcn_update_dpp(0, mb, 0x142, 0xa, 0xf, false));
    mb = max(mb, __builtin_amdgcn_update_dpp(0, mb, 0x143, 0xc, 0xf, false));
    return (unsigned)__builtin_amdgcn_readlane(mb, 63);
}
__device__ __forceinline__ unsigned wave_mean_exp(float sm) {
#define DPP_ADD(ctl, rm) sm += __builtin_bit_cast(float, __builtin_amdgcn_update_dpp(0, __builtin_bit_cast(int, sm), ctl, rm, 0xf, false))
    DPP_ADD(0x111, 0xf); DPP_ADD(0x112, 0xf); DPP_ADD(0x114, 0xf); DPP_ADD(0x118, 0xf); DPP_ADD(0x142, 0xa); DPP_ADD(0x143, 0xc);
    return ((unsigned)__builtin_amdgcn_readlane(__builtin_bit_cast(int, sm), 63) >> 23) & 255u;
}

typedef unsigned short u16x2 __attribute__((ext_vector_type(2)));
typedef unsigned u32x2 __attribute__((ext_vector_type(2)));
constexpr unsigned Q_P6 = 6144, Q_TAIL = 6656, Q_BYTES = 8192;
__device__ __forceinline__ u16x2 as16(unsigned a) { return __builtin_bit_cast(u16x2, a); }
__device__ __forceinline__ unsigned as32(u16x2 a) { return __builtin_bit_cast(unsigned, a); }
__device__ __forceinline__ unsigned pk_sub(unsigned a, unsigned b) { return as32(as16(a) - as16(b)); }
__device__ __forceinline__ unsigned pk_subs(unsigned a, unsigned b) { return as32(__builtin_elementwise_sub_sat(as16(a), as16(b))); }
__device__ __forceinline__ unsigned pk_min(unsigned a, unsigned b) { return as32(__builtin_elementwise_min(as16(a), as16(b))); }
__device__ __forceinline__ unsigned ffbl(unsigned a) { unsigned r; asm("v_ffbl_b32 %0, %1" : "=v"(r) : "v"(a)); return r; }
__device__ __forceinline__ unsigned wave_scan(unsigned x) {
    int v = (int)x;
    v += __builtin_amdgcn_update_dpp(0, v, 0x111, 0xf, 0xf, false);
    v += __builtin_amdgcn_update_dpp(0, v, 0x112, 0xf, 0xf, false);
    v += __builtin_amdgcn_update_dpp(0, v, 0x114, 0xf, 0xf, false);
    v += __builtin_amdgcn_update_dpp(0, v, 0x118, 0xf, 0xf, false);
    v += __builtin_amdgcn_update_dpp(0, v, 0x142, 0xa, 0xf, false);
    v += __builtin_amdgcn_update_dpp(0, v, 0x143, 0xc, 0xf, false);
    return (unsigned)v;
}
__device__ __forceinline__ unsigned lanes_below(unsigned long long m) { return __builtin_amdgcn_mbcnt_hi((unsigned)(m >> 32), __builtin_amdgcn_mbcnt_lo((unsigned)m, 0u)); }
template <int WPL> __device__ __forceinline__ unsigned stream_select(const LAS unsigned char* str, unsigned n, unsigned lane) {
    unsigned c = 0;
#pragma unroll
    for (int i = 0; i < WPL / 4; ++i) { const u32x4 w = *(const LAS u32x4*)(str + lane * (WPL * 4u) + i * 16u); c += __popc(w.x) + __popc(w.y) + __popc(w.z) + __popc(w.w); }
    const unsigned incl = wave_scan(c);
    unsigned b = 0, below = 0;
#pragma unroll
    for (int st = 32; st > 0; st >>= 1) {
        const unsigned v = (unsigned)__builtin_amdgcn_ds_bpermute((int)((b + st - 1u) * 4u), (int)incl);
        if (v < n) { b += st; below = v; }
    }
    unsigned r = n - below, pos = b * (WPL * 32u);
    u32x4 x = *(const LAS u32x4*)(str + b * (WPL * 4u));
    if (WPL == 8) {
        const u32x4 y = *(const LAS u32x4*)(str + b * (WPL * 4u) + 16u);
        const unsigned pc = __popc(x.x) + __popc(x.y) + __popc(x.z) + __popc(x.w); const bool up = r > pc;
        r -= up ? pc : 0u; pos += up ? 128u : 0u; x.x = up ? y.x : x.x; x.y = up ? y.y : x.y; x.z = up ? y.z : x.z; x.w = up ? y.w : x.w;
    }
    { const unsigned pc = __popc(x.x) + __popc(x.y); const bool up = r > pc; r -= up ? pc : 0u; pos += up ? 64u : 0u; x.x = up ? x.z : x.x; x.y = up ? x.w : x.y; }
    { const unsigned pc = __popc(x.x); const bool up = r > pc; r -= up ? pc : 0u; pos += up ? 32u : 0u; x.x = up ? x.y : x.x; }
    unsigned w = x.x;
#pragma unroll
    for (int st = 16; st > 0; st >>= 1) {
        const unsigned pc = __popc(w & ((1u << st) - 1u));
        if (r > pc) { r -= pc; w >>= st; pos += st; }
    }
    return n ? pos + 1u : 0u;
}
__device__ __forceinline__ unsigned q_encode(const u32x4 (&W)[8], unsigned B, LAS unsigned char* dst, unsigned lane) {
    const unsigned B1 = (B + 1u) * 0x04000400u;
    unsigned v[32], seg[4], pos[4], any = 0;
    u32x4 X[4];
#pragma unroll
    for (int s = 0; s < 4; ++s) {
        unsigned a = 0, p = 0;
        X[s].x = __builtin_amdgcn_perm(W[2 * s].y, W[2 * s].x, 0x07050301u); X[s].y = __builtin_amdgcn_perm(W[2 * s].w, W[2 * s].z, 0x07050301u);
        X[s].z = __builtin_amdgcn_perm(W[2 * s + 1].y, W[2 * s + 1].x, 0x07050301u); X[s].w = __builtin_amdgcn_perm(W[2 * s + 1].w, W[2 * s + 1].z, 0x07050301u);
#pragma unroll
        for (int k = 8 * s; k < 8 * s + 8; ++k) {
            const unsigned w = W[k >> 2][k & 3];
            const unsigned dq = as32(as16(pk_sub(B1, w & 0x7c007c00u)) >> 10);
            const unsigned ix = pk_min(dq, 0x00070007u) | 0x0c000c00u;
            const unsigned len = __builtin_amdgcn_perm(0x05040201u, 0x00000003u, ix) + pk_subs(dq, 0x00070007u);
            v[k] = (w & 0x83ff83ffu) | (__builtin_amdgcn_perm(0u, 0x03020100u, ix) << 10);
            any |= v[k];
            p += len & 0xffffu; a |= 1u << (p & 31u);
            p += len >> 16;     a |= 1u << (p & 31u);
        }
        seg[s] = a >> 1; pos[s] = p;
        asm volatile("" : "+v"(v[8 * s]), "+v"(v[8 * s + 1]), "+v"(v[8 * s + 2]), "+v"(v[8 * s + 3]), "+v"(v[8 * s + 4]), "+v"(v[8 * s + 5]), "+v"(v[8 * s + 6]), "+v"(v[8 * s + 7]),
                     "+v"(X[s]), "+v"(seg[s]), "+v"(pos[s]));
    }
    const bool ovf = max(max(pos[0], pos[1]), max(pos[2], pos[3])) > 31u || !(any & 0x0c000c00u);
    const unsigned tot = ovf ? 0u : pos[0] + pos[1] + pos[2] + pos[3];
    const unsigned incl = wave_scan(tot), total = (unsigned)__builtin_amdgcn_readlane((int)incl, 63);
    const unsigned long long om = __ballot(ovf);
    const unsigned sb = Q_TAIL + 64u * (unsigned)__popcll(om), size = sb + (((total + 127u) >> 7) << 4);
    if (size > Q_BYTES - 16u) return Q_BYTES;
    *(LAS u32x4*)(dst + Q_TAIL + lane * 16u) = (u32x4){0u, 0u, 0u, 0u};
    if (lane < 32u) *(LAS u32x4*)(dst + Q_TAIL + 1024u + lane * 16u) = (u32x4){0u, 0u, 0u, 0u};
    if (ovf) {
#pragma unroll
        for (int k = 0; k < 32; ++k) v[k] &= ~0x0c000c00u;
    }
#pragma unroll
    for (int j = 0; j < 6; ++j)
#pragma unroll
        for (int i = 0; i < 4; ++i) v[4 * j + i] |= (v[26 + j] << (12 - 3 * i)) & 0x70007000u;
#pragma unroll
    for (int j = 0; j < 6; ++j) v[24 + j / 3] |= (v[26 + j] >> (1 + j % 3)) & (0x80008000u >> (1 + j % 3));
#pragma unroll
    for (int j = 0; j < 6; ++j) *(LAS u32x4*)(dst + j * 1024u + lane * 16u) = (u32x4){v[4 * j], v[4 * j + 1], v[4 * j + 2], v[4 * j + 3]};
    *(LAS u32x2*)(dst + Q_P6 + lane * 8u) = (u32x2){v[24], v[25]};
    if (!ovf) {
        unsigned off = incl - tot;
#pragma unroll
        for (int s = 0; s < 4; ++s) {
            const unsigned long long x = (unsigned long long)seg[s] << (off & 31u);
            LAS unsigned* q = (LAS unsigned*)(dst + sb + (off >> 5) * 4u);
            __hip_atomic_fetch_or(q, (unsigned)x, __ATOMIC_RELAXED, __HIP_MEMORY_SCOPE_WORKGROUP);
            __hip_atomic_fetch_or(q + 1, (unsigned)(x >> 32), __ATOMIC_RELAXED, __HIP_MEMORY_SCOPE_WORKGROUP);
            off += pos[s];
        }
    } else {
#pragma unroll
        for (int j = 0; j < 4; ++j) *(LAS u32x4*)(dst + Q_TAIL + lanes_below(om) * 64u + j * 16u) = X[j];
    }
    return size;
}
#define RLOAD4(rs, vo, so) __builtin_bit_cast(u32x4, __builtin_amdgcn_raw_buffer_load_b128(rs, vo, so, 17))
struct QIn { unsigned v[32]; u32x4 win, X[4]; bool ovf; };
__device__ __forceinline__ void q_read(const LAS unsigned char* src, unsigned lane, QIn& q) {
#pragma unroll
    for (int j = 0; j < 6; ++j) { const u32x4 t = *(const LAS u32x4*)(src + j * 1024u + lane * 16u); q.v[4 * j] = t.x; q.v[4 * j + 1] = t.y; q.v[4 * j + 2] = t.z; q.v[4 * j + 3] = t.w; }
    { const u32x2 t = *(const LAS u32x2*)(src + Q_P6 + lane * 8u); q.v[24] = t.x; q.v[25] = t.y; }
#pragma unroll
    for (int j = 0; j < 6; ++j) {
        unsigned d = (q.v[24 + j / 3] << (1 + j % 3)) & 0x80008000u;
#pragma unroll
        for (int i = 0; i < 4; ++i) d |= (q.v[4 * j + i] & 0x70007000u) >> (12 - 3 * i);
        q.v[26 + j] = d;
    }
    unsigned ne = 64u;
#pragma unroll
    for (int k = 0; k < 32; ++k) ne -= __popc((q.v[k] | (q.v[k] >> 1)) & 0x04000400u);
    q.ovf = ne == 64u;
    ne &= 63u;
    const unsigned long long om = __ballot(q.ovf);
    const LAS unsigned char* str = src + Q_TAIL + 64u * min((unsigned)__popcll(om), 23u);
    const unsigned off = stream_select<4>(str, wave_scan(ne) - ne, lane);
    const LAS unsigned* w = (const LAS unsigned*)(str + (off >> 5) * 4u);
    const unsigned w0 = w[0], w1 = w[1], w2 = w[2], w3 = w[3], w4 = w[4];
    q.win = (u32x4){__builtin_amdgcn_alignbit(w1, w0, off), __builtin_amdgcn_alignbit(w2, w1, off), __builtin_amdgcn_alignbit(w3, w2, off), __builtin_amdgcn_alignbit(w4, w3, off)};
    if (q.ovf) {
#pragma unroll
        for (int j = 0; j < 4; ++j) q.X[j] = *(const LAS u32x4*)(src + Q_TAIL + lanes_below(om) * 64u + j * 16u);
    }
}
__device__ __forceinline__ void q_finish(unsigned B, const QIn& q, u32x4 (&W)[8]) {
    const unsigned* v = q.v;
    u32x4 win = q.win;
    const unsigned B1 = (B + 1u) * 0x04000400u;
#pragma unroll
    for (int s = 0; s < 4; ++s) {
        unsigned a = win.x << 1, used = 0;
#pragma unroll
        for (int k = 8 * s; k < 8 * s + 8; ++k) {
            const unsigned nz = v[k] | (v[k] >> 1);
            const unsigned l0 = ffbl((a & ~1u) | ((nz >> 10) & 1u));
            a >>= l0;
            const unsigned l1 = ffbl((a & ~1u) | ((nz >> 26) & 1u));
            a >>= l1;
            const unsigned len = l0 | (l1 << 16);
            used += len;
            const unsigned dq = __builtin_amdgcn_perm(0x00000006u, 0x00050400u, pk_min(len, 0x00040004u) | 0x0c000c00u) + pk_subs(len, 0x00040004u);
            W[k >> 2][k & 3] = (v[k] & 0x83ff83ffu) | (pk_sub(B1, (v[k] & 0x0c000c00u) + (dq << 10)) & 0x7c007c00u);
        }
        if (s < 3) {
            const unsigned sh = (used + (used >> 16)) & 31u;
            win.x = __builtin_amdgcn_alignbit(win.y, win.x, sh); win.y = __builtin_amdgcn_alignbit(win.z, win.y, sh);
            win.z = __builtin_amdgcn_alignbit(win.w, win.z, sh); win.w >>= sh;
        }
    }
    if (q.ovf) {
#pragma unroll
        for (int k = 0; k < 32; ++k) W[k >> 2][k & 3] = __builtin_amdgcn_perm(q.X[k >> 3][(k >> 1) & 3], W[k >> 2][k & 3], (k & 1) ? 0x07020600u : 0x05020400u);
    }
}

constexpr unsigned C_TAIL = 26624;
__device__ __forceinline__ unsigned c_encode(const u32x4 (&V)[32], unsigned B, u32x4 (&PK)[26], LAS unsigned char* scr, unsigned lane) {
    const unsigned B1 = (B + 1u) * 0x00010001u;
    unsigned seg[8], pos[8], sb = 0;
#pragma unroll
    for (int g = 0; g < 4; ++g) {
        unsigned e2[16];
#pragma unroll
        for (int v = 0; v < 8; ++v) {
            u32x4 r;
#pragma unroll
            for (int c = 0; c < 4; ++c) r[c] = __builtin_amdgcn_alignbit(V[g * 8 + v][c], V[g * 8 + v][c], 31);
#pragma unroll
            for (int i = 0; i < 3; ++i) PK[g * 6 + ((3 * v + i) >> 2)][(3 * v + i) & 3] = __builtin_amdgcn_perm(r.w, r[i], 0x04020100u + ((unsigned)i << 24));
            e2[2 * v] = __builtin_amdgcn_perm(r.y, r.x, 0x0c070c03u); e2[2 * v + 1] = __builtin_amdgcn_perm(r.w, r.z, 0x0c070c03u);
        }
#pragma unroll
        for (int s = 2 * g; s < 2 * g + 2; ++s) {
            unsigned a = 0, p = 0, cw = 0;
#pragma unroll
            for (int j = 0; j < 8; ++j) {
                const unsigned dq = pk_sub(B1, e2[(s & 1) * 8 + j]), ix = pk_min(dq, 0x00070007u) | 0x0c000c00u;
                const unsigned len = __builtin_amdgcn_perm(0x05040201u, 0x00000003u, ix) + pk_subs(dq, 0x00070007u);
                cw |= __builtin_amdgcn_perm(0u, 0x03020100u, ix) << (2 * j);
                p += len & 0xffffu; a |= 1u << (p & 31u);
                p += len >> 16;     a |= 1u << (p & 31u);
            }
            seg[s] = a >> 1; pos[s] = p; PK[24 + (s >> 2)][s & 3] = cw;
        }
        const bool ovf = max(pos[2 * g], pos[2 * g + 1]) > 31u || !(PK[24 + (g >> 1)][(2 * g) & 3] | PK[24 + (g >> 1)][(2 * g + 1) & 3]);
        const unsigned long long om = __ballot(ovf);
        if (ovf) {
            u32x4 E[2];
#pragma unroll
            for (int v = 0; v < 8; ++v) E[v >> 2][v & 3] = __builtin_amdgcn_perm(e2[2 * v + 1], e2[2 * v], 0x06040200u);
            *(LAS u32x4*)(scr + sb + lanes_below(om) * 32u) = E[0]; *(LAS u32x4*)(scr + sb + lanes_below(om) * 32u + 16u) = E[1];
            seg[2 * g] = seg[2 * g + 1] = pos[2 * g] = pos[2 * g + 1] = 0u; PK[24 + (g >> 1)][(2 * g) & 3] = 0u; PK[24 + (g >> 1)][(2 * g + 1) & 3] = 0u;
        }
        sb += 32u * (unsigned)__popcll(om);
    }
    const unsigned tot = (pos[0] + pos[1] + pos[2] + pos[3]) + (pos[4] + pos[5] + pos[6] + pos[7]);
    const unsigned incl = wave_scan(tot), total = (unsigned)__builtin_amdgcn_readlane((int)incl, 63);
    *(LAS u32x4*)(scr + sb + lane * 16u) = (u32x4){0u, 0u, 0u, 0u}; *(LAS u32x4*)(scr + sb + 1024u + lane * 16u) = (u32x4){0u, 0u, 0u, 0u};
    unsigned off = incl - tot;
#pragma unroll
    for (int s = 0; s < 8; ++s) {
        const unsigned long long x = (unsigned long long)seg[s] << (off & 31u);
        LAS unsigned* q = (LAS unsigned*)(scr + sb + (off >> 5) * 4u);
        __hip_atomic_fetch_or(q, (unsigned)x, __ATOMIC_RELAXED, __HIP_MEMORY_SCOPE_WORKGROUP);
        __hip_atomic_fetch_or(q + 1, (unsigned)(x >> 32), __ATOMIC_RELAXED, __HIP_MEMORY_SCOPE_WORKGROUP);
        off += pos[s];
    }
    return sb + ((total + 7u) >> 3);
}
__device__ __forceinline__ size_t out_off(int q) { return (size_t)((q >> 4) * 128 + ((q >> 2) & 3) * 16 + (q & 3)) * (GN * 4); }
struct CDec { unsigned win[9], a, used, hdr, xo[4], B1, E[32]; };
__device__ __forceinline__ void c_begin(CDec& d, const u32x4 (&CW)[2], const LAS unsigned char* tail, unsigned B, unsigned lane) {
    unsigned ne = 0, sb = 0;
    d.hdr = 0;
#pragma unroll
    for (int g = 0; g < 4; ++g) {
        const unsigned c0 = CW[g >> 1][(2 * g) & 3], c1 = CW[g >> 1][(2 * g + 1) & 3];
        const unsigned n = 32u - __popc((c0 | (c0 >> 1)) & 0x55555555u) - __popc((c1 | (c1 >> 1)) & 0x55555555u);
        const unsigned long long om = __ballot(n == 32u);
        ne += n & 31u; d.hdr |= (n >> 5) << g; d.xo[g] = sb + lanes_below(om) * 32u;
        sb += 32u * (unsigned)__popcll(om);
    }
    const unsigned off = stream_select<8>(tail + sb, wave_scan(ne) - ne, lane);
#pragma unroll
    for (int i = 0; i < 9; ++i) d.win[i] = *(const LAS unsigned*)(tail + sb + (off >> 5) * 4u + i * 4u);
#pragma unroll
    for (int i = 0; i < 8; ++i) d.win[i] = __builtin_amdgcn_alignbit(d.win[i + 1], d.win[i], off);
    d.B1 = (B + 1u) * 0x00010001u;
}
__device__ __forceinline__ void c_parse(CDec& d, const u32x4 (&CW)[2], int H) {
    const int s = H >> 1;
    const unsigned cw = CW[s >> 2][s & 3], nz = cw | (cw >> 1);
    const bool exc = d.hdr & (1u << (s >> 1));
    if (!(H & 1)) { d.a = d.win[0] << 1; d.used = 0; }
    unsigned e2[4];
#pragma unroll
    for (int j = (H & 1) * 4; j < (H & 1) * 4 + 4; ++j) {
        const unsigned l0 = ffbl((d.a & ~1u) | ((nz >> (2 * j)) & 1u));
        d.a >>= l0;
        const unsigned l1 = ffbl((d.a & ~1u) | ((nz >> (2 * j + 16)) & 1u));
        d.a >>= l1;
        const unsigned len = l0 | (l1 << 16);
        d.used += len;
        const unsigned dq = __builtin_amdgcn_perm(0x00000006u, 0x00050400u, pk_min(len, 0x00040004u) | 0x0c000c00u) + pk_subs(len, 0x00040004u);
        e2[j & 3] = pk_sub(d.B1, dq + ((cw >> (2 * j)) & 0x00030003u));
    }
#pragma unroll
    for (int i = 0; i < 2; ++i) d.E[2 * H + i] = __builtin_amdgcn_perm(e2[2 * i + 1], e2[2 * i], 0x06040200u);
    asm volatile("" : "+v"(d.E[2 * H]), "+v"(d.E[2 * H + 1]));
    if (H & 1) {
        const unsigned sh = exc ? 0u : d.used + (d.used >> 16);
#pragma unroll
        for (int i = 0; i < 7 - s; ++i) d.win[i] = __builtin_amdgcn_alignbit(d.win[i + 1], d.win[i], sh);
    }
}
__device__ __forceinline__ void c_patch(CDec& d, const LAS unsigned char* tail) {
#pragma unroll
    for (int g = 0; g < 4; ++g) if (d.hdr & (1u << g)) {
        const u32x4 x0 = *(const LAS u32x4*)(tail + d.xo[g]), x1 = *(const LAS u32x4*)(tail + d.xo[g] + 16u);
#pragma unroll
        for (int v = 0; v < 4; ++v) { d.E[8 * g + v] = x0[v]; d.E[8 * g + 4 + v] = x1[v]; }
    }
}
__device__ __forceinline__ void c_store(const CDec& d, const LAS unsigned char* src, const u32x4 (&A)[32], bool bad, const gchar* ob, unsigned ovo, unsigned lane) {
#pragma unroll
    for (int h = 0; h < 8; ++h) {
        u32x4 PK[3];
#pragma unroll
        for (int i = 0; i < 3; ++i) PK[i] = *(const LAS u32x4*)(src + (3 * h + i) * 1024u + lane * 16u);
#pragma unroll
        for (int v = 0; v < 4; ++v) {
            const int q = 4 * h + v;
            unsigned P[3];
#pragma unroll
            for (int i = 0; i < 3; ++i) P[i] = PK[(3 * v + i) >> 2][(3 * v + i) & 3];
            u32x4 r;
            r.x = __builtin_amdgcn_perm(d.E[q], P[0], 0x04020100u); r.y = __builtin_amdgcn_perm(d.E[q], P[1], 0x05020100u); r.z = __builtin_amdgcn_perm(d.E[q], P[2], 0x06020100u);
            r.w = __builtin_amdgcn_perm(P[1], P[0], 0x0c0c0703u) | __builtin_amdgcn_perm(d.E[q], P[2], 0x07030c0cu);
#pragma unroll
            for (int c = 0; c < 4; ++c) r[c] = bad ? 0x7fc00000u : __builtin_amdgcn_alignbit(r[c], r[c], 1);
            const f32x4 o = __builtin_bit_cast(f32x4, r) + __builtin_bit_cast(f32x4, A[q]);
            asm volatile("s_nop 4\n\tglobal_store_dwordx4 %0, %1, %2 sc0 sc1 nt\n\ts_nop 1" :: "v"(ovo), "v"(o), "s"(ob + out_off(q)) : "memory");
            asm volatile("s_waitcnt vmcnt(" STR(DEC_W) ")" ::: "memory");
        }
    }
}

constexpr int NITEM = 240;
__device__ __forceinline__ void item_decode(int i, int d, int s, int& isW, int& j, int& u) {
    constexpr int R0 = 12 + NE, R1 = 18 - NE;
    if (i < 8 * R0) {
        j = i / R0; int r = i - j * R0;
        if (i >= 6 && i < R0) r = i < 12 ? i + NE : i - 6;
        isW = r >= 6 && r < 6 + NE; u = isW ? r - 6 : r < 6 ? 2 * r : 2 * (r - 6 - NE) + 1;
        if (!isW) { if (u >= 2 * min(d, s)) u += 2; if (u >= 2 * max(d, s)) u += 2; }
    } else { const int r = i - 8 * R0; j = r / R1; const int t = r - j * R1; isW = t < 16 - NE; u = isW ? NE + t : 2 * d + t - (16 - NE); }
}
constexpr int NENC = 32 + 14 * (12 - NE);
constexpr int NTASK = 1152, NTASK_DYN0 = 200 + 3 * NENC, NTASK_DYN = NTASK - 200;
__device__ __forceinline__ void task_decode(int t, int& isW, int& j, int& u, int& dd) {
    constexpr int R0 = 16 + 8 * NE, R1 = 8 * (16 - NE);
    if (t < 8 * R0) { j = t / R0; int r = t - j * R0; isW = r >= 16; if (isW) { r -= 16; u = r >> 3; dd = r & 7; } else { u = r < 8 ? 2 * r : 2 * r - 15; dd = 7; } }
    else { int r = t - 8 * R0; j = r / R1; r -= j * R1; isW = 1; u = NE + (r >> 3); dd = r & 7; }
}

__device__ __forceinline__ void task_load(const Params& p, int rank, int i, f32x4 (&L)[16]) {
    const bool valid = i < NTASK;
    int isW, j, u, dd; task_decode(valid ? i : 0, isW, j, u, dd);
    const unsigned t = threadIdx.x;
    const gchar* b = uni(isW ? (const char*)p.w + ((size_t)(j * 64) * GN + ((rank + 1 + dd) & 7) * NL + u * 256) * 4 : (const char*)p.x + ((size_t)(u * 256) * KS + j * 64) * 4);
    unsigned vo = isW ? (t >> 6) * (4u * GN * 4u) + (t & 63u) * 16u : (t >> 3) * (KS * 4u) + ((t & 7u) ^ ((t >> 4) & 7u)) * 16u;
    if (!valid) vo = 0;
    const unsigned s0 = isW ? GN * 4u : 128u, s1 = isW ? 2u * GN * 4u : 64u * KS * 4u, s2 = isW ? 32u * GN * 4u : 128u * KS * 4u, s3 = isW ? 16u * GN * 4u : 32u * KS * 4u;
#pragma unroll
    for (int n = 0; n < 16; ++n)
        L[n] = __builtin_nontemporal_load((const GAS f32x4*)(b + ((n & 1) * s0 + ((n >> 1) & 1) * s1 + ((n >> 2) & 1) * s2 + (n >> 3) * s3) + vo));
}

__device__ __forceinline__ unsigned pk2(float a, float b, int e) {
    typedef _Float16 h2 __attribute__((ext_vector_type(2)));
    h2 h; h.x = (_Float16)__builtin_amdgcn_ldexpf(a, e); h.y = (_Float16)__builtin_amdgcn_ldexpf(b, e);
    return __builtin_bit_cast(unsigned, h);
}

#define POLL_SLEEP 8
#define FLAG_SLEEP 16
#define SEND_PRIO 3
#define MINI_LEAD 64
#define HAND_AT 20
#define PK_W 4
#define LOC_W 8
#define LINK_EXTRA 0u
#define PULL_N 7
#define PUSH_W 0
#define START_D0 0
#define ENC_DELAY 4
#define START_NOW 112
#define START_DELAY 5
#define START_SPREAD 12
#define LBAR do { asm volatile("s_waitcnt lgkmcnt(0)" ::: "memory"); __builtin_amdgcn_s_barrier(); asm volatile("" ::: "memory"); } while (0)

typedef volatile LAS unsigned vl32;
typedef volatile LAS u32x4 vl128;
#define LGKM0 asm volatile("s_waitcnt lgkmcnt(0)" ::: "memory")
__device__ __forceinline__ u32x4 lds_wait4(LAS unsigned char* lds, unsigned off, unsigned v) {
    for (;;) {
        const u32x4 q = *(vl128*)(lds + off);
        if (rfl(min(min(q.x, q.y), min(q.z, q.w))) >= v) { asm volatile("" ::: "memory"); return q; }
        __builtin_amdgcn_s_sleep(1);
    }
}
__device__ __forceinline__ unsigned lds_wait1(LAS unsigned char* lds, unsigned off, unsigned v) {
    unsigned x;
    while ((x = rfl(*(vl32*)(lds + off))) < v) __builtin_amdgcn_s_sleep(DSLEEP);
    asm volatile("" ::: "memory");
    return x;
}
__device__ __forceinline__ void task_stats(LAS unsigned char* lds, const f32x4 (&L)[16], unsigned k) {
    unsigned mu = 0; float sm = 0.f;
#pragma unroll
    for (int n = 0; n < 16; ++n) {
        const u32x4 q = __builtin_bit_cast(u32x4, L[n]) & 0x7fffffffu; mu = max(max(mu, q.x), max(q.y, max(q.z, q.w)));
        const f32x4 f = __builtin_bit_cast(f32x4, q); sm += (f.x + f.y) + (f.z + f.w);
    }
    const unsigned mb = wave_max(mu >> 23), em = wave_mean_exp(sm * (2.32f / 4096.f));
    LGKM0;
    if ((threadIdx.x & 63) == 63) *(vl32*)(lds + L_AMAX + (k & 1u) * 16 + (threadIdx.x >> 6) * 4) = ((k + 1u) << 16) | (em << 8) | mb;
}

__device__ __forceinline__ unsigned img_off(int isW, unsigned q, int c) {
    return (isW ? q * 2048u + (c >> 2) * 8192u + (c & 1) * 1024u : q * 1024u + (c >> 2) * 4096u + (c & 1) * 8192u) + ((c >> 1) & 1) * HTB;
}
__device__ __forceinline__ gu32* stage_unit(gchar* myws, int srel, int isW, int j, int u, const gchar*& ub) {
    const int idx = j * 8 + ((srel + 1) & 7);
    ub = myws + WS_STAGE + (size_t)((j * 32 + isW * 16 + u) * 8 + (idx & 7)) * UNIT;
    return (gu32*)(myws + WS_FLAG) + (isW * 16 + u) * NKT + idx;
}
__device__ __forceinline__ void own_images(gchar* myws, int rank, int isW, int j, int u, int dd, unsigned q, unsigned lane, const u32x4 (&H)[8]) {
    if (dd == 7) {
        const gchar* ub; stage_unit(myws, 7, isW, j, u, ub); ub = uni(ub);
#pragma unroll
        for (int c = 0; c < 8; ++c) store16_sys(ub + img_off(isW, q, c), lane * 16u, H[c]);
    }
    if (isW ? dd != 7 : (u >> 1) == rank) {
        const gchar* ub = uni(myws + WS_STAGE2 + (size_t)((isW ? dd * 16 + u : 112 + (u & 1)) * 8 + j) * UNIT);
#pragma unroll
        for (int c = 0; c < 8; ++c) store16_sys(ub + img_off(isW, q, c), lane * 16u, H[c]);
    }
}
__device__ __forceinline__ void own_flags(gchar* myws, int rank, int isW, int j, int u, int dd, unsigned lane, unsigned val) {
    const gchar* ub; gu32* sf = stage_unit(myws, 7, isW, j, u, ub);
    if (lane == 0 ? dd == 7 : lane == 1 && (isW ? dd != 7 : (u >> 1) == rank))
        __hip_atomic_store(lane ? (gu32*)(myws + WS_FLAG2) + (isW ? dd * 16 + u : 112 + (u & 1)) * 8 + j : sf, val, __ATOMIC_RELAXED, __HIP_MEMORY_SCOPE_SYSTEM);
}
__device__ __forceinline__ int conv_step(const Params& p, LAS unsigned char* lds, gchar* myws, gu32* tick, int rank, unsigned call, unsigned k, f32x4 (&L)[16], const f32x4 (&Ln)[16]) {
    unsigned t = threadIdx.x; asm volatile("" : "+v"(t));
    const u32x4 am = lds_wait4(lds, L_AMAX + (k & 1u) * 16, (k + 1u) << 16);
    const int i = (int)rfl(*(vl32*)(lds + L_ITEM + (k & 15u) * 4)), i1 = (int)rfl(*(vl32*)(lds + L_ITEM + ((k + 1u) & 15u) * 4)), i2 = (int)rfl(*(vl32*)(lds + L_ITEM + ((k + 2u) & 15u) * 4));
    const bool valid = i < NTASK;
    int isW, j, u, dd, cls = 0; task_decode(valid ? i : 0, isW, j, u, dd);
    if (valid) {
        const unsigned mb = rfl(max(max(am.x & 255u, am.y & 255u), max(am.z & 255u, am.w & 255u))), em = (rfl(max(max(am.x, am.y), max(am.z, am.w))) >> 8) & 255u;
        cls = (int)(mb + 2u) >> 4;
        const int e = 128 - 16 * cls;
        u32x4 H[8];
#pragma unroll
        for (int h = 0; h < 2; ++h) {
            const f32x4* Q = L + 8 * h; u32x4* G = H + 4 * h;
            if (isW) {
#pragma unroll
                for (int c = 0; c < 4; ++c) { G[c].x = pk2(Q[0][c], Q[1][c], e); G[c].y = pk2(Q[2][c], Q[3][c], e); G[c].z = pk2(Q[4][c], Q[5][c], e); G[c].w = pk2(Q[6][c], Q[7][c], e); }
            } else {
#pragma unroll
                for (int c = 0; c < 4; ++c) { G[c].x = pk2(Q[2 * c].x, Q[2 * c].y, e); G[c].y = pk2(Q[2 * c].z, Q[2 * c].w, e); G[c].z = pk2(Q[2 * c + 1].x, Q[2 * c + 1].y, e); G[c].w = pk2(Q[2 * c + 1].z, Q[2 * c + 1].w, e); }
            }
        }
        const unsigned q = rfl(t >> 6);
        own_images(myws, rank, isW, j, u, dd, q, t & 63u, H);
        if (k >= 4) lds_wait1(lds, L_CONS + q * 4, k - 3u);
#pragma unroll
        for (int c = 0; c < 8; ++c) *(LAS u32x4*)(lds + (k & 3u) * UNIT + q * Q_BYTES + c * 1024u + (t & 63u) * 16u) = H[c];
        LGKM0;
        if ((t & 63u) == 0) *(vl32*)(lds + L_QI + (k & 3u) * 16 + q * 4) = ((k + 1u) << 16) | ((unsigned)min(max((int)min(em, mb) + e - 112, 2), 31) << 5) | (unsigned)cls;
    }
    unsigned tk = NTASK_DYN - 1;
    gu32* q = tick; asm volatile("" : "+v"(q));
    if (t == 0 && valid) tk = __hip_atomic_fetch_add(q, 1u, __ATOMIC_RELAXED, __HIP_MEMORY_SCOPE_AGENT);
    task_load(p, rank, i2, L);
    __builtin_amdgcn_s_waitcnt(0x4F70);
    asm volatile("s_waitcnt vmcnt(16)" : "+v"(tk) :: "memory");
    if (valid) {
        unsigned old = 0;
        if ((t & 63u) == 0) old = __hip_atomic_fetch_add((LAS unsigned*)(lds + L_UCNT + (k & 7u) * 4), 1u, __ATOMIC_RELAXED, __HIP_MEMORY_SCOPE_WORKGROUP);
        if ((rfl(old) & 3u) == 3u) own_flags(myws, rank, isW, j, u, dd, t & 63u, (call << 8) | (unsigned)cls);
    }
    if (t == 0) *(vl32*)(lds + L_ITEM + ((k + 3u) & 15u) * 4) = (unsigned)NTASK_DYN0 + tk % (unsigned)NTASK_DYN;
    task_stats(lds, Ln, k + 1u);
    return i1;
}

__device__ __forceinline__ void lds_fetch8(const LAS unsigned char* src, unsigned lane, u32x4 (&H)[8]) {
#pragma unroll
    for (int c = 0; c < 8; ++c) H[c] = *(const LAS u32x4*)(src + c * 1024u + lane * 16u);
}

__device__ __forceinline__ unsigned first_unit(const Params& p, LAS unsigned char* lds, gchar* myws, int isW, int j, int u, int d, int first, int second, unsigned& call, u32x4 (&H)[4]) {
    const unsigned t = threadIdx.x;
    unsigned tk = 0;
    gu32* tq = (gu32*)(myws + WS_TICKET); asm volatile("" : "+v"(tq));
    if (t == 0) tk = __hip_atomic_fetch_add(tq, 1u, __ATOMIC_RELAXED, __HIP_MEMORY_SCOPE_AGENT);
    const gchar* b = uni(isW ? (const char*)p.w + ((size_t)(j * 64) * GN + d * NL + u * 256) * 4 : (const char*)p.x + ((size_t)(u * 256) * KS + j * 64) * 4);
    const unsigned vo = isW ? (t >> 6) * (4u * GN * 4u) + (t & 63u) * 16u : (t >> 3) * (KS * 4u) + ((t & 7u) ^ ((t >> 4) & 7u)) * 16u;
    const unsigned s0 = isW ? GN * 4u : 128u, s1 = isW ? 2u * GN * 4u : 64u * KS * 4u, s2 = isW ? 32u * GN * 4u : 128u * KS * 4u;
    f32x4 L[8];
#pragma unroll
    for (int n = 0; n < 8; ++n) L[n] = __builtin_nontemporal_load((const GAS f32x4*)(b + ((n & 1) * s0 + ((n >> 1) & 1) * s1 + (n >> 2) * s2) + vo));
    constexpr unsigned I0 = (L_ITEM - L_CTL) / 4;
    if (t < (L_ZERO_END - L_CTL) / 4 && t != (L_CALL - L_CTL) / 4) *(vl32*)(lds + L_CTL + t * 4) = t == I0 ? (unsigned)first : t - (I0 + 1) < 3u ? (unsigned)second + (t - (I0 + 1)) * NENC : 0u;
    unsigned mu = 0; float sm = 0.f;
#pragma unroll
    for (int n = 0; n < 8; ++n) {
        const u32x4 q = __builtin_bit_cast(u32x4, L[n]) & 0x7fffffffu; mu = max(max(mu, q.x), max(q.y, max(q.z, q.w)));
        const f32x4 f = __builtin_bit_cast(f32x4, q); sm += (f.x + f.y) + (f.z + f.w);
    }
    mu = wave_max(mu >> 23) | (wave_mean_exp(sm * (2.32f / 2048.f)) << 8);
    asm volatile("" : "+v"(tk));
    if (t == 0) *(vl32*)(lds + L_CALL) = (tk / GRID + 1u) & 0xffffffu;
    if ((t & 63u) == 0) *(vl32*)(lds + L_HDR + (t >> 6) * 4) = mu;
    LBAR;
    const u32x4 a0 = *(vl128*)(lds + L_HDR), a1 = *(vl128*)(lds + L_HDR + 16), m0 = a0 & 255u, m1 = a1 & 255u;
    call = rfl(*(vl32*)(lds + L_CALL));
    const unsigned mb = rfl(max(max(max(m0.x, m0.y), max(m0.z, m0.w)), max(max(m1.x, m1.y), max(m1.z, m1.w))));
    const unsigned em = rfl(max(max(max(a0.x, a0.y), max(a0.z, a0.w)), max(max(a1.x, a1.y), max(a1.z, a1.w)))) >> 8;
    const int cls = (int)(mb + 2u) >> 4, e = 128 - 16 * cls;
#pragma unroll
    for (int c = 0; c < 4; ++c) {
        if (isW) { H[c].x = pk2(L[0][c], L[1][c], e); H[c].y = pk2(L[2][c], L[3][c], e); H[c].z = pk2(L[4][c], L[5][c], e); H[c].w = pk2(L[6][c], L[7][c], e); }
        else { H[c].x = pk2(L[2 * c].x, L[2 * c].y, e); H[c].y = pk2(L[2 * c].z, L[2 * c].w, e); H[c].z = pk2(L[2 * c + 1].x, L[2 * c + 1].y, e); H[c].w = pk2(L[2 * c + 1].z, L[2 * c + 1].w, e); }
    }
    return ((unsigned)min(max((int)min(em, mb) + e - 112, 2), 31) << 5) | (unsigned)cls;
}

__device__ __forceinline__ void produce(const Params& p, LAS unsigned char* lds, gchar* myws, int rank, unsigned call, int second, gu32* tick, unsigned st0, const u32x4 (&H0)[4]) {
#pragma unroll
    for (int c = 0; c < 4; ++c) *(LAS u32x4*)(lds + ((threadIdx.x >> 6) & 3u) * Q_BYTES + ((threadIdx.x >> 8) * 4u + c) * 1024u + (threadIdx.x & 63u) * 16u) = H0[c];
    if (threadIdx.x < 4) *(vl32*)(lds + L_QI + threadIdx.x * 4) = (1u << 16) | st0;
    LBAR;
    if (rfl(threadIdx.x >> 6) < 4) {
        if (tick) {
            f32x4 RA[16], RB[16];
            for (int z = 0; z < ENC_DELAY; ++z) __builtin_amdgcn_s_sleep(32);
            task_load(p, rank, second, RA);
            task_stats(lds, RA, 1);
            task_load(p, rank, second + NENC, RB);
            int nx; unsigned k = 1;
            do { conv_step(p, lds, myws, tick, rank, call, k, RA, RB); nx = conv_step(p, lds, myws, tick, rank, call, k + 1u, RB, RA); k += 2; } while (nx < NTASK);
        }
    } else {
        const unsigned lane = threadIdx.x & 63u, q = rfl(threadIdx.x >> 6) - 4u;
        const gchar* peer = (const gchar*)p.ws[(rank + 1 + min(lane, 6u)) & 7];
        int ip = -1; unsigned long long infop = 0;
        auto settle = [&](unsigned kp) {
            asm volatile("s_waitcnt vmcnt(0)" ::: "memory");
            unsigned old = 0;
            if (lane == 0) old = __hip_atomic_fetch_add((LAS unsigned*)(lds + L_ECNT + (kp & 7u) * 4), 1u, __ATOMIC_RELAXED, __HIP_MEMORY_SCOPE_WORKGROUP);
            if ((rfl(old) & 3u) == 3u) {
                int isW, j, u, dd; task_decode(ip, isW, j, u, dd);
                const bool peers = isW ? dd != 7 : (u >> 1) != rank;
                const unsigned fi = (call & 1u) * 8 + (6u - lane);
                if (peers && (isW ? lane == (unsigned)dd : lane < 7u))
                    __hip_atomic_store((GAS unsigned long long*)(peer + WS_AFLAG + ((fi * 2 + isW) * 128 + j * 16 + u) * 8), ((unsigned long long)call << 40) | infop, __ATOMIC_RELAXED, __HIP_MEMORY_SCOPE_SYSTEM);
                if (kp == 0) own_flags(myws, rank, isW, j, u, dd, lane, (call << 8) | ((unsigned)infop & 31u));
            }
            ip = -1;
        };
#define UT(i) do {} while (0)
        for (unsigned k = 0;; ++k) {
            const int i = (int)rfl(*(vl32*)(lds + L_ITEM + (k & 15u) * 4));
            if (ip >= 0 && (i >= NTASK || rfl(*(vl32*)(lds + L_QI + (k & 3u) * 16 + q * 4)) < ((k + 1u) << 16))) settle(k - 1u);
            if (i >= NTASK) break;
            const unsigned st = lds_wait1(lds, L_QI + (k & 3u) * 16 + q * 4, (k + 1u) << 16);
            UT(0);
            int isW, j, u, dd; task_decode(i, isW, j, u, dd);
            LAS unsigned char* src = lds + (k & 3u) * UNIT + q * Q_BYTES;
            u32x4 H[8];
            lds_fetch8(src, lane, H);
            const bool peers = isW ? dd != 7 : (u >> 1) != rank;
            if (k == 0) own_images(myws, rank, isW, j, u, dd, q, lane, H);
            UT(1);
            unsigned size = 0;
            if (peers) {
                size = q_encode(H, (st >> 5) & 31u, src, lane);
                if (lane == 0) *(vl32*)(lds + L_SIZE + (k & 7u) * 16 + q * 4) = ((k + 1u) << 8) | ((size - Q_TAIL) >> 4);
            }
            UT(2);
            if (ip >= 0) settle(k - 1u);
            ip = i; infop = st & 1023u;
            if (peers) {
                const u32x4 sz = lds_wait4(lds, L_SIZE + (k & 7u) * 16, (k + 1u) << 8) & 127u;
                const unsigned z0 = rfl(sz.x), z1 = rfl(sz.y), z2 = rfl(sz.z), z3 = rfl(sz.w);
                infop |= (unsigned long long)(z0 | (z1 << 7) | (z2 << 14) | (z3 << 21)) << 10;
                const unsigned qo = q * Q_TAIL + ((q > 0 ? z0 : 0u) + (q > 1 ? z1 : 0u) + (q > 2 ? z2 : 0u)) * 16u, sh = qo & 127u;
                const gchar* pb = uni(myws + pin_off(call & 1u, isW, dd, j, u) + (qo - sh));
                lds_fetch8(src - sh, lane, H);
#pragma unroll
                for (int c = 0; c < 8; ++c) if ((c > 0 && c < 6) || c * 1024u + lane * 16u - sh < size) store16_sys(pb + c * 1024u, lane * 16u, H[c]);
                if (Q_BYTES + lane * 16u - sh < size) store16_sys(pb + Q_BYTES, lane * 16u, *(const LAS u32x4*)(src - sh + Q_BYTES + lane * 16u));
            }
            LGKM0;
            if (lane == 0) *(vl32*)(lds + L_CONS + q * 4) = k + 1u;
            UT(3);
        }
    }
    LBAR;
}

__device__ __forceinline__ void push_first(const Params& p, LAS unsigned char* lds, gchar* myws, int rank, unsigned call, int link, int item, unsigned st0, const u32x4 (&H)[4]) {
    const unsigned t = threadIdx.x;
    gchar* dws = uni(p.ws[(rank + 1 + link) & 7]);
    const gchar* ib = uni(dws + WS_INBOX + (size_t)(((call & 1u) * 7 + (6 - link)) * 8 + item) * UNIT);
#pragma unroll
    for (int c = 0; c < 4; ++c) {
        if (c > 1 && rfl(t >> 6) != 1u) asm volatile("s_waitcnt vmcnt(" STR(PUSH_W) ")" ::: "memory");
        store16_sys(ib + c * 8192u, t * 16u, H[c]);
    }
    asm volatile("s_waitcnt vmcnt(0)" ::: "memory");
    if ((t & 63u) == 0 && __hip_atomic_fetch_add((LAS unsigned*)(lds + L_ECNT), 1u, __ATOMIC_RELAXED, __HIP_MEMORY_SCOPE_WORKGROUP) == 7u) {
        __hip_atomic_store((GAS unsigned long long*)(dws + WS_IFLAG) + ((call & 1u) * 8 + (6 - link)) * 8 + item, ((unsigned long long)call << 40) | st0, __ATOMIC_RELAXED, __HIP_MEMORY_SCOPE_SYSTEM);
    }
}

template <int SLEEP> __device__ __forceinline__ bool flag_wait(const gchar* f, gu32* tmo, unsigned call, unsigned& info, unsigned& hi) {
    const unsigned long long t0 = __builtin_amdgcn_s_memrealtime();
    for (;;) {
        const unsigned long long fv = __hip_atomic_load((const GAS unsigned long long*)f, __ATOMIC_RELAXED, __HIP_MEMORY_SCOPE_SYSTEM);
        info = rfl((unsigned)fv); hi = rfl((unsigned)(fv >> 32));
        if ((hi >> 8) >= call) break;
        if (rfl(ld_agt(tmo)) == call || __builtin_amdgcn_s_memrealtime() - t0 > DEADLINE) {
            if ((threadIdx.x & 63u) == 0) __hip_atomic_store(tmo, call, __ATOMIC_RELAXED, __HIP_MEMORY_SCOPE_AGENT);
            return false;
        }
        __builtin_amdgcn_s_sleep(SLEEP);
    }
    asm volatile("" ::: "memory");
    return true;
}

__device__ __forceinline__ void fetch_inputs(const Params& p, LAS unsigned char* lds, gchar* myws, int rank, unsigned call, int srel, int first) {
    const unsigned lane = threadIdx.x & 63u, wv = rfl(threadIdx.x >> 6);
    gu32* tmo = (gu32*)(myws + WS_TMO);
    const unsigned fi = (call & 1u) * 8 + srel;
    const int src = (rank + 1 + srel) & 7;
    if (wv == 1) {
        unsigned k = 0;
        for (;;) {
            unsigned tk = 0;
            if (lane == 0) tk = __hip_atomic_fetch_add((gu32*)(myws + WS_LINKQ + srel * 128), 1u, __ATOMIC_RELAXED, __HIP_MEMORY_SCOPE_AGENT);
            const int i = 8 + (int)(rfl(tk) % (unsigned)NITEM);
            unsigned info = 0, hi = 0; bool ok = true;
            if (i < NITEM) { int isW, j, u; item_decode(i, rank, src, isW, j, u); ok = flag_wait<2>(myws + WS_AFLAG + ((fi * 2 + isW) * 128 + j * 16 + u) * 8, tmo, call, info, hi); }
            if (ok) { if (lane == 0) *(vl128*)(lds + L_DESC + (k & 7u) * 16) = (u32x4){k + 1u, (unsigned)i, info & 1023u, (info >> 10) | ((hi & 63u) << 22)}; ++k; }
            if (i >= NITEM) break;
            if (ok) lds_wait1(lds, L_LST, k);
        }
    } else if (wv == 0) {
        const gchar* sws = uni(p.ws[src]);
        __builtin_amdgcn_s_setprio(SEND_PRIO);
#define LWAIT(off, v, n) do { if (rfl(*(vl32*)(lds + (off))) < (v)) { asm volatile("s_waitcnt vmcnt(0)" ::: "memory"); if (lane == 0) *(vl32*)(lds + L_LAND) = (n); lds_wait1(lds, off, v); } } while (0)
        unsigned k = 0; bool handed = false;
        for (;; ++k) {
            LWAIT(L_DESC + (k & 7u) * 16, k + 1u, 32u * k);
            const u32x4 de = *(vl128*)(lds + L_DESC + (k & 7u) * 16);
            const int i = (int)rfl(de.y); const unsigned zs = rfl(de.w);
            if (i >= NITEM) break;
            if (lane == 0) *(vl32*)(lds + L_LST) = k + 1u;
            int isW, j, u; item_decode(i, rank, src, isW, j, u);
            const gchar* ub = uni(sws + pin_off(call & 1u, isW, 6 - srel, j, u));
            if (k >= 4) LWAIT(L_FREE + (k & 3u) * 4, k & ~3u, 32u * k);
            const unsigned size = min(4u * Q_TAIL + ((zs & 127u) + ((zs >> 7) & 127u) + ((zs >> 14) & 127u) + (zs >> 21)) * 16u, UNIT), dst = (unsigned)(uintptr_t)lds + (k & 3u) * UNIT;
#pragma unroll
            for (int n = 0; n < 32; ++n) {
                if (n == HAND_AT) {
                    const u32x4 dn = *(vl128*)(lds + L_DESC + ((k + 1u) & 7u) * 16);
                    handed = rfl(dn.x) == k + 2u && rfl(dn.y) >= (unsigned)NITEM;
                    if (handed && lane == 0) __hip_atomic_fetch_add((gu32*)(myws + WS_OREL + srel * 128), 1u, __ATOMIC_RELAXED, __HIP_MEMORY_SCOPE_AGENT);
                }
                if (n >= 26 && n * 1024u >= size) break;
                if (n < 26 || n * 1024u + lane * 16u < size) glds16_sys(dst + n * 1024u, lane * 16u, ub + n * 1024u);
                asm volatile("s_waitcnt vmcnt(" STR(PULL_N) ")" ::: "memory");
                if (n + 1 >= PULL_N && lane == 0) *(vl32*)(lds + L_LAND) = 32u * k + (n + 1 - PULL_N);
            }
        }
        if (!handed && lane == 0) __hip_atomic_fetch_add((gu32*)(myws + WS_OREL + srel * 128), 1u, __ATOMIC_RELAXED, __HIP_MEMORY_SCOPE_AGENT);
        asm volatile("s_waitcnt vmcnt(0)" ::: "memory");
        if (lane == 0) *(vl32*)(lds + L_LAND) = 32u * k;
        __builtin_amdgcn_s_setprio(0);
    } else {
        unsigned inf0, hi0;
        const int it = first;
            if (wv >= 4 && flag_wait<2>(myws + WS_IFLAG + (fi * 8 + it) * 8, tmo, call, inf0, hi0)) {
                const __amdgpu_buffer_rsrc_t rs = __builtin_amdgcn_make_buffer_rsrc((void*)uni(myws + WS_INBOX + (size_t)(((call & 1u) * 7 + srel) * 8 + it) * UNIT), 0, UNIT, 0x00020000);
                u32x4 W[8];
#pragma unroll
                for (int c = 0; c < 8; ++c) W[c] = RLOAD4(rs, lane * 16u, (wv - 4u) * Q_BYTES + c * 1024u);
                int isW, j, u; item_decode(it, rank, src, isW, j, u);
                const gchar* ub; gu32* f = stage_unit(myws, srel, isW, j, u, ub); ub = uni(ub);
                __builtin_amdgcn_s_waitcnt(0x0F70);
#pragma unroll
                for (int c = 0; c < 8; ++c) store16_sys(ub + ((wv - 4u) * Q_BYTES + c * 1024u), lane * 16u, W[c]);
                asm volatile("s_waitcnt vmcnt(0)" ::: "memory");
                if (lane == 0 && __hip_atomic_fetch_add((LAS unsigned*)(lds + L_QI), 1u, __ATOMIC_RELAXED, __HIP_MEMORY_SCOPE_WORKGROUP) == 3u)
                    __hip_atomic_store(f, (call << 8) | (inf0 & 31u), __ATOMIC_RELAXED, __HIP_MEMORY_SCOPE_SYSTEM);
            }
        gu32* pf = nullptr; unsigned pk = 0, pv = 0;
        auto ack = [&]() {
            asm volatile("s_waitcnt vmcnt(0)" ::: "memory");
            if (lane == 0) {
                const unsigned old = __hip_atomic_fetch_add((LAS unsigned*)(lds + L_UCNT + (pk & 7u) * 4), 1u, __ATOMIC_RELAXED, __HIP_MEMORY_SCOPE_WORKGROUP);
                if ((old & 3u) == 3u) __hip_atomic_store(pf, pv, __ATOMIC_RELAXED, __HIP_MEMORY_SCOPE_SYSTEM);
            }
            pf = nullptr;
        };
        for (unsigned n = wv - 2u;; n += 6u) {
            const unsigned k = n >> 2, q = n & 3u;
            if (pf && rfl(*(vl32*)(lds + L_LAND)) < 32u * k + 8u * q + 8u) ack();
            unsigned kd = k;
            for (;;) {
                const u32x4 d0 = *(vl128*)(lds + L_DESC + (k & 7u) * 16), d1 = *(vl128*)(lds + L_DESC + ((k + 7u) & 7u) * 16);
                if (rfl(d0.x) >= k + 1u) break;
                if (k > 0 && rfl(d1.x) == k && rfl(d1.y) >= (unsigned)NITEM) { kd = k - 1u; break; }
                __builtin_amdgcn_s_sleep(DSLEEP);
            }
            const u32x4 de = *(vl128*)(lds + L_DESC + (kd & 7u) * 16);
            const int i = (int)rfl(de.y); const unsigned info = rfl(de.z);
            if (i >= NITEM) { if (pf) ack(); break; }
            const unsigned zs = rfl(de.w), zq = (zs >> (7u * q)) & 127u;
            const unsigned qo = min(q * Q_TAIL + ((q > 0 ? zs & 127u : 0u) + (q > 1 ? (zs >> 7) & 127u : 0u) + (q > 2 ? (zs >> 14) & 127u : 0u)) * 16u, UNIT - Q_TAIL);
            lds_wait1(lds, L_LAND, 32u * k + min((qo + Q_TAIL + zq * 16u + 1023u) >> 10, 32u));
            const LAS unsigned char* qs = lds + (k & 3u) * UNIT + qo;
            u32x4 W[8];
#define SLOT_READ do { LGKM0; if (lane == 0) __hip_atomic_fetch_add((LAS unsigned*)(lds + L_FREE + (k & 3u) * 4), 1u, __ATOMIC_RELAXED, __HIP_MEMORY_SCOPE_WORKGROUP); if (pf) ack(); } while (0)
            if (zq == 96u) { lds_fetch8(qs, lane, W); SLOT_READ; }
            else { QIn Q; q_read(qs, lane, Q); SLOT_READ; q_finish((info >> 5) & 31u, Q, W); }
            int isW, j, u; item_decode(i, rank, src, isW, j, u);
            const gchar* ub; gu32* f = stage_unit(myws, srel, isW, j, u, ub); ub = uni(ub);
#pragma unroll
            for (int c = 0; c < 8; ++c) store16_sys(ub + img_off(isW, q, c), lane * 16u, W[c]);
            pf = f; pk = k; pv = (call << 8) | (info & 31u);
        }
    }
    LBAR;
}

__device__ __forceinline__ void tile_order(unsigned T, unsigned& row, unsigned& col) { row = T < 2u * NE ? T / NE : T & 1u; col = T < 2u * NE ? T % NE : T >> 1; }
constexpr unsigned NCT = 256u + NOW;

__device__ __forceinline__ void fetch_outputs(const Params& p, LAS unsigned char* lds, gchar* myws, int rank, unsigned call, int link) {
    unsigned lane = threadIdx.x; asm volatile("" : "+v"(lane)); lane &= 63u;
    const unsigned wv = rfl(threadIdx.x >> 6);
    gu32* tmo = (gu32*)(myws + WS_TMO);
    const int src = (rank + 1 + link) & 7;
    const gchar* sws = uni(p.ws[src]);
    asm volatile("s_waitcnt vmcnt(0)" ::: "memory");
    LBAR;
    if (wv == 1) {
        for (unsigned k = 0;; ++k) {
            unsigned ct = 0;
            if (lane == 0) ct = __hip_atomic_fetch_add((gu32*)(myws + WS_OTK + link * 128 + 64), 1u, __ATOMIC_RELAXED, __HIP_MEMORY_SCOPE_AGENT);
            ct = rfl(ct) % NCT;
            unsigned info = 0, hi, ci = 256u; bool ok = true;
            if (ct < 256u) { unsigned row, col; tile_order(ct >> 3, row, col); ci = (row * 16u + col) * 8u + (ct & 7u); ok = flag_wait<2>(myws + WS_CFLAG + (src * 256 + ci) * 8, tmo, call, info, hi); }
            else if (lane == 0) *(vl32*)(lds + L_TURN) = k + 1u;
            if (lane == 0) *(vl128*)(lds + L_DESC + (k & 7u) * 16) = (u32x4){k + 1u, ci, ok ? info : 0u, ct};
            if (ct >= 256u) break;
            lds_wait1(lds, L_LST, k + 1u);
        }
    } else if (wv == 0) {
        {
            unsigned tk = 0;
            if (lane == 0) tk = __hip_atomic_fetch_add((gu32*)(myws + WS_OTK + link * 128), 1u, __ATOMIC_RELAXED, __HIP_MEMORY_SCOPE_AGENT);
            const unsigned need = (call - 1u) * 8u + (rfl(tk) % NOW) * 8u / NOW + 1u - LINK_EXTRA;
            const unsigned long long t0 = __builtin_amdgcn_s_memrealtime();
            while ((int)(need - rfl(ld_agt((const gu32*)(myws + WS_OREL + link * 128)))) > 0 && __builtin_amdgcn_s_memrealtime() - t0 < DEADLINE) __builtin_amdgcn_s_sleep(4);
        }
        __builtin_amdgcn_s_setprio(SEND_PRIO);
        unsigned k = 0;
        for (;; ++k) {
            LWAIT(L_DESC + (k & 7u) * 16, k + 1u, 2u * k);
            const u32x4 de = *(vl128*)(lds + L_DESC + (k & 7u) * 16);
            const unsigned ci = rfl(de.y), tail = min(rfl(de.z) >> 8, 8192u);
            if (ci >= 256u) break;
            if (lane == 0) *(vl32*)(lds + L_LST) = k + 1u;
            if (k >= 4) LWAIT(L_FREE + (k & 3u) * 4, k - 3u, 2u * k);
            const gchar* rb = uni(sws + WS_PACK + (size_t)(rank * 256 + ci) * CHUNK);
            const unsigned dst = (unsigned)(uintptr_t)lds + (k & 3u) * CHUNK;
            for (unsigned o = 0; o < tail; o += 1024u) {
                if (o + lane * 16u < tail) glds16_sys(dst + C_TAIL + o, lane * 16u, rb + (C_TAIL + o));
                asm volatile("s_waitcnt vmcnt(" STR(OPULL_N) ")" ::: "memory");
            }
#pragma unroll
            for (int n = 0; n < 26; ++n) {
                const unsigned o = (n < 2 ? 24 + n : n - 2) * 1024u;
                glds16_sys(dst + o, lane * 16u, rb + o);
                asm volatile("s_waitcnt vmcnt(" STR(OPULL_N) ")" ::: "memory");
                if ((n == OPULL_N - 1 || n == OPULL_N + 1) && lane == 0) *(vl32*)(lds + L_LAND) = 2u * k + (n > OPULL_N);
            }
        }
        asm volatile("s_waitcnt vmcnt(0)" ::: "memory");
        if (lane == 0) *(vl32*)(lds + L_LAND) = 2u * k;
    } else if (wv >= 4) {
        for (unsigned k = wv - 4u;; k += 4u) {
            bool end = false;
            while (rfl(*(vl32*)(lds + L_DESC + (k & 7u) * 16)) < k + 1u && !(end = rfl(*(vl32*)(lds + L_TURN)) - 1u <= k)) __builtin_amdgcn_s_sleep(2);
            if (end) break;
            const u32x4 de = *(vl128*)(lds + L_DESC + (k & 7u) * 16);
            const unsigned ci = rfl(de.y), info = rfl(de.z);
            if (ci >= 256u) break;
            unsigned pv, ph;
            const bool bad = !flag_wait<FLAG_SLEEP>(myws + WS_PFLAG + (link * 256 + ci) * 8, tmo, call, pv, ph) || info == 0u || rfl(ld_agt(tmo)) == call;
            const gchar* ob = uni((const char*)p.out[rank] + ((size_t)((ci >> 7) * 256) * GN + src * NL + ((ci >> 3) & 15u) * 256) * 4);
            const unsigned ovo = (((ci >> 2) & 1u) * 64u + 4u * (lane >> 4)) * (GN * 4u) + ((ci & 3u) * 16u + (lane & 15u)) * 16u;
            const __amdgpu_buffer_rsrc_t ro = __builtin_amdgcn_make_buffer_rsrc((void*)ob, 0, 255u * GN * 4u + 1024u, 0x00020000);
            u32x4 A[32];
#pragma unroll
            for (int q = 0; q < 32; ++q) { A[q] = RLOAD4(ro, ovo, (unsigned)out_off(q)); asm volatile("s_waitcnt vmcnt(" STR(DEC_W) ")" ::: "memory"); }
            const LAS unsigned char* slot = lds + (k & 3u) * CHUNK;
            lds_wait1(lds, L_LAND, 2u * k + 1u);
            CDec D; u32x4 CW[2];
            CW[0] = *(const LAS u32x4*)(slot + 24 * 1024u + lane * 16u); CW[1] = *(const LAS u32x4*)(slot + 25 * 1024u + lane * 16u);
            c_begin(D, CW, slot + C_TAIL, info & 255u, lane);
#pragma unroll
            for (int h = 0; h < 16; ++h) c_parse(D, CW, h);
            c_patch(D, slot + C_TAIL);
            __builtin_amdgcn_s_waitcnt(0x0F70);
            lds_wait1(lds, L_LAND, 2u * k + 2u);
            c_store(D, slot, A, bad, ob, ovo, lane);
            LGKM0;
            if (lane == 0) *(vl32*)(lds + L_FREE + (k & 3u) * 4) = k + 1u;
        }
    }
}

__global__ void __launch_bounds__(NTHR, 2) gemm_rs_kernel(const Params p) {
    extern __shared__ __attribute__((aligned(16))) unsigned char smem[];
    LAS unsigned char* lds = (LAS unsigned char*)smem;
    const int tid = threadIdx.x;
    const int rank = (int)rfl(p.rank), bid = (int)rfl(blockIdx.x);
    gchar* myws = uni(p.ws[rank]);
    STAMP(0);
    const int xcd = bid & 7, slot = bid >> 3;
    gu32* tmo = (gu32*)(myws + WS_TMO);

    int pm, pn;
    {
        int ar = -1;
        if (slot < 11) {
            const int c = slot < 7 ? xcd * 7 + slot : xcd * 4 + slot - 7, o = slot < 7 ? c - 28 : c;
            if (c < 28) { ar = c >> 1; pn = (slot < 7 ? 14 : 12) + (c & 1); } else { pm = 2 * rank + (o & 1); pn = o >> 1; }
        } else {
            const int li = slot - 11, q = li / 7;
            pn = 3 * (xcd & 3) + q; ar = 2 * (li - q * 7) + (xcd >> 2);
        }
        if (ar >= 0) pm = ar + (ar >= 2 * rank ? 2 : 0);
    }
    const int enc = slot < 7 ? -1 : slot < 11 ? bid - 56 : pn >= NE ? 32 + (pn - NE) * 14 + pm - (pm > 2 * rank ? 2 : 0) : -1;
    unsigned call;
    {
        const int first = slot < 11 ? bid + 24 : slot < 21 ? bid - 88 : bid - 56, second = enc >= 0 ? 200 + enc : NTASK;
        int isW, j, u, dd = slot;
        if (slot < 7) item_decode(xcd, (rank + 1 + slot) & 7, rank, isW, j, u); else task_decode(first, isW, j, u, dd);
        if (slot >= 7) for (int z = 0; z < (first >= START_NOW ? 2 * START_DELAY + (first - START_NOW) * START_SPREAD / 64 : START_D0); ++z) __builtin_amdgcn_s_sleep(16);
        u32x4 H0[4];
        const unsigned st0 = first_unit(p, lds, myws, isW, j, u, (rank + 1 + dd) & 7, first, second, call, H0);
        if (slot < 7) { push_first(p, lds, myws, rank, call, slot, xcd, st0, H0); STAMP(5); fetch_inputs(p, lds, myws, rank, call, slot, xcd); }
        else { produce(p, lds, myws, rank, call, second, enc >= 0 ? (gu32*)(myws + WS_TASKQ) : nullptr, st0, H0); STAMP(5); }
    }
    STAMP(1);

    const int wid = rfl(tid >> 6), wr = wid >> 2, wc = wid & 3;
#define LANE_VALS int lane = tid; asm volatile("" : "+v"(lane)); lane &= 63; const int fr = lane & 15, fq = lane >> 4
    const int dest = pm >> 1, link = (dest - rank - 1) & 7;
    constexpr int NMW = NOW > NE ? 2 * NE - NOW : NE;
    const bool early = slot >= 11 && pn < NE, odd = (pm ^ pn) & 1, worker = early && (odd ? pn < NOW : pn >= NMW);
    const int mini0 = enc >= 0 ? enc : early && !odd && !worker ? NENC + pn * 7 + link : 224, ministep = enc >= 0 ? 224 : 7 * NMW;
    const int njob = 1 + (224 - mini0 + ministep - 1) / ministep, jmain = enc >= 0 ? 1 : 0;
    gchar* dws = uni(p.ws[dest]);
    LAS unsigned char* scr = lds + wid * HTB;

#define KIDX(t) ((t) + ((((t) + 6 - link) * kmag) >> 8))
#define KOFF(t) ((unsigned)(KIDX(t) >> 3) * (8u * KT_BYTES) + (unsigned)(KIDX(t) & 7) * UNIT)
#define SA(b, h) (((b) * 2 + (h)) * HTB)
#define SB(b, h) ((4 + (b) * 2 + (h)) * HTB)
#define STAGE(bufoff, gbase) do { _Pragma("unroll") for (int _i = 0; _i < 2; ++_i) \
        __builtin_amdgcn_global_load_lds((const gu32*)((gbase) + voff[_i]), (LAS unsigned*)(lds + (bufoff) + ldsw + _i * 8192), 16, 0, GLDS_AUX); } while (0)
#define LDA(dst, b, h) do { _Pragma("unroll") for (int m = 0; m < 4; ++m) { dst[m][0] = *(const LAS h16x8*)(lds + SA(b, h) + aoff0 + m * 2048); dst[m][1] = *(const LAS h16x8*)(lds + SA(b, h) + aoff1 + m * 2048); } } while (0)
#define LDB(dst, b, h) do { _Pragma("unroll") for (int n = 0; n < 2; ++n) _Pragma("unroll") for (int k = 0; k < 2; ++k) dst[n][k] = *(const LAS h16x8*)(lds + SB(b, h) + boff + n * 1024 + k * 8192); } while (0)
#define MMA(ai, bj, At, Bt) do { __builtin_amdgcn_s_setprio(1); _Pragma("unroll") for (int m = 0; m < 4; ++m) _Pragma("unroll") for (int n = 0; n < 2; ++n) _Pragma("unroll") for (int k = 0; k < 2; ++k) \
        acc[ai][bj][m][n] = __builtin_amdgcn_mfma_f32_16x16x32_f16(At[m][k], Bt[n][k], acc[ai][bj][m][n], 0, 0, 0); __builtin_amdgcn_s_setprio(0); } while (0)
#define WAIT_V(n) asm volatile("s_waitcnt vmcnt(" #n ")" ::: "memory")
#define WAIT_L(n) asm volatile("s_waitcnt lgkmcnt(" #n ")" ::: "memory")
#define BAR __builtin_amdgcn_s_barrier()
#define SCHED __builtin_amdgcn_sched_barrier(0)
#define HOOK(need) do { if (ready < (need)) { \
        if (wid == 0) { \
            const unsigned long long t0 = __builtin_amdgcn_s_memrealtime(); unsigned fa, fb; int rd; \
            unsigned ln = voff[0]; asm volatile("" : "+v"(ln)); ln >>= 4;     \
            for (;;) { \
                { const int li_ = KIDX(min((int)ln, nk - 1)); fa = ld_sys(fA + li_); fb = ld_sys(fB + li_); } \
                const unsigned long long okm = __ballot((fa >> 8) >= call && (fb >> 8) >= call); \
                rd = (okm == ~0ull) ? 64 : (int)__builtin_ctzll(~okm); \
                if (rd >= (need)) break; \
                if (rfl(ld_agt(tmo)) == call || __builtin_amdgcn_s_memrealtime() - t0 > DEADLINE) { \
                    if (ln == 0) __hip_atomic_store(tmo, call, __ATOMIC_RELAXED, __HIP_MEMORY_SCOPE_AGENT); \
                    rd = 64; break; } \
                __builtin_amdgcn_s_sleep(POLL_SLEEP); \
            } \
            const int sv = (int)((fa & 31u) + (fb & 31u)); \
            const int sp = __builtin_amdgcn_update_dpp(-1, sv, 0x138, 0xf, 0xf, false);     \
            const unsigned long long cg = __ballot(sv != sp); \
            *(LAS int*)(lds + L_TAB + ln * 4) = sv; \
            if (ln == 0) { u32x4 h; h.x = (unsigned)rd; h.y = (unsigned)cg; h.z = (unsigned)(cg >> 32); h.w = 0; *(LAS u32x4*)(lds + L_HDR) = h; } \
            DIAG_WAIT(t0); \
            asm volatile("s_waitcnt vmcnt(0) lgkmcnt(0)" ::: "memory"); \
        } \
        asm volatile("" ::: "memory"); BAR; asm volatile("" ::: "memory"); \
        { const u32x4 h = *(const LAS u32x4*)(lds + L_HDR); ready = (int)rfl(h.x); chg = ((unsigned long long)rfl(h.z) << 32) | rfl(h.y); } \
    } } while (0)
#define DIAG_WAIT(t0) do {} while (0)
#define PKMUL4(op) ".set rs_i, 0\n\t.set rs_d, 0\n\t.irpc c, \"" op "\"\n\t.ifc \\c,:\n\t.set rs_d, 1\n\t.endif\n\t" \
        ".if rs_d == 0\n\t.ifnc \\c,v\n\t.ifnc \\c,[\n\t.set rs_i, rs_i*10 + \\c\n\t.endif\n\t.endif\n\t.endif\n\t.endr\n\t" \
        "v_pk_mul_f32 v[rs_i:rs_i+1], v[rs_i:rs_i+1], %16\n\tv_pk_mul_f32 v[rs_i+2:rs_i+3], v[rs_i+2:rs_i+3], %16\n\t"
#define RS16(a) asm volatile("s_cmp_eq_u32 %17, 0\n\ts_cbranch_scc1 .Lrs%=\n\ts_nop 15\n\ts_nop 3\n\t" \
            PKMUL4("%0") PKMUL4("%1") PKMUL4("%2") PKMUL4("%3") PKMUL4("%4") PKMUL4("%5") PKMUL4("%6") PKMUL4("%7") \
            PKMUL4("%8") PKMUL4("%9") PKMUL4("%10") PKMUL4("%11") PKMUL4("%12") PKMUL4("%13") PKMUL4("%14") PKMUL4("%15") \
            "s_nop 4\n.Lrs%=:" \
            : "+v"(acc[a][0][0][0]), "+v"(acc[a][0][0][1]), "+v"(acc[a][0][1][0]), "+v"(acc[a][0][1][1]), "+v"(acc[a][0][2][0]), "+v"(acc[a][0][2][1]), "+v"(acc[a][0][3][0]), "+v"(acc[a][0][3][1]), \
              "+v"(acc[a][1][0][0]), "+v"(acc[a][1][0][1]), "+v"(acc[a][1][1][0]), "+v"(acc[a][1][1][1]), "+v"(acc[a][1][2][0]), "+v"(acc[a][1][2][1]), "+v"(acc[a][1][3][0]), "+v"(acc[a][1][3][1]) \
            : "s"(fac), "s"(de) : "scc")
#define RS_PREP(kt) do { de = 0; \
        if ((chg >> ((kt) & 63)) & 1ull) { \
            const int s = (int)rfl(*(const LAS unsigned*)(lds + L_TAB + ((kt) & 63) * 4)); \
            smax = max(smax, s); const int sn = max(s, smax - 4); de = max(16 * (scur - sn), -126); scur = sn; } \
        fac = (unsigned long long)(unsigned)((de + 127) << 23) * 0x100000001ull; \
    } while (0)
#define TRIP do { \
        LDB(B0, 0, 0); SCHED; LDA(At, 0, 0); STAGE(SA(1, 1), cA + o1 + HTB); SCHED; RS16(0); RS16(1); \
        WAIT_L(8); BAR; WAIT_L(0); MMA(0, 0, At, B0); BAR; SCHED; \
        LDB(B1, 0, 1); STAGE(SB(0, 0), cB + o2); \
        BAR; WAIT_L(0); MMA(0, 1, At, B1); BAR; \
        LDA(At, 0, 1); STAGE(SA(0, 0), cA + o2); \
        BAR; WAIT_L(0); MMA(1, 0, At, B0); BAR; SCHED; \
        STAGE(SB(0, 1), cB + o2 + HTB); RS_PREP(t + 1); SCHED; \
        WAIT_V(6); BAR; MMA(1, 1, At, B1); BAR; SCHED; \
        LDB(B0, 1, 0); SCHED; LDA(At, 1, 0); STAGE(SA(0, 1), cA + o2 + HTB); SCHED; RS16(0); RS16(1); \
        WAIT_L(8); BAR; WAIT_L(0); MMA(0, 0, At, B0); BAR; SCHED; \
        LDB(B1, 1, 1); STAGE(SB(1, 0), cB + o3); \
        BAR; WAIT_L(0); MMA(0, 1, At, B1); BAR; \
        LDA(At, 1, 1); STAGE(SA(1, 0), cA + o3); \
        BAR; WAIT_L(0); MMA(1, 0, At, B0); BAR; SCHED; \
        STAGE(SB(1, 1), cB + o3 + HTB); \
        if (t + 2 < nk) RS_PREP(t + 2); \
        { const int t4 = t + 4 < nk ? t + 4 : 0; o1 = o3; o2 = KOFF(t4); o3 = KOFF(t4 + 1); asm volatile("" : "+s"(o2), "+s"(o3), "+s"(fac), "+s"(de)); } SCHED; \
        WAIT_V(6); BAR; MMA(1, 1, At, B1); BAR; SCHED; \
    } while (0)

    for (int job = 0; job < njob; ++job) {
    const bool mini = job != jmain;
    const int mi = mini0 + (job - (job > jmain)) * ministep, mT = mi / 7, ml = mi - mT * 7;
    const int mrow = mT < 2 * NE ? mT / NE : mT & 1, mcol = mT < 2 * NE ? mT % NE : mT >> 1;
    const int ua = mini ? 112 + mrow : pm, ub_ = mini ? ml * 16 + mcol : 16 + pn;
    const int nk = mini ? 8 : dest == rank ? NKT : NKT - 8, kmag = nk == NKT - 8 ? 37 : 0;
    const gchar* cA = myws + (mini ? WS_STAGE2 : WS_STAGE) + (size_t)ua * (8 * UNIT);
    const gchar* cB = myws + (mini ? WS_STAGE2 : WS_STAGE) + (size_t)ub_ * (8 * UNIT);
    const gu32* fA = (const gu32*)(myws + (mini ? WS_FLAG2 : WS_FLAG)) + ua * (mini ? 8 : NKT);
    const gu32* fB = (const gu32*)(myws + (mini ? WS_FLAG2 : WS_FLAG)) + ub_ * (mini ? 8 : NKT);
    int ready = 0; unsigned long long chg = 0; int scur = 0, smax = 0;
    u32x4 V[32];
    {
    LANE_VALS;
    const unsigned voff[2] = {(unsigned)(wid * 64 + lane) * 16u, (unsigned)(wid * 64 + lane) * 16u + 8192u}; const unsigned ldsw = (unsigned)wid * 1024u;
    const unsigned aoff0 = (unsigned)((wr * 64 + fr) * 128 + ((fq ^ (fr >> 1)) * 16)), aoff1 = aoff0 ^ 64u;
    const unsigned boff = (unsigned)(fq * 2048 + (wc * 16 + fr) * 16);
    f32x4 acc[2][2][4][2];
#pragma unroll
    for (int a = 0; a < 2; ++a)
#pragma unroll
        for (int b = 0; b < 2; ++b)
#pragma unroll
            for (int m = 0; m < 4; ++m)
#pragma unroll
                for (int n = 0; n < 2; ++n) acc[a][b][m][n] = (f32x4){0.f, 0.f, 0.f, 0.f};
    h16x8 At[4][2], B0[2][2], B1[2][2];

    WAIT_V(0);
    if (mini && job > jmain && wid == 0) {
        const unsigned long long t0 = __builtin_amdgcn_s_memrealtime();
        while ((int)(rfl(ld_agt((const gu32*)(myws + WS_OTK + ml * 128 + 64))) % NCT) < mT * 8 - MINI_LEAD && __builtin_amdgcn_s_memrealtime() - t0 < 100000ull)
            __builtin_amdgcn_s_sleep(64);
    }
    HOOK(2);
    if (!mini) STAMP(2); else STAMP(20 + 4 * (job & 1));
    unsigned o1 = KOFF(1), o2 = KOFF(2), o3 = KOFF(3); int de; unsigned long long fac;
    RS_PREP(0);
    STAGE(SB(0, 0), cB); STAGE(SA(0, 0), cA); STAGE(SB(0, 1), cB + HTB); STAGE(SA(0, 1), cA + HTB);
    if (wr == 1) BAR;
    WAIT_V(4); BAR;
    STAGE(SB(1, 0), cB + o1); STAGE(SA(1, 0), cA + o1); STAGE(SB(1, 1), cB + o1 + HTB);
    WAIT_V(6); BAR;
    if (mini) STAMP(21 + 4 * (job & 1));
    for (int t = 0; t < nk; t += 2) {
        HOOK(min(t + 4, nk));
        TRIP;
    }
    WAIT_V(0);
    if (wr == 0) BAR;
    BAR;
    if (!mini) STAMP(3); else STAMP(22 + 4 * (job & 1));

    const float qn = __builtin_nanf("");
    {
        const bool bad = rfl(ld_agt(tmo)) == call;
        const int oe = 16 * scur - 256;
#pragma unroll
        for (int a = 0; a < 2; ++a)
#pragma unroll
            for (int m = 0; m < 4; ++m)
#pragma unroll
                for (int r = 0; r < 4; ++r) {
                    f32x4 v;
                    v.x = __builtin_amdgcn_ldexpf(acc[a][0][m][0][r], oe); v.y = __builtin_amdgcn_ldexpf(acc[a][0][m][1][r], oe);
                    v.z = __builtin_amdgcn_ldexpf(acc[a][1][m][0][r], oe); v.w = __builtin_amdgcn_ldexpf(acc[a][1][m][1][r], oe);
                    if (bad) v = (f32x4){qn, qn, qn, qn};
                    V[(a * 4 + m) * 4 + r] = __builtin_bit_cast(u32x4, v);
                }
    }
    }
    LANE_VALS;
    const unsigned ovo = (unsigned)(wr * 64 + 4 * fq) * (GN * 4u) + (unsigned)(wc * 16 + fr) * 16u;
#define OSTAMP(k) do {} while (0)
#define DIAG_END do {} while (0)
    if (mini || dest == rank) {
        const gchar* ob = uni((const char*)p.out[rank] + ((size_t)((mini ? mrow : pm & 1) * 256) * GN + (mini ? ((rank + 1 + ml) & 7) * NL + mcol * 256 : rank * NL + pn * 256)) * 4);
#pragma unroll
        for (int q = 0; q < 32; ++q) { store16_sys(ob + out_off(q), ovo, V[q]); asm volatile("s_waitcnt vmcnt(" STR(LOC_W) ")" ::: "memory"); }
        WAIT_V(0);
        if (mini && lane == 0) __hip_atomic_store((GAS unsigned long long*)(myws + WS_PFLAG) + ml * 256 + (mrow * 16 + mcol) * 8 + wid, (unsigned long long)call << 40, __ATOMIC_RELAXED, __HIP_MEMORY_SCOPE_SYSTEM);
    }
    if (mini) STAMP(23 + 4 * (job & 1));
    int packed = mini ? 0 : dest - rank;
    asm volatile("" : "+v"(packed)); packed = (int)rfl(packed);
    if (packed) {
        const unsigned cidx = (unsigned)(((pm & 1) * 16 + pn) * 8 + wid);
        const gchar* pb = uni(myws + WS_PACK + (size_t)(dest * 256 + cidx) * CHUNK);
        unsigned mu = 0; float sm = 0.f;
#pragma unroll
        for (int q = 0; q < 32; ++q) {
            const u32x4 w = V[q] & 0x7fffffffu; mu = max(max(mu, w.x), max(w.y, max(w.z, w.w)));
            const f32x4 f = __builtin_bit_cast(f32x4, w); sm += (f.x + f.y) + (f.z + f.w);
        }
        const unsigned myB = min(max(min(wave_mean_exp(sm * (2.32f / 8192.f)), wave_max(mu >> 23)), 2u), 254u);
        u32x4 PK[26];
        const unsigned tail = c_encode(V, myB, PK, scr, lane);
        asm volatile("" ::: "memory");
        for (unsigned o = 0; o < tail; o += 1024u) if (o + lane * 16u < tail) store16_sys(pb + (C_TAIL + o), lane * 16u, *(const LAS u32x4*)(scr + o + lane * 16u));
#pragma unroll
        for (int q = 0; q < 26; ++q) { store16_sys(pb + q * 1024u, lane * 16u, PK[q]); asm volatile("s_waitcnt vmcnt(" STR(PK_W) ")" ::: "memory"); }
        WAIT_V(0);
        if (lane == 0) __hip_atomic_store((GAS unsigned long long*)(dws + WS_CFLAG) + rank * 256 + cidx, ((unsigned long long)call << 40) | (tail << 8) | myB, __ATOMIC_RELAXED, __HIP_MEMORY_SCOPE_SYSTEM);
        OSTAMP(31);
    }
    }
    if (worker) fetch_outputs(p, lds, myws, rank, call, link);
    DIAG_END;
}

extern "C" void kernel_launch(int rank, int nranks, ncclComm_t comm,
                              hipStream_t stream,
                              void* const* d_in, const int* in_sizes, int n_in,
                              void* d_out, int out_size,
                              const int* in_dtypes, int out_dtype,
                              void* const* d_out_all,
                              void* const* d_ws_all, size_t ws_bytes) {
    (void)comm; (void)in_dtypes; (void)out_dtype; (void)d_out;
    if (nranks != NR || (unsigned)rank >= (unsigned)NR || n_in < 2 || ws_bytes < WS_NEED ||
        in_sizes[0] != GM * KS || in_sizes[1] != KS * GN || out_size != (GM / NR) * GN) {
        fprintf(stderr, "gemm_rs: unexpected problem shape\n");
        return;
    }
    HIP_CHECK(hipFuncSetAttribute((const void*)gemm_rs_kernel, hipFuncAttributeMaxDynamicSharedMemorySize, (int)LDS_BYTES));
    Params p;
    p.x = (const float*)d_in[0]; p.w = (const float*)d_in[1];
    for (int i = 0; i < NR; ++i) { p.out[i] = (float*)d_out_all[i]; p.ws[i] = (char*)d_ws_all[i]; }
    p.rank = rank; p.pad = 0;
    hipLaunchKernelGGL(gemm_rs_kernel, dim3(GRID), dim3(NTHR), LDS_BYTES, stream, p);
    HIP_CHECK(hipGetLastError());
}
```

```cpp
#include <hip/hip_runtime.h>
#include <rccl/rccl.h>
#include <cstdio>
#include <cstdint>

#define HIP_CHECK(expr) do { hipError_t _e = (expr); if (_e != hipSuccess) { \
    fprintf(stderr, "HIP error %s at %s:%d\n", hipGetErrorString(_e), __FILE__, __LINE__); \
    return; } } while (0)

#define LAS __attribute__((address_space(3)))
#define GAS __attribute__((address_space(1)))
typedef GAS char gchar;
typedef GAS unsigned gu32;
typedef _Float16 h16x8 __attribute__((ext_vector_type(8)));
typedef float f32x4 __attribute__((ext_vector_type(4)));
typedef unsigned u32x4 __attribute__((ext_vector_type(4)));

constexpr int NR = 8;
constexpr int GM = 4096, GN = 32768, KS = 512, NL = 4096;
constexpr int GRID = 256, NTHR = 512;
constexpr unsigned UNIT = 32768;
constexpr unsigned HTB = 16384;
constexpr int NKT = 64;
constexpr size_t WS_TMO = 128, WS_TASKQ = 256, WS_LINKQ = 1024, WS_OTK = 2048, WS_OREL = 3072;
constexpr size_t WS_FLAG = 4096;
constexpr size_t WS_CFLAG = 16384;
constexpr size_t WS_AFLAG = 32768;
constexpr size_t WS_IFLAG = 12288;
constexpr size_t WS_FLAG2 = 65536;
constexpr size_t WS_PFLAG = 73728;
constexpr size_t WS_CALLS = 98304;
constexpr size_t WS_INBOX = 131072;
constexpr size_t WS_PX = WS_INBOX + (size_t)112 * UNIT;
constexpr size_t WS_PW = WS_PX + (size_t)256 * UNIT;
constexpr size_t WS_STAGE = WS_PW + (size_t)14 * 128 * UNIT;
constexpr unsigned KT_BYTES = 32 * UNIT;
constexpr unsigned CHUNK = 34816;
constexpr size_t WS_STAGE2 = WS_STAGE + (size_t)NKT * KT_BYTES;
constexpr size_t WS_PACK = WS_STAGE2 + (size_t)114 * 8 * UNIT;
constexpr size_t WS_NEED = WS_PACK + (size_t)NR * 256 * CHUNK;
__device__ __forceinline__ size_t pin_off(unsigned par, int isW, int dd, int j, int u) {
    return isW ? WS_PW + (size_t)((par * 7 + dd) * 128 + j * 16 + u) * UNIT : WS_PX + (size_t)(par * 128 + j * 16 + u) * UNIT;
}
#define STR2(x) #x
#define STR(x) STR2(x)
#define PULL_N 7
#define OPULL_N 4
#define DEC_W 7
#define PK_W 4
#define LOC_W 8
#define PUSH_W 0
constexpr int NOW = 12;
constexpr int HAND_AT = 20, MINI_LEAD = 64, ENC_DELAY = 4, START_NOW = 112, START_DELAY = 5, START_SPREAD = 12;
constexpr int DSLEEP = 2, POLL_SLEEP = 8, FLAG_SLEEP = 16, SEND_PRIO = 3;
constexpr int NE = 10;
constexpr unsigned long long DEADLINE = 100000000ull;
constexpr unsigned L_CTL = 4 * CHUNK + 4096;
constexpr unsigned L_AMAX = L_CTL, L_CONS = L_CTL + 32, L_TURN = L_CTL + 48, L_CALL = L_CTL + 52, L_LAND = L_CTL + 56, L_LST = L_CTL + 60, L_ITEM = L_CTL + 64, L_QI = L_CTL + 128;
constexpr unsigned L_ECNT = L_CTL + 192, L_UCNT = L_CTL + 224, L_FREE = L_CTL + 256, L_DESC = L_CTL + 320, L_SIZE = L_CTL + 448, L_ZERO_END = L_CTL + 576;
constexpr unsigned L_HDR = L_CTL + 576, L_TAB = L_CTL + 640, LDS_BYTES = L_CTL + 1024;

struct Params {
    const float* x; const float* w; char* myws; int rank; int pad;
    char* peer[NR];
    float* out;
};

__device__ __forceinline__ void store16_sys(const gchar* sbase, unsigned voff, u32x4 v) {
    asm volatile("s_nop 4\n\tglobal_store_dwordx4 %0, %1, %2 sc0 sc1\n\ts_nop 1" :: "v"(voff), "v"(v), "s"(sbase) : "memory");
}
__device__ __forceinline__ void glds16_sys(unsigned dst, unsigned voff, const gchar* sbase) {
    unsigned keep;
    asm volatile("s_mov_b32 %0, m0\n\ts_mov_b32 m0, %1\n\ts_nop 2\n\tglobal_load_lds_dwordx4 %2, %3 sc0 sc1\n\ts_mov_b32 m0, %0" : "=&s"(keep) : "s"(dst), "v"(voff), "s"(sbase) : "memory");
}
__device__ __forceinline__ unsigned ld_sys(const gu32* q) { return __hip_atomic_load(q, __ATOMIC_RELAXED, __HIP_MEMORY_SCOPE_SYSTEM); }
__device__ __forceinline__ unsigned ld_agt(const gu32* q) { return __hip_atomic_load(q, __ATOMIC_RELAXED, __HIP_MEMORY_SCOPE_AGENT); }
__device__ __forceinline__ unsigned rfl(unsigned v) { return (unsigned)__builtin_amdgcn_readfirstlane((int)v); }

template <class T> __device__ __forceinline__ gchar* uni(T q) {
    unsigned lo = rfl((unsigned)(uintptr_t)q), hi = rfl((unsigned)((uintptr_t)q >> 32));
    asm volatile("" : "+s"(lo), "+s"(hi));
    return (gchar*)(((uintptr_t)hi << 32) | lo);
}

__device__ __forceinline__ unsigned wave_max(unsigned x) {
    int mb = (int)x;
    mb = max(mb, __builtin_amdgcn_update_dpp(0, mb, 0x111, 0xf, 0xf, false));
    mb = max(mb, __builtin_amdgcn_update_dpp(0, mb, 0x112, 0xf, 0xf, false));
    mb = max(mb, __builtin_amdgcn_update_dpp(0, mb, 0x114, 0xf, 0xf, false));
    mb = max(mb, __builtin_amdgcn_update_dpp(0, mb, 0x118, 0xf, 0xf, false));
    mb = max(mb, __builtin_amdgcn_update_dpp(0, mb, 0x142, 0xa, 0xf, false));
    mb = max(mb, __builtin_amdgcn_update_dpp(0, mb, 0x143, 0xc, 0xf, false));
    return (unsigned)__builtin_amdgcn_readlane(mb, 63);
}
__device__ __forceinline__ unsigned wave_mean_exp(float sm) {
#define DPP_ADD(ctl, rm) sm += __builtin_bit_cast(float, __builtin_amdgcn_update_dpp(0, __builtin_bit_cast(int, sm), ctl, rm, 0xf, false))
    DPP_ADD(0x111, 0xf); DPP_ADD(0x112, 0xf); DPP_ADD(0x114, 0xf); DPP_ADD(0x118, 0xf); DPP_ADD(0x142, 0xa); DPP_ADD(0x143, 0xc);
    return ((unsigned)__builtin_amdgcn_readlane(__builtin_bit_cast(int, sm), 63) >> 23) & 255u;
}

typedef unsigned short u16x2 __attribute__((ext_vector_type(2)));
typedef unsigned u32x2 __attribute__((ext_vector_type(2)));
constexpr unsigned Q_P6 = 6144, Q_TAIL = 6656, Q_BYTES = 8192;
__device__ __forceinline__ u16x2 as16(unsigned a) { return __builtin_bit_cast(u16x2, a); }
__device__ __forceinline__ unsigned as32(u16x2 a) { return __builtin_bit_cast(unsigned, a); }
__device__ __forceinline__ unsigned pk_sub(unsigned a, unsigned b) { return as32(as16(a) - as16(b)); }
__device__ __forceinline__ unsigned pk_subs(unsigned a, unsigned b) { return as32(__builtin_elementwise_sub_sat(as16(a), as16(b))); }
__device__ __forceinline__ unsigned pk_min(unsigned a, unsigned b) { return as32(__builtin_elementwise_min(as16(a), as16(b))); }
__device__ __forceinline__ unsigned ffbl(unsigned a) { unsigned r; asm("v_ffbl_b32 %0, %1" : "=v"(r) : "v"(a)); return r; }
__device__ __forceinline__ unsigned wave_scan(unsigned x) {
    int v = (int)x;
    v += __builtin_amdgcn_update_dpp(0, v, 0x111, 0xf, 0xf, false);
    v += __builtin_amdgcn_update_dpp(0, v, 0x112, 0xf, 0xf, false);
    v += __builtin_amdgcn_update_dpp(0, v, 0x114, 0xf, 0xf, false);
    v += __builtin_amdgcn_update_dpp(0, v, 0x118, 0xf, 0xf, false);
    v += __builtin_amdgcn_update_dpp(0, v, 0x142, 0xa, 0xf, false);
    v += __builtin_amdgcn_update_dpp(0, v, 0x143, 0xc, 0xf, false);
    return (unsigned)v;
}
__device__ __forceinline__ unsigned lanes_below(unsigned long long m) { return __builtin_amdgcn_mbcnt_hi((unsigned)(m >> 32), __builtin_amdgcn_mbcnt_lo((unsigned)m, 0u)); }
template <int WPL> __device__ __forceinline__ unsigned stream_select(const LAS unsigned char* str, unsigned n, unsigned lane) {
    unsigned c = 0;
#pragma unroll
    for (int i = 0; i < WPL / 4; ++i) { const u32x4 w = *(const LAS u32x4*)(str + lane * (WPL * 4u) + i * 16u); c += __popc(w.x) + __popc(w.y) + __popc(w.z) + __popc(w.w); }
    const unsigned incl = wave_scan(c);
    unsigned b = 0, below = 0;
#pragma unroll
    for (int st = 32; st > 0; st >>= 1) {
        const unsigned v = (unsigned)__builtin_amdgcn_ds_bpermute((int)((b + st - 1u) * 4u), (int)incl);
        if (v < n) { b += st; below = v; }
    }
    unsigned r = n - below, pos = b * (WPL * 32u);
    u32x4 x = *(const LAS u32x4*)(str + b * (WPL * 4u));
    if (WPL == 8) {
        const u32x4 y = *(const LAS u32x4*)(str + b * (WPL * 4u) + 16u);
        const unsigned pc = __popc(x.x) + __popc(x.y) + __popc(x.z) + __popc(x.w); const bool up = r > pc;
        r -= up ? pc : 0u; pos += up ? 128u : 0u; x.x = up ? y.x : x.x; x.y = up ? y.y : x.y; x.z = up ? y.z : x.z; x.w = up ? y.w : x.w;
    }
    { const unsigned pc = __popc(x.x) + __popc(x.y); const bool up = r > pc; r -= up ? pc : 0u; pos += up ? 64u : 0u; x.x = up ? x.z : x.x; x.y = up ? x.w : x.y; }
    { const unsigned pc = __popc(x.x); const bool up = r > pc; r -= up ? pc : 0u; pos += up ? 32u : 0u; x.x = up ? x.y : x.x; }
    unsigned w = x.x;
#pragma unroll
    for (int st = 16; st > 0; st >>= 1) {
        const unsigned pc = __popc(w & ((1u << st) - 1u));
        if (r > pc) { r -= pc; w >>= st; pos += st; }
    }
    return n ? pos + 1u : 0u;
}
__device__ __forceinline__ unsigned q_encode(const u32x4 (&W)[8], unsigned B, LAS unsigned char* dst, unsigned lane) {
    const unsigned B1 = (B + 1u) * 0x04000400u;
    unsigned v[32], seg[4], pos[4], any = 0;
    u32x4 X[4];
#pragma unroll
    for (int s = 0; s < 4; ++s) {
        unsigned a = 0, p = 0;
        X[s].x = __builtin_amdgcn_perm(W[2 * s].y, W[2 * s].x, 0x07050301u); X[s].y = __builtin_amdgcn_perm(W[2 * s].w, W[2 * s].z, 0x07050301u);
        X[s].z = __builtin_amdgcn_perm(W[2 * s + 1].y, W[2 * s + 1].x, 0x07050301u); X[s].w = __builtin_amdgcn_perm(W[2 * s + 1].w, W[2 * s + 1].z, 0x07050301u);
#pragma unroll
        for (int k = 8 * s; k < 8 * s + 8; ++k) {
            const unsigned w = W[k >> 2][k & 3];
            const unsigned dq = as32(as16(pk_sub(B1, w & 0x7c007c00u)) >> 10);
            const unsigned ix = pk_min(dq, 0x00070007u) | 0x0c000c00u;
            const unsigned len = __builtin_amdgcn_perm(0x05040201u, 0x00000003u, ix) + pk_subs(dq, 0x00070007u);
            v[k] = (w & 0x83ff83ffu) | (__builtin_amdgcn_perm(0u, 0x03020100u, ix) << 10);
            any |= v[k];
            p += len & 0xffffu; a |= 1u << (p & 31u);
            p += len >> 16;     a |= 1u << (p & 31u);
        }
        seg[s] = a >> 1; pos[s] = p;
        asm volatile("" : "+v"(v[8 * s]), "+v"(v[8 * s + 1]), "+v"(v[8 * s + 2]), "+v"(v[8 * s + 3]), "+v"(v[8 * s + 4]), "+v"(v[8 * s + 5]), "+v"(v[8 * s + 6]), "+v"(v[8 * s + 7]),
                     "+v"(X[s]), "+v"(seg[s]), "+v"(pos[s]));
    }
    const bool ovf = max(max(pos[0], pos[1]), max(pos[2], pos[3])) > 31u || !(any & 0x0c000c00u);
    const unsigned tot = ovf ? 0u : pos[0] + pos[1] + pos[2] + pos[3];
    const unsigned incl = wave_scan(tot), total = (unsigned)__builtin_amdgcn_readlane((int)incl, 63);
    const unsigned long long om = __ballot(ovf);
    const unsigned sb = Q_TAIL + 64u * (unsigned)__popcll(om), size = sb + (((total + 127u) >> 7) << 4);
    if (size > Q_BYTES - 16u) return Q_BYTES;
    *(LAS u32x4*)(dst + Q_TAIL + lane * 16u) = (u32x4){0u, 0u, 0u, 0u};
    if (lane < 32u) *(LAS u32x4*)(dst + Q_TAIL + 1024u + lane * 16u) = (u32x4){0u, 0u, 0u, 0u};
    if (ovf) {
#pragma unroll
        for (int k = 0; k < 32; ++k) v[k] &= ~0x0c000c00u;
    }
#pragma unroll
    for (int j = 0; j < 6; ++j)
#pragma unroll
        for (int i = 0; i < 4; ++i) v[4 * j + i] |= (v[26 + j] << (12 - 3 * i)) & 0x70007000u;
#pragma unroll
    for (int j = 0; j < 6; ++j) v[24 + j / 3] |= (v[26 + j] >> (1 + j % 3)) & (0x80008000u >> (1 + j % 3));
#pragma unroll
    for (int j = 0; j < 6; ++j) *(LAS u32x4*)(dst + j * 1024u + lane * 16u) = (u32x4){v[4 * j], v[4 * j + 1], v[4 * j + 2], v[4 * j + 3]};
    *(LAS u32x2*)(dst + Q_P6 + lane * 8u) = (u32x2){v[24], v[25]};
    if (!ovf) {
        unsigned off = incl - tot;
#pragma unroll
        for (int s = 0; s < 4; ++s) {
            const unsigned long long x = (unsigned long long)seg[s] << (off & 31u);
            LAS unsigned* q = (LAS unsigned*)(dst + sb + (off >> 5) * 4u);
            __hip_atomic_fetch_or(q, (unsigned)x, __ATOMIC_RELAXED, __HIP_MEMORY_SCOPE_WORKGROUP);
            __hip_atomic_fetch_or(q + 1, (unsigned)(x >> 32), __ATOMIC_RELAXED, __HIP_MEMORY_SCOPE_WORKGROUP);
            off += pos[s];
        }
    } else {
#pragma unroll
        for (int j = 0; j < 4; ++j) *(LAS u32x4*)(dst + Q_TAIL + lanes_below(om) * 64u + j * 16u) = X[j];
    }
    return size;
}
#define RLOAD4(rs, vo, so) __builtin_bit_cast(u32x4, __builtin_amdgcn_raw_buffer_load_b128(rs, vo, so, 17))
struct QIn { unsigned v[32]; u32x4 win, X[4]; bool ovf; };
__device__ __forceinline__ void q_read(const LAS unsigned char* src, unsigned lane, QIn& q) {
#pragma unroll
    for (int j = 0; j < 6; ++j) { const u32x4 t = *(const LAS u32x4*)(src + j * 1024u + lane * 16u); q.v[4 * j] = t.x; q.v[4 * j + 1] = t.y; q.v[4 * j + 2] = t.z; q.v[4 * j + 3] = t.w; }
    { const u32x2 t = *(const LAS u32x2*)(src + Q_P6 + lane * 8u); q.v[24] = t.x; q.v[25] = t.y; }
#pragma unroll
    for (int j = 0; j < 6; ++j) {
        unsigned d = (q.v[24 + j / 3] << (1 + j % 3)) & 0x80008000u;
#pragma unroll
        for (int i = 0; i < 4; ++i) d |= (q.v[4 * j + i] & 0x70007000u) >> (12 - 3 * i);
        q.v[26 + j] = d;
    }
    unsigned ne = 64u;
#pragma unroll
    for (int k = 0; k < 32; ++k) ne -= __popc((q.v[k] | (q.v[k] >> 1)) & 0x04000400u);
    q.ovf = ne == 64u;
    ne &= 63u;
    const unsigned long long om = __ballot(q.ovf);
    const LAS unsigned char* str = src + Q_TAIL + 64u * min((unsigned)__popcll(om), 23u);
    const unsigned off = stream_select<4>(str, wave_scan(ne) - ne, lane);
    const LAS unsigned* w = (const LAS unsigned*)(str + (off >> 5) * 4u);
    const unsigned w0 = w[0], w1 = w[1], w2 = w[2], w3 = w[3], w4 = w[4];
    q.win = (u32x4){__builtin_amdgcn_alignbit(w1, w0, off), __builtin_amdgcn_alignbit(w2, w1, off), __builtin_amdgcn_alignbit(w3, w2, off), __builtin_amdgcn_alignbit(w4, w3, off)};
    if (q.ovf) {
#pragma unroll
        for (int j = 0; j < 4; ++j) q.X[j] = *(const LAS u32x4*)(src + Q_TAIL + lanes_below(om) * 64u + j * 16u);
    }
}
__device__ __forceinline__ void q_finish(unsigned B, const QIn& q, u32x4 (&W)[8]) {
    const unsigned* v = q.v;
    u32x4 win = q.win;
    const unsigned B1 = (B + 1u) * 0x04000400u;
#pragma unroll
    for (int s = 0; s < 4; ++s) {
        unsigned a = win.x << 1, used = 0;
#pragma unroll
        for (int k = 8 * s; k < 8 * s + 8; ++k) {
            const unsigned nz = v[k] | (v[k] >> 1);
            const unsigned l0 = ffbl((a & ~1u) | ((nz >> 10) & 1u));
            a >>= l0;
            const unsigned l1 = ffbl((a & ~1u) | ((nz >> 26) & 1u));
            a >>= l1;
            const unsigned len = l0 | (l1 << 16);
            used += len;
            const unsigned dq = __builtin_amdgcn_perm(0x00000006u, 0x00050400u, pk_min(len, 0x00040004u) | 0x0c000c00u) + pk_subs(len, 0x00040004u);
            W[k >> 2][k & 3] = (v[k] & 0x83ff83ffu) | (pk_sub(B1, (v[k] & 0x0c000c00u) + (dq << 10)) & 0x7c007c00u);
        }
        if (s < 3) {
            const unsigned sh = (used + (used >> 16)) & 31u;
            win.x = __builtin_amdgcn_alignbit(win.y, win.x, sh); win.y = __builtin_amdgcn_alignbit(win.z, win.y, sh);
            win.z = __builtin_amdgcn_alignbit(win.w, win.z, sh); win.w >>= sh;
        }
    }
    if (q.ovf) {
#pragma unroll
        for (int k = 0; k < 32; ++k) W[k >> 2][k & 3] = __builtin_amdgcn_perm(q.X[k >> 3][(k >> 1) & 3], W[k >> 2][k & 3], (k & 1) ? 0x07020600u : 0x05020400u);
    }
}

constexpr unsigned C_TAIL = 26624;
__device__ __forceinline__ unsigned c_encode(const u32x4 (&V)[32], unsigned B, u32x4 (&PK)[26], LAS unsigned char* scr, unsigned lane) {
    const unsigned B1 = (B + 1u) * 0x00010001u;
    unsigned seg[8], pos[8], sb = 0;
#pragma unroll
    for (int g = 0; g < 4; ++g) {
        unsigned e2[16];
#pragma unroll
        for (int v = 0; v < 8; ++v) {
            u32x4 r;
#pragma unroll
            for (int c = 0; c < 4; ++c) r[c] = __builtin_amdgcn_alignbit(V[g * 8 + v][c], V[g * 8 + v][c], 31);
#pragma unroll
            for (int i = 0; i < 3; ++i) PK[g * 6 + ((3 * v + i) >> 2)][(3 * v + i) & 3] = __builtin_amdgcn_perm(r.w, r[i], 0x04020100u + ((unsigned)i << 24));
            e2[2 * v] = __builtin_amdgcn_perm(r.y, r.x, 0x0c070c03u); e2[2 * v + 1] = __builtin_amdgcn_perm(r.w, r.z, 0x0c070c03u);
        }
#pragma unroll
        for (int s = 2 * g; s < 2 * g + 2; ++s) {
            unsigned a = 0, p = 0, cw = 0;
#pragma unroll
            for (int j = 0; j < 8; ++j) {
                const unsigned dq = pk_sub(B1, e2[(s & 1) * 8 + j]), ix = pk_min(dq, 0x00070007u) | 0x0c000c00u;
                const unsigned len = __builtin_amdgcn_perm(0x05040201u, 0x00000003u, ix) + pk_subs(dq, 0x00070007u);
                cw |= __builtin_amdgcn_perm(0u, 0x03020100u, ix) << (2 * j);
                p += len & 0xffffu; a |= 1u << (p & 31u);
                p += len >> 16;     a |= 1u << (p & 31u);
            }
            seg[s] = a >> 1; pos[s] = p; PK[24 + (s >> 2)][s & 3] = cw;
        }
        const bool ovf = max(pos[2 * g], pos[2 * g + 1]) > 31u || !(PK[24 + (g >> 1)][(2 * g) & 3] | PK[24 + (g >> 1)][(2 * g + 1) & 3]);
        const unsigned long long om = __ballot(ovf);
        if (ovf) {
            u32x4 E[2];
#pragma unroll
            for (int v = 0; v < 8; ++v) E[v >> 2][v & 3] = __builtin_amdgcn_perm(e2[2 * v + 1], e2[2 * v], 0x06040200u);
            *(LAS u32x4*)(scr + sb + lanes_below(om) * 32u) = E[0]; *(LAS u32x4*)(scr + sb + lanes_below(om) * 32u + 16u) = E[1];
            seg[2 * g] = seg[2 * g + 1] = pos[2 * g] = pos[2 * g + 1] = 0u; PK[24 + (g >> 1)][(2 * g) & 3] = 0u; PK[24 + (g >> 1)][(2 * g + 1) & 3] = 0u;
        }
        sb += 32u * (unsigned)__popcll(om);
    }
    const unsigned tot = (pos[0] + pos[1] + pos[2] + pos[3]) + (pos[4] + pos[5] + pos[6] + pos[7]);
    const unsigned incl = wave_scan(tot), total = (unsigned)__builtin_amdgcn_readlane((int)incl, 63);
    *(LAS u32x4*)(scr + sb + lane * 16u) = (u32x4){0u, 0u, 0u, 0u}; *(LAS u32x4*)(scr + sb + 1024u + lane * 16u) = (u32x4){0u, 0u, 0u, 0u};
    unsigned off = incl - tot;
#pragma unroll
    for (int s = 0; s < 8; ++s) {
        const unsigned long long x = (unsigned long long)seg[s] << (off & 31u);
        LAS unsigned* q = (LAS unsigned*)(scr + sb + (off >> 5) * 4u);
        __hip_atomic_fetch_or(q, (unsigned)x, __ATOMIC_RELAXED, __HIP_MEMORY_SCOPE_WORKGROUP);
        __hip_atomic_fetch_or(q + 1, (unsigned)(x >> 32), __ATOMIC_RELAXED, __HIP_MEMORY_SCOPE_WORKGROUP);
        off += pos[s];
    }
    return sb + ((total + 7u) >> 3);
}
__device__ __forceinline__ size_t out_off(int q) { return (size_t)((q >> 4) * 128 + ((q >> 2) & 3) * 16 + (q & 3)) * (GN * 4); }
__device__ __forceinline__ void c_parse(unsigned (&E)[16], const LAS unsigned char* src, unsigned hf, unsigned B, unsigned lane) {
    const u32x4 C0 = *(const LAS u32x4*)(src + 24 * 1024u + lane * 16u), C1 = *(const LAS u32x4*)(src + 25 * 1024u + lane * 16u), CW = hf ? C1 : C0;
    unsigned ne = 0, nb = 0, sb = 0, xo[2] = {0u, 0u}; bool exc[2] = {false, false};
#pragma unroll
    for (int g = 0; g < 4; ++g) {
        const unsigned c0 = (g < 2 ? C0 : C1)[(2 * g) & 3], c1 = (g < 2 ? C0 : C1)[(2 * g + 1) & 3];
        const unsigned n = 32u - __popc((c0 | (c0 >> 1)) & 0x55555555u) - __popc((c1 | (c1 >> 1)) & 0x55555555u);
        const unsigned long long om = __ballot(n == 32u);
        if (g == 2) nb = hf ? ne : 0u;
        ne += n & 31u;
        if ((unsigned)(g >> 1) == hf) { exc[g & 1] = n == 32u; xo[g & 1] = sb + lanes_below(om) * 32u; }
        sb += 32u * (unsigned)__popcll(om);
    }
    const LAS unsigned char* tail = src + C_TAIL;
    const unsigned off = stream_select<8>(tail + sb, wave_scan(ne) - ne + nb, lane);
    unsigned win[5];
#pragma unroll
    for (int i = 0; i < 5; ++i) win[i] = *(const LAS unsigned*)(tail + sb + (off >> 5) * 4u + i * 4u);
#pragma unroll
    for (int i = 0; i < 4; ++i) win[i] = __builtin_amdgcn_alignbit(win[i + 1], win[i], off);
    const unsigned B1 = (B + 1u) * 0x00010001u;
#pragma unroll
    for (int s = 0; s < 4; ++s) {
        const unsigned cw = CW[s], nz = cw | (cw >> 1);
        unsigned a = win[0] << 1, used = 0, e2[8];
#pragma unroll
        for (int j = 0; j < 8; ++j) {
            const unsigned l0 = ffbl((a & ~1u) | ((nz >> (2 * j)) & 1u));
            a >>= l0;
            const unsigned l1 = ffbl((a & ~1u) | ((nz >> (2 * j + 16)) & 1u));
            a >>= l1;
            const unsigned len = l0 | (l1 << 16);
            used += len;
            const unsigned dq = __builtin_amdgcn_perm(0x00000006u, 0x00050400u, pk_min(len, 0x00040004u) | 0x0c000c00u) + pk_subs(len, 0x00040004u);
            e2[j] = pk_sub(B1, dq + ((cw >> (2 * j)) & 0x00030003u));
        }
#pragma unroll
        for (int i = 0; i < 4; ++i) E[4 * s + i] = __builtin_amdgcn_perm(e2[2 * i + 1], e2[2 * i], 0x06040200u);
        asm volatile("" : "+v"(E[4 * s]), "+v"(E[4 * s + 1]), "+v"(E[4 * s + 2]), "+v"(E[4 * s + 3]));
        const unsigned sh = exc[s >> 1] ? 0u : used + (used >> 16);
#pragma unroll
        for (int i = 0; i < 3 - s; ++i) win[i] = __builtin_amdgcn_alignbit(win[i + 1], win[i], sh);
    }
#pragma unroll
    for (int g = 0; g < 2; ++g) if (exc[g]) {
        const u32x4 x0 = *(const LAS u32x4*)(tail + xo[g]), x1 = *(const LAS u32x4*)(tail + xo[g] + 16u);
#pragma unroll
        for (int v = 0; v < 4; ++v) { E[8 * g + v] = x0[v]; E[8 * g + 4 + v] = x1[v]; }
    }
}
__device__ __forceinline__ void c_store(const unsigned (&E)[16], const LAS unsigned char* src, const u32x4 (&A)[16], bool bad, const gchar* ob, unsigned ovo, unsigned lane) {
#pragma unroll
    for (int h = 0; h < 4; ++h) {
        u32x4 PK[3];
#pragma unroll
        for (int i = 0; i < 3; ++i) PK[i] = *(const LAS u32x4*)(src + (3 * h + i) * 1024u + lane * 16u);
#pragma unroll
        for (int v = 0; v < 4; ++v) {
            const int q = 4 * h + v;
            unsigned P[3];
#pragma unroll
            for (int i = 0; i < 3; ++i) P[i] = PK[(3 * v + i) >> 2][(3 * v + i) & 3];
            u32x4 r;
            r.x = __builtin_amdgcn_perm(E[q], P[0], 0x04020100u); r.y = __builtin_amdgcn_perm(E[q], P[1], 0x05020100u); r.z = __builtin_amdgcn_perm(E[q], P[2], 0x06020100u);
            r.w = __builtin_amdgcn_perm(P[1], P[0], 0x0c0c0703u) | __builtin_amdgcn_perm(E[q], P[2], 0x07030c0cu);
#pragma unroll
            for (int c = 0; c < 4; ++c) r[c] = bad ? 0x7fc00000u : __builtin_amdgcn_alignbit(r[c], r[c], 1);
            const f32x4 o = __builtin_bit_cast(f32x4, r) + __builtin_bit_cast(f32x4, A[q]);
            asm volatile("s_nop 4\n\tglobal_store_dwordx4 %0, %1, %2 sc0 sc1 nt\n\ts_nop 1" :: "v"(ovo), "v"(o), "s"(ob + out_off(q)) : "memory");
            asm volatile("s_waitcnt vmcnt(" STR(DEC_W) ")" ::: "memory");
        }
    }
}

constexpr int NITEM = 240;
__device__ __forceinline__ void item_decode(int i, int d, int s, int& isW, int& j, int& u) {
    constexpr int R0 = 12 + NE, R1 = 18 - NE;
    if (i < 8 * R0) {
        j = i / R0; int r = i - j * R0;
        if (i >= 6 && i < R0) r = i < 12 ? i + NE : i - 6;
        isW = r >= 6 && r < 6 + NE; u = isW ? r - 6 : r < 6 ? 2 * r : 2 * (r - 6 - NE) + 1;
        if (!isW) { if (u >= 2 * min(d, s)) u += 2; if (u >= 2 * max(d, s)) u += 2; }
    } else { const int r = i - 8 * R0; j = r / R1; const int t = r - j * R1; isW = t < 16 - NE; u = isW ? NE + t : 2 * d + t - (16 - NE); }
}
constexpr int NENC = 32 + 14 * (12 - NE);
constexpr int NTASK = 1152, NTASK_DYN0 = 200 + 3 * NENC, NTASK_DYN = NTASK - 200;
__device__ __forceinline__ void task_decode(int t, int& isW, int& j, int& u, int& dd) {
    constexpr int R0 = 16 + 8 * NE, R1 = 8 * (16 - NE);
    if (t < 8 * R0) { j = t / R0; int r = t - j * R0; isW = r >= 16; if (isW) { r -= 16; u = r >> 3; dd = r & 7; } else { u = r < 8 ? 2 * r : 2 * r - 15; dd = 7; } }
    else { int r = t - 8 * R0; j = r / R1; r -= j * R1; isW = 1; u = NE + (r >> 3); dd = r & 7; }
}

__device__ __forceinline__ void task_load(const Params& p, int rank, int i, f32x4 (&L)[16]) {
    const bool valid = i < NTASK;
    int isW, j, u, dd; task_decode(valid ? i : 0, isW, j, u, dd);
    const unsigned t = threadIdx.x;
    const gchar* b = uni(isW ? (const char*)p.w + ((size_t)(j * 64) * GN + ((rank + 1 + dd) & 7) * NL + u * 256) * 4 : (const char*)p.x + ((size_t)(u * 256) * KS + j * 64) * 4);
    unsigned vo = isW ? (t >> 6) * (4u * GN * 4u) + (t & 63u) * 16u : (t >> 3) * (KS * 4u) + ((t & 7u) ^ ((t >> 4) & 7u)) * 16u;
    if (!valid) vo = 0;
    const unsigned s0 = isW ? GN * 4u : 128u, s1 = isW ? 2u * GN * 4u : 64u * KS * 4u, s2 = isW ? 32u * GN * 4u : 128u * KS * 4u, s3 = isW ? 16u * GN * 4u : 32u * KS * 4u;
#pragma unroll
    for (int n = 0; n < 16; ++n)
        L[n] = __builtin_nontemporal_load((const GAS f32x4*)(b + ((n & 1) * s0 + ((n >> 1) & 1) * s1 + ((n >> 2) & 1) * s2 + (n >> 3) * s3) + vo));
}

__device__ __forceinline__ unsigned pk2(float a, float b, int e) {
    typedef _Float16 h2 __attribute__((ext_vector_type(2)));
    h2 h; h.x = (_Float16)__builtin_amdgcn_ldexpf(a, e); h.y = (_Float16)__builtin_amdgcn_ldexpf(b, e);
    return __builtin_bit_cast(unsigned, h);
}

#define LBAR do { asm volatile("s_waitcnt lgkmcnt(0)" ::: "memory"); __builtin_amdgcn_s_barrier(); asm volatile("" ::: "memory"); } while (0)

typedef volatile LAS unsigned vl32;
typedef volatile LAS u32x4 vl128;
#define LGKM0 asm volatile("s_waitcnt lgkmcnt(0)" ::: "memory")
__device__ __forceinline__ u32x4 lds_wait4(LAS unsigned char* lds, unsigned off, unsigned v) {
    for (;;) {
        const u32x4 q = *(vl128*)(lds + off);
        if (rfl(min(min(q.x, q.y), min(q.z, q.w))) >= v) { asm volatile("" ::: "memory"); return q; }
        __builtin_amdgcn_s_sleep(1);
    }
}
__device__ __forceinline__ unsigned lds_wait1(LAS unsigned char* lds, unsigned off, unsigned v) {
    unsigned x;
    while ((x = rfl(*(vl32*)(lds + off))) < v) __builtin_amdgcn_s_sleep(DSLEEP);
    asm volatile("" ::: "memory");
    return x;
}
template <int N> __device__ __forceinline__ unsigned wave_stats(const f32x4 (&L)[N]) {
    unsigned mu = 0; float sm = 0.f;
#pragma unroll
    for (int n = 0; n < N; ++n) {
        const u32x4 q = __builtin_bit_cast(u32x4, L[n]) & 0x7fffffffu; mu = max(max(mu, q.x), max(q.y, max(q.z, q.w)));
        const f32x4 f = __builtin_bit_cast(f32x4, q); sm += (f.x + f.y) + (f.z + f.w);
    }
    unsigned mb = wave_max(mu >> 23);
    if (mb == 255u) {
        mu = 0;
#pragma unroll
        for (int n = 0; n < N; ++n)
#pragma unroll
            for (int c = 0; c < 4; ++c) mu = max(mu, (__builtin_bit_cast(unsigned, L[n][c]) + 0x00800000u) & 0x7f800000u);
        mb = max(wave_max(mu >> 23), 1u) - 1u;
    }
    return mb | (wave_mean_exp(sm * (2.32f / (256.f * N))) << 8);
}
__device__ __forceinline__ void task_stats(LAS unsigned char* lds, const f32x4 (&L)[16], unsigned k) {
    const unsigned ws = wave_stats(L);
    LGKM0;
    if ((threadIdx.x & 63) == 63) *(vl32*)(lds + L_AMAX + (k & 1u) * 16 + (threadIdx.x >> 6) * 4) = ((k + 1u) << 16) | ws;
}

__device__ __forceinline__ unsigned img_off(int isW, unsigned q, int c) {
    return (isW ? q * 2048u + (c >> 2) * 8192u + (c & 1) * 1024u : q * 1024u + (c >> 2) * 4096u + (c & 1) * 8192u) + ((c >> 1) & 1) * HTB;
}
__device__ __forceinline__ gu32* stage_unit(gchar* myws, int srel, int isW, int j, int u, const gchar*& ub) {
    const int idx = j * 8 + ((srel + 1) & 7);
    ub = myws + WS_STAGE + (size_t)((j * 32 + isW * 16 + u) * 8 + (idx & 7)) * UNIT;
    return (gu32*)(myws + WS_FLAG) + (isW * 16 + u) * NKT + idx;
}
__device__ __forceinline__ void own_images(gchar* myws, int rank, int isW, int j, int u, int dd, unsigned q, unsigned lane, const u32x4 (&H)[8]) {
    if (dd == 7) {
        const gchar* ub; stage_unit(myws, 7, isW, j, u, ub); ub = uni(ub);
#pragma unroll
        for (int c = 0; c < 8; ++c) store16_sys(ub + img_off(isW, q, c), lane * 16u, H[c]);
    }
    if (isW ? dd != 7 : (u >> 1) == rank) {
        const gchar* ub = uni(myws + WS_STAGE2 + (size_t)((isW ? dd * 16 + u : 112 + (u & 1)) * 8 + j) * UNIT);
#pragma unroll
        for (int c = 0; c < 8; ++c) store16_sys(ub + img_off(isW, q, c), lane * 16u, H[c]);
    }
}
__device__ __forceinline__ void own_flags(gchar* myws, int rank, int isW, int j, int u, int dd, unsigned lane, unsigned val) {
    const gchar* ub; gu32* sf = stage_unit(myws, 7, isW, j, u, ub);
    if (lane == 0 ? dd == 7 : lane == 1 && (isW ? dd != 7 : (u >> 1) == rank))
        __hip_atomic_store(lane ? (gu32*)(myws + WS_FLAG2) + (isW ? dd * 16 + u : 112 + (u & 1)) * 8 + j : sf, val, __ATOMIC_RELAXED, __HIP_MEMORY_SCOPE_SYSTEM);
}
__device__ __forceinline__ int conv_step(const Params& p, LAS unsigned char* lds, gchar* myws, gu32* tick, int rank, unsigned call, unsigned k, f32x4 (&L)[16], const f32x4 (&Ln)[16]) {
    unsigned t = threadIdx.x; asm volatile("" : "+v"(t));
    const u32x4 am = lds_wait4(lds, L_AMAX + (k & 1u) * 16, (k + 1u) << 16);
    const int i = (int)rfl(*(vl32*)(lds + L_ITEM + (k & 15u) * 4)), i1 = (int)rfl(*(vl32*)(lds + L_ITEM + ((k + 1u) & 15u) * 4)), i2 = (int)rfl(*(vl32*)(lds + L_ITEM + ((k + 2u) & 15u) * 4));
    const bool valid = i < NTASK;
    if (valid) {
        int isW, j, u, dd; task_decode(i, isW, j, u, dd);
        const unsigned mb = rfl(max(max(am.x & 255u, am.y & 255u), max(am.z & 255u, am.w & 255u))), em = (rfl(max(max(am.x, am.y), max(am.z, am.w))) >> 8) & 255u;
        const int cls = (int)(mb + 2u) >> 4;
        const int e = 128 - 16 * cls;
        u32x4 H[8];
#pragma unroll
        for (int h = 0; h < 2; ++h) {
            const f32x4* Q = L + 8 * h; u32x4* G = H + 4 * h;
            if (isW) {
#pragma unroll
                for (int c = 0; c < 4; ++c) { G[c].x = pk2(Q[0][c], Q[1][c], e); G[c].y = pk2(Q[2][c], Q[3][c], e); G[c].z = pk2(Q[4][c], Q[5][c], e); G[c].w = pk2(Q[6][c], Q[7][c], e); }
            } else {
#pragma unroll
                for (int c = 0; c < 4; ++c) { G[c].x = pk2(Q[2 * c].x, Q[2 * c].y, e); G[c].y = pk2(Q[2 * c].z, Q[2 * c].w, e); G[c].z = pk2(Q[2 * c + 1].x, Q[2 * c + 1].y, e); G[c].w = pk2(Q[2 * c + 1].z, Q[2 * c + 1].w, e); }
            }
        }
        const unsigned q = rfl(t >> 6);
        own_images(myws, rank, isW, j, u, dd, q, t & 63u, H);
        if (k >= 4) lds_wait1(lds, L_CONS + q * 4, k - 3u);
#pragma unroll
        for (int c = 0; c < 8; ++c) *(LAS u32x4*)(lds + (k & 3u) * UNIT + q * Q_BYTES + c * 1024u + (t & 63u) * 16u) = H[c];
        LGKM0;
        if ((t & 63u) == 0) *(vl32*)(lds + L_QI + (k & 3u) * 16 + q * 4) = ((k + 1u) << 16) | ((unsigned)min(max((int)min(em, mb) + e - 112, 2), 31) << 5) | (unsigned)cls;
        asm volatile("s_waitcnt vmcnt(0)" ::: "memory");
        unsigned old = 0;
        if ((t & 63u) == 0) old = __hip_atomic_fetch_add((LAS unsigned*)(lds + L_UCNT + (k & 7u) * 4), 1u, __ATOMIC_RELAXED, __HIP_MEMORY_SCOPE_WORKGROUP);
        if ((rfl(old) & 3u) == 3u) own_flags(myws, rank, isW, j, u, dd, t & 63u, (call << 8) | (unsigned)cls);
    }
    unsigned tk = NTASK_DYN - 1;
    gu32* q = tick; asm volatile("" : "+v"(q));
    if (t == 0 && valid) tk = __hip_atomic_fetch_add(q, 1u, __ATOMIC_RELAXED, __HIP_MEMORY_SCOPE_AGENT);
    task_load(p, rank, i2, L);
    __builtin_amdgcn_s_waitcnt(0x4F70);
    asm volatile("" : "+v"(tk));
    if (t == 0) *(vl32*)(lds + L_ITEM + ((k + 3u) & 15u) * 4) = (unsigned)NTASK_DYN0 + tk % (unsigned)NTASK_DYN;
    task_stats(lds, Ln, k + 1u);
    return i1;
}

__device__ __forceinline__ void lds_fetch8(const LAS unsigned char* src, unsigned lane, u32x4 (&H)[8]) {
#pragma unroll
    for (int c = 0; c < 8; ++c) H[c] = *(const LAS u32x4*)(src + c * 1024u + lane * 16u);
}

__device__ __forceinline__ void first_load(const gchar* px, const gchar* pw, int isW, int j, int u, int d, f32x4 (&L)[8]) {
    const unsigned t = threadIdx.x;
    const gchar* b = uni(isW ? pw + ((size_t)(j * 64) * GN + d * NL + u * 256) * 4 : px + ((size_t)(u * 256) * KS + j * 64) * 4);
    const unsigned vo = isW ? (t >> 6) * (4u * GN * 4u) + (t & 63u) * 16u : (t >> 3) * (KS * 4u) + ((t & 7u) ^ ((t >> 4) & 7u)) * 16u;
    const unsigned s0 = isW ? GN * 4u : 128u, s1 = isW ? 2u * GN * 4u : 64u * KS * 4u, s2 = isW ? 32u * GN * 4u : 128u * KS * 4u;
#pragma unroll
    for (int n = 0; n < 8; ++n) L[n] = __builtin_nontemporal_load((const GAS f32x4*)(b + ((n & 1) * s0 + ((n >> 1) & 1) * s1 + (n >> 2) * s2) + vo));
}
__device__ __forceinline__ unsigned first_unit(LAS unsigned char* lds, gchar* myws, int isW, const f32x4 (&L)[8], int first, int second, unsigned& call, u32x4 (&H)[4]) {
    const unsigned t = threadIdx.x;
    unsigned tk = 0;
    gu32* tq = (gu32*)(myws + WS_CALLS + blockIdx.x * 128u); asm volatile("" : "+v"(tq));
    if (t == 0) tk = __hip_atomic_fetch_add(tq, 1u, __ATOMIC_RELAXED, __HIP_MEMORY_SCOPE_AGENT);
    constexpr unsigned I0 = (L_ITEM - L_CTL) / 4;
    if (t < (L_ZERO_END - L_CTL) / 4 && t != (L_CALL - L_CTL) / 4) *(vl32*)(lds + L_CTL + t * 4) = t == I0 ? (unsigned)first : t - (I0 + 1) < 3u ? (unsigned)second + (t - (I0 + 1)) * NENC : 0u;
    const unsigned mu = wave_stats(L);
    asm volatile("" : "+v"(tk));
    if (t == 0) *(vl32*)(lds + L_CALL) = (tk + 1u) & 0xffffffu;
    if ((t & 63u) == 0) *(vl32*)(lds + L_HDR + (t >> 6) * 4) = mu;
    LBAR;
    const u32x4 a0 = *(vl128*)(lds + L_HDR), a1 = *(vl128*)(lds + L_HDR + 16), m0 = a0 & 255u, m1 = a1 & 255u;
    call = rfl(*(vl32*)(lds + L_CALL));
    const unsigned mb = rfl(max(max(max(m0.x, m0.y), max(m0.z, m0.w)), max(max(m1.x, m1.y), max(m1.z, m1.w))));
    const unsigned em = rfl(max(max(max(a0.x, a0.y), max(a0.z, a0.w)), max(max(a1.x, a1.y), max(a1.z, a1.w)))) >> 8;
    const int cls = (int)(mb + 2u) >> 4, e = 128 - 16 * cls;
#pragma unroll
    for (int c = 0; c < 4; ++c) {
        if (isW) { H[c].x = pk2(L[0][c], L[1][c], e); H[c].y = pk2(L[2][c], L[3][c], e); H[c].z = pk2(L[4][c], L[5][c], e); H[c].w = pk2(L[6][c], L[7][c], e); }
        else { H[c].x = pk2(L[2 * c].x, L[2 * c].y, e); H[c].y = pk2(L[2 * c].z, L[2 * c].w, e); H[c].z = pk2(L[2 * c + 1].x, L[2 * c + 1].y, e); H[c].w = pk2(L[2 * c + 1].z, L[2 * c + 1].w, e); }
    }
    return ((unsigned)min(max((int)min(em, mb) + e - 112, 2), 31) << 5) | (unsigned)cls;
}

__device__ __forceinline__ void produce(const Params& p, LAS unsigned char* lds, gchar* myws, int rank, unsigned call, int second, gu32* tick, unsigned st0, const u32x4 (&H0)[4]) {
#pragma unroll
    for (int c = 0; c < 4; ++c) *(LAS u32x4*)(lds + ((threadIdx.x >> 6) & 3u) * Q_BYTES + ((threadIdx.x >> 8) * 4u + c) * 1024u + (threadIdx.x & 63u) * 16u) = H0[c];
    if (threadIdx.x < 4) *(vl32*)(lds + L_QI + threadIdx.x * 4) = (1u << 16) | st0;
    LBAR;
    if (rfl(threadIdx.x >> 6) < 4) {
        if (tick) {
            f32x4 RA[16], RB[16];
            for (int z = 0; z < ENC_DELAY; ++z) __builtin_amdgcn_s_sleep(32);
            task_load(p, rank, second, RA);
            task_stats(lds, RA, 1);
            task_load(p, rank, second + NENC, RB);
            int nx; unsigned k = 1;
            do { conv_step(p, lds, myws, tick, rank, call, k, RA, RB); nx = conv_step(p, lds, myws, tick, rank, call, k + 1u, RB, RA); k += 2; } while (nx < NTASK);
        }
    } else {
        const unsigned lane = threadIdx.x & 63u, q = rfl(threadIdx.x >> 6) - 4u;
        const gchar* peer = (const gchar*)p.peer[min(lane, 6u)];
        int ip = -1; unsigned long long infop = 0;
        auto settle = [&](unsigned kp) {
            asm volatile("s_waitcnt vmcnt(0)" ::: "memory");
            unsigned old = 0;
            if (lane == 0) old = __hip_atomic_fetch_add((LAS unsigned*)(lds + L_ECNT + (kp & 7u) * 4), 1u, __ATOMIC_RELAXED, __HIP_MEMORY_SCOPE_WORKGROUP);
            if ((rfl(old) & 3u) == 3u) {
                int isW, j, u, dd; task_decode(ip, isW, j, u, dd);
                const bool peers = isW ? dd != 7 : (u >> 1) != rank;
                const unsigned fi = (call & 1u) * 8 + (6u - lane);
                if (peers && (isW ? lane == (unsigned)dd : lane < 7u))
                    __hip_atomic_store((GAS unsigned long long*)(peer + WS_AFLAG + ((fi * 2 + isW) * 128 + j * 16 + u) * 8), ((unsigned long long)call << 40) | infop, __ATOMIC_RELAXED, __HIP_MEMORY_SCOPE_SYSTEM);
                if (kp == 0) own_flags(myws, rank, isW, j, u, dd, lane, (call << 8) | ((unsigned)infop & 31u));
            }
            ip = -1;
        };
        for (unsigned k = 0;; ++k) {
            const int i = (int)rfl(*(vl32*)(lds + L_ITEM + (k & 15u) * 4));
            if (ip >= 0 && (i >= NTASK || rfl(*(vl32*)(lds + L_QI + (k & 3u) * 16 + q * 4)) < ((k + 1u) << 16))) settle(k - 1u);
            if (i >= NTASK) break;
            const unsigned st = lds_wait1(lds, L_QI + (k & 3u) * 16 + q * 4, (k + 1u) << 16);
            int isW, j, u, dd; task_decode(i, isW, j, u, dd);
            LAS unsigned char* src = lds + (k & 3u) * UNIT + q * Q_BYTES;
            u32x4 H[8];
            lds_fetch8(src, lane, H);
            const bool peers = isW ? dd != 7 : (u >> 1) != rank;
            if (k == 0) own_images(myws, rank, isW, j, u, dd, q, lane, H);
            unsigned size = 0;
            if (peers) {
                size = q_encode(H, (st >> 5) & 31u, src, lane);
                if (lane == 0) *(vl32*)(lds + L_SIZE + (k & 7u) * 16 + q * 4) = ((k + 1u) << 8) | ((size - Q_TAIL) >> 4);
            }
            if (ip >= 0) settle(k - 1u);
            ip = i; infop = st & 1023u;
            if (peers) {
                const u32x4 sz = lds_wait4(lds, L_SIZE + (k & 7u) * 16, (k + 1u) << 8) & 127u;
                const unsigned z0 = rfl(sz.x), z1 = rfl(sz.y), z2 = rfl(sz.z), z3 = rfl(sz.w);
                infop |= (unsigned long long)(z0 | (z1 << 7) | (z2 << 14) | (z3 << 21)) << 10;
                const unsigned qo = q * Q_TAIL + ((q > 0 ? z0 : 0u) + (q > 1 ? z1 : 0u) + (q > 2 ? z2 : 0u)) * 16u, sh = qo & 127u;
                const gchar* pb = uni(myws + pin_off(call & 1u, isW, dd, j, u) + (qo - sh));
                lds_fetch8(src - sh, lane, H);
#pragma unroll
                for (int c = 0; c < 8; ++c) if ((c > 0 && c < 6) || c * 1024u + lane * 16u - sh < size) store16_sys(pb + c * 1024u, lane * 16u, H[c]);
                if (Q_BYTES + lane * 16u - sh < size) store16_sys(pb + Q_BYTES, lane * 16u, *(const LAS u32x4*)(src - sh + Q_BYTES + lane * 16u));
            }
            LGKM0;
            if (lane == 0) *(vl32*)(lds + L_CONS + q * 4) = k + 1u;
        }
    }
    LBAR;
}

__device__ __forceinline__ void push_first(LAS unsigned char* lds, gchar* myws, gchar* dws, unsigned call, int link, int item, unsigned st0, const u32x4 (&H)[4]) {
    const unsigned t = threadIdx.x;
    const gchar* ib = uni(dws + WS_INBOX + (size_t)(((call & 1u) * 7 + (6 - link)) * 8 + item) * UNIT);
#pragma unroll
    for (int c = 0; c < 4; ++c) {
        if (c > 1 && rfl(t >> 6) != 1u) asm volatile("s_waitcnt vmcnt(" STR(PUSH_W) ")" ::: "memory");
        store16_sys(ib + c * 8192u, t * 16u, H[c]);
    }
    asm volatile("s_waitcnt vmcnt(0)" ::: "memory");
    if ((t & 63u) == 0 && __hip_atomic_fetch_add((LAS unsigned*)(lds + L_ECNT), 1u, __ATOMIC_RELAXED, __HIP_MEMORY_SCOPE_WORKGROUP) == 7u) {
        __hip_atomic_store((GAS unsigned long long*)(dws + WS_IFLAG) + ((call & 1u) * 8 + (6 - link)) * 8 + item, ((unsigned long long)call << 40) | st0, __ATOMIC_RELAXED, __HIP_MEMORY_SCOPE_SYSTEM);
    }
}

template <int SLEEP> __device__ __forceinline__ bool flag_wait(const gchar* f, gu32* tmo, unsigned call, unsigned& info, unsigned& hi) {
    const unsigned long long t0 = __builtin_amdgcn_s_memrealtime();
    for (;;) {
        const unsigned long long fv = __hip_atomic_load((const GAS unsigned long long*)f, __ATOMIC_RELAXED, __HIP_MEMORY_SCOPE_SYSTEM);
        info = rfl((unsigned)fv); hi = rfl((unsigned)(fv >> 32));
        if ((hi >> 8) >= call) break;
        if (rfl(ld_agt(tmo)) == call || __builtin_amdgcn_s_memrealtime() - t0 > DEADLINE) {
            if ((threadIdx.x & 63u) == 0) __hip_atomic_store(tmo, call, __ATOMIC_RELAXED, __HIP_MEMORY_SCOPE_AGENT);
            return false;
        }
        __builtin_amdgcn_s_sleep(SLEEP);
    }
    asm volatile("" ::: "memory");
    return true;
}

__device__ __forceinline__ void fetch_inputs(LAS unsigned char* lds, gchar* myws, const gchar* sws, int rank, unsigned call, int srel, int first) {
    const unsigned lane = threadIdx.x & 63u, wv = rfl(threadIdx.x >> 6);
    gu32* tmo = (gu32*)(myws + WS_TMO);
    const unsigned fi = (call & 1u) * 8 + srel;
    const int src = (rank + 1 + srel) & 7;
    if (wv == 1) {
        unsigned k = 0;
        for (;;) {
            unsigned tk = 0;
            if (lane == 0) tk = __hip_atomic_fetch_add((gu32*)(myws + WS_LINKQ + srel * 128), 1u, __ATOMIC_RELAXED, __HIP_MEMORY_SCOPE_AGENT);
            const int i = 8 + (int)(rfl(tk) % (unsigned)NITEM);
            unsigned info = 0, hi = 0; bool ok = true;
            if (i < NITEM) { int isW, j, u; item_decode(i, rank, src, isW, j, u); ok = flag_wait<2>(myws + WS_AFLAG + ((fi * 2 + isW) * 128 + j * 16 + u) * 8, tmo, call, info, hi); }
            if (ok) { if (lane == 0) *(vl128*)(lds + L_DESC + (k & 7u) * 16) = (u32x4){k + 1u, (unsigned)i, info & 1023u, (info >> 10) | ((hi & 63u) << 22)}; ++k; }
            if (i >= NITEM) break;
            if (ok) lds_wait1(lds, L_LST, k);
        }
    } else if (wv == 0) {
        __builtin_amdgcn_s_setprio(SEND_PRIO);
#define LWAIT(off, v, n) do { if (rfl(*(vl32*)(lds + (off))) < (v)) { asm volatile("s_waitcnt vmcnt(0)" ::: "memory"); if (lane == 0) *(vl32*)(lds + L_LAND) = (n); lds_wait1(lds, off, v); } } while (0)
        unsigned k = 0; bool handed = false;
        for (;; ++k) {
            LWAIT(L_DESC + (k & 7u) * 16, k + 1u, 32u * k);
            const u32x4 de = *(vl128*)(lds + L_DESC + (k & 7u) * 16);
            const int i = (int)rfl(de.y); const unsigned zs = rfl(de.w);
            if (i >= NITEM) break;
            if (lane == 0) *(vl32*)(lds + L_LST) = k + 1u;
            int isW, j, u; item_decode(i, rank, src, isW, j, u);
            const gchar* ub = uni(sws + pin_off(call & 1u, isW, 6 - srel, j, u));
            if (k >= 4) LWAIT(L_FREE + (k & 3u) * 4, k & ~3u, 32u * k);
            const unsigned size = min(4u * Q_TAIL + ((zs & 127u) + ((zs >> 7) & 127u) + ((zs >> 14) & 127u) + (zs >> 21)) * 16u, UNIT), dst = (unsigned)(uintptr_t)lds + (k & 3u) * UNIT;
#pragma unroll
            for (int n = 0; n < 32; ++n) {
                if (n == HAND_AT) {
                    const u32x4 dn = *(vl128*)(lds + L_DESC + ((k + 1u) & 7u) * 16);
                    handed = rfl(dn.x) == k + 2u && rfl(dn.y) >= (unsigned)NITEM;
                    if (handed && lane == 0) __hip_atomic_fetch_add((gu32*)(myws + WS_OREL + srel * 128), 1u, __ATOMIC_RELAXED, __HIP_MEMORY_SCOPE_AGENT);
                }
                if (n >= 26 && n * 1024u >= size) break;
                if (n < 26 || n * 1024u + lane * 16u < size) glds16_sys(dst + n * 1024u, lane * 16u, ub + n * 1024u);
                asm volatile("s_waitcnt vmcnt(" STR(PULL_N) ")" ::: "memory");
                if (n + 1 >= PULL_N && lane == 0) *(vl32*)(lds + L_LAND) = 32u * k + (n + 1 - PULL_N);
            }
        }
        if (!handed && lane == 0) __hip_atomic_fetch_add((gu32*)(myws + WS_OREL + srel * 128), 1u, __ATOMIC_RELAXED, __HIP_MEMORY_SCOPE_AGENT);
        asm volatile("s_waitcnt vmcnt(0)" ::: "memory");
        if (lane == 0) *(vl32*)(lds + L_LAND) = 32u * k;
        __builtin_amdgcn_s_setprio(0);
    } else {
        unsigned inf0, hi0;
        if (wv >= 4 && flag_wait<2>(myws + WS_IFLAG + (fi * 8 + first) * 8, tmo, call, inf0, hi0)) {
            const __amdgpu_buffer_rsrc_t rs = __builtin_amdgcn_make_buffer_rsrc((void*)uni(myws + WS_INBOX + (size_t)(((call & 1u) * 7 + srel) * 8 + first) * UNIT), 0, UNIT, 0x00020000);
            u32x4 W[8];
#pragma unroll
            for (int c = 0; c < 8; ++c) W[c] = RLOAD4(rs, lane * 16u, (wv - 4u) * Q_BYTES + c * 1024u);
            int isW, j, u; item_decode(first, rank, src, isW, j, u);
            const gchar* ub; gu32* f = stage_unit(myws, srel, isW, j, u, ub); ub = uni(ub);
            __builtin_amdgcn_s_waitcnt(0x0F70);
#pragma unroll
            for (int c = 0; c < 8; ++c) store16_sys(ub + ((wv - 4u) * Q_BYTES + c * 1024u), lane * 16u, W[c]);
            asm volatile("s_waitcnt vmcnt(0)" ::: "memory");
            if (lane == 0 && __hip_atomic_fetch_add((LAS unsigned*)(lds + L_QI), 1u, __ATOMIC_RELAXED, __HIP_MEMORY_SCOPE_WORKGROUP) == 3u)
                __hip_atomic_store(f, (call << 8) | (inf0 & 31u), __ATOMIC_RELAXED, __HIP_MEMORY_SCOPE_SYSTEM);
        }
        gu32* pf = nullptr; unsigned pk = 0, pv = 0;
        auto ack = [&]() {
            asm volatile("s_waitcnt vmcnt(0)" ::: "memory");
            if (lane == 0) {
                const unsigned old = __hip_atomic_fetch_add((LAS unsigned*)(lds + L_UCNT + (pk & 7u) * 4), 1u, __ATOMIC_RELAXED, __HIP_MEMORY_SCOPE_WORKGROUP);
                if ((old & 3u) == 3u) __hip_atomic_store(pf, pv, __ATOMIC_RELAXED, __HIP_MEMORY_SCOPE_SYSTEM);
            }
            pf = nullptr;
        };
        for (unsigned n = wv - 2u;; n += 6u) {
            const unsigned k = n >> 2, q = n & 3u;
            if (pf && rfl(*(vl32*)(lds + L_LAND)) < 32u * k + 8u * q + 8u) ack();
            unsigned kd = k;
            for (;;) {
                const u32x4 d0 = *(vl128*)(lds + L_DESC + (k & 7u) * 16), d1 = *(vl128*)(lds + L_DESC + ((k + 7u) & 7u) * 16);
                if (rfl(d0.x) >= k + 1u) break;
                if (k > 0 && rfl(d1.x) == k && rfl(d1.y) >= (unsigned)NITEM) { kd = k - 1u; break; }
                __builtin_amdgcn_s_sleep(DSLEEP);
            }
            const u32x4 de = *(vl128*)(lds + L_DESC + (kd & 7u) * 16);
            const int i = (int)rfl(de.y); const unsigned info = rfl(de.z);
            if (i >= NITEM) { if (pf) ack(); break; }
            const unsigned zs = rfl(de.w), zq = (zs >> (7u * q)) & 127u;
            const unsigned qo = min(q * Q_TAIL + ((q > 0 ? zs & 127u : 0u) + (q > 1 ? (zs >> 7) & 127u : 0u) + (q > 2 ? (zs >> 14) & 127u : 0u)) * 16u, UNIT - Q_TAIL);
            lds_wait1(lds, L_LAND, 32u * k + min((qo + Q_TAIL + zq * 16u + 1023u) >> 10, 32u));
            const LAS unsigned char* qs = lds + (k & 3u) * UNIT + qo;
            u32x4 W[8];
#define SLOT_READ do { LGKM0; if (lane == 0) __hip_atomic_fetch_add((LAS unsigned*)(lds + L_FREE + (k & 3u) * 4), 1u, __ATOMIC_RELAXED, __HIP_MEMORY_SCOPE_WORKGROUP); if (pf) ack(); } while (0)
            if (zq == 96u) { lds_fetch8(qs, lane, W); SLOT_READ; }
            else { QIn Q; q_read(qs, lane, Q); SLOT_READ; q_finish((info >> 5) & 31u, Q, W); }
            int isW, j, u; item_decode(i, rank, src, isW, j, u);
            const gchar* ub; gu32* f = stage_unit(myws, srel, isW, j, u, ub); ub = uni(ub);
#pragma unroll
            for (int c = 0; c < 8; ++c) store16_sys(ub + img_off(isW, q, c), lane * 16u, W[c]);
            pf = f; pk = k; pv = (call << 8) | (info & 31u);
        }
    }
    LBAR;
}

__device__ __forceinline__ void tile_order(unsigned T, unsigned& row, unsigned& col) { row = T < 2u * NE ? T / NE : T & 1u; col = T < 2u * NE ? T % NE : T >> 1; }
constexpr unsigned NCT = 256u + NOW;

__device__ __forceinline__ void fetch_outputs(const Params& p, LAS unsigned char* lds, gchar* myws, int rank, unsigned call, int link) {
    unsigned lane = threadIdx.x; asm volatile("" : "+v"(lane)); lane &= 63u;
    const unsigned wv = rfl(threadIdx.x >> 6);
    gu32* tmo = (gu32*)(myws + WS_TMO);
    const int src = (rank + 1 + link) & 7;
    const gchar* sws = uni(p.peer[link]);
    asm volatile("s_waitcnt vmcnt(0)" ::: "memory");
    LBAR;
    if (wv == 1) {
        for (unsigned k = 0;; ++k) {
            unsigned ct = 0;
            if (lane == 0) ct = __hip_atomic_fetch_add((gu32*)(myws + WS_OTK + link * 128 + 64), 1u, __ATOMIC_RELAXED, __HIP_MEMORY_SCOPE_AGENT);
            ct = rfl(ct) % NCT;
            unsigned info = 0, hi, ci = 256u; bool ok = true;
            if (ct < 256u) { unsigned row, col; tile_order(ct >> 3, row, col); ci = (row * 16u + col) * 8u + (ct & 7u); ok = flag_wait<2>(myws + WS_CFLAG + (src * 256 + ci) * 8, tmo, call, info, hi); }
            else if (lane == 0) *(vl32*)(lds + L_TURN) = k + 1u;
            if (lane == 0) *(vl128*)(lds + L_DESC + (k & 7u) * 16) = (u32x4){k + 1u, ci, ok ? info : 0u, ct};
            if (ct >= 256u) break;
            lds_wait1(lds, L_LST, k + 1u);
        }
    } else if (wv == 0) {
        {
            unsigned tk = 0;
            if (lane == 0) tk = __hip_atomic_fetch_add((gu32*)(myws + WS_OTK + link * 128), 1u, __ATOMIC_RELAXED, __HIP_MEMORY_SCOPE_AGENT);
            const unsigned need = (call - 1u) * 8u + (rfl(tk) % NOW) * 8u / NOW + 1u;
            const unsigned long long t0 = __builtin_amdgcn_s_memrealtime();
            while ((int)(need - rfl(ld_agt((const gu32*)(myws + WS_OREL + link * 128)))) > 0 && __builtin_amdgcn_s_memrealtime() - t0 < DEADLINE) __builtin_amdgcn_s_sleep(4);
        }
        __builtin_amdgcn_s_setprio(SEND_PRIO);
        unsigned k = 0;
        for (;; ++k) {
            LWAIT(L_DESC + (k & 7u) * 16, k + 1u, 3u * k);
            const u32x4 de = *(vl128*)(lds + L_DESC + (k & 7u) * 16);
            const unsigned ci = rfl(de.y), tail = min(rfl(de.z) >> 8, 8192u);
            if (ci >= 256u) break;
            if (lane == 0) *(vl32*)(lds + L_LST) = k + 1u;
            if (k >= 4) LWAIT(L_FREE + (k & 3u) * 4, (k >> 2) * 2u, 3u * k);
            const gchar* rb = uni(sws + WS_PACK + (size_t)(rank * 256 + ci) * CHUNK);
            const unsigned dst = (unsigned)(uintptr_t)lds + (k & 3u) * CHUNK;
            for (unsigned o = 0; o < tail; o += 1024u) {
                if (o + lane * 16u < tail) glds16_sys(dst + C_TAIL + o, lane * 16u, rb + (C_TAIL + o));
                asm volatile("s_waitcnt vmcnt(" STR(OPULL_N) ")" ::: "memory");
            }
#pragma unroll
            for (int n = 0; n < 26; ++n) {
                const unsigned o = (n < 2 ? 24 + n : n - 2) * 1024u;
                glds16_sys(dst + o, lane * 16u, rb + o);
                asm volatile("s_waitcnt vmcnt(" STR(OPULL_N) ")" ::: "memory");
                if ((n == OPULL_N - 1 || n == OPULL_N + 1 || n == OPULL_N + 13) && lane == 0) *(vl32*)(lds + L_LAND) = 3u * k + (n > OPULL_N) + (n > OPULL_N + 1);
            }
        }
        asm volatile("s_waitcnt vmcnt(0)" ::: "memory");
        if (lane == 0) *(vl32*)(lds + L_LAND) = 3u * k;
    } else {
        for (unsigned n = wv - 2u;; n += 6u) {
            const unsigned k = n >> 1, hf = n & 1u;
            bool end = false;
            while (rfl(*(vl32*)(lds + L_DESC + (k & 7u) * 16)) < k + 1u && !(end = rfl(*(vl32*)(lds + L_TURN)) - 1u <= k)) __builtin_amdgcn_s_sleep(2);
            if (end) break;
            const u32x4 de = *(vl128*)(lds + L_DESC + (k & 7u) * 16);
            const unsigned ci = rfl(de.y), info = rfl(de.z);
            if (ci >= 256u) break;
            unsigned pv, ph;
            const bool bad = !flag_wait<FLAG_SLEEP>(myws + WS_PFLAG + (link * 256 + ci) * 8, tmo, call, pv, ph) || info == 0u || rfl(ld_agt(tmo)) == call;
            const gchar* ob = uni((const char*)p.out + ((size_t)((ci >> 7) * 256 + hf * 128) * GN + src * NL + ((ci >> 3) & 15u) * 256) * 4);
            const unsigned ovo = (((ci >> 2) & 1u) * 64u + 4u * (lane >> 4)) * (GN * 4u) + ((ci & 3u) * 16u + (lane & 15u)) * 16u;
            const __amdgpu_buffer_rsrc_t ro = __builtin_amdgcn_make_buffer_rsrc((void*)ob, 0, 127u * GN * 4u + 1024u, 0x00020000);
            u32x4 A[16];
#pragma unroll
            for (int q = 0; q < 16; ++q) { A[q] = RLOAD4(ro, ovo, (unsigned)out_off(q)); asm volatile("s_waitcnt vmcnt(" STR(DEC_W) ")" ::: "memory"); }
            const LAS unsigned char* slot = lds + (k & 3u) * CHUNK;
            lds_wait1(lds, L_LAND, 3u * k + 1u);
            unsigned E[16];
            c_parse(E, slot, hf, info & 255u, lane);
            __builtin_amdgcn_s_waitcnt(0x0F70);
            lds_wait1(lds, L_LAND, 3u * k + 2u + hf);
            c_store(E, slot + hf * 12288u, A, bad, ob, ovo, lane);
            LGKM0;
            if (lane == 0) __hip_atomic_fetch_add((LAS unsigned*)(lds + L_FREE + (k & 3u) * 4), 1u, __ATOMIC_RELAXED, __HIP_MEMORY_SCOPE_WORKGROUP);
        }
    }
}

__global__ void __launch_bounds__(NTHR, 2) gemm_rs_kernel(const Params p) {
    extern __shared__ __attribute__((aligned(16))) unsigned char smem[];
    LAS unsigned char* lds = (LAS unsigned char*)smem;
    const int tid = threadIdx.x;
    int rank = p.rank, bid = blockIdx.x;
    gchar* myws = (gchar*)p.myws; const gchar* px = (const gchar*)p.x; const gchar* pw = (const gchar*)p.w;
    gchar* lws = (gchar*)p.peer[(bid >> 3) & 7];
    asm volatile("" : "+s"(rank), "+s"(bid), "+s"(myws), "+s"(px), "+s"(pw), "+s"(lws));
    const int xcd = bid & 7, slot = bid >> 3;
    gu32* tmo = (gu32*)(myws + WS_TMO);

    const int first = slot < 11 ? bid + 24 : slot < 21 ? bid - 88 : bid - 56;
    int isW0, j, u, dd = slot;
    if (slot < 7) item_decode(xcd, (rank + 1 + slot) & 7, rank, isW0, j, u); else task_decode(first, isW0, j, u, dd);
    if (slot >= 7) for (int z = 0; z < (first >= START_NOW ? 2 * START_DELAY + (first - START_NOW) * START_SPREAD / 64 : 0); ++z) __builtin_amdgcn_s_sleep(16);
    f32x4 L0[8];
    first_load(px, pw, isW0, j, u, (rank + 1 + dd) & 7, L0);
    __builtin_amdgcn_sched_barrier(0);

    int pm, pn;
    {
        int ar = -1;
        if (slot < 11) {
            const int c = slot < 7 ? xcd * 7 + slot : xcd * 4 + slot - 7, o = slot < 7 ? c - 28 : c;
            if (c < 28) { ar = c >> 1; pn = (slot < 7 ? 14 : 12) + (c & 1); } else { pm = 2 * rank + (o & 1); pn = o >> 1; }
        } else {
            const int li = slot - 11, q = li / 7;
            pn = 3 * (xcd & 3) + q; ar = 2 * (li - q * 7) + (xcd >> 2);
        }
        if (ar >= 0) pm = ar + (ar >= 2 * rank ? 2 : 0);
    }
    const int enc = slot < 7 ? -1 : slot < 11 ? bid - 56 : pn >= NE ? 32 + (pn - NE) * 14 + pm - (pm > 2 * rank ? 2 : 0) : -1;
    unsigned call;
    {
        const int second = enc >= 0 ? 200 + enc : NTASK;
        u32x4 H0[4];
        const unsigned st0 = first_unit(lds, myws, isW0, L0, first, second, call, H0);
        if (slot < 7) { push_first(lds, myws, lws, call, slot, xcd, st0, H0); fetch_inputs(lds, myws, lws, rank, call, slot, xcd); }
        else { produce(p, lds, myws, rank, call, second, enc >= 0 ? (gu32*)(myws + WS_TASKQ) : nullptr, st0, H0); }
    }

    const int wid = rfl(tid >> 6), wr = wid >> 2, wc = wid & 3;
#define LANE_VALS int lane = tid; asm volatile("" : "+v"(lane)); lane &= 63; const int fr = lane & 15, fq = lane >> 4
    const int dest = pm >> 1, link = (dest - rank - 1) & 7;
    constexpr int NMW = NOW > NE ? 2 * NE - NOW : NE;
    const bool early = slot >= 11 && pn < NE, odd = (pm ^ pn) & 1, worker = early && (odd ? pn < NOW : pn >= NMW);
    const int mini0 = enc >= 0 ? enc : early && !odd && !worker ? NENC + pn * 7 + link : 224, ministep = enc >= 0 ? 224 : 7 * NMW;
    const int njob = 1 + (224 - mini0 + ministep - 1) / ministep, jmain = enc >= 0 ? 1 : 0;
    gchar* dws = uni(p.peer[link]);
    LAS unsigned char* scr = lds + wid * HTB;

#define KIDX(t) ((t) + ((((t) + 6 - link) * kmag) >> 8))
#define KOFF(t) ((unsigned)(KIDX(t) >> 3) * (8u * KT_BYTES) + (unsigned)(KIDX(t) & 7) * UNIT)
#define SA(b, h) (((b) * 2 + (h)) * HTB)
#define SB(b, h) ((4 + (b) * 2 + (h)) * HTB)
#define STAGE(bufoff, gbase) do { _Pragma("unroll") for (int _i = 0; _i < 2; ++_i) \
        __builtin_amdgcn_global_load_lds((const gu32*)((gbase) + voff[_i]), (LAS unsigned*)(lds + (bufoff) + ldsw + _i * 8192), 16, 0, 17); } while (0)
#define LDA(dst, b, h) do { _Pragma("unroll") for (int m = 0; m < 4; ++m) { dst[m][0] = *(const LAS h16x8*)(lds + SA(b, h) + aoff0 + m * 2048); dst[m][1] = *(const LAS h16x8*)(lds + SA(b, h) + aoff1 + m * 2048); } } while (0)
#define LDB(dst, b, h) do { _Pragma("unroll") for (int n = 0; n < 2; ++n) _Pragma("unroll") for (int k = 0; k < 2; ++k) dst[n][k] = *(const LAS h16x8*)(lds + SB(b, h) + boff + n * 1024 + k * 8192); } while (0)
#define MMA(ai, bj, At, Bt) do { __builtin_amdgcn_s_setprio(1); _Pragma("unroll") for (int m = 0; m < 4; ++m) _Pragma("unroll") for (int n = 0; n < 2; ++n) _Pragma("unroll") for (int k = 0; k < 2; ++k) \
        acc[ai][bj][m][n] = __builtin_amdgcn_mfma_f32_16x16x32_f16(At[m][k], Bt[n][k], acc[ai][bj][m][n], 0, 0, 0); __builtin_amdgcn_s_setprio(0); } while (0)
#define WAIT_V(n) asm volatile("s_waitcnt vmcnt(" #n ")" ::: "memory")
#define WAIT_L(n) asm volatile("s_waitcnt lgkmcnt(" #n ")" ::: "memory")
#define BAR __builtin_amdgcn_s_barrier()
#define SCHED __builtin_amdgcn_sched_barrier(0)
#define HOOK(need) do { if (ready < (need)) { \
        if (wid == 0) { \
            const unsigned long long t0 = __builtin_amdgcn_s_memrealtime(); unsigned fa, fb; int rd; \
            unsigned ln = voff[0]; asm volatile("" : "+v"(ln)); ln >>= 4;     \
            for (;;) { \
                { const int li_ = KIDX(min((int)ln, nk - 1)); fa = ld_sys(fA + li_); fb = ld_sys(fB + li_); } \
                const unsigned long long okm = __ballot((fa >> 8) >= call && (fb >> 8) >= call); \
                rd = (okm == ~0ull) ? 64 : (int)__builtin_ctzll(~okm); \
                if (rd >= (need)) break; \
                if (rfl(ld_agt(tmo)) == call || __builtin_amdgcn_s_memrealtime() - t0 > DEADLINE) { \
                    if (ln == 0) __hip_atomic_store(tmo, call, __ATOMIC_RELAXED, __HIP_MEMORY_SCOPE_AGENT); \
                    rd = 64; break; } \
                __builtin_amdgcn_s_sleep(POLL_SLEEP); \
            } \
            const int sv = (int)((fa & 31u) + (fb & 31u)); \
            const int sp = __builtin_amdgcn_update_dpp(-1, sv, 0x138, 0xf, 0xf, false);     \
            const unsigned long long cg = __ballot(sv != sp); \
            *(LAS int*)(lds + L_TAB + ln * 4) = sv; \
            if (ln == 0) { u32x4 h; h.x = (unsigned)rd; h.y = (unsigned)cg; h.z = (unsigned)(cg >> 32); h.w = 0; *(LAS u32x4*)(lds + L_HDR) = h; } \
            asm volatile("s_waitcnt vmcnt(0) lgkmcnt(0)" ::: "memory"); \
        } \
        asm volatile("" ::: "memory"); BAR; asm volatile("" ::: "memory"); \
        { const u32x4 h = *(const LAS u32x4*)(lds + L_HDR); ready = (int)rfl(h.x); chg = ((unsigned long long)rfl(h.z) << 32) | rfl(h.y); } \
    } } while (0)
#define PKMUL4(op) ".set rs_i, 0\n\t.set rs_d, 0\n\t.irpc c, \"" op "\"\n\t.ifc \\c,:\n\t.set rs_d, 1\n\t.endif\n\t" \
        ".if rs_d == 0\n\t.ifnc \\c,v\n\t.ifnc \\c,[\n\t.set rs_i, rs_i*10 + \\c\n\t.endif\n\t.endif\n\t.endif\n\t.endr\n\t" \
        "v_pk_mul_f32 v[rs_i:rs_i+1], v[rs_i:rs_i+1], %16\n\tv_pk_mul_f32 v[rs_i+2:rs_i+3], v[rs_i+2:rs_i+3], %16\n\t"
#define RS16(a) asm volatile("s_cmp_eq_u32 %17, 0\n\ts_cbranch_scc1 .Lrs%=\n\ts_nop 15\n\ts_nop 3\n\t" \
            PKMUL4("%0") PKMUL4("%1") PKMUL4("%2") PKMUL4("%3") PKMUL4("%4") PKMUL4("%5") PKMUL4("%6") PKMUL4("%7") \
            PKMUL4("%8") PKMUL4("%9") PKMUL4("%10") PKMUL4("%11") PKMUL4("%12") PKMUL4("%13") PKMUL4("%14") PKMUL4("%15") \
            "s_nop 4\n.Lrs%=:" \
            : "+v"(acc[a][0][0][0]), "+v"(acc[a][0][0][1]), "+v"(acc[a][0][1][0]), "+v"(acc[a][0][1][1]), "+v"(acc[a][0][2][0]), "+v"(acc[a][0][2][1]), "+v"(acc[a][0][3][0]), "+v"(acc[a][0][3][1]), \
              "+v"(acc[a][1][0][0]), "+v"(acc[a][1][0][1]), "+v"(acc[a][1][1][0]), "+v"(acc[a][1][1][1]), "+v"(acc[a][1][2][0]), "+v"(acc[a][1][2][1]), "+v"(acc[a][1][3][0]), "+v"(acc[a][1][3][1]) \
            : "s"(fac), "s"(de) : "scc")
#define RS_PREP(kt) do { de = 0; \
        if ((chg >> ((kt) & 63)) & 1ull) { \
            const int s = (int)rfl(*(const LAS unsigned*)(lds + L_TAB + ((kt) & 63) * 4)); \
            smax = max(smax, s); const int sn = max(s, smax - 4); de = max(16 * (scur - sn), -126); scur = sn; } \
        fac = (unsigned long long)(unsigned)((de + 127) << 23) * 0x100000001ull; \
    } while (0)
#define TRIP do { \
        LDB(B0, 0, 0); SCHED; LDA(At, 0, 0); STAGE(SA(1, 1), cA + o1 + HTB); SCHED; RS16(0); RS16(1); \
        WAIT_L(8); BAR; WAIT_L(0); MMA(0, 0, At, B0); BAR; SCHED; \
        LDB(B1, 0, 1); STAGE(SB(0, 0), cB + o2); \
        BAR; WAIT_L(0); MMA(0, 1, At, B1); BAR; \
        LDA(At, 0, 1); STAGE(SA(0, 0), cA + o2); \
        BAR; WAIT_L(0); MMA(1, 0, At, B0); BAR; SCHED; \
        STAGE(SB(0, 1), cB + o2 + HTB); RS_PREP(t + 1); SCHED; \
        WAIT_V(6); BAR; MMA(1, 1, At, B1); BAR; SCHED; \
        LDB(B0, 1, 0); SCHED; LDA(At, 1, 0); STAGE(SA(0, 1), cA + o2 + HTB); SCHED; RS16(0); RS16(1); \
        WAIT_L(8); BAR; WAIT_L(0); MMA(0, 0, At, B0); BAR; SCHED; \
        LDB(B1, 1, 1); STAGE(SB(1, 0), cB + o3); \
        BAR; WAIT_L(0); MMA(0, 1, At, B1); BAR; \
        LDA(At, 1, 1); STAGE(SA(1, 0), cA + o3); \
        BAR; WAIT_L(0); MMA(1, 0, At, B0); BAR; SCHED; \
        STAGE(SB(1, 1), cB + o3 + HTB); \
        if (t + 2 < nk) RS_PREP(t + 2); \
        { const int t4 = t + 4 < nk ? t + 4 : 0; o1 = o3; o2 = KOFF(t4); o3 = KOFF(t4 + 1); asm volatile("" : "+s"(o2), "+s"(o3), "+s"(fac), "+s"(de)); } SCHED; \
        WAIT_V(6); BAR; MMA(1, 1, At, B1); BAR; SCHED; \
    } while (0)

    for (int job = 0; job < njob; ++job) {
    const bool mini = job != jmain;
    const int mi = mini0 + (job - (job > jmain)) * ministep, mT = mi / 7, ml = mi - mT * 7;
    const int mrow = mT < 2 * NE ? mT / NE : mT & 1, mcol = mT < 2 * NE ? mT % NE : mT >> 1;
    const int ua = mini ? 112 + mrow : pm, ub_ = mini ? ml * 16 + mcol : 16 + pn;
    const int nk = mini ? 8 : dest == rank ? NKT : NKT - 8, kmag = nk == NKT - 8 ? 37 : 0;
    const gchar* cA = myws + (mini ? WS_STAGE2 : WS_STAGE) + (size_t)ua * (8 * UNIT);
    const gchar* cB = myws + (mini ? WS_STAGE2 : WS_STAGE) + (size_t)ub_ * (8 * UNIT);
    const gu32* fA = (const gu32*)(myws + (mini ? WS_FLAG2 : WS_FLAG)) + ua * (mini ? 8 : NKT);
    const gu32* fB = (const gu32*)(myws + (mini ? WS_FLAG2 : WS_FLAG)) + ub_ * (mini ? 8 : NKT);
    int ready = 0; unsigned long long chg = 0; int scur = 0, smax = 0;
    u32x4 V[32];
    {
    LANE_VALS;
    const unsigned voff[2] = {(unsigned)(wid * 64 + lane) * 16u, (unsigned)(wid * 64 + lane) * 16u + 8192u}; const unsigned ldsw = (unsigned)wid * 1024u;
    const unsigned aoff0 = (unsigned)((wr * 64 + fr) * 128 + ((fq ^ (fr >> 1)) * 16)), aoff1 = aoff0 ^ 64u;
    const unsigned boff = (unsigned)(fq * 2048 + (wc * 16 + fr) * 16);
    f32x4 acc[2][2][4][2];
#pragma unroll
    for (int a = 0; a < 2; ++a)
#pragma unroll
        for (int b = 0; b < 2; ++b)
#pragma unroll
            for (int m = 0; m < 4; ++m)
#pragma unroll
                for (int n = 0; n < 2; ++n) acc[a][b][m][n] = (f32x4){0.f, 0.f, 0.f, 0.f};
    h16x8 At[4][2], B0[2][2], B1[2][2];

    WAIT_V(0);
    if (mini && job > jmain && wid == 0) {
        const unsigned long long t0 = __builtin_amdgcn_s_memrealtime();
        while ((int)(rfl(ld_agt((const gu32*)(myws + WS_OTK + ml * 128 + 64))) % NCT) < mT * 8 - MINI_LEAD && __builtin_amdgcn_s_memrealtime() - t0 < 100000ull)
            __builtin_amdgcn_s_sleep(64);
    }
    HOOK(2);
    unsigned o1 = KOFF(1), o2 = KOFF(2), o3 = KOFF(3); int de; unsigned long long fac;
    RS_PREP(0);
    STAGE(SB(0, 0), cB); STAGE(SA(0, 0), cA); STAGE(SB(0, 1), cB + HTB); STAGE(SA(0, 1), cA + HTB);
    if (wr == 1) BAR;
    WAIT_V(4); BAR;
    STAGE(SB(1, 0), cB + o1); STAGE(SA(1, 0), cA + o1); STAGE(SB(1, 1), cB + o1 + HTB);
    WAIT_V(6); BAR;
    for (int t = 0; t < nk; t += 2) {
        HOOK(min(t + 4, nk));
        TRIP;
    }
    WAIT_V(0);
    if (wr == 0) BAR;
    BAR;

    const float qn = __builtin_nanf("");
    {
        const bool bad = rfl(ld_agt(tmo)) == call;
        const int oe = 16 * scur - 256;
#pragma unroll
        for (int a = 0; a < 2; ++a)
#pragma unroll
            for (int m = 0; m < 4; ++m)
#pragma unroll
                for (int r = 0; r < 4; ++r) {
                    f32x4 v;
                    v.x = __builtin_amdgcn_ldexpf(acc[a][0][m][0][r], oe); v.y = __builtin_amdgcn_ldexpf(acc[a][0][m][1][r], oe);
                    v.z = __builtin_amdgcn_ldexpf(acc[a][1][m][0][r], oe); v.w = __builtin_amdgcn_ldexpf(acc[a][1][m][1][r], oe);
                    if (bad) v = (f32x4){qn, qn, qn, qn};
                    V[(a * 4 + m) * 4 + r] = __builtin_bit_cast(u32x4, v);
                }
    }
    }
    LANE_VALS;
    const unsigned ovo = (unsigned)(wr * 64 + 4 * fq) * (GN * 4u) + (unsigned)(wc * 16 + fr) * 16u;
    if (mini || dest == rank) {
        const gchar* ob = uni((const char*)p.out + ((size_t)((mini ? mrow : pm & 1) * 256) * GN + (mini ? ((rank + 1 + ml) & 7) * NL + mcol * 256 : rank * NL + pn * 256)) * 4);
#pragma unroll
        for (int q = 0; q < 32; ++q) { store16_sys(ob + out_off(q), ovo, V[q]); asm volatile("s_waitcnt vmcnt(" STR(LOC_W) ")" ::: "memory"); }
        WAIT_V(0);
        if (mini && lane == 0) __hip_atomic_store((GAS unsigned long long*)(myws + WS_PFLAG) + ml * 256 + (mrow * 16 + mcol) * 8 + wid, (unsigned long long)call << 40, __ATOMIC_RELAXED, __HIP_MEMORY_SCOPE_SYSTEM);
    }
    int packed = mini ? 0 : dest - rank;
    asm volatile("" : "+v"(packed)); packed = (int)rfl(packed);
    if (packed) {
        const unsigned cidx = (unsigned)(((pm & 1) * 16 + pn) * 8 + wid);
        const gchar* pb = uni(myws + WS_PACK + (size_t)(dest * 256 + cidx) * CHUNK);
        unsigned mu = 0; float sm = 0.f;
#pragma unroll
        for (int q = 0; q < 32; ++q) {
            const u32x4 w = V[q] & 0x7fffffffu; mu = max(max(mu, w.x), max(w.y, max(w.z, w.w)));
            const f32x4 f = __builtin_bit_cast(f32x4, w); sm += (f.x + f.y) + (f.z + f.w);
        }
        const unsigned myB = min(max(min(wave_mean_exp(sm * (2.32f / 8192.f)), wave_max(mu >> 23)), 2u), 254u);
        u32x4 PK[26];
        const unsigned tail = c_encode(V, myB, PK, scr, lane);
        asm volatile("" ::: "memory");
        for (unsigned o = 0; o < tail; o += 1024u) if (o + lane * 16u < tail) store16_sys(pb + (C_TAIL + o), lane * 16u, *(const LAS u32x4*)(scr + o + lane * 16u));
#pragma unroll
        for (int q = 0; q < 26; ++q) { store16_sys(pb + q * 1024u, lane * 16u, PK[q]); asm volatile("s_waitcnt vmcnt(" STR(PK_W) ")" ::: "memory"); }
        WAIT_V(0);
        if (lane == 0) __hip_atomic_store((GAS unsigned long long*)(dws + WS_CFLAG) + rank * 256 + cidx, ((unsigned long long)call << 40) | (tail << 8) | myB, __ATOMIC_RELAXED, __HIP_MEMORY_SCOPE_SYSTEM);
    }
    }
    if (worker) fetch_outputs(p, lds, myws, rank, call, link);
}

extern "C" void kernel_launch(int rank, int nranks, ncclComm_t comm,
                              hipStream_t stream,
                              void* const* d_in, const int* in_sizes, int n_in,
                              void* d_out, int out_size,
                              const int* in_dtypes, int out_dtype,
                              void* const* d_out_all,
                              void* const* d_ws_all, size_t ws_bytes) {
    (void)comm; (void)in_dtypes; (void)out_dtype; (void)d_out_all;
    if (nranks != NR || (unsigned)rank >= (unsigned)NR || n_in < 2 || ws_bytes < WS_NEED ||
        in_sizes[0] != GM * KS || in_sizes[1] != KS * GN || out_size != (GM / NR) * GN) {
        fprintf(stderr, "gemm_rs: unexpected problem shape\n");
        return;
    }
    HIP_CHECK(hipFuncSetAttribute((const void*)gemm_rs_kernel, hipFuncAttributeMaxDynamicSharedMemorySize, (int)LDS_BYTES));
    Params p;
    p.x = (const float*)d_in[0]; p.w = (const float*)d_in[1];
    for (int i = 0; i < NR; ++i) p.peer[i] = (char*)d_ws_all[(rank + 1 + i) & (NR - 1)];
    p.myws = (char*)d_ws_all[rank]; p.out = (float*)d_out;
    p.rank = rank; p.pad = 0;
    hipLaunchKernelGGL(gemm_rs_kernel, dim3(GRID), dim3(NTHR), LDS_BYTES, stream, p);
    HIP_CHECK(hipGetLastError());
}
```
